```python
import numpy as np

kernel_name = "gpu_gemm_ar_m4096_k4096_n16384_f32_relu"


ABSMAX_RTOL = 0.001

COMPUTE_DTYPE = "float32"
EPILOGUE = "relu"
IN_QUANT = None
OUT_QUANT = None
IN_DTYPES = {"x": "float32", "w_mat": "float32"}
WORLD_SIZE = 8

def setup_inputs():

    x = np.random.randn(4096, 4096).astype(np.float32)
    w_mat = (np.random.randn(4096, 16384) / np.sqrt(4096)).astype(np.float32)
    return {"x": x, "w_mat": w_mat}

def _epilogue(y):
    return np.maximum(y, 0.0).astype(np.float32)

def reference(x, w_mat):
    y = (x @ w_mat).astype(np.float32)
    return _epilogue(y)
```

```cpp
#include <hip/hip_runtime.h>
#include <rccl/rccl.h>
#include <cstdio>

#define HIP_CHECK(expr) do { hipError_t _e = (expr); if (_e != hipSuccess) { \
    fprintf(stderr, "HIP error %s at %s:%d\n", hipGetErrorString(_e), __FILE__, __LINE__); \
    return; } } while (0)

typedef unsigned u32;
typedef unsigned long long u64;
typedef u32 u32x2 __attribute__((ext_vector_type(2)));
typedef u32 u32x4 __attribute__((ext_vector_type(4)));
typedef float f32x4 __attribute__((ext_vector_type(4)));
typedef _Float16 f16x8 __attribute__((ext_vector_type(8)));
#define LAS __attribute__((address_space(3)))
#define GAS __attribute__((address_space(1)))

extern "C" __device__ u32 llvm_writelane(u32, u32, u32) __asm("llvm.amdgcn.writelane.i32");

namespace {

constexpr int NR = 8, GM = 4096, GN = 16384, KL = 512;
constexpr int NWG = 256, NTHR = 512;
constexpr int FS = 14;
constexpr int LPP = FS / 7 * 8;
constexpr int WIN = 4;
constexpr int N_IN_ITEMS = 384, N_ITEMS = N_IN_ITEMS + 1024;
constexpr int N_JOBS = 640;
constexpr u64 DEADLINE = 30000000ull;

constexpr size_t OFF_CNT = 0;
constexpr size_t OFF_TMO = 16384;
constexpr size_t OFF_TICK = 16384 + 256;
constexpr size_t OFF_FA = 32768;
constexpr size_t OFF_FB = 49152;
constexpr size_t OFF_REC = 65536;
constexpr size_t OFF_DBG = 917504;
constexpr size_t OFF_A = 1ull << 20;
constexpr size_t OFF_B = OFF_A + (32ull << 20);
constexpr size_t OFF_BX = OFF_B + (16ull << 20);
constexpr size_t OFF_O = OFF_BX + (16ull << 20);
constexpr size_t QSLOT = 49152;
constexpr size_t WS_NEED = OFF_O + 1024 * QSLOT;
static_assert(OFF_REC + (size_t)NR * N_ITEMS * 64 <= OFF_DBG, "records overlap");

constexpr u32 RING = 65536;
constexpr u32 L_SCR = RING, SCR_B = 8448;
constexpr u32 L_DESC = L_SCR + 6 * SCR_B, NQ = 64;
constexpr u32 L_ITEM = L_DESC + NQ * 16;
constexpr u32 L_FCTL = L_ITEM + 64;
constexpr u32 STG = 49152;
constexpr u32 PK_B = 19456, PK_IMG = 8320;
constexpr u32 L_TAB = 8 * PK_B;
constexpr u32 L_ALLOC = L_TAB + 256;
constexpr u32 L_CALL = L_ALLOC + 16;
constexpr u32 LDS_BYTES = L_CALL + 48;
static_assert(L_FCTL + 64 <= L_TAB && 3 * STG <= L_TAB && LDS_BYTES <= 163840, "LDS map");

struct Params {
    const float* x; const float* w; float* out;
    char* ws[NR];
    int rank; int pad;
};

#define VMCNT(n) asm volatile("s_waitcnt vmcnt(" #n ")" ::: "memory")
#define LGKM0() asm volatile("s_waitcnt lgkmcnt(0)" ::: "memory")
#define CFENCE() asm volatile("" ::: "memory")
#define BAR() do { CFENCE(); __builtin_amdgcn_s_barrier(); CFENCE(); } while (0)

__device__ __forceinline__ void st16_sys(void* p, u32x4 v) {
    asm volatile("global_store_dwordx4 %0, %1, off sc0 sc1\n\ts_nop 1" :: "v"(p), "v"(v) : "memory");
}
__device__ __forceinline__ void st4_sys(void* p, u32 v) {
    asm volatile("global_store_dword %0, %1, off sc0 sc1" :: "v"(p), "v"(v) : "memory");
}
__device__ __forceinline__ u32 ld4_sys(const void* p) {
    u32 v; asm volatile("global_load_dword %0, %1, off sc0 sc1\n\ts_waitcnt vmcnt(0)" : "=&v"(v) : "v"(p) : "memory"); return v;
}
__device__ __forceinline__ u32x2 ld8_sys(const void* p) {
    u32x2 v; asm volatile("global_load_dwordx2 %0, %1, off sc0 sc1\n\ts_waitcnt vmcnt(0)" : "=&v"(v) : "v"(p) : "memory"); return v;
}
__device__ __forceinline__ void glds16_sys(const void* g, u32 lds) {
    u32 keep;
    asm volatile("s_mov_b32 %0, m0\n\ts_mov_b32 m0, %2\n\ts_nop 0\n\tglobal_load_lds_dwordx4 %1, off sc0 sc1\n\ts_mov_b32 m0, %0"
                 : "=&s"(keep) : "v"(g), "s"(lds) : "memory");
}
__device__ __forceinline__ void lds_st(u32 a, u32 v) { asm volatile("ds_write_b32 %0, %1" :: "v"(a), "v"(v) : "memory"); }
__device__ __forceinline__ u32 lds_ld(u32 a) {
    u32 v; asm volatile("ds_read_b32 %0, %1\n\ts_waitcnt lgkmcnt(0)" : "=&v"(v) : "v"(a) : "memory"); return v;
}
__device__ __forceinline__ u32x4 lds_ld16(u32 a) {
    u32x4 v; asm volatile("ds_read_b128 %0, %1\n\ts_waitcnt lgkmcnt(0)" : "=&v"(v) : "v"(a) : "memory"); return v;
}
__device__ __forceinline__ u32 uni(u32 v) { return __builtin_amdgcn_readfirstlane(v); }
__device__ __forceinline__ u32 lds_ldu(u32 a) { return uni(lds_ld(a)); }
__device__ __forceinline__ u64 now() { return __builtin_amdgcn_s_memrealtime(); }
__device__ __forceinline__ u32 prefix(u64 m) { return __builtin_amdgcn_mbcnt_hi((u32)(m >> 32), __builtin_amdgcn_mbcnt_lo((u32)m, 0u)); }

#define STAMP(k) do { } while (0)

__device__ __forceinline__ bool give_up(u32& spins, u64 t0, u32* tmo, u32 call) {
    if ((++spins & 63u) != 0u) return false;
    if (uni(ld4_sys(tmo)) == call) return true;
    if (now() - t0 > DEADLINE) { st4_sys(tmo, call); return true; }
    return false;
}

struct Job { int isA, pidx, j; };
__device__ __forceinline__ Job job_of(int jb) {
    Job o;
    if (jb < 576) { o.j = jb / 72; const int t = jb % 72; o.isA = t < 8; o.pidx = t < 8 ? t : t - 8; }
    else { o.j = (jb - 576) >> 3; o.isA = 1; o.pidx = 8 + ((jb - 576) & 7); }
    return o;
}
__device__ __forceinline__ void job_load(const Params& P, const Job& J, int wave, int lane, f32x4 (&v)[8]) {
    const int c = lane & 15, q = lane >> 4;
    if (J.isA) {
        const GAS float* p = (const GAS float*)P.x + (size_t)(J.pidx * 256 + wave * 32 + c) * KL + J.j * 64 + q * 4;
#pragma unroll
        for (int i = 0; i < 8; ++i) v[i] = __builtin_nontemporal_load((const GAS f32x4*)(p + (i >> 2) * 16 * KL + (i & 3) * 16));
    } else {
        const GAS float* p = (const GAS float*)P.w + (size_t)(J.j * 64 + (wave & 1) * 32 + q * 4) * GN + J.pidx * 256 + (wave >> 1) * 64 + c * 4;
#pragma unroll
        for (int i = 0; i < 8; ++i) v[i] = __builtin_nontemporal_load((const GAS f32x4*)(p + (size_t)((i >> 2) * 16 + (i & 3)) * GN));
    }
}
__device__ __forceinline__ void job_store(const Params& P, const Job& J, int wave, int lane, const f32x4 (&v)[8]) {
    char* ws = P.ws[P.rank];
    if (J.isA) {
        char* d = ws + OFF_A + (size_t)(((J.pidx * 2 + (wave >> 2)) * 8 + J.j) * 8 + P.rank) * 16384 + (wave & 3) * 4096 + lane * 16;
#pragma unroll
        for (int f = 0; f < 4; ++f) {
            f16x8 h;
#pragma unroll
            for (int e = 0; e < 8; ++e) h[e] = (_Float16)v[f * 2 + (e >> 2)][e & 3];
            st16_sys(d + f * 1024, __builtin_bit_cast(u32x4, h));
        }
    } else {
        const int owner = J.pidx >> 3, pn = J.pidx & 7;
        char* d = (owner == P.rank ? ws + OFF_B + (size_t)((pn * 8 + J.j) * 8 + P.rank) * 32768
                                   : ws + OFF_BX + (size_t)(J.pidx * 8 + J.j) * 32768) + (wave >> 1) * 8192 + (wave & 1) * 1024 + lane * 16;
#pragma unroll
        for (int i = 0; i < 4; ++i) {
            f16x8 h;
#pragma unroll
            for (int e = 0; e < 8; ++e) h[e] = (_Float16)v[e][i];
            st16_sys(d + i * 2048, __builtin_bit_cast(u32x4, h));
        }
    }
}
__device__ __forceinline__ void job_publish(const Params& P, const Job& J, int tid, u32 call) {
    char* ws = P.ws[P.rank];
    if (J.isA) {
        if (tid < 16) {
            const int p = tid >> 1, pm = J.pidx * 2 + (tid & 1);
            if (p != P.rank) {
                const int item = pm < 16 ? J.j * 32 + pm : 256 + J.j * 16 + (pm - 16);
                char* wp = P.ws[0];
#pragma unroll
                for (int i = 1; i < NR; ++i) wp = p == i ? P.ws[i] : wp;
                st4_sys(wp + OFF_REC + (size_t)(P.rank * N_ITEMS + item) * 64, call);
            } else {
                char* f = ws + OFF_FA + (size_t)(((pm * 8 + J.j) * 8 + P.rank) * 2) * 4;
                st4_sys(f, call); st4_sys(f + 4, call);
            }
        }
    } else {
        const int owner = J.pidx >> 3, pn = J.pidx & 7;
        if (owner == P.rank) { if (tid < 4) st4_sys(ws + OFF_FB + (size_t)(((pn * 8 + J.j) * 8 + P.rank) * 4 + tid) * 4, call); }
        else if (tid < 2) st4_sys(P.ws[owner & 7] + OFF_REC + (size_t)(P.rank * N_ITEMS + J.j * 32 + 16 + pn * 2 + tid) * 64, call);
    }
}

__device__ __forceinline__ void scout(const Params& P, int src, int li, int lane, u32 call) {
    char* ws = P.ws[P.rank];
    u32* tmo = (u32*)(ws + OFF_TMO);
    u32* tick = (u32*)(ws + OFF_TICK + src * 256);
    const char* rec = ws + OFF_REC + (size_t)src * N_ITEMS * 64;
    const u32 tbase = (call - 1u) * (u32)N_ITEMS - (u32)LPP;
    u32 item = li, pieces = 0, nq = 0, k = 0;
    while (item < (u32)N_ITEMS) {
        const bool isin = item < (u32)N_IN_ITEMS;
        u32x2 r; bool dead = false;
        {
            const u64 t0 = now(); u32 sp = 0;
            for (;;) {
                r = ld8_sys(rec + (size_t)item * 64 + (isin ? 0 : (lane & 7) * 8));
                if (__builtin_amdgcn_ballot_w64(r.x >= call) == __builtin_amdgcn_ballot_w64(true)) break;
                if (give_up(sp, t0, tmo, call)) { dead = true; break; }
                __builtin_amdgcn_s_sleep(2);
            }
        }
        u32 soff, bytes, nsub; u32x4 d;
        if (isin) {
            u32 dest;
            if (item < 256u && (item & 31u) >= 16u) {
                const u32 j = item >> 5, pn = ((item & 31u) - 16u) >> 1, h = item & 1u;
                soff = (u32)OFF_BX + ((P.rank * 8 + pn) * 8 + j) * 32768u + h * 16384u;
                dest = 4096u + ((pn * 8 + j) * 8 + src) * 4 + h * 2;
            } else {
                const u32 j = item < 256u ? item >> 5 : (item - 256u) >> 4, pm = item < 256u ? item & 31u : 16u + (item & 15u);
                soff = (u32)OFF_A + ((pm * 8 + j) * 8 + src) * 16384u;
                dest = ((pm * 8 + j) * 8 + src) * 2;
            }
            bytes = dead ? 0u : 16384u; nsub = 2;
            const u32 s = lane & 1;
            d.x = pieces * 1024u + (dead ? 0u : s * 8192u); d.y = pieces + (dead ? 0u : (s + 1) * 8u); d.z = (dead ? 2u : 0u) << 24; d.w = (dest + s) << 16;
        } else {
            const u32 o = item - N_IN_ITEMS;
            soff = (u32)OFF_O + o * (u32)QSLOT;
            const u32 a = __builtin_amdgcn_ds_bpermute((lane & 3) * 8, r.y), b = __builtin_amdgcn_ds_bpermute((lane & 3) * 8 + 4, r.y);
            u32 off16 = a & 0x3fffu, n = (a >> 14) & 0xfffu, id = (a >> 26) & 3u, e = b & 0xfffu, b16 = (b >> 12) & 0x7fffu;
            n = n > 2048u ? 2048u : n; e = e > 2048u ? 2048u : e;
            u32 end = off16 * 16u + ((256u + ((n + 7u) >> 3) * 26u + e * 2u + 15u) & ~15u);
            if (dead || __builtin_amdgcn_ballot_w64(end > 44032u)) { dead = true; off16 = 0; end = 0; id = lane & 3; }
            bytes = __builtin_amdgcn_readlane(end, 3); nsub = 4;
            d.x = pieces * 1024u + off16 * 16u; d.y = pieces + ((end + 1023u) >> 10); d.z = n | (e << 12) | ((dead ? 3u : 1u) << 24);
            d.w = b16 | ((o * 4u + id) << 16);
        }
        const u32 np = (bytes + 1023u) >> 10;
        {
            const u64 t0 = now(); u32 sp = 0;
            for (;;) {
                u32 nx = lds_ld(L_FCTL + 32 + (lane < 6 ? lane : 0) * 4);
                u32 qm = __builtin_amdgcn_readlane(nx, 0);
#pragma unroll
                for (int i = 1; i < 6; ++i) qm = min(qm, (u32)__builtin_amdgcn_readlane(nx, i));
                const u32 first = qm < nq ? lds_ldu(L_DESC + (qm % NQ) * 16) >> 10 : pieces;
                if (pieces + np - first <= RING / 1024 && (qm >= nq || nq + nsub - qm <= NQ)) break;
                if ((++sp & 1023u) == 0u && now() - t0 > 4 * DEADLINE) { st4_sys(tmo, call); break; }
                __builtin_amdgcn_s_sleep(1);
            }
        }
        if (lane < (int)nsub) asm volatile("ds_write_b128 %0, %1" :: "v"(L_DESC + ((nq + lane) % NQ) * 16), "v"(d) : "memory");
        nq += nsub;
        if (lane == 0) {
            u32x2 it = {soff, bytes};
            asm volatile("ds_write_b64 %0, %1" :: "v"(L_ITEM + (k & 3u) * 16), "v"(it) : "memory");
            lds_st(L_FCTL + 4, nq);
            lds_st(L_FCTL + 8, k + 1);
        }
        pieces += np; ++k;
        {
            const u64 t0 = now(); u32 sp = 0;
            while (lds_ldu(L_FCTL + 12) < k) {
                if ((++sp & 1023u) == 0u && now() - t0 > 4 * DEADLINE) { st4_sys(tmo, call); break; }
                __builtin_amdgcn_s_sleep(1);
            }
        }
        u32 t = 0;
        if (lane == 0) t = __hip_atomic_fetch_add(tick, 1u, __ATOMIC_RELAXED, __HIP_MEMORY_SCOPE_AGENT);
        item = uni(t) - tbase;
    }
    if (lane == 0) { lds_st(L_FCTL + 4, nq | 0x80000000u); lds_st(L_FCTL + 8, k | 0x80000000u); }
}

__device__ __forceinline__ void loader(const Params& P, int src, int lane, u32 call) {
    const char* base = P.ws[src] + lane * 16;
    u32* tmo = (u32*)(P.ws[P.rank] + OFF_TMO);
    u32 issued = 0, k = 0, landed = 0;
    for (;;) {
        u32 pub = lds_ldu(L_FCTL + 8);
        if ((pub & 0x7fffffffu) <= k) {
            VMCNT(0);
            landed = issued;
            if (lane == 0) lds_st(L_FCTL, issued);
            const u64 t0 = now(); u32 sp = 0;
            while ((pub & 0x7fffffffu) <= k && !(pub >> 31)) {
                if ((++sp & 1023u) == 0u && now() - t0 > 4 * DEADLINE) { st4_sys(tmo, call); VMCNT(0); return; }
                __builtin_amdgcn_s_sleep(1);
                pub = lds_ldu(L_FCTL + 8);
            }
            if ((pub & 0x7fffffffu) <= k) return;
        }
        const u32x4 it = lds_ld16(L_ITEM + (k & 3u) * 16);
        const u32 soff = uni(it.x), bytes = uni(it.y);
        ++k;
        if (lane == 0) lds_st(L_FCTL + 12, k);
        const char* s = base + soff;
        for (u32 o = 0; o < bytes; o += 1024u) {
            asm volatile("s_waitcnt vmcnt(%0)" :: "n"(WIN - 1) : "memory");
            if (issued > landed + (WIN - 1) && lane == 0) lds_st(L_FCTL, issued - (WIN - 1));
            if (o + lane * 16u < bytes) glds16_sys(s + o, (issued * 1024u) & (RING - 1));
            ++issued;
        }
    }
}

__device__ __forceinline__ void decoder(const Params& P, LAS char* L, int src, int dw, int lane, u32 call) {
    char* ws = P.ws[P.rank];
    u32* tmo = (u32*)(ws + OFF_TMO);
    LAS char* scr = L + L_SCR + dw * SCR_B;
    for (u32 q = dw;; q += 6) {
        {
            const u64 t0 = now(); u32 sp = 0;
            u32 nq = lds_ldu(L_FCTL + 4);
            while ((nq & 0x7fffffffu) <= q && !(nq >> 31)) {
                if ((++sp & 1023u) == 0u && now() - t0 > 4 * DEADLINE) { st4_sys(tmo, call); return; }
                __builtin_amdgcn_s_sleep(2);
                nq = lds_ldu(L_FCTL + 4);
            }
            if ((nq & 0x7fffffffu) <= q) return;
        }
        const u32x4 dv = lds_ld16(L_DESC + (q % NQ) * 16);
        const u32 pos = uni(dv.x), endp = uni(dv.y), z = uni(dv.z), wv = uni(dv.w);
        {
            const u64 t0 = now(); u32 sp = 0;
            while (lds_ldu(L_FCTL) < endp) {
                if ((++sp & 1023u) == 0u && now() - t0 > 4 * DEADLINE) { st4_sys(tmo, call); break; }
                __builtin_amdgcn_s_sleep(1);
            }
        }
        const u32 type = z >> 24, dest = wv >> 16;
        if (type == 0u || type == 2u) {
            if (type == 0u) {
                u32x4 v[8];
#pragma unroll
                for (int i = 0; i < 8; ++i) v[i] = *(LAS u32x4*)(L + ((pos + i * 1024u) & (RING - 1)) + lane * 16);
                LGKM0();
#pragma unroll
                for (int i = 0; i < 8; ++i) asm volatile("" : "+v"(v[i]));
                if (lane == 0) lds_st(L_FCTL + 32 + dw * 4, q + 6);
                char* d = ws + OFF_A + (size_t)dest * 8192 + lane * 16;
#pragma unroll
                for (int i = 0; i < 8; ++i) st16_sys(d + i * 1024, v[i]);
                VMCNT(0);
            } else if (lane == 0) lds_st(L_FCTL + 32 + dw * 4, q + 6);
            if (lane == 0) st4_sys(ws + OFF_FA + (size_t)dest * 4, call);
            continue;
        }
        const u32 tile = dest >> 4, wq = (dest >> 1) & 7u, h = dest & 1u;
        const u32 row0 = (tile >> 3) * 128u + (wq >> 2) * 64u + h * 32u + (lane >> 4) * 4u;
        const u32 col0 = src * 2048u + (tile & 7u) * 256u + (wq & 3u) * 64u + (lane & 15) * 4u;
        GAS float* o = (GAS float*)P.out + (size_t)row0 * GN + col0;
        if (type == 3u) {
            if (lane == 0) lds_st(L_FCTL + 32 + dw * 4, q + 6);
            const float qn = __builtin_nanf("");
#pragma unroll
            for (int g = 0; g < 8; ++g) *(GAS f32x4*)(o + (size_t)((g >> 2) * 16 + (g & 3)) * GN) = (f32x4){qn, qn, qn, qn};
            continue;
        }
        const u32 n = z & 0xfffu, e = (z >> 12) & 0xfffu, b2 = (wv & 0xffffu) * 0x10001u, n8 = (n + 7u) >> 3;
        const u32x2 mk = *(LAS u32x2*)(L + ((pos + (lane & 31) * 8u) & (RING - 1)));
        const u32 o_lo = pos + 256u, o_mid = o_lo + n8 * 16u, o_hi = o_mid + n8 * 8u, o_esc = o_hi + n8 * 2u;
        for (u32 g = lane; g < n8; g += 64u) {
            const u32x4 lo = *(LAS u32x4*)(L + ((o_lo + g * 16u) & (RING - 1)));
            const u32x2 mid = *(LAS u32x2*)(L + ((o_mid + g * 8u) & (RING - 1)));
            const u32 H = *(LAS unsigned short*)(L + ((o_hi + g * 2u) & (RING - 1)));
            const u32 T = H | (H << 12), ta = T & 0x03030303u, tb = (T >> 2) & 0x03030303u;
            const u32 u0 = __builtin_amdgcn_perm(ta, mid.x, 0x05010400u) + b2, u1 = __builtin_amdgcn_perm(ta, mid.x, 0x07030602u) + b2;
            const u32 u2 = __builtin_amdgcn_perm(tb, mid.y, 0x05010400u) + b2, u3 = __builtin_amdgcn_perm(tb, mid.y, 0x07030602u) + b2;
            u32x4 a, b;
            a.x = __builtin_amdgcn_perm(u0, lo.x, 0x05040100u); a.y = __builtin_amdgcn_perm(u0, lo.x, 0x07060302u);
            a.z = __builtin_amdgcn_perm(u1, lo.y, 0x05040100u); a.w = __builtin_amdgcn_perm(u1, lo.y, 0x07060302u);
            b.x = __builtin_amdgcn_perm(u2, lo.z, 0x05040100u); b.y = __builtin_amdgcn_perm(u2, lo.z, 0x07060302u);
            b.z = __builtin_amdgcn_perm(u3, lo.w, 0x05040100u); b.w = __builtin_amdgcn_perm(u3, lo.w, 0x07060302u);
            *(LAS u32x4*)(scr + g * 32u) = a; *(LAS u32x4*)(scr + g * 32u + 16u) = b;
        }
        CFENCE();
        for (u32 g = lane; g < e; g += 64u) {
            const u32 rr = *(LAS unsigned short*)(L + ((o_esc + g * 2u) & (RING - 1)));
            LAS u32* t = (LAS u32*)(scr + (rr >> 5) * 4u);
            *t = (*t + ((rr & 31u) << 26)) & 0x7fffffffu;
        }
        u32 ml = mk.x, mh = mk.y;
        LGKM0();
        asm volatile("" : "+v"(ml), "+v"(mh));
        if (lane == 0) lds_st(L_FCTL + 32 + dw * 4, q + 6);
        u32 basep = 0;
#pragma unroll
        for (int g = 0; g < 8; ++g) {
            f32x4 v;
#pragma unroll
            for (int i = 0; i < 4; ++i) {
                const u64 m = ((u64)(u32)__builtin_amdgcn_readlane(mh, g * 4 + i) << 32) | (u32)__builtin_amdgcn_readlane(ml, g * 4 + i);
                const u32 p = min(basep + prefix(m), 2048u);
                const float t = *(LAS float*)(scr + p * 4u);
                v[i] = __builtin_amdgcn_inverse_ballot_w64(m) ? t : 0.0f;
                basep += __builtin_popcountll(m);
            }
            __builtin_nontemporal_store(v, (GAS f32x4*)(o + (size_t)((g >> 2) * 16 + (g & 3)) * GN));
        }
        CFENCE();
    }
}

__device__ __forceinline__ void wait_group(const Params& P, int pm, int pn, int j, int lane, u32 call) {
    const char* ws = P.ws[P.rank];
    u32* tmo = (u32*)(P.ws[P.rank] + OFF_TMO);
    const char* f = lane < 16 ? ws + OFF_FA + (size_t)((pm * 8 + j) * 16 + lane) * 4 : ws + OFF_FB + (size_t)((pn * 8 + j) * 32 + ((lane - 16) & 31)) * 4;
    const u64 t0 = now(); u32 sp = 0;
    for (;;) {
        const u32 v = ld4_sys(f);
        if (__builtin_amdgcn_ballot_w64(v >= call) == __builtin_amdgcn_ballot_w64(true)) break;
        if (give_up(sp, t0, tmo, call)) break;
        __builtin_amdgcn_s_sleep(4);
    }
}

__device__ __forceinline__ void gemm_tile(const Params& P, LAS char* L, int pm, int pn, int wave, int lane, u32 call) {
    char* ws = P.ws[P.rank];
    const int wr = wave >> 2, wc = wave & 3;
    const char* gA = ws + OFF_A + (size_t)pm * 64 * 16384 + wave * 2048 + lane * 16;
    const char* gB = ws + OFF_B + (size_t)pn * 64 * 32768 + wave * 4096 + lane * 16;
    f32x4 acc[4][4];
#pragma unroll
    for (int m = 0; m < 4; ++m)
#pragma unroll
        for (int i = 0; i < 4; ++i) acc[m][i] = (f32x4){0.f, 0.f, 0.f, 0.f};
#define STAGE(t, s) do { \
        const char* a_ = gA + (size_t)(t) * 16384; const char* b_ = gB + (size_t)(t) * 32768; LAS char* l_ = L + (s) * STG; \
        _Pragma("unroll") for (int i_ = 0; i_ < 2; ++i_) __builtin_amdgcn_global_load_lds((const GAS u32*)(a_ + i_ * 1024), (LAS u32*)(l_ + wave * 2048 + i_ * 1024), 16, 0, 17); \
        _Pragma("unroll") for (int i_ = 0; i_ < 4; ++i_) __builtin_amdgcn_global_load_lds((const GAS u32*)(b_ + i_ * 1024), (LAS u32*)(l_ + 16384 + wave * 4096 + i_ * 1024), 16, 0, 17); \
    } while (0)
    if (wave == 0) wait_group(P, pm, pn, 0, lane, call);
    BAR();
    STAGE(0, 0); STAGE(1, 1);
    int s = 0;
    for (int t = 0; t < 64; ++t) {
        if (t < 63) VMCNT(6); else VMCNT(0);
        if (wave == 0 && ((t + 2) & 7) == 0 && t + 2 < 64) wait_group(P, pm, pn, (t + 2) >> 3, lane, call);
        LGKM0();
        BAR();
        const int s2 = s == 0 ? 2 : s - 1;
        if (t + 2 < 64) STAGE(t + 2, s2);
        const LAS char* la = L + s * STG + wr * 8192 + lane * 16;
        const LAS char* lb = L + s * STG + 16384 + wc * 8192 + lane * 16;
#pragma unroll
        for (int kk = 0; kk < 2; ++kk) {
            f16x8 a[4], b[4];
#pragma unroll
            for (int m = 0; m < 4; ++m) a[m] = *(const LAS f16x8*)(la + m * 2048 + kk * 1024);
#pragma unroll
            for (int i = 0; i < 4; ++i) b[i] = *(const LAS f16x8*)(lb + i * 2048 + kk * 1024);
#pragma unroll
            for (int m = 0; m < 4; ++m)
#pragma unroll
                for (int i = 0; i < 4; ++i) acc[m][i] = __builtin_amdgcn_mfma_f32_16x16x32_f16(a[m], b[i], acc[m][i], 0, 0, 0);
        }
        s = s == 2 ? 0 : s + 1;
    }
#undef STAGE
    STAMP(pm < 16 ? 4 : 6);
    if (wave == 0) {
        const u32 bad = ld4_sys(ws + OFF_TMO) == call;
        if (lane < 4) lds_st(L_ALLOC + lane * 4, 0u);
        if (lane == 0) lds_st(L_CALL + 4, bad);
    }
    LGKM0();
    BAR();
    const bool bad = lds_ldu(L_CALL + 4) != 0u;
    const int tile = pm * 8 + pn;
    LAS char* dense = L + wave * PK_B;
    LAS char* img = dense + PK_IMG;
    GAS float* o = (GAS float*)P.out + (size_t)(pm * 128 + wr * 64 + (lane >> 4) * 4) * GN + P.rank * 2048 + pn * 256 + wc * 64 + (lane & 15) * 4;
    char* og = ws + OFF_O + (size_t)(tile * 4 + (wave >> 1)) * QSLOT;
#pragma unroll
    for (int h = 0; h < 2; ++h) {
        u32 v[32]; u32 mx = 0;
#pragma unroll
        for (int g = 0; g < 8; ++g) {
#pragma unroll
            for (int i = 0; i < 4; ++i) {
                const float y = __builtin_elementwise_maximum(acc[h * 2 + (g >> 2)][i][g & 3], 0.0f);
                u32 b = __builtin_bit_cast(u32, y) & 0x7fffffffu;
                if (bad) b = 0x7fc00000u;
                v[g * 4 + i] = b; mx = max(mx, b);
            }
            const u32x4 t = {v[g * 4], v[g * 4 + 1], v[g * 4 + 2], v[g * 4 + 3]};
            __builtin_nontemporal_store(t, (GAS u32x4*)(o + (size_t)(h * 32 + (g >> 2) * 16 + (g & 3)) * GN));
        }
#pragma unroll
        for (int d = 32; d >= 1; d >>= 1) mx = max(mx, (u32)__shfl_xor((int)mx, d));
        mx = uni(mx) >> 16;
        const u32 b16 = mx >= 1023u ? mx - 1023u : 0u, B = b16 << 16;
        u32 n = 0, ml = 0, mh = 0;
#pragma unroll
        for (int r = 0; r < 32; ++r) {
            const u64 m = __builtin_amdgcn_ballot_w64(v[r] != 0u);
            if (v[r] != 0u) *(LAS u32*)(dense + (n + prefix(m)) * 4u) = (v[r] - B) & 0x7fffffffu;
            ml = llvm_writelane((u32)m, r, ml); mh = llvm_writelane((u32)(m >> 32), r, mh);
            n += __builtin_popcountll(m);
        }
        if (lane < 8) *(LAS u32*)(dense + (n + lane) * 4u) = 0u;
        if (lane < 32) *(LAS u32x2*)(img + lane * 8) = (u32x2){ml, mh};
        CFENCE();
        const u32 n8 = (n + 7u) >> 3, o_mid = 256u + n8 * 16u, o_hi = o_mid + n8 * 8u, o_esc = o_hi + n8 * 2u;
        u32 e = 0;
        for (u32 g0 = 0; g0 < n8; g0 += 64u) {
            const u32 g = g0 + lane;
            u64 W = 0;
            if (g < n8) {
                const u32x4 a = *(LAS u32x4*)(dense + g * 32u), b = *(LAS u32x4*)(dense + g * 32u + 16u);
                u32x4 lo;
                lo.x = __builtin_amdgcn_perm(a.y, a.x, 0x05040100u); lo.y = __builtin_amdgcn_perm(a.w, a.z, 0x05040100u);
                lo.z = __builtin_amdgcn_perm(b.y, b.x, 0x05040100u); lo.w = __builtin_amdgcn_perm(b.w, b.z, 0x05040100u);
                const u32 u0 = __builtin_amdgcn_perm(a.y, a.x, 0x07060302u), u1 = __builtin_amdgcn_perm(a.w, a.z, 0x07060302u);
                const u32 u2 = __builtin_amdgcn_perm(b.y, b.x, 0x07060302u), u3 = __builtin_amdgcn_perm(b.w, b.z, 0x07060302u);
                const u32x2 mid = {__builtin_amdgcn_perm(u1, u0, 0x06040200u), __builtin_amdgcn_perm(u3, u2, 0x06040200u)};
                const u32 ta = __builtin_amdgcn_perm(u1, u0, 0x07050301u), tb = __builtin_amdgcn_perm(u3, u2, 0x07050301u);
                const u32 T = (ta & 0x03030303u) | ((tb & 0x03030303u) << 2);
                *(LAS u32x4*)(img + 256u + g * 16u) = lo;
                *(LAS u32x2*)(img + o_mid + g * 8u) = mid;
                *(LAS unsigned short*)(img + o_hi + g * 2u) = (unsigned short)(T | (T >> 12));
                W = ((u64)(tb & 0x7c7c7c7cu) << 32) | (ta & 0x7c7c7c7cu);
            }
            for (;;) {
                const u64 em = __builtin_amdgcn_ballot_w64(W != 0ull);
                if (em == 0ull) break;
                if (W != 0ull) {
                    const u32 sh = (u32)__builtin_ctzll(W) & ~7u;
                    const u32 es = (u32)(W >> (sh + 2u)) & 31u;
                    *(LAS unsigned short*)(img + o_esc + (e + prefix(em)) * 2u) = (unsigned short)(((g * 8u + (sh >> 3)) << 5) | es);
                    W &= ~(0xffull << sh);
                }
                e += __builtin_popcountll(em);
            }
        }
        CFENCE();
        const u32 size16 = (o_esc + e * 2u + 15u) >> 4;
        u32 old = 0;
        if (lane == 0) {
            old = __hip_atomic_fetch_add((LAS u32*)(L + L_ALLOC + (wave >> 1) * 4), 0x10000u | size16, __ATOMIC_RELAXED, __HIP_MEMORY_SCOPE_WORKGROUP);
            const u32x4 ent = {call, (old & 0xffffu) | (n << 14) | ((u32)((wave & 1) * 2 + h) << 26), call, e | (b16 << 12)};
            *(LAS u32x4*)(L + L_TAB + ((wave >> 1) * 4 + ((old >> 16) & 3u)) * 16) = ent;
        }
        const u32 off = (uni(old) & 0xffffu) * 16u;
        for (u32 g = lane; g < size16; g += 64u) st16_sys(og + off + g * 16u, *(LAS u32x4*)(img + g * 16u));
        LGKM0();
        CFENCE();
    }
    VMCNT(0);
    LGKM0();
    BAR();
    const int tid = wave * 64 + lane;
    if (tid < 128 && (tid >> 4) != P.rank) {
        const int p = tid >> 4;
        char* wp = P.ws[0];
#pragma unroll
        for (int i = 1; i < NR; ++i) wp = p == i ? P.ws[i] : wp;
        st16_sys(wp + OFF_REC + (size_t)(P.rank * N_ITEMS + N_IN_ITEMS + tile * 4) * 64 + (tid & 15) * 16, *(LAS u32x4*)(L + L_TAB + (tid & 15) * 16));
    }
    LGKM0();
    STAMP(pm < 16 ? 5 : 7);
}

__global__ __launch_bounds__(NTHR, 2) void fused_kernel(Params P) {
    extern __shared__ __attribute__((aligned(16))) char smem[];
    LAS char* L = (LAS char*)smem;
    const int tid = threadIdx.x, lane = tid & 63, wave = __builtin_amdgcn_readfirstlane(tid >> 6);
    const int b = blockIdx.x, xg = b & 7, s = b >> 3;
    const bool fetch = s < FS;
    if (wave == 0) STAMP(0);
    f32x4 v[8];
    Job J = job_of(b);
    job_load(P, J, wave, lane, v);
    if (tid == 0) {
        const u32 c = __hip_atomic_fetch_add((u32*)(P.ws[P.rank] + OFF_CNT + b * 64), 1u, __ATOMIC_RELAXED, __HIP_MEMORY_SCOPE_AGENT) + 1u;
        lds_st(L_CALL, c);
    }
    if (fetch && tid < 16) lds_st(L_FCTL + tid * 4, tid >= 8 ? (u32)(tid - 8) : 0u);
    int jb = fetch ? N_JOBS : 256 + (s - FS) * 8 + xg;
    u32 call;
    for (;;) {
        job_store(P, J, wave, lane, v);
        const Job Jc = J;
        const bool more = jb < N_JOBS;
        VMCNT(0);
        if (more) { J = job_of(jb); job_load(P, J, wave, lane, v); jb += NWG - FS * 8; }
        LGKM0();
        BAR();
        call = lds_ldu(L_CALL);
        job_publish(P, Jc, tid, call);
        if (!more) break;
    }
    if (wave == 0) STAMP(1);
    if (fetch) {
        const int src = (P.rank + 1 + s % 7) & 7, li = (s / 7) * 8 + xg;
        if (wave == 0) { loader(P, src, lane, call); STAMP(2); }
        else if (wave == 1) scout(P, src, li, lane, call);
        else { decoder(P, L, src, wave - 2, lane, call); STAMP(8 + wave); }
    } else if (s < FS + 16) {
        const int g = s - FS, pm = (xg >> 1) * 4 + (g >> 2), pn = (xg & 1) * 4 + (g & 3);
#pragma unroll 1
        for (int bt = 0; bt < 2; ++bt) gemm_tile(P, L, pm + bt * 16, pn, wave, lane, call);
    }
}

}

extern "C" void kernel_launch(int rank, int nranks, ncclComm_t comm, hipStream_t stream,
                              void* const* d_in, const int* in_sizes, int n_in,
                              void* d_out, int out_size,
                              const int* in_dtypes, int out_dtype,
                              void* const* d_out_all,
                              void* const* d_ws_all, size_t ws_bytes) {
    (void)comm; (void)in_sizes; (void)n_in; (void)out_size; (void)in_dtypes; (void)out_dtype; (void)d_out_all;
    if (nranks != NR || (unsigned)rank >= (unsigned)NR || ws_bytes < WS_NEED) { fprintf(stderr, "unsupported configuration\n"); return; }
    static bool ready[NR];
    if (!ready[rank]) {
        HIP_CHECK(hipFuncSetAttribute((const void*)fused_kernel, hipFuncAttributeMaxDynamicSharedMemorySize, (int)LDS_BYTES));
        ready[rank] = true;
    }
    Params P;
    P.x = (const float*)d_in[0]; P.w = (const float*)d_in[1]; P.out = (float*)d_out;
    for (int i = 0; i < NR; ++i) P.ws[i] = (char*)d_ws_all[i];
    P.rank = rank; P.pad = 0;
    hipLaunchKernelGGL(fused_kernel, dim3(NWG), dim3(NTHR), LDS_BYTES, stream, P);
    HIP_CHECK(hipGetLastError());
}
```

```cpp
#include <hip/hip_runtime.h>
#include <rccl/rccl.h>
#include <cstdio>

#define HIP_CHECK(expr) do { hipError_t _e = (expr); if (_e != hipSuccess) { \
    fprintf(stderr, "HIP error %s at %s:%d\n", hipGetErrorString(_e), __FILE__, __LINE__); \
    return; } } while (0)

typedef unsigned u32;
typedef unsigned long long u64;
typedef u32 u32x2 __attribute__((ext_vector_type(2)));
typedef u32 u32x4 __attribute__((ext_vector_type(4)));
typedef float f32x4 __attribute__((ext_vector_type(4)));
typedef _Float16 f16x8 __attribute__((ext_vector_type(8)));
#define LAS __attribute__((address_space(3)))
#define GAS __attribute__((address_space(1)))

extern "C" __device__ u32 llvm_writelane(u32, u32, u32) __asm("llvm.amdgcn.writelane.i32");

namespace {

constexpr int NR = 8, GM = 4096, GN = 16384, KL = 512;
constexpr int NWG = 256, NTHR = 512;
#define WINV 8
#define AHEAD 0
#define OSTORE 2
constexpr int FS = 7, GS = 24;
constexpr int LPP = 8;
constexpr int WIN = WINV;
constexpr int NLP = 36;
constexpr int N_IN_ITEMS = 832, N_OUT_ITEMS = 512, N_ITEMS = N_IN_ITEMS + N_OUT_ITEMS;
constexpr int N_TASKS = 2560;
#define PACE1 250
#define PACE2 450
#define PACE3 120
constexpr u64 DEADLINE = 30000000ull;

constexpr size_t OFF_CNT = 0;
constexpr size_t OFF_TMO = 16384;
constexpr size_t OFF_TICK = 16384 + 256;
constexpr size_t OFF_OF = 20480;
constexpr size_t OFF_FA = 32768;
constexpr size_t OFF_FB = OFF_FA + 32768;
constexpr size_t OFF_DBG = 262144;
constexpr size_t OFF_REC = 1ull << 20;
constexpr size_t REC_PAR = 1ull << 20;
constexpr size_t OFF_A = 4ull << 20;
constexpr size_t OFF_B = OFF_A + (32ull << 20);
constexpr size_t OFF_AX = OFF_B + (72ull << 20);
constexpr size_t OFF_BX = OFF_AX + (4ull << 20);
constexpr size_t X_PAR = 20ull << 20;
constexpr size_t OFF_O = OFF_AX + 2 * X_PAR;
constexpr size_t QSLOT = 49152;
constexpr size_t WS_NEED = OFF_O + N_OUT_ITEMS * QSLOT;
static_assert((size_t)NR * N_ITEMS * 64 <= REC_PAR && OFF_FB + NLP * 2048 <= OFF_DBG, "workspace map");

constexpr u32 RING = 65536;
constexpr u32 L_SCR = RING, SCR_B = 8448;
constexpr u32 L_DESC = L_SCR + 6 * SCR_B, NQ = 64;
constexpr u32 L_ITEM = L_DESC + NQ * 16;
constexpr u32 L_FCTL = L_ITEM + 64;
constexpr u32 L_T1 = 131072, L_T2 = L_T1 + 512;
constexpr u32 PK_B = 19456, PK_IMG = 8320;
constexpr u32 LDS_BYTES = 8 * PK_B;
static_assert(L_FCTL + 64 <= LDS_BYTES && L_T2 + 512 <= LDS_BYTES && LDS_BYTES <= 163840, "LDS map");

struct Params {
    const float* x; const float* w; float* out;
    char* me;
    char* ws[NR];
    int rank; int pad;
};

#define VMCNT(n) asm volatile("s_waitcnt vmcnt(" #n ")" ::: "memory")
#define LGKM0() asm volatile("s_waitcnt lgkmcnt(0)" ::: "memory")
#define CFENCE() asm volatile("" ::: "memory")
#define BAR() do { CFENCE(); __builtin_amdgcn_s_barrier(); CFENCE(); } while (0)

__device__ __forceinline__ void st16_sys(void* p, u32x4 v) {
    asm volatile("global_store_dwordx4 %0, %1, off sc0 sc1\n\ts_nop 1" :: "v"(p), "v"(v) : "memory");
}
__device__ __forceinline__ void st_out(GAS float* p, f32x4 v) {
    asm volatile("global_store_dwordx4 %0, %1, off sc0 sc1 nt\n\ts_nop 1" :: "v"(p), "v"(v) : "memory");
}
__device__ __forceinline__ void st4_sys(void* p, u32 v) {
    asm volatile("global_store_dword %0, %1, off sc0 sc1" :: "v"(p), "v"(v) : "memory");
}
__device__ __forceinline__ void st8_sys(void* p, u32x2 v) {
    asm volatile("global_store_dwordx2 %0, %1, off sc0 sc1" :: "v"(p), "v"(v) : "memory");
}
__device__ __forceinline__ u32 ld4_sys(const void* p) {
    u32 v; asm volatile("global_load_dword %0, %1, off sc0 sc1\n\ts_waitcnt vmcnt(0)" : "=&v"(v) : "v"(p) : "memory"); return v;
}
__device__ __forceinline__ u32x2 ld8_sys(const void* p) {
    u32x2 v; asm volatile("global_load_dwordx2 %0, %1, off sc0 sc1\n\ts_waitcnt vmcnt(0)" : "=&v"(v) : "v"(p) : "memory"); return v;
}
__device__ __forceinline__ void glds16_sys(const void* g, u32 lds) {
    u32 keep;
    asm volatile("s_mov_b32 %0, m0\n\ts_mov_b32 m0, %2\n\ts_nop 0\n\tglobal_load_lds_dwordx4 %1, off sc0 sc1\n\ts_mov_b32 m0, %0"
                 : "=&s"(keep) : "v"(g), "s"(lds) : "memory");
}
__device__ __forceinline__ void lds_st(u32 a, u32 v) { asm volatile("ds_write_b32 %0, %1" :: "v"(a), "v"(v) : "memory"); }
__device__ __forceinline__ u32 lds_ld(u32 a) {
    u32 v; asm volatile("ds_read_b32 %0, %1\n\ts_waitcnt lgkmcnt(0)" : "=&v"(v) : "v"(a) : "memory"); return v;
}
__device__ __forceinline__ u32x4 lds_ld16(u32 a) {
    u32x4 v; asm volatile("ds_read_b128 %0, %1\n\ts_waitcnt lgkmcnt(0)" : "=&v"(v) : "v"(a) : "memory"); return v;
}
__device__ __forceinline__ u32 uni(u32 v) { return __builtin_amdgcn_readfirstlane(v); }
__device__ __forceinline__ u32 lds_ldu(u32 a) { return uni(lds_ld(a)); }
__device__ __forceinline__ u64 now() { return __builtin_amdgcn_s_memrealtime(); }
__device__ __forceinline__ u32 prefix(u64 m) { return __builtin_amdgcn_mbcnt_hi((u32)(m >> 32), __builtin_amdgcn_mbcnt_lo((u32)m, 0u)); }

__device__ __forceinline__ u32 wave_umax(u32 v) {
    v = max(v, (u32)__builtin_amdgcn_update_dpp(0, (int)v, 0x111, 0xf, 0xf, true));
    v = max(v, (u32)__builtin_amdgcn_update_dpp(0, (int)v, 0x112, 0xf, 0xf, true));
    v = max(v, (u32)__builtin_amdgcn_update_dpp(0, (int)v, 0x114, 0xf, 0xf, true));
    v = max(v, (u32)__builtin_amdgcn_update_dpp(0, (int)v, 0x118, 0xf, 0xf, true));
    v = max(v, (u32)__builtin_amdgcn_update_dpp(0, (int)v, 0x142, 0xa, 0xf, false));
    v = max(v, (u32)__builtin_amdgcn_update_dpp(0, (int)v, 0x143, 0xc, 0xf, false));
    return __builtin_amdgcn_readlane(v, 63);
}
typedef unsigned short u16x2 __attribute__((ext_vector_type(2)));
__device__ __forceinline__ u32 pk_add16(u32 a, u32 b) { return __builtin_bit_cast(u32, (u16x2)(__builtin_bit_cast(u16x2, a) + __builtin_bit_cast(u16x2, b))); }
__device__ __forceinline__ u32 pk_sub16(u32 a, u32 b) { return __builtin_bit_cast(u32, (u16x2)(__builtin_bit_cast(u16x2, a) - __builtin_bit_cast(u16x2, b))); }
__device__ __forceinline__ char* peer_ws(const Params& P, int p) {
    u64 wp = (u64)P.ws[0];
#pragma unroll
    for (int i = 1; i < NR; ++i) { u64 q = (u64)P.ws[i]; asm volatile("" : "+s"(q)); wp = p == i ? q : wp; }
    return (char*)wp;
}

#define STAMP(k) do { } while (0)
#define STAMPV(k, v) do { } while (0)

__device__ __forceinline__ bool give_up(u32& spins, u64 t0, u32* tmo, u32 call) {
    if ((++spins & 63u) != 0u) return false;
    if (uni(ld4_sys(tmo)) == call) return true;
    if (now() - t0 > DEADLINE) { st4_sys(tmo, call); return true; }
    return false;
}

__device__ __forceinline__ void stage_task(const Params& P, LAS char* scr, int t, int lane, GAS u32* cnt) {
    int j, idx; bool isA;
    if (t < 1792) { j = t / 224; const int u = t % 224; isA = u < 64; idx = isA ? u : (u < 192 ? u + 64 : u - 192); }
    else { const int u = t - 1792, v = u % 384; j = v / 48; isA = false; idx = 32 + (u / 384) * 48 + v % 48; }
    const int c = lane & 15, q = lane >> 4;
    f32x4 v[16];
    if (isA) {
        const GAS float* p = (const GAS float*)P.x + (size_t)(idx * 64 + c) * KL + j * 64 + q * 4;
#pragma unroll
        for (int i = 0; i < 16; ++i) v[i] = __builtin_nontemporal_load((const GAS f32x4*)(p + (i >> 2) * 16 * KL + (i & 3) * 16));
    } else {
        const GAS float* p = (const GAS float*)P.w + (size_t)(j * 64 + q * 4) * GN + idx * 64 + c * 4;
#pragma unroll
        for (int i = 0; i < 16; ++i) v[i] = __builtin_nontemporal_load((const GAS f32x4*)(p + (size_t)((i >> 2) * 16 + (i & 3)) * GN));
    }
    const u32 craw = __hip_atomic_load(cnt, __ATOMIC_RELAXED, __HIP_MEMORY_SCOPE_AGENT);
    __builtin_amdgcn_sched_barrier(0);
    float mf = 0.f;
#pragma unroll
    for (int i = 0; i < 16; ++i) mf = fmaxf(fmaxf(mf, fmaxf(fabsf(v[i][0]), fabsf(v[i][1]))), fmaxf(fabsf(v[i][2]), fabsf(v[i][3])));
    const u32 mb = wave_umax(__builtin_bit_cast(u32, mf));
    u32 code = ((mb >> 23) + 2u) >> 3; code = code < 2u ? 2u : code;
    const float sc = __builtin_bit_cast(float, (263u - 8u * code) << 23);
    const u32 hm = __builtin_bit_cast(unsigned short, (_Float16)(__builtin_bit_cast(float, mb) * sc));
    const u32 bb = uni(hm >= 8191u ? hm - 8191u : 0u), b2 = bb * 0x10001u;
    if (mb == 0u) code = 63u;
    u32x4 p[8];
    if (isA) {
#pragma unroll
        for (int f = 0; f < 8; ++f) { f16x8 h;
#pragma unroll
            for (int e = 0; e < 8; ++e) h[e] = (_Float16)(v[f * 2 + (e >> 2)][e & 3] * sc);
            p[f] = __builtin_bit_cast(u32x4, h); }
    } else {
#pragma unroll
        for (int f = 0; f < 8; ++f) { f16x8 h;
#pragma unroll
            for (int e = 0; e < 8; ++e) h[e] = (_Float16)(v[(f & 1) * 8 + e][f >> 1] * sc);
            p[f] = __builtin_bit_cast(u32x4, h); }
    }
    char* ws = P.me;
    const u32 call = uni(craw) + 1u;
    const bool all = isA || idx < 128;
    const int owner = (idx - 128) >> 4, lp = idx < 128 ? 4 + (idx >> 2) : (idx >> 2) & 3;
    const bool local = all || owner == P.rank, exprt = all || owner != P.rank;
    const u32 dest = isA ? (((idx >> 1) * 8 + j) * 8 + P.rank) * 2 + (idx & 1) : 4096 + (((lp * 8 + j) * 8 + P.rank) * 4 + (idx & 3));
    u32 e = 0;
    if (exprt) {
        char* d = ws + (call & 1u) * X_PAR + lane * 16 +
                  (isA ? OFF_AX + (size_t)(((idx >> 1) * 8 + j) * 2 + (idx & 1)) * 8192 : OFF_BX + (size_t)(((idx >> 2) * 8 + j) * 4 + (idx & 3)) * 8192);
        u32x4 E, C;
#pragma unroll
        for (int k = 0; k < 4; ++k) { const u32 d7 = pk_sub16(p[7][k], b2); E[k] = (d7 << 1) & 0xc000c000u; C[k] = (d7 & 0x1fff1fffu) | ((d7 >> 2) & 0x20002000u); }
#pragma unroll
        for (int f = 0; f < 7; ++f) { u32x4 o;
#pragma unroll
            for (int k = 0; k < 4; ++k) { const u32 df = pk_sub16(p[f][k], b2);
                E[k] |= (df >> (13 - 2 * f)) & (0x00030003u << (2 * f));
                o[k] = (df & 0x9fff9fffu) | ((C[k] << (13 - 2 * f)) & 0x60006000u); }
            st16_sys(d + f * 1024, o); }
#pragma unroll
        for (int k = 0; k < 4; ++k) {
            u32 x = E[k];
            for (;;) {
                const u64 em = __builtin_amdgcn_ballot_w64(x != 0u);
                if (em == 0ull || e >= 448u) { if (em != 0ull) e = 1023u; break; }
                if (x != 0u) {
                    const u32 bit = (u32)__builtin_ctz(x) & ~1u;
                    *(LAS unsigned short*)(scr + (e + prefix(em)) * 2u) = (unsigned short)((lane << 8) | (k << 6) | (bit << 1) | ((x >> bit) & 3u));
                    x &= ~(3u << bit);
                }
                e += __builtin_popcountll(em);
            }
        }
        CFENCE();
        if (e == 1023u) st16_sys(d + 7168, E);
        else if (lane * 8u < e) st16_sys(d + 7168, *(LAS u32x4*)(scr + lane * 16));
    }
    if (local) {
        char* d = ws + OFF_A + (size_t)dest * 8192 + lane * 16;
#pragma unroll
        for (int f = 0; f < 8; ++f) st16_sys(d + f * 1024, p[f]);
    }
    VMCNT(0);
    if (lane < NR) {
        if (lane == P.rank) { if (local) st8_sys(ws + OFF_FA + (size_t)dest * 8, (u32x2){call, code}); }
        else if (all || lane == owner) {
            const int hb = (lp % 12) * 2 + ((idx >> 1) & 1);
            const int item = isA ? j * 56 + (idx >> 1) : (lp < 12 ? j * 56 + 32 + hb : 256 + (lp / 12) * 192 + j * 24 + hb);
            st8_sys(peer_ws(P, lane) + OFF_REC + (call & 1u) * REC_PAR + (size_t)(P.rank * N_ITEMS + item) * 64 + (idx & 1) * 8,
                    (u32x2){call, e | (bb << 10) | (code << 25)});
        }
    }
    CFENCE();
}

__device__ __forceinline__ void scout(const Params& P, int src, int li, int lane, u32 call) {
    char* ws = P.me;
    u32* tmo = (u32*)(ws + OFF_TMO);
    u32* tick = (u32*)(ws + OFF_TICK + src * 256);
    const char* rec = ws + OFF_REC + (call & 1u) * REC_PAR + (size_t)src * N_ITEMS * 64;
    const u32 xpar = (call & 1u) * (u32)X_PAR;
    const u32 tbase = (call - 1u) * (u32)N_ITEMS - (u32)LPP;
    u32 item = li, pieces = 0, nq = 0, k = 0;
    while (item < (u32)N_ITEMS) {
        const bool isin = item < (u32)N_IN_ITEMS;
        u32x2 r; bool dead = false;
        u32 tk;
        if (lane == 0) asm volatile("global_atomic_add %0, %1, %2, off sc0" : "=v"(tk) : "v"(tick), "v"(1u) : "memory");
        {
            const u64 t0 = now(); u32 sp = 0;
            for (;;) {
                r = ld8_sys(rec + (size_t)item * 64 + (lane & (isin ? 1 : 7)) * 8);
                if (__builtin_amdgcn_ballot_w64(r.x >= call) == __builtin_amdgcn_ballot_w64(true)) break;
                if (give_up(sp, t0, tmo, call)) { dead = true; break; }
                __builtin_amdgcn_s_sleep(2);
            }
        }
        asm volatile("" : "+v"(tk));
        const u32 next = uni(tk) - tbase;
        u32 soff, bytes, bytes1 = 0, nsub; u32x4 d;
        if (isin) {
            u32 dest;
            u32 j, u;
            if (item < 448u) { j = item / 56u; u = item % 56u; }
            else { const u32 v = item - 448u, w = v % 192u; j = w / 24u; u = 56u + (v / 192u) * 24u + w % 24u; }
            if (u >= 32u) {
                const u32 lp = (u - 32u) >> 1, h = u & 1u, gp = lp < 4u ? 32u + P.rank * 4u + lp : lp - 4u;
                soff = (u32)OFF_BX + xpar + (gp * 8 + j) * 32768u + h * 16384u;
                dest = 4096u + ((lp * 8 + j) * 8 + src) * 4 + h * 2;
            } else {
                soff = (u32)OFF_AX + xpar + (u * 8 + j) * 16384u;
                dest = ((u * 8 + j) * 8 + src) * 2;
            }
            const u32 s = lane & 1, e = r.y & 1023u;
            const u32 sb = dead ? 0u : 7168u + (e >= 512u ? 1024u : (e * 2u + 15u) & ~15u);
            bytes = __builtin_amdgcn_readlane(sb, 0); bytes1 = __builtin_amdgcn_readlane(sb, 1); nsub = 2;
            const u32 p0 = pieces + (s ? (bytes + 1023u) >> 10 : 0u);
            d.x = p0 * 1024u; d.y = p0 + ((sb + 1023u) >> 10); d.z = dead ? (62u << 12) | (2u << 24) : e | ((r.y >> 25) << 12); d.w = ((r.y >> 10) & 0x7fffu) | ((dest + s) << 16);
        } else {
            const u32 o = item - N_IN_ITEMS;
            soff = (u32)OFF_O + o * (u32)QSLOT;
            const u32 a = __builtin_amdgcn_ds_bpermute((lane & 3) * 8, r.y), b = __builtin_amdgcn_ds_bpermute((lane & 3) * 8 + 4, r.y);
            u32 off16 = a & 0x3fffu, n = (a >> 14) & 0xfffu, e = b & 0xfffu, b16 = (b >> 12) & 0x7fffu;
            n = n > 2048u ? 2048u : n; e = e > 2048u ? 2048u : e;
            u32 end = off16 * 16u + ((256u + ((n + 7u) >> 3) * 26u + e * 2u + 15u) & ~15u);
            if (dead || __builtin_amdgcn_ballot_w64(end > 44032u)) { dead = true; off16 = 0; end = 0; }
            bytes = __builtin_amdgcn_readlane(end, 3); nsub = 4;
            d.x = pieces * 1024u + off16 * 16u; d.y = pieces + ((end + 1023u) >> 10); d.z = n | (e << 12) | ((dead ? 3u : 1u) << 24);
            d.w = b16 | ((o * 4u + (lane & 3)) << 16);
        }
        const u32 np = ((bytes + 1023u) >> 10) + ((bytes1 + 1023u) >> 10);
        {
            const u64 t0 = now(); u32 sp = 0;
            for (;;) {
                u32 nx = lds_ld(L_FCTL + 32 + (lane < 6 ? lane : 0) * 4);
                u32 qm = __builtin_amdgcn_readlane(nx, 0);
#pragma unroll
                for (int i = 1; i < 6; ++i) qm = min(qm, (u32)__builtin_amdgcn_readlane(nx, i));
                const u32 first = qm < nq ? lds_ldu(L_DESC + (qm % NQ) * 16) >> 10 : pieces;
                if (pieces + np - first <= RING / 1024 && (qm >= nq || nq + nsub - qm <= NQ)) break;
                if ((++sp & 1023u) == 0u && now() - t0 > 4 * DEADLINE) { st4_sys(tmo, call); break; }
                __builtin_amdgcn_s_sleep(1);
            }
        }
        if (lane < (int)nsub) asm volatile("ds_write_b128 %0, %1" :: "v"(L_DESC + ((nq + lane) % NQ) * 16), "v"(d) : "memory");
        nq += nsub;
        if (lane == 0) {
            const u32x4 it = {soff, bytes, bytes1, item};
            asm volatile("ds_write_b128 %0, %1" :: "v"(L_ITEM + (k & 3u) * 16), "v"(it) : "memory");
            lds_st(L_FCTL + 4, nq);
            lds_st(L_FCTL + 8, k + 1);
        }
        pieces += np; ++k;
        {
            const u64 t0 = now(); u32 sp = 0;
            while (lds_ldu(L_FCTL + 12) + AHEAD < k) {
                if ((++sp & 1023u) == 0u && now() - t0 > 4 * DEADLINE) { st4_sys(tmo, call); break; }
                __builtin_amdgcn_s_sleep(1);
            }
        }
        item = next;
    }
    if (lane == 0) { lds_st(L_FCTL + 4, nq | 0x80000000u); lds_st(L_FCTL + 8, k | 0x80000000u); }
}

__device__ __forceinline__ void loader(const Params& P, int src, int lane, u32 call) {
    const char* base = P.ws[src] + lane * 16;
    u32* tmo = (u32*)(P.me + OFF_TMO);
    u32 issued = 0, k = 0, landed = 0;
    for (;;) {
        u32 pub = lds_ldu(L_FCTL + 8);
        if ((pub & 0x7fffffffu) <= k) {
            VMCNT(0);
            landed = issued;
            if (lane == 0) lds_st(L_FCTL, issued);
            const u64 t0 = now(); u32 sp = 0;
            while ((pub & 0x7fffffffu) <= k && !(pub >> 31)) {
                if ((++sp & 1023u) == 0u && now() - t0 > 4 * DEADLINE) { st4_sys(tmo, call); VMCNT(0); return; }
                __builtin_amdgcn_s_sleep(1);
                pub = lds_ldu(L_FCTL + 8);
            }
            if ((pub & 0x7fffffffu) <= k) return;
        }
        const u32x4 it = lds_ld16(L_ITEM + (k & 3u) * 16);
        const u32 soff = uni(it.x), bytes0 = uni(it.y), bytes1 = uni(it.z);
        ++k;
        if (lane == 0) lds_st(L_FCTL + 12, k);
#pragma unroll 1
        for (int sg = 0; sg < 2; ++sg) {
            const char* s = base + soff + sg * 8192;
            const u32 bytes = sg ? bytes1 : bytes0;
            for (u32 o = 0; o < bytes; o += 1024u) {
                asm volatile("s_waitcnt vmcnt(%0)" :: "n"(WIN - 1) : "memory");
                if (issued > landed + (WIN - 1) && lane == 0) lds_st(L_FCTL, issued - (WIN - 1));
                if (o + lane * 16u < bytes) glds16_sys(s + o, (issued * 1024u) & (RING - 1));
                ++issued;
            }
        }
    }
}

__device__ __forceinline__ void decoder(const Params& P, LAS char* L, int src, int dw, int lane, u32 call) {
    char* ws = P.me;
    u32* tmo = (u32*)(ws + OFF_TMO);
    LAS char* scr = L + L_SCR + dw * SCR_B;
    for (u32 q = dw;; q += 6) {
        {
            const u64 t0 = now(); u32 sp = 0;
            u32 nq = lds_ldu(L_FCTL + 4);
            while ((nq & 0x7fffffffu) <= q && !(nq >> 31)) {
                if ((++sp & 1023u) == 0u && now() - t0 > 4 * DEADLINE) { st4_sys(tmo, call); return; }
                __builtin_amdgcn_s_sleep(2);
                nq = lds_ldu(L_FCTL + 4);
            }
            if ((nq & 0x7fffffffu) <= q) return;
        }
        const u32x4 dv = lds_ld16(L_DESC + (q % NQ) * 16);
        const u32 pos = uni(dv.x), endp = uni(dv.y), z = uni(dv.z), wv = uni(dv.w);
        {
            const u64 t0 = now(); u32 sp = 0;
            while (lds_ldu(L_FCTL) < endp) {
                if ((++sp & 1023u) == 0u && now() - t0 > 4 * DEADLINE) { st4_sys(tmo, call); break; }
                __builtin_amdgcn_s_sleep(1);
            }
        }
        const u32 type = z >> 24, dest = wv >> 16;
        if (type == 0u || type == 2u) {
            if (type == 0u) {
                const u32 e = z & 0xfffu, b2 = (wv & 0xffffu) * 0x10001u;
                u32x4 v[8], E;
#pragma unroll
                for (int i = 0; i < 7; ++i) v[i] = *(LAS u32x4*)(L + ((pos + i * 1024u) & (RING - 1)) + lane * 16);
                LAS char* er = L + ((pos + 7168u) & (RING - 1));
                if (e >= 512u) E = *(LAS u32x4*)(er + lane * 16);
                else {
                    *(LAS u32x4*)(scr + lane * 16) = (u32x4){0u, 0u, 0u, 0u};
                    CFENCE();
                    for (u32 g = lane; g < e; g += 64u) {
                        const u32 rr = *(LAS unsigned short*)(er + g * 2u);
                        __hip_atomic_fetch_or((LAS u32*)(scr + ((rr >> 6) & 255u) * 4u), (rr & 3u) << ((rr >> 1) & 30u), __ATOMIC_RELAXED, __HIP_MEMORY_SCOPE_WAVEFRONT);
                    }
                    CFENCE();
                    E = *(LAS u32x4*)(scr + lane * 16);
                }
                LGKM0();
#pragma unroll
                for (int i = 0; i < 7; ++i) asm volatile("" : "+v"(v[i]));
                asm volatile("" : "+v"(E));
                if (lane == 0) lds_st(L_FCTL + 32 + dw * 4, q + 6);
                char* d = ws + OFF_A + (size_t)dest * 8192 + lane * 16;
                u32x4 C = {0u, 0u, 0u, 0u};
#pragma unroll
                for (int f = 0; f < 7; ++f) { u32x4 o;
#pragma unroll
                    for (int k = 0; k < 4; ++k) {
                        C[k] |= (v[f][k] >> (13 - 2 * f)) & (0x00030003u << (2 * f));
                        o[k] = pk_add16((v[f][k] & 0x9fff9fffu) | ((E[k] << (13 - 2 * f)) & 0x60006000u), b2); }
                    st16_sys(d + f * 1024, o); }
                u32x4 o;
#pragma unroll
                for (int k = 0; k < 4; ++k) o[k] = pk_add16((C[k] & 0x1fff1fffu) | ((C[k] << 2) & 0x80008000u) | ((E[k] >> 1) & 0x60006000u), b2);
                st16_sys(d + 7168, o);
                VMCNT(0);
            } else if (lane == 0) lds_st(L_FCTL + 32 + dw * 4, q + 6);
            if (lane == 0) st8_sys(ws + OFF_FA + (size_t)dest * 8, (u32x2){call, (z >> 12) & 63u});
            continue;
        }
        const u32 tile = dest >> 5, wq = (dest >> 2) & 7u, su = dest & 3u;
        const u32 row0 = (tile >> 2) * 256u + (su >> 1) * 128u + (wq >> 2) * 64u + (su & 1u) * 32u + (lane >> 4) * 4u;
        const u32 col0 = 8192u + src * 1024u + (tile & 3u) * 256u + (wq & 3u) * 64u + (lane & 15) * 4u;
        GAS float* o = (GAS float*)P.out + (size_t)row0 * GN + col0;
        if (type == 3u) {
            if (lane == 0) lds_st(L_FCTL + 32 + dw * 4, q + 6);
            const float qn = __builtin_nanf("");
#pragma unroll
            for (int g = 0; g < 8; ++g) *(GAS f32x4*)(o + (size_t)((g >> 2) * 16 + (g & 3)) * GN) = (f32x4){qn, qn, qn, qn};
            continue;
        }
        const u32 n = z & 0xfffu, e = (z >> 12) & 0xfffu, b2 = (wv & 0xffffu) * 0x10001u, n8 = (n + 7u) >> 3;
        const u32x2 mk = *(LAS u32x2*)(L + ((pos + (lane & 31) * 8u) & (RING - 1)));
        const u32 o_lo = pos + 256u, o_mid = o_lo + n8 * 16u, o_hi = o_mid + n8 * 8u, o_esc = o_hi + n8 * 2u;
        for (u32 g = lane; g < n8; g += 64u) {
            const u32x4 lo = *(LAS u32x4*)(L + ((o_lo + g * 16u) & (RING - 1)));
            const u32x2 mid = *(LAS u32x2*)(L + ((o_mid + g * 8u) & (RING - 1)));
            const u32 H = *(LAS unsigned short*)(L + ((o_hi + g * 2u) & (RING - 1)));
            const u32 T = H | (H << 12), ta = T & 0x03030303u, tb = (T >> 2) & 0x03030303u;
            const u32 u0 = __builtin_amdgcn_perm(ta, mid.x, 0x05010400u) + b2, u1 = __builtin_amdgcn_perm(ta, mid.x, 0x07030602u) + b2;
            const u32 u2 = __builtin_amdgcn_perm(tb, mid.y, 0x05010400u) + b2, u3 = __builtin_amdgcn_perm(tb, mid.y, 0x07030602u) + b2;
            u32x4 a, b;
            a.x = __builtin_amdgcn_perm(u0, lo.x, 0x05040100u); a.y = __builtin_amdgcn_perm(u0, lo.x, 0x07060302u);
            a.z = __builtin_amdgcn_perm(u1, lo.y, 0x05040100u); a.w = __builtin_amdgcn_perm(u1, lo.y, 0x07060302u);
            b.x = __builtin_amdgcn_perm(u2, lo.z, 0x05040100u); b.y = __builtin_amdgcn_perm(u2, lo.z, 0x07060302u);
            b.z = __builtin_amdgcn_perm(u3, lo.w, 0x05040100u); b.w = __builtin_amdgcn_perm(u3, lo.w, 0x07060302u);
            *(LAS u32x4*)(scr + g * 32u) = a; *(LAS u32x4*)(scr + g * 32u + 16u) = b;
        }
        CFENCE();
        for (u32 g = lane; g < e; g += 64u) {
            const u32 rr = *(LAS unsigned short*)(L + ((o_esc + g * 2u) & (RING - 1)));
            LAS u32* t = (LAS u32*)(scr + (rr >> 5) * 4u);
            *t = (*t + ((rr & 31u) << 26)) & 0x7fffffffu;
        }
        u32 ml = mk.x, mh = mk.y;
        LGKM0();
        asm volatile("" : "+v"(ml), "+v"(mh));
        if (lane == 0) lds_st(L_FCTL + 32 + dw * 4, q + 6);
        u32 basep = 0;
#pragma unroll
        for (int g = 0; g < 8; ++g) {
            f32x4 v;
#pragma unroll
            for (int i = 0; i < 4; ++i) {
                const u64 m = ((u64)(u32)__builtin_amdgcn_readlane(mh, g * 4 + i) << 32) | (u32)__builtin_amdgcn_readlane(ml, g * 4 + i);
                const u32 p = min(basep + prefix(m), 2048u);
                const float t = *(LAS float*)(scr + p * 4u);
                v[i] = __builtin_amdgcn_inverse_ballot_w64(m) ? t : 0.0f;
                basep += __builtin_popcountll(m);
            }
            st_out(o + (size_t)((g >> 2) * 16 + (g & 3)) * GN, v);
        }
        CFENCE();
    }
}

__device__ __forceinline__ void gemm_wg(const Params& P, LAS char* L, int pm, int lp0, int wave, int lane, u32 call) {
    char* ws = P.me;
    const int wr = wave >> 2, wc = wave & 3;
    const u32 voffA = wave * 2048 + lane * 16, voffB = (wave >> 1) * 8192 + (wave & 1) * 2048 + lane * 16;
    const u32 aoff = wr * 8192 + lane * 16, boff = wc * 4096 + lane * 16;
    constexpr size_t KA = 16384, KB = 32768, HA = 64 * KA, HB = 4096;
#define SA(b, h) (((b) * 2 + (h)) * 16384)
#define SB(b, h) ((4 + (b) * 2 + (h)) * 16384)
#define STAGE(bufoff, gbase, voff) do { \
        __builtin_amdgcn_global_load_lds((const GAS u32*)((gbase) + (voff)), (LAS u32*)(L + (bufoff) + wave * 2048), 16, 0, 17); \
        __builtin_amdgcn_global_load_lds((const GAS u32*)((gbase) + (voff)), (LAS u32*)(L + (bufoff) + wave * 2048), 16, 1024, 17); } while (0)
#define LDA(dst, b, h) do { _Pragma("unroll") for (int m = 0; m < 4; ++m) _Pragma("unroll") for (int k = 0; k < 2; ++k) \
        dst[m][k] = *(const LAS f16x8*)(L + SA(b, h) + aoff + m * 2048 + k * 1024); } while (0)
#define LDB(dst, b, h) do { _Pragma("unroll") for (int n = 0; n < 2; ++n) _Pragma("unroll") for (int k = 0; k < 2; ++k) \
        dst[n][k] = *(const LAS f16x8*)(L + SB(b, h) + boff + n * 2048 + k * 1024); } while (0)
#define MMA(ai, bj, At, Bt) do { __builtin_amdgcn_s_setprio(1); \
        _Pragma("unroll") for (int m = 0; m < 4; ++m) _Pragma("unroll") for (int n = 0; n < 2; ++n) _Pragma("unroll") for (int k = 0; k < 2; ++k) \
            acc[ai][bj][m][n] = __builtin_amdgcn_mfma_f32_16x16x32_f16(At[m][k], Bt[n][k], acc[ai][bj][m][n], 0, 0, 0); \
        __builtin_amdgcn_s_setprio(0); } while (0)
#define ZA(ai) do { if (skip & (1u << (ai))) { _Pragma("unroll") for (int m = 0; m < 4; ++m) _Pragma("unroll") for (int k = 0; k < 2; ++k) \
        At[m][k] = (f16x8){0, 0, 0, 0, 0, 0, 0, 0}; } } while (0)
#define WAIT_L(n) asm volatile("s_waitcnt lgkmcnt(" #n ")" ::: "memory")
#define RBAR __builtin_amdgcn_s_barrier()
#define SCHED __builtin_amdgcn_sched_barrier(0)
    auto flag_ptr = [&](int lp, int j, int ent) -> const char* {
        return ent < 32 ? ws + OFF_FA + (size_t)(((pm * 2 + (ent >> 4)) * 8 + j) * 16 + (ent & 15)) * 8 : ws + OFF_FB + (size_t)((lp * 8 + j) * 32 + (ent - 32)) * 8; };
    auto poll = [&](int lp, int j) {
        const char* f = flag_ptr(lp, j, lane);
        const u64 t0 = now(); u32 sp = 0; u32x2 v;
        for (;;) {
            v = ld8_sys(f);
            if (__builtin_amdgcn_ballot_w64(v.x >= call) == ~0ull) break;
            if (give_up(sp, t0, (u32*)(ws + OFF_TMO), call)) { v.y = 62u; break; }
            __builtin_amdgcn_s_sleep(4);
        }
        asm volatile("ds_write_b64 %0, %1\n\ts_waitcnt lgkmcnt(0)" :: "v"(L_T2 + lane * 8), "v"(v) : "memory");
    };
    auto codes = [&](u32 tb) -> u32 {
        const u32 r = lane & 7;
        const u32 a0 = *(const LAS u32*)(L + tb + (r * 2 + wr) * 8 + 4) & 63u, a1 = *(const LAS u32*)(L + tb + (16 + r * 2 + wr) * 8 + 4) & 63u;
        const u32 b = *(const LAS u32*)(L + tb + (32 + r * 4 + wc) * 8 + 4) & 63u;
        const u32 c0 = (a0 == 62u || b == 62u) ? 254u : (a0 == 63u || b == 63u) ? 255u : a0 + b;
        const u32 c1 = (a1 == 62u || b == 62u) ? 254u : (a1 == 63u || b == 63u) ? 255u : a1 + b;
        return c0 | (c1 << 8);
    };
    f32x4 acc[2][2][4][2];
    f16x8 At[4][2], B0[2][2], B1[2][2];
    u32 cur = 0, cmax = 0, skip = 0, bad = 0;
#define ADJUST(r) do { const u32 cn_ = __builtin_amdgcn_readlane(pk, (r)); \
        if (cn_ != cur || skip) { skip = 0; \
            _Pragma("unroll") for (int ai = 0; ai < 2; ++ai) { \
                const u32 c_ = (cn_ >> (8 * ai)) & 255u, o_ = (cur >> (8 * ai)) & 255u, x_ = (cmax >> (8 * ai)) & 255u; \
                if (c_ == 254u) bad = 1; \
                else if (c_ != 255u && c_ != o_) { \
                    if (c_ + 10u < x_) skip |= 1u << ai;          \
                    else { const int sh_ = 8 * ((int)o_ - (int)c_); \
                        _Pragma("unroll") for (int bj = 0; bj < 2; ++bj) _Pragma("unroll") for (int m = 0; m < 4; ++m) _Pragma("unroll") for (int n = 0; n < 2; ++n) \
                            _Pragma("unroll") for (int e_ = 0; e_ < 4; ++e_) acc[ai][bj][m][n][e_] = __builtin_ldexpf(acc[ai][bj][m][n][e_], sh_); \
                        cur = (cur & ~(255u << (8 * ai))) | (c_ << (8 * ai)); \
                        if (c_ > x_) cmax = (cmax & ~(255u << (8 * ai))) | (c_ << (8 * ai)); } } } } } while (0)
#pragma unroll
    for (int a = 0; a < 2; ++a)
#pragma unroll
        for (int b = 0; b < 2; ++b)
#pragma unroll
            for (int m = 0; m < 4; ++m)
#pragma unroll
                for (int n = 0; n < 2; ++n) acc[a][b][m][n] = (f32x4){0.f, 0.f, 0.f, 0.f};
    const char* const cA = ws + OFF_A + (size_t)pm * 2 * HA;
    const char* cB = ws + OFF_B + (size_t)lp0 * 64 * KB;
    if (wave == 0) poll(lp0, 0);
    BAR();
    u32 pk = codes(L_T2), pkn = pk;
    STAGE(SB(0, 0), cB, voffB); STAGE(SA(0, 0), cA, voffA); STAGE(SB(0, 1), cB + HB, voffB); STAGE(SA(0, 1), cA + HA, voffA);
    if (wr == 1) RBAR;
    VMCNT(4); RBAR;
    STAGE(SB(1, 0), cB + KB, voffB); STAGE(SA(1, 0), cA + KA, voffA); STAGE(SB(1, 1), cB + HB + KB, voffB);
    VMCNT(6); RBAR;
#pragma unroll 1
    for (int rd = 0; rd < 3; ++rd) {
        const int lp = lp0 + rd * 12;
        const char* nB = rd < 2 ? cB + 12 * 64 * KB : cB;
#pragma unroll 1
        for (int t = 0; t < 64; t += 2) {
            const bool last = t == 62;
            const char* a1 = cA + (size_t)(t + 1) * KA;
            const char* a2 = last ? cA : cA + (size_t)(t + 2) * KA; const char* b2 = last ? nB : cB + (size_t)(t + 2) * KB;
            const char* a3 = a2 + KA; const char* b3 = b2 + KB;
            if ((t & 7) >= 4 && (t < 56 || rd < 2)) {
                const int nlp = t < 56 ? lp : lp + 12, nj = ((t >> 3) + 1) & 7;
                if ((t & 7) == 4) {
                    if (wave == 0 && lane < 32) __builtin_amdgcn_global_load_lds((const GAS u32*)flag_ptr(nlp, nj, lane * 2), (LAS u32*)(L + L_T1), 16, 0, 17);
                } else {
                    u32 tb = L_T1;
                    if (__builtin_amdgcn_ballot_w64(*(const LAS u32*)(L + L_T1 + lane * 8) >= call) != ~0ull) {
                        if (wave == 0) poll(nlp, nj);
                        BAR();
                        tb = L_T2;
                    }
                    pkn = codes(tb);
                    asm volatile("" : "+v"(pkn));
                }
            }
            if ((t & 7) == 0) pk = pkn;
            ADJUST(t & 7);
            LDB(B0, 0, 0); SCHED; LDA(At, 0, 0); STAGE(SA(1, 1), a1 + HA, voffA);
            WAIT_L(8); RBAR; WAIT_L(0); ZA(0); MMA(0, 0, At, B0); RBAR; SCHED;
            LDB(B1, 0, 1); STAGE(SB(0, 0), b2, voffB);
            RBAR; WAIT_L(0); MMA(0, 1, At, B1); RBAR;
            LDA(At, 0, 1); STAGE(SA(0, 0), a2, voffA);
            RBAR; WAIT_L(0); ZA(1); MMA(1, 0, At, B0); RBAR; SCHED;
            STAGE(SB(0, 1), b2 + HB, voffB);
            VMCNT(6); RBAR; MMA(1, 1, At, B1); RBAR;
            ADJUST((t & 7) + 1);
            LDB(B0, 1, 0); SCHED; LDA(At, 1, 0); STAGE(SA(0, 1), a2 + HA, voffA);
            WAIT_L(8); RBAR; WAIT_L(0); ZA(0); MMA(0, 0, At, B0); RBAR; SCHED;
            LDB(B1, 1, 1); STAGE(SB(1, 0), b3, voffB);
            RBAR; WAIT_L(0); MMA(0, 1, At, B1); RBAR;
            LDA(At, 1, 1); STAGE(SA(1, 0), a3, voffA);
            RBAR; WAIT_L(0); ZA(1); MMA(1, 0, At, B0); RBAR; SCHED;
            STAGE(SB(1, 1), b3 + HB, voffB);
            VMCNT(6); RBAR; MMA(1, 1, At, B1); RBAR;
        }
        if (wave == 0) STAMP(4 + rd * 2);
        {
            int ln = lane; asm volatile("" : "+v"(ln));
            const int gp = lp < 4 ? 32 + P.rank * 4 + lp : lp - 4;
            GAS float* o = (GAS float*)P.out + (size_t)(pm * 256 + wr * 64 + (ln >> 4) * 4) * GN + gp * 256 + wc * 64 + (ln & 15) * 4;
#pragma unroll
            for (int ai = 0; ai < 2; ++ai) {
                const int sh = 8 * (int)((cur >> (8 * ai)) & 255u) - 272;
#pragma unroll
                for (int m = 0; m < 4; ++m)
#pragma unroll
                    for (int r = 0; r < 4; ++r) {
                        f32x4 v;
#pragma unroll
                        for (int i = 0; i < 4; ++i) {
                            const float y = __builtin_elementwise_maximum(__builtin_ldexpf(acc[ai][i >> 1][m][i & 1][r], sh), 0.0f);
                            v[i] = __uint_as_float(bad ? 0x7fc00000u : __float_as_uint(y) & 0x7fffffffu);
                            acc[ai][i >> 1][m][i & 1][r] = 0.f;
                        }
                        st_out(o + (size_t)(ai * 128 + m * 16 + r) * GN, v);
                    }
            }
            if (lp < 4) {
                VMCNT(0);
                if (ln == 0) st4_sys(ws + OFF_OF + (size_t)((pm * 4 + lp) * 8 + wave) * 4, call);
            }
            cur = 0; cmax = 0; skip = 0; bad = 0;
        }
        if (wave == 0) STAMP(5 + rd * 2);
        cB = nB;
    }
    VMCNT(0);
    if (wr == 0) RBAR;
#undef SA
#undef SB
#undef STAGE
#undef LDA
#undef LDB
#undef MMA
#undef ZA
#undef ADJUST
}

__device__ __forceinline__ void packer(const Params& P, LAS char* L, int pw, int wave, int lane, u32 call) {
    char* ws = P.me;
    LAS char* dense = L + wave * PK_B;
    LAS char* img = dense + PK_IMG;
    const auto rs = __builtin_amdgcn_make_buffer_rsrc((void*)P.out, 0, (u32)GM * GN * 4u, 0x00020000);
#pragma unroll 1
    for (int it = pw; it < N_OUT_ITEMS; it += 64) {
        bool bad = false;
        {
            const u64 t0 = now(); u32 sp = 0;
            while (uni(ld4_sys(ws + OFF_OF + it * 4)) < call) {
                if (give_up(sp, t0, (u32*)(ws + OFF_TMO), call)) { bad = true; break; }
                __builtin_amdgcn_s_sleep(8);
            }
        }
        const int tile = it >> 3, wq = it & 7;
        const u32 voff = ((u32)((tile >> 2) * 256 + (wq >> 2) * 64 + (lane >> 4) * 4) * GN + 8192 + P.rank * 1024 + (tile & 3) * 256 + (wq & 3) * 64 + (lane & 15) * 4) * 4u;
        char* og = ws + OFF_O + (size_t)it * QSLOT;
        u32 off16 = 0, ea = 0, eb = 0;
        u32x4 nx[8];
#pragma unroll
        for (int g = 0; g < 8; ++g) nx[g] = __builtin_amdgcn_raw_buffer_load_b128(rs, voff, ((g >> 2) * 16 + (g & 3)) * GN * 4, 17);
#pragma unroll 1
        for (int su = 0; su < 4; ++su) {
            u32 v[32]; u32 mx = 0;
#pragma unroll
            for (int r = 0; r < 32; ++r) { v[r] = bad ? 0x7fc00000u : nx[r >> 2][r & 3]; mx = max(mx, v[r]); }
            {
                const int sn = su < 3 ? su + 1 : 3;
#pragma unroll
                for (int g = 0; g < 8; ++g) nx[g] = __builtin_amdgcn_raw_buffer_load_b128(rs, voff, ((sn >> 1) * 128 + (sn & 1) * 32 + (g >> 2) * 16 + (g & 3)) * GN * 4, 17);
            }
            mx = wave_umax(mx) >> 16;
            const u32 b16 = mx >= 1023u ? mx - 1023u : 0u, B = b16 << 16;
            u32 n = 0, ml = 0, mh = 0;
#pragma unroll
            for (int r = 0; r < 32; ++r) {
                const u64 m = __builtin_amdgcn_ballot_w64(v[r] != 0u);
                const u32 ad = v[r] != 0u ? (n + prefix(m)) * 4u : 8240u;
                *(LAS u32*)(dense + ad) = v[r] - B;
                ml = llvm_writelane((u32)m, r, ml); mh = llvm_writelane((u32)(m >> 32), r, mh);
                n += __builtin_popcountll(m);
            }
            if (lane < 8) *(LAS u32*)(dense + (n + lane) * 4u) = 0u;
            if (lane < 32) *(LAS u32x2*)(img + lane * 8) = (u32x2){ml, mh};
            CFENCE();
            const u32 n8 = (n + 7u) >> 3, o_mid = 256u + n8 * 16u, o_hi = o_mid + n8 * 8u, o_esc = o_hi + n8 * 2u;
            u32 e = 0;
            for (u32 g0 = 0; g0 < n8; g0 += 64u) {
                const u32 g = g0 + lane;
                u64 W = 0;
                if (g < n8) {
                    const u32x4 a = *(LAS u32x4*)(dense + g * 32u), b = *(LAS u32x4*)(dense + g * 32u + 16u);
                    u32x4 lo;
                    lo.x = __builtin_amdgcn_perm(a.y, a.x, 0x05040100u); lo.y = __builtin_amdgcn_perm(a.w, a.z, 0x05040100u);
                    lo.z = __builtin_amdgcn_perm(b.y, b.x, 0x05040100u); lo.w = __builtin_amdgcn_perm(b.w, b.z, 0x05040100u);
                    const u32 u0 = __builtin_amdgcn_perm(a.y, a.x, 0x07060302u), u1 = __builtin_amdgcn_perm(a.w, a.z, 0x07060302u);
                    const u32 u2 = __builtin_amdgcn_perm(b.y, b.x, 0x07060302u), u3 = __builtin_amdgcn_perm(b.w, b.z, 0x07060302u);
                    const u32x2 mid = {__builtin_amdgcn_perm(u1, u0, 0x06040200u), __builtin_amdgcn_perm(u3, u2, 0x06040200u)};
                    const u32 ta = __builtin_amdgcn_perm(u1, u0, 0x07050301u), tb = __builtin_amdgcn_perm(u3, u2, 0x07050301u);
                    const u32 T = (ta & 0x03030303u) | ((tb & 0x03030303u) << 2);
                    *(LAS u32x4*)(img + 256u + g * 16u) = lo;
                    *(LAS u32x2*)(img + o_mid + g * 8u) = mid;
                    *(LAS unsigned short*)(img + o_hi + g * 2u) = (unsigned short)(T | (T >> 12));
                    W = ((u64)(tb & 0x7c7c7c7cu) << 32) | (ta & 0x7c7c7c7cu);
                }
                for (;;) {
                    const u64 em = __builtin_amdgcn_ballot_w64(W != 0ull);
                    if (em == 0ull) break;
                    if (W != 0ull) {
                        const u32 sh = (u32)__builtin_ctzll(W) & ~7u;
                        const u32 es = (u32)(W >> (sh + 2u)) & 31u;
                        *(LAS unsigned short*)(img + o_esc + (e + prefix(em)) * 2u) = (unsigned short)(((g * 8u + (sh >> 3)) << 5) | es);
                        W &= ~(0xffull << sh);
                    }
                    e += __builtin_popcountll(em);
                }
            }
            CFENCE();
            const u32 size16 = (o_esc + e * 2u + 15u) >> 4;
            if ((lane & 3) == su) { ea = off16 | (n << 14); eb = e | (b16 << 12); }
            for (u32 g = lane; g < size16; g += 64u) st16_sys(og + (off16 + g) * 16u, *(LAS u32x4*)(img + g * 16u));
            off16 += size16;
            LGKM0();
            CFENCE();
        }
        VMCNT(0);
        if (lane < 32 && (lane >> 2) != P.rank)
            st16_sys(peer_ws(P, lane >> 2) + OFF_REC + (call & 1u) * REC_PAR + (size_t)(P.rank * N_ITEMS + N_IN_ITEMS + it) * 64 + (lane & 3) * 16, (u32x4){call, ea, call, eb});
        if (it < 64) STAMP(16);
    }
    STAMP(17);
}

__global__ __launch_bounds__(NTHR, 2) __attribute__((amdgpu_waves_per_eu(2, 2))) void fused_kernel(Params P) {
    extern __shared__ __attribute__((aligned(16))) char smem[];
    LAS char* L = (LAS char*)smem;
    const int tid = threadIdx.x, lane = tid & 63, wave = __builtin_amdgcn_readfirstlane(tid >> 6);
    const int b = blockIdx.x, xg = b & 7, s = b >> 3;
    const bool fetch = s < FS;
    constexpr int NF = FS * 8, NG = NWG - NF;
    if (wave == 0) STAMP(0);
    GAS u32* const cnt = (GAS u32*)(P.me + OFF_CNT + (b * 8 + wave) * 8);
    if (fetch) {
        if (tid < 16) lds_st(L_FCTL + tid * 4, tid >= 8 ? (u32)(tid - 8) : 0u);
        LGKM0();
        BAR();
    }
    if (!fetch) {
        const u64 tstart = now();
#pragma unroll 1
        for (int t = wave * NG + (b - NF); t < N_TASKS; t += 8 * NG) {
            const u64 go = t < 32 ? 0ull : (t < 224 ? (u64)PACE1 : (u64)(PACE2 + (t - 224) * PACE3 / 224));
            while (now() - tstart < go) __builtin_amdgcn_s_sleep(4);
            stage_task(P, L + wave * 1024, t, lane, cnt);
        }
    }
    const u32 call = uni(__hip_atomic_load(cnt, __ATOMIC_RELAXED, __HIP_MEMORY_SCOPE_AGENT)) + 1u;
    if (lane == 0) st4_sys((void*)cnt, call);
    STAMP(fetch ? 1 : (wave == 0 ? 3 : 18));
    if (fetch) {
        const int src = (P.rank + 1 + s) & 7;
        if (wave == 0) { loader(P, src, lane, call); STAMP(2); }
        else if (wave == 1) scout(P, src, xg, lane, call);
        else { decoder(P, L, src, wave - 2, lane, call); STAMP(8 + wave); }
    } else if (s < FS + GS) {
        const int g = s - FS;
        gemm_wg(P, L, (xg >> 1) * 4 + g / 6, (xg & 1) * 6 + g % 6, wave, lane, call);
    } else {
        BAR();
        packer(P, L, wave * 8 + xg, wave, lane, call);
    }
}

}

extern "C" void kernel_launch(int rank, int nranks, ncclComm_t comm, hipStream_t stream,
                              void* const* d_in, const int* in_sizes, int n_in,
                              void* d_out, int out_size,
                              const int* in_dtypes, int out_dtype,
                              void* const* d_out_all,
                              void* const* d_ws_all, size_t ws_bytes) {
    (void)comm; (void)in_sizes; (void)n_in; (void)out_size; (void)in_dtypes; (void)out_dtype; (void)d_out_all;
    if (nranks != NR || (unsigned)rank >= (unsigned)NR || ws_bytes < WS_NEED) { fprintf(stderr, "unsupported configuration\n"); return; }
    static bool ready[NR];
    if (!ready[rank]) {
        HIP_CHECK(hipFuncSetAttribute((const void*)fused_kernel, hipFuncAttributeMaxDynamicSharedMemorySize, (int)LDS_BYTES));
        ready[rank] = true;
    }
    Params P;
    P.x = (const float*)d_in[0]; P.w = (const float*)d_in[1]; P.out = (float*)d_out;
    for (int i = 0; i < NR; ++i) P.ws[i] = (char*)d_ws_all[i];
    P.me = P.ws[rank]; P.rank = rank; P.pad = 0;
    hipLaunchKernelGGL(fused_kernel, dim3(NWG), dim3(NTHR), LDS_BYTES, stream, P);
    HIP_CHECK(hipGetLastError());
}
```

```cpp
#include <hip/hip_runtime.h>
#include <rccl/rccl.h>
#include <cstdio>

#define HIP_CHECK(expr) do { hipError_t _e = (expr); if (_e != hipSuccess) { \
    fprintf(stderr, "HIP error %s at %s:%d\n", hipGetErrorString(_e), __FILE__, __LINE__); \
    return; } } while (0)

typedef unsigned u32;
typedef unsigned long long u64;
typedef u32 u32x2 __attribute__((ext_vector_type(2)));
typedef u32 u32x4 __attribute__((ext_vector_type(4)));
typedef float f32x4 __attribute__((ext_vector_type(4)));
typedef _Float16 f16x8 __attribute__((ext_vector_type(8)));
#define LAS __attribute__((address_space(3)))
#define GAS __attribute__((address_space(1)))

extern "C" __device__ u32 llvm_writelane(u32, u32, u32) __asm("llvm.amdgcn.writelane.i32");

namespace {

constexpr int NR = 8, GM = 4096, GN = 16384, KL = 512;
constexpr int NWG = 256, NTHR = 512;
#define WINV 8
#define GAUX 17
#define AHEAD 0
#define OSTORE 2
constexpr int FS = 7, GS = 24;
constexpr int LPP = 8;
constexpr int WIN = WINV;
constexpr int NLP = 36;
constexpr int N_IN_ITEMS = 832, N_OUT_ITEMS = 512, N_ITEMS = N_IN_ITEMS + N_OUT_ITEMS;
constexpr int N_TASKS = 2560;
#define PACE1 250
#define PACE2 450
#define PACE3 120
constexpr u64 DEADLINE = 30000000ull;

constexpr size_t OFF_CNT = 0;
constexpr size_t OFF_TMO = 16384;
constexpr size_t OFF_TICK = 16384 + 256;
constexpr size_t OFF_OF = 20480;
constexpr size_t OFF_FA = 32768;
constexpr size_t OFF_FB = OFF_FA + 32768;
constexpr size_t OFF_DBG = 262144;
constexpr size_t OFF_REC = 1ull << 20;
constexpr size_t REC_PAR = 1ull << 20;
constexpr size_t OFF_A = 4ull << 20;
constexpr size_t OFF_B = OFF_A + (32ull << 20);
constexpr size_t OFF_AX = OFF_B + (72ull << 20);
constexpr size_t OFF_BX = OFF_AX + (4ull << 20);
constexpr size_t X_PAR = 20ull << 20;
constexpr size_t OFF_O = OFF_AX + 2 * X_PAR;
constexpr size_t QSLOT = 49152;
constexpr size_t WS_NEED = OFF_O + N_OUT_ITEMS * QSLOT;
static_assert((size_t)NR * N_ITEMS * 64 <= REC_PAR && OFF_FB + NLP * 2048 <= OFF_DBG, "workspace map");

constexpr u32 RING = 65536;
constexpr u32 L_SCR = RING, SCR_B = 8448;
constexpr u32 L_DESC = L_SCR + 6 * SCR_B, NQ = 64;
constexpr u32 L_ITEM = L_DESC + NQ * 16;
constexpr u32 L_FCTL = L_ITEM + 64;
constexpr u32 L_LAND = L_FCTL + 64;
constexpr u32 L_T1 = 131072, L_T2 = L_T1 + 512, L_ZERO = L_T2 + 512;
constexpr u32 PK_B = 19456, PK_IMG = 8320;
constexpr u32 LDS_BYTES = 8 * PK_B;
static_assert(L_LAND + 256 <= LDS_BYTES && L_ZERO + 8192 <= LDS_BYTES && LDS_BYTES <= 163840, "LDS map");

struct Params {
    const float* x; const float* w; float* out;
    char* me;
    char* ws[NR];
    int rank; int pad;
};

#define VMCNT(n) asm volatile("s_waitcnt vmcnt(" #n ")" ::: "memory")
#define LGKM0() asm volatile("s_waitcnt lgkmcnt(0)" ::: "memory")
#define CFENCE() asm volatile("" ::: "memory")
#define BAR() do { CFENCE(); __builtin_amdgcn_s_barrier(); CFENCE(); } while (0)

__device__ __forceinline__ void st16_sys(void* p, u32x4 v) {
    asm volatile("global_store_dwordx4 %0, %1, off sc0 sc1\n\ts_nop 1" :: "v"(p), "v"(v) : "memory");
}
__device__ __forceinline__ void st_out(GAS float* p, f32x4 v) {
    asm volatile("global_store_dwordx4 %0, %1, off sc0 sc1 nt\n\ts_nop 1" :: "v"(p), "v"(v) : "memory");
}
__device__ __forceinline__ void st4_sys(void* p, u32 v) {
    asm volatile("global_store_dword %0, %1, off sc0 sc1" :: "v"(p), "v"(v) : "memory");
}
__device__ __forceinline__ void st8_sys(void* p, u32x2 v) {
    asm volatile("global_store_dwordx2 %0, %1, off sc0 sc1" :: "v"(p), "v"(v) : "memory");
}
__device__ __forceinline__ u32 ld4_sys(const void* p) {
    u32 v; asm volatile("global_load_dword %0, %1, off sc0 sc1\n\ts_waitcnt vmcnt(0)" : "=&v"(v) : "v"(p) : "memory"); return v;
}
__device__ __forceinline__ u32x2 ld8_sys(const void* p) {
    u32x2 v; asm volatile("global_load_dwordx2 %0, %1, off sc0 sc1\n\ts_waitcnt vmcnt(0)" : "=&v"(v) : "v"(p) : "memory"); return v;
}
__device__ __forceinline__ void lds_st(u32 a, u32 v) { asm volatile("ds_write_b32 %0, %1" :: "v"(a), "v"(v) : "memory"); }
__device__ __forceinline__ u32 lds_ld(u32 a) {
    u32 v; asm volatile("ds_read_b32 %0, %1\n\ts_waitcnt lgkmcnt(0)" : "=&v"(v) : "v"(a) : "memory"); return v;
}
__device__ __forceinline__ u32x4 lds_ld16(u32 a) {
    u32x4 v; asm volatile("ds_read_b128 %0, %1\n\ts_waitcnt lgkmcnt(0)" : "=&v"(v) : "v"(a) : "memory"); return v;
}
__device__ __forceinline__ u32 uni(u32 v) { return __builtin_amdgcn_readfirstlane(v); }
__device__ __forceinline__ u32 lds_ldu(u32 a) { return uni(lds_ld(a)); }
__device__ __forceinline__ u64 now() { return __builtin_amdgcn_s_memrealtime(); }
__device__ __forceinline__ u32 prefix(u64 m) { return __builtin_amdgcn_mbcnt_hi((u32)(m >> 32), __builtin_amdgcn_mbcnt_lo((u32)m, 0u)); }

__device__ __forceinline__ u32 wave_umax(u32 v) {
    v = max(v, (u32)__builtin_amdgcn_update_dpp(0, (int)v, 0x111, 0xf, 0xf, true));
    v = max(v, (u32)__builtin_amdgcn_update_dpp(0, (int)v, 0x112, 0xf, 0xf, true));
    v = max(v, (u32)__builtin_amdgcn_update_dpp(0, (int)v, 0x114, 0xf, 0xf, true));
    v = max(v, (u32)__builtin_amdgcn_update_dpp(0, (int)v, 0x118, 0xf, 0xf, true));
    v = max(v, (u32)__builtin_amdgcn_update_dpp(0, (int)v, 0x142, 0xa, 0xf, false));
    v = max(v, (u32)__builtin_amdgcn_update_dpp(0, (int)v, 0x143, 0xc, 0xf, false));
    return __builtin_amdgcn_readlane(v, 63);
}
typedef unsigned short u16x2 __attribute__((ext_vector_type(2)));
__device__ __forceinline__ u32 pk_add16(u32 a, u32 b) { return __builtin_bit_cast(u32, (u16x2)(__builtin_bit_cast(u16x2, a) + __builtin_bit_cast(u16x2, b))); }
__device__ __forceinline__ u32 pk_sub16(u32 a, u32 b) { return __builtin_bit_cast(u32, (u16x2)(__builtin_bit_cast(u16x2, a) - __builtin_bit_cast(u16x2, b))); }
__device__ __forceinline__ char* peer_ws(const Params& P, int p) {
    u64 wp = (u64)P.ws[0];
#pragma unroll
    for (int i = 1; i < NR; ++i) { u64 q = (u64)P.ws[i]; asm volatile("" : "+s"(q)); wp = p == i ? q : wp; }
    return (char*)wp;
}

#define STAMP(k) do { } while (0)
#define STAMPV(k, v) do { } while (0)

__device__ __forceinline__ bool give_up(u32& spins, u64 t0, u32* tmo, u32 call) {
    if ((++spins & 63u) != 0u) return false;
    if (uni(ld4_sys(tmo)) == call) return true;
    if (now() - t0 > DEADLINE) { st4_sys(tmo, call); return true; }
    return false;
}

__device__ __forceinline__ void stage_task(const Params& P, LAS char* scr, int t, int lane, GAS u32* cnt) {
    int j, idx; bool isA;
    if (t < 1792) { j = t / 224; const int u = t % 224; isA = u < 64; idx = isA ? u : (u < 192 ? u + 64 : u - 192); }
    else { const int u = t - 1792, v = u % 384; j = v / 48; isA = false; idx = 32 + (u / 384) * 48 + v % 48; }
    const int c = lane & 15, q = lane >> 4;
    f32x4 v[16];
    if (isA) {
        const GAS float* p = (const GAS float*)P.x + (size_t)(idx * 64 + c) * KL + j * 64 + q * 4;
#pragma unroll
        for (int i = 0; i < 16; ++i) v[i] = __builtin_nontemporal_load((const GAS f32x4*)(p + (i >> 2) * 16 * KL + (i & 3) * 16));
    } else {
        const GAS float* p = (const GAS float*)P.w + (size_t)(j * 64 + q * 4) * GN + idx * 64 + c * 4;
#pragma unroll
        for (int i = 0; i < 16; ++i) v[i] = __builtin_nontemporal_load((const GAS f32x4*)(p + (size_t)((i >> 2) * 16 + (i & 3)) * GN));
    }
    const u32 craw = __hip_atomic_load(cnt, __ATOMIC_RELAXED, __HIP_MEMORY_SCOPE_AGENT);
    __builtin_amdgcn_sched_barrier(0);
    float mf = 0.f;
#pragma unroll
    for (int i = 0; i < 16; ++i) mf = fmaxf(fmaxf(mf, fmaxf(fabsf(v[i][0]), fabsf(v[i][1]))), fmaxf(fabsf(v[i][2]), fabsf(v[i][3])));
    const u32 mb = wave_umax(__builtin_bit_cast(u32, mf));
    u32 code = ((mb >> 23) + 2u) >> 3; code = code < 2u ? 2u : code;
    const float sc = __builtin_bit_cast(float, (263u - 8u * code) << 23);
    const u32 hm = __builtin_bit_cast(unsigned short, (_Float16)(__builtin_bit_cast(float, mb) * sc));
    const u32 bb = uni(hm >= 8191u ? hm - 8191u : 0u), b2 = bb * 0x10001u;
    if (mb == 0u) code = 63u;
    u32x4 p[8];
    if (isA) {
#pragma unroll
        for (int f = 0; f < 8; ++f) { f16x8 h;
#pragma unroll
            for (int e = 0; e < 8; ++e) h[e] = (_Float16)(v[f * 2 + (e >> 2)][e & 3] * sc);
            p[f] = __builtin_bit_cast(u32x4, h); }
    } else {
#pragma unroll
        for (int f = 0; f < 8; ++f) { f16x8 h;
#pragma unroll
            for (int e = 0; e < 8; ++e) h[e] = (_Float16)(v[(f & 1) * 8 + e][f >> 1] * sc);
            p[f] = __builtin_bit_cast(u32x4, h); }
    }
    char* ws = P.me;
    const u32 call = uni(craw) + 1u;
    const bool all = isA || idx < 128;
    const int owner = (idx - 128) >> 4, lp = idx < 128 ? 4 + (idx >> 2) : (idx >> 2) & 3;
    const bool local = all || owner == P.rank, exprt = all || owner != P.rank;
    const u32 dest = isA ? (((idx >> 1) * 8 + j) * 8 + P.rank) * 2 + (idx & 1) : 4096 + (((lp * 8 + j) * 8 + P.rank) * 4 + (idx & 3));
    u32 e = 0;
    if (exprt) {
        char* d = ws + (call & 1u) * X_PAR + lane * 16 +
                  (isA ? OFF_AX + (size_t)(((idx >> 1) * 8 + j) * 2 + (idx & 1)) * 8192 : OFF_BX + (size_t)(((idx >> 2) * 8 + j) * 4 + (idx & 3)) * 8192);
        u32x4 E, C;
#pragma unroll
        for (int k = 0; k < 4; ++k) { const u32 d7 = pk_sub16(p[7][k], b2); E[k] = (d7 << 1) & 0xc000c000u; C[k] = (d7 & 0x1fff1fffu) | ((d7 >> 2) & 0x20002000u); }
#pragma unroll
        for (int f = 0; f < 7; ++f) { u32x4 o;
#pragma unroll
            for (int k = 0; k < 4; ++k) { const u32 df = pk_sub16(p[f][k], b2);
                E[k] |= (df >> (13 - 2 * f)) & (0x00030003u << (2 * f));
                o[k] = (df & 0x9fff9fffu) | ((C[k] << (13 - 2 * f)) & 0x60006000u); }
            st16_sys(d + f * 1024, o); }
#pragma unroll
        for (int k = 0; k < 4; ++k) {
            u32 x = E[k];
            for (;;) {
                const u64 em = __builtin_amdgcn_ballot_w64(x != 0u);
                if (em == 0ull || e >= 448u) { if (em != 0ull) e = 1023u; break; }
                if (x != 0u) {
                    const u32 bit = (u32)__builtin_ctz(x) & ~1u;
                    *(LAS unsigned short*)(scr + (e + prefix(em)) * 2u) = (unsigned short)((lane << 8) | (k << 6) | (bit << 1) | ((x >> bit) & 3u));
                    x &= ~(3u << bit);
                }
                e += __builtin_popcountll(em);
            }
        }
        CFENCE();
        if (e == 1023u) st16_sys(d + 7168, E);
        else if (lane * 8u < e) st16_sys(d + 7168, *(LAS u32x4*)(scr + lane * 16));
    }
    if (local) {
        char* d = ws + OFF_A + (size_t)dest * 8192 + lane * 16;
#pragma unroll
        for (int f = 0; f < 8; ++f) st16_sys(d + f * 1024, p[f]);
    }
    VMCNT(0);
    if (lane < NR) {
        if (lane == P.rank) { if (local) st8_sys(ws + OFF_FA + (size_t)dest * 8, (u32x2){call, code}); }
        else if (all || lane == owner) {
            const int hb = (lp % 12) * 2 + ((idx >> 1) & 1);
            const int item = isA ? j * 56 + (idx >> 1) : (lp < 12 ? j * 56 + 32 + hb : 256 + (lp / 12) * 192 + j * 24 + hb);
            st8_sys(peer_ws(P, lane) + OFF_REC + (call & 1u) * REC_PAR + (size_t)(P.rank * N_ITEMS + item) * 64 + (idx & 1) * 8,
                    (u32x2){call, e | (bb << 10) | (code << 25)});
        }
    }
    CFENCE();
}

__device__ __forceinline__ void scout(const Params& P, int src, int li, int lane, u32 call) {
    char* ws = P.me;
    u32* tmo = (u32*)(ws + OFF_TMO);
    u32* tick = (u32*)(ws + OFF_TICK + src * 256);
    const char* rec = ws + OFF_REC + (call & 1u) * REC_PAR + (size_t)src * N_ITEMS * 64;
    const u32 xpar = (call & 1u) * (u32)X_PAR;
    const u32 tbase = (call - 1u) * (u32)N_ITEMS - (u32)LPP;
    u32 item = li, pieces = 0, nq = 0, k = 0;
    __builtin_amdgcn_s_setprio(2);
    while (item < (u32)N_ITEMS) {
        const bool isin = item < (u32)N_IN_ITEMS;
        u32x2 r; bool dead = false;
        u32 tk;
        if (lane == 0) asm volatile("global_atomic_add %0, %1, %2, off sc0" : "=v"(tk) : "v"(tick), "v"(1u) : "memory");
        {
            const u64 t0 = now(); u32 sp = 0;
            for (;;) {
                r = ld8_sys(rec + (size_t)item * 64 + (lane & (isin ? 1 : 7)) * 8);
                if (__builtin_amdgcn_ballot_w64(r.x >= call) == __builtin_amdgcn_ballot_w64(true)) break;
                if (give_up(sp, t0, tmo, call)) { dead = true; break; }
                __builtin_amdgcn_s_sleep(2);
            }
        }
        asm volatile("" : "+v"(tk));
        const u32 next = uni(tk) - tbase;
        u32 soff, bytes, bytes1 = 0, nsub; u32x4 d;
        if (isin) {
            u32 dest;
            u32 j, u;
            if (item < 448u) { j = item / 56u; u = item % 56u; }
            else { const u32 v = item - 448u, w = v % 192u; j = w / 24u; u = 56u + (v / 192u) * 24u + w % 24u; }
            if (u >= 32u) {
                const u32 lp = (u - 32u) >> 1, h = u & 1u, gp = lp < 4u ? 32u + P.rank * 4u + lp : lp - 4u;
                soff = (u32)OFF_BX + xpar + (gp * 8 + j) * 32768u + h * 16384u;
                dest = 4096u + ((lp * 8 + j) * 8 + src) * 4 + h * 2;
            } else {
                soff = (u32)OFF_AX + xpar + (u * 8 + j) * 16384u;
                dest = ((u * 8 + j) * 8 + src) * 2;
            }
            const u32 s = lane & 1, e = r.y & 1023u;
            const u32 sb = dead ? 0u : 7168u + (e >= 512u ? 1024u : (e * 2u + 15u) & ~15u);
            bytes = __builtin_amdgcn_readlane(sb, 0); bytes1 = __builtin_amdgcn_readlane(sb, 1); nsub = 2;
            const u32 p0 = pieces + (s ? (bytes + 1023u) >> 10 : 0u);
            d.x = p0 * 1024u; d.y = p0 + ((sb + 1023u) >> 10); d.z = dead ? (62u << 12) | (2u << 24) : e | ((r.y >> 25) << 12); d.w = ((r.y >> 10) & 0x7fffu) | ((dest + s) << 16);
        } else {
            const u32 o = item - N_IN_ITEMS;
            soff = (u32)OFF_O + o * (u32)QSLOT;
            const u32 a = __builtin_amdgcn_ds_bpermute((lane & 3) * 8, r.y), b = __builtin_amdgcn_ds_bpermute((lane & 3) * 8 + 4, r.y);
            u32 off16 = a & 0x3fffu, n = (a >> 14) & 0xfffu, e = b & 0xfffu, b16 = (b >> 12) & 0x7fffu;
            n = n > 2048u ? 2048u : n; e = e > 2048u ? 2048u : e;
            u32 end = off16 * 16u + ((256u + ((n + 7u) >> 3) * 26u + e * 2u + 15u) & ~15u);
            if (dead || __builtin_amdgcn_ballot_w64(end > 44032u)) { dead = true; off16 = 0; end = 0; }
            bytes = __builtin_amdgcn_readlane(end, 3); nsub = 4;
            d.x = pieces * 1024u + off16 * 16u; d.y = pieces + ((end + 1023u) >> 10); d.z = n | (e << 12) | ((dead ? 3u : 1u) << 24);
            d.w = b16 | ((o * 4u + (lane & 3)) << 16);
        }
        const u32 np = ((bytes + 1023u) >> 10) + ((bytes1 + 1023u) >> 10);
        {
            const u64 t0 = now(); u32 sp = 0;
            for (;;) {
                u32 nx = lds_ld(L_FCTL + (lane < 6 ? 32 + lane * 4 : 12));
                u32 qm = __builtin_amdgcn_readlane(nx, 0);
#pragma unroll
                for (int i = 1; i < 6; ++i) qm = min(qm, (u32)__builtin_amdgcn_readlane(nx, i));
                const u32 first = qm < nq ? lds_ldu(L_DESC + (qm % NQ) * 16) >> 10 : pieces;
                if ((u32)__builtin_amdgcn_readlane(nx, 6) + AHEAD >= k && pieces + np - first <= RING / 1024 && (qm >= nq || nq + nsub - qm <= NQ)) break;
                if ((++sp & 1023u) == 0u && now() - t0 > 4 * DEADLINE) { st4_sys(tmo, call); break; }
                __builtin_amdgcn_s_sleep(1);
            }
        }
        if (lane < (int)nsub) asm volatile("ds_write_b128 %0, %1" :: "v"(L_DESC + ((nq + lane) % NQ) * 16), "v"(d) : "memory");
        nq += nsub;
        if (lane == 0) {
            const u32x4 it = {soff, bytes, bytes1, k + 1};
            asm volatile("ds_write_b128 %0, %1" :: "v"(L_ITEM + (k & 3u) * 16), "v"(it) : "memory");
            lds_st(L_FCTL + 4, nq);
        }
        pieces += np; ++k;
        item = next;
    }
    if (lane == 0) {
        const u32x4 it = {0u, 0xffffffffu, 0u, k + 1};
        asm volatile("ds_write_b128 %0, %1" :: "v"(L_ITEM + (k & 3u) * 16), "v"(it) : "memory");
        lds_st(L_FCTL + 4, nq | 0x80000000u);
    }
}

__device__ __forceinline__ void loader(const Params& P, int src, int lane, u32 call) {
    const u64 base = (u64)P.ws[src];
    const u32 voff = lane * 16, vland = L_LAND + lane * 4;
    u32* tmo = (u32*)(P.me + OFF_TMO);
    int issued = 0, landed = 0;
    u32 k = 0;
    __builtin_amdgcn_s_setprio(3);
#define PIECE(sp) do { asm volatile("s_mov_b32 m0, %2\n\ts_waitcnt vmcnt(%3)\n\tglobal_load_lds_dwordx4 %0, %1 sc0 sc1" \
                                    :: "v"(voff), "s"(sp), "s"(((u32)issued * 1024u) & (RING - 1)), "n"(WIN - 1) : "memory"); } while (0)
    for (;;) {
        u32x4 it = lds_ld16(L_ITEM + (k & 3u) * 16);
        if (uni(it.w) != k + 1 || uni(it.y) == 0xffffffffu) {
            VMCNT(0);
            landed = issued;
            lds_st(vland, (u32)issued);
            const u64 t0 = now(); u32 sp = 0;
            while (uni(it.w) != k + 1) {
                if ((++sp & 1023u) == 0u && now() - t0 > 4 * DEADLINE) { st4_sys(tmo, call); return; }
                __builtin_amdgcn_s_sleep(1);
                it = lds_ld16(L_ITEM + (k & 3u) * 16);
            }
            if (uni(it.y) == 0xffffffffu) return;
        }
        const u32 soff = uni(it.x), bytes0 = uni(it.y), bytes1 = uni(it.z);
        ++k;
        if (lane == 0) lds_st(L_FCTL + 12, k);
#pragma unroll 1
        for (int sg = 0; sg < 2; ++sg) {
            u64 sp = base + soff + sg * 8192;
            u32 left = sg ? bytes1 : bytes0;
#pragma unroll 1
            for (; left >= 1024u; left -= 1024u, sp += 1024) {
                PIECE(sp);
                lds_st(vland, (u32)max(landed, issued - (WIN - 1)));
                ++issued;
            }
            if (left) {
                if (voff < left) PIECE(sp);
                lds_st(vland, (u32)max(landed, issued - (WIN - 1)));
                ++issued;
            }
        }
    }
#undef PIECE
}

__device__ __forceinline__ void decoder(const Params& P, LAS char* L, int src, int dw, int lane, u32 call) {
    char* ws = P.me;
    u32* tmo = (u32*)(ws + OFF_TMO);
    LAS char* scr = L + L_SCR + dw * SCR_B;
    for (u32 q = dw;; q += 6) {
        {
            const u64 t0 = now(); u32 sp = 0;
            u32 nq = lds_ldu(L_FCTL + 4);
            while ((nq & 0x7fffffffu) <= q && !(nq >> 31)) {
                if ((++sp & 1023u) == 0u && now() - t0 > 4 * DEADLINE) { st4_sys(tmo, call); return; }
                __builtin_amdgcn_s_sleep(2);
                nq = lds_ldu(L_FCTL + 4);
            }
            if ((nq & 0x7fffffffu) <= q) return;
        }
        const u32x4 dv = lds_ld16(L_DESC + (q % NQ) * 16);
        const u32 pos = uni(dv.x), endp = uni(dv.y), z = uni(dv.z), wv = uni(dv.w);
        {
            const u64 t0 = now(); u32 sp = 0;
            while (lds_ldu(L_LAND) < endp) {
                if ((++sp & 1023u) == 0u && now() - t0 > 4 * DEADLINE) { st4_sys(tmo, call); break; }
                __builtin_amdgcn_s_sleep(1);
            }
        }
        const u32 type = z >> 24, dest = wv >> 16;
        if (type == 0u || type == 2u) {
            if (type == 0u) {
                const u32 e = z & 0xfffu, b2 = (wv & 0xffffu) * 0x10001u;
                u32x4 v[8], E;
#pragma unroll
                for (int i = 0; i < 7; ++i) v[i] = *(LAS u32x4*)(L + ((pos + i * 1024u) & (RING - 1)) + lane * 16);
                LAS char* er = L + ((pos + 7168u) & (RING - 1));
                if (e >= 512u) E = *(LAS u32x4*)(er + lane * 16);
                else {
                    *(LAS u32x4*)(scr + lane * 16) = (u32x4){0u, 0u, 0u, 0u};
                    CFENCE();
                    for (u32 g = lane; g < e; g += 64u) {
                        const u32 rr = *(LAS unsigned short*)(er + g * 2u);
                        __hip_atomic_fetch_or((LAS u32*)(scr + ((rr >> 6) & 255u) * 4u), (rr & 3u) << ((rr >> 1) & 30u), __ATOMIC_RELAXED, __HIP_MEMORY_SCOPE_WAVEFRONT);
                    }
                    CFENCE();
                    E = *(LAS u32x4*)(scr + lane * 16);
                }
                LGKM0();
#pragma unroll
                for (int i = 0; i < 7; ++i) asm volatile("" : "+v"(v[i]));
                asm volatile("" : "+v"(E));
                if (lane == 0) lds_st(L_FCTL + 32 + dw * 4, q + 6);
                char* d = ws + OFF_A + (size_t)dest * 8192 + lane * 16;
                u32x4 C = {0u, 0u, 0u, 0u};
#pragma unroll
                for (int f = 0; f < 7; ++f) { u32x4 o;
#pragma unroll
                    for (int k = 0; k < 4; ++k) {
                        C[k] |= (v[f][k] >> (13 - 2 * f)) & (0x00030003u << (2 * f));
                        o[k] = pk_add16((v[f][k] & 0x9fff9fffu) | ((E[k] << (13 - 2 * f)) & 0x60006000u), b2); }
                    st16_sys(d + f * 1024, o); }
                u32x4 o;
#pragma unroll
                for (int k = 0; k < 4; ++k) o[k] = pk_add16((C[k] & 0x1fff1fffu) | ((C[k] << 2) & 0x80008000u) | ((E[k] >> 1) & 0x60006000u), b2);
                st16_sys(d + 7168, o);
                VMCNT(0);
            } else if (lane == 0) lds_st(L_FCTL + 32 + dw * 4, q + 6);
            if (lane == 0) st8_sys(ws + OFF_FA + (size_t)dest * 8, (u32x2){call, (z >> 12) & 63u});
            continue;
        }
        const u32 tile = dest >> 5, wq = (dest >> 2) & 7u, su = dest & 3u;
        const u32 row0 = (tile >> 2) * 256u + (su >> 1) * 128u + (wq >> 2) * 64u + (su & 1u) * 32u + (lane >> 4) * 4u;
        const u32 col0 = 8192u + src * 1024u + (tile & 3u) * 256u + (wq & 3u) * 64u + (lane & 15) * 4u;
        GAS float* o = (GAS float*)P.out + (size_t)row0 * GN + col0;
        if (type == 3u) {
            if (lane == 0) lds_st(L_FCTL + 32 + dw * 4, q + 6);
            const float qn = __builtin_nanf("");
#pragma unroll
            for (int g = 0; g < 8; ++g) *(GAS f32x4*)(o + (size_t)((g >> 2) * 16 + (g & 3)) * GN) = (f32x4){qn, qn, qn, qn};
            continue;
        }
        const u32 n = z & 0xfffu, e = (z >> 12) & 0xfffu, b2 = (wv & 0xffffu) * 0x10001u, n8 = (n + 7u) >> 3;
        const u32x2 mk = *(LAS u32x2*)(L + ((pos + (lane & 31) * 8u) & (RING - 1)));
        const u32 o_lo = pos + 256u, o_mid = o_lo + n8 * 16u, o_hi = o_mid + n8 * 8u, o_esc = o_hi + n8 * 2u;
        for (u32 g = lane; g < n8; g += 64u) {
            const u32x4 lo = *(LAS u32x4*)(L + ((o_lo + g * 16u) & (RING - 1)));
            const u32x2 mid = *(LAS u32x2*)(L + ((o_mid + g * 8u) & (RING - 1)));
            const u32 H = *(LAS unsigned short*)(L + ((o_hi + g * 2u) & (RING - 1)));
            const u32 T = H | (H << 12), ta = T & 0x03030303u, tb = (T >> 2) & 0x03030303u;
            const u32 u0 = __builtin_amdgcn_perm(ta, mid.x, 0x05010400u) + b2, u1 = __builtin_amdgcn_perm(ta, mid.x, 0x07030602u) + b2;
            const u32 u2 = __builtin_amdgcn_perm(tb, mid.y, 0x05010400u) + b2, u3 = __builtin_amdgcn_perm(tb, mid.y, 0x07030602u) + b2;
            u32x4 a, b;
            a.x = __builtin_amdgcn_perm(u0, lo.x, 0x05040100u); a.y = __builtin_amdgcn_perm(u0, lo.x, 0x07060302u);
            a.z = __builtin_amdgcn_perm(u1, lo.y, 0x05040100u); a.w = __builtin_amdgcn_perm(u1, lo.y, 0x07060302u);
            b.x = __builtin_amdgcn_perm(u2, lo.z, 0x05040100u); b.y = __builtin_amdgcn_perm(u2, lo.z, 0x07060302u);
            b.z = __builtin_amdgcn_perm(u3, lo.w, 0x05040100u); b.w = __builtin_amdgcn_perm(u3, lo.w, 0x07060302u);
            *(LAS u32x4*)(scr + g * 32u) = a; *(LAS u32x4*)(scr + g * 32u + 16u) = b;
        }
        CFENCE();
        for (u32 g = lane; g < e; g += 64u) {
            const u32 rr = *(LAS unsigned short*)(L + ((o_esc + g * 2u) & (RING - 1)));
            LAS u32* t = (LAS u32*)(scr + (rr >> 5) * 4u);
            *t = (*t + ((rr & 31u) << 26)) & 0x7fffffffu;
        }
        u32 ml = mk.x, mh = mk.y;
        LGKM0();
        asm volatile("" : "+v"(ml), "+v"(mh));
        if (lane == 0) lds_st(L_FCTL + 32 + dw * 4, q + 6);
        u32 basep = 0;
#pragma unroll
        for (int g = 0; g < 8; ++g) {
            f32x4 v;
#pragma unroll
            for (int i = 0; i < 4; ++i) {
                const u64 m = ((u64)(u32)__builtin_amdgcn_readlane(mh, g * 4 + i) << 32) | (u32)__builtin_amdgcn_readlane(ml, g * 4 + i);
                const u32 p = min(basep + prefix(m), 2048u);
                const float t = *(LAS float*)(scr + p * 4u);
                v[i] = __builtin_amdgcn_inverse_ballot_w64(m) ? t : 0.0f;
                basep += __builtin_popcountll(m);
            }
            st_out(o + (size_t)((g >> 2) * 16 + (g & 3)) * GN, v);
        }
        CFENCE();
    }
}

__device__ __forceinline__ void gemm_wg(const Params& P, LAS char* L, int pm, int lp0, int wave, int lane, u32 call) {
    char* ws = P.me;
    const int wr = wave >> 2, wc = wave & 3;
    const u32 voffA = wave * 2048 + lane * 16, voffB = (wave >> 1) * 8192 + (wave & 1) * 2048 + lane * 16;
    const u32 aoff = wr * 8192 + lane * 16, boff = wc * 4096 + lane * 16;
    constexpr size_t KA = 16384, KB = 32768, HA = 64 * KA, HB = 4096;
#define SA(b, h) (((b) * 2 + (h)) * 16384)
#define SB(b, h) ((4 + (b) * 2 + (h)) * 16384)
#define STAGE(bufoff, gbase, voff) do { \
        __builtin_amdgcn_global_load_lds((const GAS u32*)((gbase) + (voff)), (LAS u32*)(L + (bufoff) + wave * 2048), 16, 0, GAUX); \
        __builtin_amdgcn_global_load_lds((const GAS u32*)((gbase) + (voff)), (LAS u32*)(L + (bufoff) + wave * 2048), 16, 1024, GAUX); } while (0)
#define LDA(dst, b, h) do { _Pragma("unroll") for (int m = 0; m < 4; ++m) _Pragma("unroll") for (int k = 0; k < 2; ++k) \
        dst[m][k] = *(const LAS f16x8*)(L + SA(b, h) + ((h) ? aoff1 : aoff0) + m * 2048 + k * 1024); } while (0)
#define LDB(dst, b, h) do { _Pragma("unroll") for (int n = 0; n < 2; ++n) _Pragma("unroll") for (int k = 0; k < 2; ++k) \
        dst[n][k] = *(const LAS f16x8*)(L + SB(b, h) + boff + n * 2048 + k * 1024); } while (0)
#define MMA(ai, bj, At, Bt) do { __builtin_amdgcn_s_setprio(1); \
        _Pragma("unroll") for (int m = 0; m < 4; ++m) _Pragma("unroll") for (int n = 0; n < 2; ++n) _Pragma("unroll") for (int k = 0; k < 2; ++k) \
            acc[ai][bj][m][n] = __builtin_amdgcn_mfma_f32_16x16x32_f16(At[m][k], Bt[n][k], acc[ai][bj][m][n], 0, 0, 0); \
        __builtin_amdgcn_s_setprio(0); } while (0)
#define WAIT_L(n) asm volatile("s_waitcnt lgkmcnt(" #n ")" ::: "memory")
#define RBAR __builtin_amdgcn_s_barrier()
#define SCHED __builtin_amdgcn_sched_barrier(0)
    auto flag_ptr = [&](int lp, int j, int ent) -> const char* {
        return ent < 32 ? ws + OFF_FA + (size_t)(((pm * 2 + (ent >> 4)) * 8 + j) * 16 + (ent & 15)) * 8 : ws + OFF_FB + (size_t)((lp * 8 + j) * 32 + (ent - 32)) * 8; };
    auto poll = [&](int lp, int j) {
        const char* f = flag_ptr(lp, j, lane);
        const u64 t0 = now(); u32 sp = 0; u32x2 v;
        for (;;) {
            v = ld8_sys(f);
            if (__builtin_amdgcn_ballot_w64(v.x >= call) == ~0ull) break;
            if (give_up(sp, t0, (u32*)(ws + OFF_TMO), call)) { v.y = 62u; break; }
            __builtin_amdgcn_s_sleep(4);
        }
        asm volatile("ds_write_b64 %0, %1\n\ts_waitcnt lgkmcnt(0)" :: "v"(L_T2 + lane * 8), "v"(v) : "memory");
    };
    auto codes = [&](u32 tb) -> u32 {
        const u32 r = lane & 7;
        const u32 a0 = *(const LAS u32*)(L + tb + (r * 2 + wr) * 8 + 4) & 63u, a1 = *(const LAS u32*)(L + tb + (16 + r * 2 + wr) * 8 + 4) & 63u;
        const u32 b = *(const LAS u32*)(L + tb + (32 + r * 4 + wc) * 8 + 4) & 63u;
        const u32 c0 = (a0 == 62u || b == 62u) ? 254u : (a0 == 63u || b == 63u) ? 255u : a0 + b;
        const u32 c1 = (a1 == 62u || b == 62u) ? 254u : (a1 == 63u || b == 63u) ? 255u : a1 + b;
        return c0 | (c1 << 8);
    };
    f32x4 acc[2][2][4][2];
    f16x8 At[4][2], B0[2][2], B1[2][2];
    u32 cur = 0, cmax = 0, bad = 0;
    u32 aoff0 = aoff, aoff1 = aoff;
    u32 nm, nmn = 0;
    auto changes = [&](u32 nw, u32 old) -> u32 {
        const u32 prev = __builtin_amdgcn_update_dpp((int)__builtin_amdgcn_readlane(old, 7), (int)nw, 0x111, 0xf, 0xf, false);
        return (u32)__builtin_amdgcn_ballot_w64(nw != prev) & 255u; };
#define ADJUST(r, buf) do { if (__builtin_expect((nm >> (buf)) & 1u, 0)) { const u32 cn_ = __builtin_amdgcn_readlane(pk, (r)); \
            aoff0 = aoff; aoff1 = aoff; \
            _Pragma("unroll") for (int ai = 0; ai < 2; ++ai) { \
                const u32 c_ = (cn_ >> (8 * ai)) & 255u, o_ = (cur >> (8 * ai)) & 255u, x_ = (cmax >> (8 * ai)) & 255u; \
                if (c_ == 254u) bad = 1; \
                else if (c_ != 255u && c_ != o_) { \
                    if (c_ + 10u < x_) {                          \
                        _Pragma("unroll") for (int z_ = 0; z_ < 8; ++z_) *(LAS u32x4*)(L + L_ZERO + z_ * 1024 + lane * 16) = (u32x4){0u, 0u, 0u, 0u}; \
                        (ai ? aoff1 : aoff0) = L_ZERO - SA(buf, ai) + lane * 16; nm = 255u; nmn |= 1u; \
                    } else { const int sh_ = 8 * ((int)o_ - (int)c_); \
                        _Pragma("unroll") for (int bj = 0; bj < 2; ++bj) _Pragma("unroll") for (int m = 0; m < 4; ++m) _Pragma("unroll") for (int n = 0; n < 2; ++n) \
                            _Pragma("unroll") for (int e_ = 0; e_ < 4; ++e_) acc[ai][bj][m][n][e_] = __builtin_ldexpf(acc[ai][bj][m][n][e_], sh_); \
                        cur = (cur & ~(255u << (8 * ai))) | (c_ << (8 * ai)); \
                        if (c_ > x_) cmax = (cmax & ~(255u << (8 * ai))) | (c_ << (8 * ai)); } } } \
            WAIT_L(0); } } while (0)
#pragma unroll
    for (int a = 0; a < 2; ++a)
#pragma unroll
        for (int b = 0; b < 2; ++b)
#pragma unroll
            for (int m = 0; m < 4; ++m)
#pragma unroll
                for (int n = 0; n < 2; ++n) acc[a][b][m][n] = (f32x4){0.f, 0.f, 0.f, 0.f};
    const char* const cA = ws + OFF_A + (size_t)pm * 2 * HA;
    const char* cB = ws + OFF_B + (size_t)lp0 * 64 * KB;
    if (wave == 0) poll(lp0, 0);
    BAR();
    u32 pk = codes(L_T2), pkn = pk;
    nm = changes(pk, pk) | 1u;
    STAGE(SB(0, 0), cB, voffB); STAGE(SA(0, 0), cA, voffA); STAGE(SB(0, 1), cB + HB, voffB); STAGE(SA(0, 1), cA + HA, voffA);
    if (wr == 1) RBAR;
    VMCNT(4); RBAR;
    STAGE(SB(1, 0), cB + KB, voffB); STAGE(SA(1, 0), cA + KA, voffA); STAGE(SB(1, 1), cB + HB + KB, voffB);
    VMCNT(6); RBAR;
#pragma unroll 1
    for (int rd = 0; rd < 3; ++rd) {
        const int lp = lp0 + rd * 12;
        const char* nB = rd < 2 ? cB + 12 * 64 * KB : cB;
#pragma unroll 1
        for (int t = 0; t < 64; t += 2) {
            const bool last = t == 62;
            const char* a1 = cA + (size_t)(t + 1) * KA;
            const char* a2 = last ? cA : cA + (size_t)(t + 2) * KA; const char* b2 = last ? nB : cB + (size_t)(t + 2) * KB;
            const char* a3 = a2 + KA; const char* b3 = b2 + KB;
            ADJUST(t & 7, 0);
            LDB(B0, 0, 0); SCHED; LDA(At, 0, 0); STAGE(SA(1, 1), a1 + HA, voffA);
            WAIT_L(8); RBAR; WAIT_L(0); MMA(0, 0, At, B0); RBAR; SCHED;
            LDB(B1, 0, 1); STAGE(SB(0, 0), b2, voffB);
            RBAR; WAIT_L(0); MMA(0, 1, At, B1); RBAR;
            LDA(At, 0, 1); STAGE(SA(0, 0), a2, voffA);
            RBAR; WAIT_L(0); MMA(1, 0, At, B0); RBAR; SCHED;
            STAGE(SB(0, 1), b2 + HB, voffB);
            VMCNT(6); RBAR; MMA(1, 1, At, B1); RBAR;
            ADJUST((t & 7) + 1, 1);
            LDB(B0, 1, 0); SCHED; LDA(At, 1, 0); STAGE(SA(0, 1), a2 + HA, voffA);
            WAIT_L(8); RBAR; WAIT_L(0); MMA(0, 0, At, B0); RBAR; SCHED;
            LDB(B1, 1, 1); STAGE(SB(1, 0), b3, voffB);
            RBAR; WAIT_L(0); MMA(0, 1, At, B1); RBAR;
            LDA(At, 1, 1); STAGE(SA(1, 0), a3, voffA);
            RBAR; WAIT_L(0); MMA(1, 0, At, B0); RBAR; SCHED;
            STAGE(SB(1, 1), b3 + HB, voffB);
            nm >>= 2;
            if (__builtin_expect((t & 7) != 0, 0)) {
                if ((t & 7) == 6) { pk = pkn; nm = nmn; nmn = 0; }
                else if (t < 56 || rd < 2) {
                    const int nlp = t < 56 ? lp : lp + 12, nj = ((t >> 3) + 1) & 7;
                    if ((t & 7) == 2) {
                        if (wave == 0 && lane < 32) __builtin_amdgcn_global_load_lds((const GAS u32*)flag_ptr(nlp, nj, lane * 2), (LAS u32*)(L + L_T1), 16, 0, 17);
                    } else {
                        const u32 tag = *(const LAS u32*)(L + L_T1 + lane * 8);
                        pkn = codes(L_T1);
                        if (__builtin_amdgcn_ballot_w64(tag >= call) != ~0ull) {
                            if (wave == 0) poll(nlp, nj);
                            BAR();
                            pkn = codes(L_T2);
                        }
                        nmn |= changes(pkn, pk);
                    }
                }
            }
            VMCNT(6); RBAR; MMA(1, 1, At, B1); RBAR;
        }
        if (wave == 0) { STAMP(4 + rd * 2); STAMPV(50 + rd, __builtin_amdgcn_s_memtime()); }
        {
            int ln = lane; asm volatile("" : "+v"(ln));
            const int gp = lp < 4 ? 32 + P.rank * 4 + lp : lp - 4;
            GAS float* o = (GAS float*)P.out + (size_t)(pm * 256 + wr * 64 + (ln >> 4) * 4) * GN + gp * 256 + wc * 64 + (ln & 15) * 4;
#pragma unroll
            for (int ai = 0; ai < 2; ++ai) {
                const int sh = 8 * (int)((cur >> (8 * ai)) & 255u) - 272;
#pragma unroll
                for (int m = 0; m < 4; ++m)
#pragma unroll
                    for (int r = 0; r < 4; ++r) {
                        f32x4 v;
#pragma unroll
                        for (int i = 0; i < 4; ++i) {
                            const float y = __builtin_elementwise_maximum(__builtin_ldexpf(acc[ai][i >> 1][m][i & 1][r], sh), 0.0f);
                            v[i] = __uint_as_float(bad ? 0x7fc00000u : __float_as_uint(y) & 0x7fffffffu);
                            acc[ai][i >> 1][m][i & 1][r] = 0.f;
                        }
                        st_out(o + (size_t)(ai * 128 + m * 16 + r) * GN, v);
                    }
            }
            if (lp < 4) {
                VMCNT(0);
                if (ln == 0) st4_sys(ws + OFF_OF + (size_t)((pm * 4 + lp) * 8 + wave) * 4, call);
            }
            cur = 0; cmax = 0; bad = 0; nm |= 1u;
        }
        if (wave == 0) STAMP(5 + rd * 2);
        cB = nB;
    }
    VMCNT(0);
    if (wr == 0) RBAR;
#undef SA
#undef SB
#undef STAGE
#undef LDA
#undef LDB
#undef MMA
#undef ADJUST
}

__device__ __forceinline__ void packer(const Params& P, LAS char* L, int pw, int wave, int lane, u32 call) {
    char* ws = P.me;
    LAS char* dense = L + wave * PK_B;
    LAS char* img = dense + PK_IMG;
    const auto rs = __builtin_amdgcn_make_buffer_rsrc((void*)P.out, 0, (u32)GM * GN * 4u, 0x00020000);
#pragma unroll 1
    for (int it = pw; it < N_OUT_ITEMS; it += 64) {
        bool bad = false;
        {
            const u64 t0 = now(); u32 sp = 0;
            while (uni(ld4_sys(ws + OFF_OF + it * 4)) < call) {
                if (give_up(sp, t0, (u32*)(ws + OFF_TMO), call)) { bad = true; break; }
                __builtin_amdgcn_s_sleep(8);
            }
        }
        const int tile = it >> 3, wq = it & 7;
        const u32 voff = ((u32)((tile >> 2) * 256 + (wq >> 2) * 64 + (lane >> 4) * 4) * GN + 8192 + P.rank * 1024 + (tile & 3) * 256 + (wq & 3) * 64 + (lane & 15) * 4) * 4u;
        char* og = ws + OFF_O + (size_t)it * QSLOT;
        u32 off16 = 0, ea = 0, eb = 0;
        u32x4 nx[8];
#pragma unroll
        for (int g = 0; g < 8; ++g) nx[g] = __builtin_amdgcn_raw_buffer_load_b128(rs, voff, ((g >> 2) * 16 + (g & 3)) * GN * 4, 17);
#pragma unroll 1
        for (int su = 0; su < 4; ++su) {
            u32 v[32]; u32 mx = 0;
#pragma unroll
            for (int r = 0; r < 32; ++r) { v[r] = bad ? 0x7fc00000u : nx[r >> 2][r & 3]; mx = max(mx, v[r]); }
            {
                const int sn = su < 3 ? su + 1 : 3;
#pragma unroll
                for (int g = 0; g < 8; ++g) nx[g] = __builtin_amdgcn_raw_buffer_load_b128(rs, voff, ((sn >> 1) * 128 + (sn & 1) * 32 + (g >> 2) * 16 + (g & 3)) * GN * 4, 17);
            }
            mx = wave_umax(mx) >> 16;
            const u32 b16 = mx >= 1023u ? mx - 1023u : 0u, B = b16 << 16;
            u32 n = 0, ml = 0, mh = 0;
#pragma unroll
            for (int r = 0; r < 32; ++r) {
                const u64 m = __builtin_amdgcn_ballot_w64(v[r] != 0u);
                const u32 ad = v[r] != 0u ? (n + prefix(m)) * 4u : 8240u;
                *(LAS u32*)(dense + ad) = v[r] - B;
                ml = llvm_writelane((u32)m, r, ml); mh = llvm_writelane((u32)(m >> 32), r, mh);
                n += __builtin_popcountll(m);
            }
            if (lane < 8) *(LAS u32*)(dense + (n + lane) * 4u) = 0u;
            if (lane < 32) *(LAS u32x2*)(img + lane * 8) = (u32x2){ml, mh};
            CFENCE();
            const u32 n8 = (n + 7u) >> 3, o_mid = 256u + n8 * 16u, o_hi = o_mid + n8 * 8u, o_esc = o_hi + n8 * 2u;
            u32 e = 0;
            for (u32 g0 = 0; g0 < n8; g0 += 64u) {
                const u32 g = g0 + lane;
                u64 W = 0;
                if (g < n8) {
                    const u32x4 a = *(LAS u32x4*)(dense + g * 32u), b = *(LAS u32x4*)(dense + g * 32u + 16u);
                    u32x4 lo;
                    lo.x = __builtin_amdgcn_perm(a.y, a.x, 0x05040100u); lo.y = __builtin_amdgcn_perm(a.w, a.z, 0x05040100u);
                    lo.z = __builtin_amdgcn_perm(b.y, b.x, 0x05040100u); lo.w = __builtin_amdgcn_perm(b.w, b.z, 0x05040100u);
                    const u32 u0 = __builtin_amdgcn_perm(a.y, a.x, 0x07060302u), u1 = __builtin_amdgcn_perm(a.w, a.z, 0x07060302u);
                    const u32 u2 = __builtin_amdgcn_perm(b.y, b.x, 0x07060302u), u3 = __builtin_amdgcn_perm(b.w, b.z, 0x07060302u);
                    const u32x2 mid = {__builtin_amdgcn_perm(u1, u0, 0x06040200u), __builtin_amdgcn_perm(u3, u2, 0x06040200u)};
                    const u32 ta = __builtin_amdgcn_perm(u1, u0, 0x07050301u), tb = __builtin_amdgcn_perm(u3, u2, 0x07050301u);
                    const u32 T = (ta & 0x03030303u) | ((tb & 0x03030303u) << 2);
                    *(LAS u32x4*)(img + 256u + g * 16u) = lo;
                    *(LAS u32x2*)(img + o_mid + g * 8u) = mid;
                    *(LAS unsigned short*)(img + o_hi + g * 2u) = (unsigned short)(T | (T >> 12));
                    W = ((u64)(tb & 0x7c7c7c7cu) << 32) | (ta & 0x7c7c7c7cu);
                }
                for (;;) {
                    const u64 em = __builtin_amdgcn_ballot_w64(W != 0ull);
                    if (em == 0ull) break;
                    if (W != 0ull) {
                        const u32 sh = (u32)__builtin_ctzll(W) & ~7u;
                        const u32 es = (u32)(W >> (sh + 2u)) & 31u;
                        *(LAS unsigned short*)(img + o_esc + (e + prefix(em)) * 2u) = (unsigned short)(((g * 8u + (sh >> 3)) << 5) | es);
                        W &= ~(0xffull << sh);
                    }
                    e += __builtin_popcountll(em);
                }
            }
            CFENCE();
            const u32 size16 = (o_esc + e * 2u + 15u) >> 4;
            if ((lane & 3) == su) { ea = off16 | (n << 14); eb = e | (b16 << 12); }
            for (u32 g = lane; g < size16; g += 64u) st16_sys(og + (off16 + g) * 16u, *(LAS u32x4*)(img + g * 16u));
            off16 += size16;
            LGKM0();
            CFENCE();
        }
        VMCNT(0);
        if (lane < 32 && (lane >> 2) != P.rank)
            st16_sys(peer_ws(P, lane >> 2) + OFF_REC + (call & 1u) * REC_PAR + (size_t)(P.rank * N_ITEMS + N_IN_ITEMS + it) * 64 + (lane & 3) * 16, (u32x4){call, ea, call, eb});
        if (it < 64) STAMP(16);
    }
    STAMP(17);
}

__global__ __launch_bounds__(NTHR, 2) __attribute__((amdgpu_waves_per_eu(2, 2))) void fused_kernel(Params P) {
    extern __shared__ __attribute__((aligned(16))) char smem[];
    LAS char* L = (LAS char*)smem;
    const int tid = threadIdx.x, lane = tid & 63, wave = __builtin_amdgcn_readfirstlane(tid >> 6);
    const int b = blockIdx.x, xg = b & 7, s = b >> 3;
    const bool fetch = s < FS;
    constexpr int NF = FS * 8, NG = NWG - NF;
    if (wave == 0) STAMP(0);
    GAS u32* const cnt = (GAS u32*)(P.me + OFF_CNT + (b * 8 + wave) * 8);
    if (fetch) {
        if (tid < 96) lds_st(L_ITEM + tid * 4, tid >= 24 && tid < 32 ? (u32)(tid - 24) : 0u);
        LGKM0();
        BAR();
    }
    if (!fetch) {
        const u64 tstart = now();
#pragma unroll 1
        for (int t = wave * NG + (b - NF); t < N_TASKS; t += 8 * NG) {
            const u64 go = t < 32 ? 0ull : (t < 224 ? (u64)PACE1 : (u64)(PACE2 + (t - 224) * PACE3 / 224));
            while (now() - tstart < go) __builtin_amdgcn_s_sleep(4);
            stage_task(P, L + wave * 1024, t, lane, cnt);
        }
    }
    const u32 call = uni(__hip_atomic_load(cnt, __ATOMIC_RELAXED, __HIP_MEMORY_SCOPE_AGENT)) + 1u;
    if (lane == 0) st4_sys((void*)cnt, call);
    STAMP(fetch ? 1 : (wave == 0 ? 3 : 18));
    if (fetch) {
        const int src = (P.rank + 1 + s) & 7;
        if (wave == 0) { loader(P, src, lane, call); STAMP(2); }
        else if (wave == 1) scout(P, src, xg, lane, call);
        else { decoder(P, L, src, wave - 2, lane, call); STAMP(8 + wave); }
    } else if (s < FS + GS) {
        const int g = s - FS;
        gemm_wg(P, L, (xg >> 1) * 4 + g / 6, (xg & 1) * 6 + g % 6, wave, lane, call);
    } else {
        BAR();
        packer(P, L, wave * 8 + xg, wave, lane, call);
    }
}

}

extern "C" void kernel_launch(int rank, int nranks, ncclComm_t comm, hipStream_t stream,
                              void* const* d_in, const int* in_sizes, int n_in,
                              void* d_out, int out_size,
                              const int* in_dtypes, int out_dtype,
                              void* const* d_out_all,
                              void* const* d_ws_all, size_t ws_bytes) {
    (void)comm; (void)in_sizes; (void)n_in; (void)out_size; (void)in_dtypes; (void)out_dtype; (void)d_out_all;
    if (nranks != NR || (unsigned)rank >= (unsigned)NR || ws_bytes < WS_NEED) { fprintf(stderr, "unsupported configuration\n"); return; }
    static bool ready[NR];
    if (!ready[rank]) {
        HIP_CHECK(hipFuncSetAttribute((const void*)fused_kernel, hipFuncAttributeMaxDynamicSharedMemorySize, (int)LDS_BYTES));
        ready[rank] = true;
    }
    Params P;
    P.x = (const float*)d_in[0]; P.w = (const float*)d_in[1]; P.out = (float*)d_out;
    for (int i = 0; i < NR; ++i) P.ws[i] = (char*)d_ws_all[i];
    P.me = P.ws[rank]; P.rank = rank; P.pad = 0;
    hipLaunchKernelGGL(fused_kernel, dim3(NWG), dim3(NTHR), LDS_BYTES, stream, P);
    HIP_CHECK(hipGetLastError());
}
```

```cpp
#include <hip/hip_runtime.h>
#include <rccl/rccl.h>
#include <cstdio>

#define HIP_CHECK(expr) do { hipError_t _e = (expr); if (_e != hipSuccess) { \
    fprintf(stderr, "HIP error %s at %s:%d\n", hipGetErrorString(_e), __FILE__, __LINE__); \
    return; } } while (0)

typedef unsigned u32;
typedef unsigned long long u64;
typedef u32 u32x2 __attribute__((ext_vector_type(2)));
typedef u32 u32x4 __attribute__((ext_vector_type(4)));
typedef float f32x4 __attribute__((ext_vector_type(4)));
typedef _Float16 f16x8 __attribute__((ext_vector_type(8)));
#define LAS __attribute__((address_space(3)))
#define GAS __attribute__((address_space(1)))

extern "C" __device__ u32 llvm_writelane(u32, u32, u32) __asm("llvm.amdgcn.writelane.i32");

namespace {

constexpr int NR = 8, GM = 4096, GN = 16384, KL = 512;
constexpr int NWG = 256, NTHR = 512;
#define WINV 8
#define NPUSH 36
#define GSTORE 0
#define GSLEEP 16
#define AHEAD 0
#define OSTORE 2
constexpr int FS = 7, GS = 24;
constexpr int LPP = 8;
constexpr int WIN = WINV;
constexpr int NLP = 36;
constexpr int N_IN_ITEMS = 832, N_OUT_ITEMS = 512, N_ITEMS = N_IN_ITEMS + N_OUT_ITEMS;
constexpr int N_PAIRS = 1280, ITEM0 = NPUSH / 2;
#define PACE2 450
#define PACE3 120
constexpr u64 DEADLINE = 30000000ull;

constexpr size_t OFF_CNT = 0;
constexpr size_t OFF_TMO = 16384;
constexpr size_t OFF_TICK = 16384 + 256;
constexpr size_t OFF_OF = 20480;
constexpr u32 U_B = 8192, N_UNITS = U_B + NLP * 256;
constexpr size_t OFF_FA = 32768;
constexpr size_t OFF_DBG = 262144;
constexpr size_t OFF_REC = 1ull << 20;
constexpr size_t REC_PAR = 1ull << 20;
constexpr size_t OFF_A = 4ull << 20;
constexpr size_t OFF_AX = OFF_A + (size_t)N_UNITS * 8192;
constexpr size_t OFF_BX = OFF_AX + (4ull << 20);
constexpr size_t X_PAR = 20ull << 20;
constexpr size_t OFF_O = OFF_AX + 2 * X_PAR;
constexpr size_t QSLOT = 49152;
constexpr size_t WS_NEED = OFF_O + N_OUT_ITEMS * QSLOT;
static_assert((size_t)NR * N_ITEMS * 64 <= REC_PAR && OFF_FA + N_UNITS * 8 <= OFF_DBG, "workspace map");

constexpr u32 RING = 65536;
constexpr u32 L_SCR = RING, SCR_B = 8448;
constexpr u32 L_DESC = L_SCR + 6 * SCR_B, NQ = 64;
constexpr u32 L_ITEM = L_DESC + NQ * 16;
constexpr u32 L_FCTL = L_ITEM + 64;
constexpr u32 L_LAND = L_FCTL + 64;
constexpr u32 L_T1 = 131072, L_T2 = L_T1 + 512, L_ZERO = L_T2 + 512;
constexpr u32 PK_B = 19456, PK_IMG = 8320;
constexpr u32 LDS_BYTES = 8 * PK_B;
static_assert(L_LAND + 256 <= LDS_BYTES && L_ZERO + 8192 <= LDS_BYTES && LDS_BYTES <= 163840, "LDS map");

struct Params {
    const float* x; const float* w; float* out;
    char* me;
    char* ws[NR];
    int rank; int pad;
};

#define VMCNT(n) asm volatile("s_waitcnt vmcnt(" #n ")" ::: "memory")
#define LGKM0() asm volatile("s_waitcnt lgkmcnt(0)" ::: "memory")
#define CFENCE() asm volatile("" ::: "memory")
#define BAR() do { CFENCE(); __builtin_amdgcn_s_barrier(); CFENCE(); } while (0)

__device__ __forceinline__ void st16_sys(void* p, u32x4 v) {
    asm volatile("global_store_dwordx4 %0, %1, off sc0 sc1\n\ts_nop 1" :: "v"(p), "v"(v) : "memory");
}
__device__ __forceinline__ void st_out(GAS float* p, f32x4 v) {
    asm volatile("global_store_dwordx4 %0, %1, off sc0 sc1 nt\n\ts_nop 1" :: "v"(p), "v"(v) : "memory");
}
__device__ __forceinline__ void st4_sys(void* p, u32 v) {
    asm volatile("global_store_dword %0, %1, off sc0 sc1" :: "v"(p), "v"(v) : "memory");
}
__device__ __forceinline__ void st8_sys(void* p, u32x2 v) {
    asm volatile("global_store_dwordx2 %0, %1, off sc0 sc1" :: "v"(p), "v"(v) : "memory");
}
__device__ __forceinline__ u32 ld4_sys(const void* p) {
    u32 v; asm volatile("global_load_dword %0, %1, off sc0 sc1\n\ts_waitcnt vmcnt(0)" : "=&v"(v) : "v"(p) : "memory"); return v;
}
__device__ __forceinline__ u32x2 ld8_sys(const void* p) {
    u32x2 v; asm volatile("global_load_dwordx2 %0, %1, off sc0 sc1\n\ts_waitcnt vmcnt(0)" : "=&v"(v) : "v"(p) : "memory"); return v;
}
__device__ __forceinline__ void lds_st(u32 a, u32 v) { asm volatile("ds_write_b32 %0, %1" :: "v"(a), "v"(v) : "memory"); }
__device__ __forceinline__ u32 lds_ld(u32 a) {
    u32 v; asm volatile("ds_read_b32 %0, %1\n\ts_waitcnt lgkmcnt(0)" : "=&v"(v) : "v"(a) : "memory"); return v;
}
__device__ __forceinline__ u32x4 lds_ld16(u32 a) {
    u32x4 v; asm volatile("ds_read_b128 %0, %1\n\ts_waitcnt lgkmcnt(0)" : "=&v"(v) : "v"(a) : "memory"); return v;
}
__device__ __forceinline__ u32 uni(u32 v) { return __builtin_amdgcn_readfirstlane(v); }
__device__ __forceinline__ u32 lds_ldu(u32 a) { return uni(lds_ld(a)); }
__device__ __forceinline__ u64 now() { return __builtin_amdgcn_s_memrealtime(); }
__device__ __forceinline__ u32 prefix(u64 m) { return __builtin_amdgcn_mbcnt_hi((u32)(m >> 32), __builtin_amdgcn_mbcnt_lo((u32)m, 0u)); }

__device__ __forceinline__ u32 wave_umax(u32 v) {
    v = max(v, (u32)__builtin_amdgcn_update_dpp(0, (int)v, 0x111, 0xf, 0xf, true));
    v = max(v, (u32)__builtin_amdgcn_update_dpp(0, (int)v, 0x112, 0xf, 0xf, true));
    v = max(v, (u32)__builtin_amdgcn_update_dpp(0, (int)v, 0x114, 0xf, 0xf, true));
    v = max(v, (u32)__builtin_amdgcn_update_dpp(0, (int)v, 0x118, 0xf, 0xf, true));
    v = max(v, (u32)__builtin_amdgcn_update_dpp(0, (int)v, 0x142, 0xa, 0xf, false));
    v = max(v, (u32)__builtin_amdgcn_update_dpp(0, (int)v, 0x143, 0xc, 0xf, false));
    return __builtin_amdgcn_readlane(v, 63);
}
__device__ __forceinline__ u32 wave_scan(u32 v) {
    v += (u32)__builtin_amdgcn_update_dpp(0, (int)v, 0x111, 0xf, 0xf, true);
    v += (u32)__builtin_amdgcn_update_dpp(0, (int)v, 0x112, 0xf, 0xf, true);
    v += (u32)__builtin_amdgcn_update_dpp(0, (int)v, 0x114, 0xf, 0xf, true);
    v += (u32)__builtin_amdgcn_update_dpp(0, (int)v, 0x118, 0xf, 0xf, true);
    v += (u32)__builtin_amdgcn_update_dpp(0, (int)v, 0x142, 0xa, 0xf, false);
    v += (u32)__builtin_amdgcn_update_dpp(0, (int)v, 0x143, 0xc, 0xf, false);
    return v;
}
typedef unsigned short u16x2 __attribute__((ext_vector_type(2)));
typedef short i16x2 __attribute__((ext_vector_type(2)));
#define PU(x) __builtin_bit_cast(u16x2, (u32)(x))
#define PI(x) __builtin_bit_cast(i16x2, (u32)(x))
#define PW(x) __builtin_bit_cast(u32, (x))
__device__ __forceinline__ u32 pk_min1(u32 a) { u32 r; asm("v_pk_min_u16 %0, %1, 1 op_sel_hi:[1,0]" : "=v"(r) : "v"(a)); return r; }
__device__ __forceinline__ u32 pk_mad3(u32 a, u32 c) { u32 r; asm("v_pk_mad_u16 %0, %1, 3, %2 op_sel_hi:[1,0,1]" : "=v"(r) : "v"(a), "v"(c)); return r; }
__device__ __forceinline__ u32 pk_mad(u32 a, u32 k, u32 c) { u32 r; asm("v_pk_mad_u16 %0, %1, %2, %3" : "=v"(r) : "v"(a), "s"(k), "v"(c)); return r; }
__device__ __forceinline__ float amax3(float m, float a, float b) { float r; asm("v_max3_f32 %0, %1, |%2|, |%3|" : "=v"(r) : "v"(m), "v"(a), "v"(b)); return r; }
__device__ __forceinline__ char* peer_ws(const Params& P, int p) {
    u64 wp = (u64)P.ws[0];
#pragma unroll
    for (int i = 1; i < NR; ++i) { u64 q = (u64)P.ws[i]; asm volatile("" : "+s"(q)); wp = p == i ? q : wp; }
    return (char*)wp;
}

#define STAMP(k) do { } while (0)
#define STAMPV(k, v) do { } while (0)

__device__ __forceinline__ bool give_up(u32& spins, u64 t0, u32* tmo, u32 call) {
    if ((++spins & 63u) != 0u) return false;
    if (uni(ld4_sys(tmo)) == call) return true;
    if (now() - t0 > DEADLINE) { st4_sys(tmo, call); return true; }
    return false;
}

__device__ __forceinline__ float block_scale(const f32x4 (&v)[16], u32& code, u32& hm) {
    float mf = 0.f;
#pragma unroll
    for (int i = 0; i < 16; ++i) mf = amax3(amax3(mf, v[i][0], v[i][1]), v[i][2], v[i][3]);
    const u32 mb = wave_umax(__float_as_uint(mf));
    code = ((mb >> 23) + 2u) >> 3; code = code < 2u ? 2u : code;
    const float sc = __uint_as_float((263u - 8u * code) << 23);
    hm = uni(__builtin_bit_cast(unsigned short, (_Float16)(__uint_as_float(mb) * sc)));
    if (mb == 0u) code = 63u;
    return sc;
}
__device__ __forceinline__ void load_x(const Params& P, int idx, int j, int lane, f32x4 (&v)[16]) {
    const GAS float* p = (const GAS float*)P.x + (size_t)(idx * 64 + (lane & 15)) * KL + j * 64 + (lane >> 4) * 4;
#pragma unroll
    for (int i = 0; i < 16; ++i) v[i] = __builtin_nontemporal_load((const GAS f32x4*)(p + (i >> 2) * 16 * KL + (i & 3) * 16));
}
__device__ __forceinline__ void push_task(const Params& P, int idx, int lane, u32 craw, const f32x4 (&v)[16]) {
    u32 code, hm;
    const float sc = block_scale(v, code, hm);
    const u32 call = uni(craw) + 1u;
    const u32 dest = (call & 1u) * 4096u + ((idx >> 1) * 64 + P.rank) * 2 + (idx & 1);
    const u32 voff = (u32)OFF_A + dest * 8192u + lane * 16u;
#pragma unroll
    for (int f = 0; f < 8; ++f) { f16x8 h;
#pragma unroll
        for (int e = 0; e < 8; ++e) h[e] = (_Float16)(v[f * 2 + (e >> 2)][e & 3] * sc);
        const u32x4 hv = __builtin_bit_cast(u32x4, h);
        const u32 vo = voff + f * 1024;
#pragma unroll
        for (int r = 0; r < NR; ++r) asm volatile("global_store_dwordx4 %0, %1, %2 sc0 sc1\n\ts_nop 1" :: "v"(vo), "v"(hv), "s"((u64)P.ws[r]) : "memory");
    }
    VMCNT(0);
    if (lane < NR) st8_sys(peer_ws(P, lane) + OFF_FA + (size_t)dest * 8, (u32x2){call, code});
}
__device__ __forceinline__ void load_w(const Params& P, int idx, int j, int lane, f32x4 (&v)[16]) {
    const GAS float* p = (const GAS float*)P.w + (size_t)(j * 64 + (lane >> 4) * 4) * GN + idx * 64 + (lane & 15) * 4;
#pragma unroll
    for (int i = 0; i < 16; ++i) v[i] = __builtin_nontemporal_load((const GAS f32x4*)(p + (size_t)((i >> 2) * 16 + (i & 3)) * GN));
}
__device__ __forceinline__ void task_of(int q, int half, int& j, int& idx, bool& isA) {
    if (q < 896) { j = q / 112; const int u = q % 112; isA = u < 32; idx = isA ? u : (u < 96 ? u + 32 : u - 96); }
    else { const int u = q - 896, v = u % 192; j = v / 24; isA = false; idx = 16 + (u / 192) * 24 + v % 24; }
    idx = idx * 2 + half;
}
__device__ __forceinline__ void stage_task(const Params& P, LAS char* scr, u32 lpair, int q, int half, int lane, GAS u32* cnt, const f32x4 (&v)[16]) {
    int j, idx; bool isA;
    task_of(q, half, j, idx, isA);
    const u32 craw = __hip_atomic_load(cnt, __ATOMIC_RELAXED, __HIP_MEMORY_SCOPE_AGENT);
    *(LAS u32x4*)(scr + lane * 16) = (u32x4){0u, 0u, 0u, 0u}; *(LAS u32x4*)(scr + 1024 + lane * 16) = (u32x4){0u, 0u, 0u, 0u};
    __builtin_amdgcn_sched_barrier(0);
    u32 code, hm;
    const float sc = block_scale(v, code, hm);
    const u32 bb = hm >= 4095u ? hm - 4095u : 0u, b2 = bb * 0x10001u;
    u32x4 p[8];
    if (isA) {
#pragma unroll
        for (int f = 0; f < 8; ++f) { f16x8 h;
#pragma unroll
            for (int e = 0; e < 8; ++e) h[e] = (_Float16)(v[f * 2 + (e >> 2)][e & 3] * sc);
            p[f] = __builtin_bit_cast(u32x4, h); }
    } else {
#pragma unroll
        for (int f = 0; f < 8; ++f) { f16x8 h;
#pragma unroll
            for (int e = 0; e < 8; ++e) h[e] = (_Float16)(v[(f & 1) * 8 + e][f >> 1] * sc);
            p[f] = __builtin_bit_cast(u32x4, h); }
    }
    char* ws = P.me;
    const u32 call = uni(craw) + 1u;
    const bool all = isA || idx < 128;
    const int owner = (idx - 128) >> 4, lp = idx < 128 ? 4 + (idx >> 2) : (idx >> 2) & 3;
    const bool local = all || owner == P.rank, exprt = all || owner != P.rank;
    const u32 dest = isA ? (call & 1u) * 4096u + (((idx >> 1) * 8 + j) * 8 + P.rank) * 2 + half : U_B + (((lp * 8 + j) * 8 + P.rank) * 4 + (idx & 3));
    if (local) {
        char* d = ws + OFF_A + (size_t)dest * 8192 + lane * 16;
#pragma unroll
        for (int f = 0; f < 8; ++f) st16_sys(d + f * 1024, p[f]);
    }
    u32 size16 = 0;
    if (exprt) {
        u32 w[32], T[8], len[8], e = 0, tot = 0;
#pragma unroll
        for (int f = 0; f < 8; ++f) {
            u32 TT = 0, acc = 0, g[4];
#pragma unroll
            for (int k = 0; k < 4; ++k) {
                const u32 h = p[f][k];
                const u32 x = PW((u16x2)(PU(h & 0x7fff7fffu) - PU(b2)));
                const u32 o = PW((i16x2)(PI(x) >> (i16x2){10, 10}));
                const u32 sv = PW(__builtin_elementwise_max((i16x2)(-PI(o)), (i16x2){0, 0}));
                const u32 c = PW(__builtin_elementwise_min(PU(o), (u16x2){3, 3}));
                const u32 fl = PW((u16x2)(PU(c + 0x00010001u) >> (u16x2){2, 2}));
                w[f * 4 + k] = (h & 0x80008000u) | (c << 10) | (x & 0x03ff03ffu);
                TT |= PW((u16x2)(PU(sv & 0x00070007u) << PU(acc)));
                acc = pk_mad3(fl, acc);
                g[k] = PW(__builtin_elementwise_sub_sat((u16x2)(PU(o) + (u16x2){7, 7}), (u16x2){10, 10}));
            }
            T[f] = (TT & 0xffffu) | ((TT >> 16) << (acc & 31u));
            len[f] = (acc & 0xffffu) + (acc >> 16);
            tot += len[f];
            if (__builtin_amdgcn_ballot_w64((g[0] | g[1] | g[2] | g[3]) != 0u) != 0ull) {
#pragma unroll
                for (int k = 0; k < 8; ++k) {
                    const bool es = ((g[k >> 1] >> ((k & 1) * 16)) & 0xffffu) != 0u;
                    const u64 em = __builtin_amdgcn_ballot_w64(es);
                    if (es && e < 384u) *(LAS u32*)(scr + 4 + (e + prefix(em)) * 4u) = (u32)(f * 512 + k) + lane * 8u + ((p[f][k >> 1] >> ((k & 1) * 16)) << 16);
                    e += __builtin_popcountll(em);
                }
            }
        }
        const u32 inc = wave_scan(tot);
        size16 = (6656u + 4u + e * 4u + (((u32)__builtin_amdgcn_readlane(inc, 63) + 31u) >> 5) * 4u + 15u) >> 4;
        if (size16 >= 512u) size16 = 512u;
        u32 off16 = 0;
        if (half == 0) { if (lane == 0) lds_st(lpair, size16); }
        else {
            const u64 t0 = now(); u32 sp = 0;
            while ((off16 = lds_ldu(lpair)) == 0u) {
                if ((++sp & 1023u) == 0u && now() - t0 > 4 * DEADLINE) { st4_sys(ws + OFF_TMO, call); break; }
                __builtin_amdgcn_s_sleep(1);
            }
        }
        char* d = ws + (call & 1u) * X_PAR + off16 * 16u + lane * 16 +
                  (isA ? OFF_AX + (size_t)((idx >> 1) * 8 + j) * 16384 : OFF_BX + (size_t)(((idx >> 2) * 8 + j) * 2 + ((idx >> 1) & 1)) * 16384);
        if (size16 == 512u) {
#pragma unroll
            for (int f = 0; f < 8; ++f) st16_sys(d + f * 1024, p[f]);
        } else {
            u32 bp = (e + 1u) * 32u + inc - tot;
#pragma unroll
            for (int f = 0; f < 8; ++f) {
                const u64 t = (u64)T[f] << (bp & 31u);
                LAS u32* a = (LAS u32*)(scr + (bp >> 5) * 4u);
                __hip_atomic_fetch_or(a, (u32)t, __ATOMIC_RELAXED, __HIP_MEMORY_SCOPE_WAVEFRONT);
                __hip_atomic_fetch_or(a + 1, (u32)(t >> 32), __ATOMIC_RELAXED, __HIP_MEMORY_SCOPE_WAVEFRONT);
                bp += len[f];
            }
            if (lane == 0) *(LAS u32*)scr = e;
#pragma unroll
            for (int i = 0; i < 24; ++i) w[i] |= (w[26 + (i >> 2)] << (12 - 3 * (i & 3))) & 0x70007000u;
#pragma unroll
            for (int i = 0; i < 6; ++i) w[24 + i / 3] |= (w[26 + i] >> (1 + i % 3)) & (0x40004000u >> (i % 3));
#pragma unroll
            for (int f = 0; f < 6; ++f) st16_sys(d + f * 1024, (u32x4){w[f * 4], w[f * 4 + 1], w[f * 4 + 2], w[f * 4 + 3]});
            st8_sys(d + 6144 - lane * 8, (u32x2){w[24], w[25]});
            CFENCE();
            for (u32 g = lane; g + 416u < size16; g += 64u) st16_sys(d + 6656 + (g - lane) * 16u, *(LAS u32x4*)(scr + g * 16u));
        }
    }
    VMCNT(0);
    if (lane < NR) {
        if (lane == P.rank) { if (local) st8_sys(ws + OFF_FA + (size_t)dest * 8, (u32x2){call, code}); }
        else if (all || lane == owner) {
            const int hb = (lp % 12) * 2 + ((idx >> 1) & 1);
            const int item = isA ? j * 56 + (idx >> 1) : (lp < 12 ? j * 56 + 32 + hb : 256 + (lp / 12) * 192 + j * 24 + hb);
            st8_sys(peer_ws(P, lane) + OFF_REC + (call & 1u) * REC_PAR + (size_t)(P.rank * N_ITEMS + item) * 64 + half * 8,
                    (u32x2){call, size16 | (bb << 10) | (code << 25)});
        }
    }
    CFENCE();
}

__device__ __forceinline__ void scout(const Params& P, int src, int li, int lane, u32 call) {
    char* ws = P.me;
    u32* tmo = (u32*)(ws + OFF_TMO);
    u32* tick = (u32*)(ws + OFF_TICK + src * 256);
    const char* rec = ws + OFF_REC + (call & 1u) * REC_PAR + (size_t)src * N_ITEMS * 64;
    const u32 xpar = (call & 1u) * (u32)X_PAR;
    const u32 tbase = (call - 1u) * (u32)(N_ITEMS - ITEM0) - (u32)(LPP + ITEM0);
    u32 item = ITEM0 + li, pieces = 0, nq = 0, k = 0;
    __builtin_amdgcn_s_setprio(2);
    while (item < (u32)N_ITEMS) {
        const bool isin = item < (u32)N_IN_ITEMS;
        u32x2 r; bool dead = false;
        u32 tk;
        if (lane == 0) asm volatile("global_atomic_add %0, %1, %2, off sc0" : "=v"(tk) : "v"(tick), "v"(1u) : "memory");
        {
            const u64 t0 = now(); u32 sp = 0;
            for (;;) {
                r = ld8_sys(rec + (size_t)item * 64 + (lane & (isin ? 1 : 7)) * 8);
                if (__builtin_amdgcn_ballot_w64(r.x >= call) == __builtin_amdgcn_ballot_w64(true)) break;
                if (give_up(sp, t0, tmo, call)) { dead = true; break; }
                __builtin_amdgcn_s_sleep(2);
            }
        }
        asm volatile("" : "+v"(tk));
        const u32 next = uni(tk) - tbase;
        u32 soff, bytes, nsub; u32x4 d;
        if (isin) {
            u32 dest;
            u32 j, u;
            if (item < 448u) { j = item / 56u; u = item % 56u; }
            else { const u32 v = item - 448u, w = v % 192u; j = w / 24u; u = 56u + (v / 192u) * 24u + w % 24u; }
            if (u >= 32u) {
                const u32 lp = (u - 32u) >> 1, h = u & 1u, gp = lp < 4u ? 32u + P.rank * 4u + lp : lp - 4u;
                soff = (u32)OFF_BX + xpar + (gp * 8 + j) * 32768u + h * 16384u;
                dest = U_B + ((lp * 8 + j) * 8 + src) * 4 + h * 2;
            } else {
                soff = (u32)OFF_AX + xpar + (u * 8 + j) * 16384u;
                dest = (call & 1u) * 4096u + ((u * 8 + j) * 8 + src) * 2;
            }
            const u32 s = lane & 1, sz = min(r.y & 1023u, 512u), sb = dead ? 0u : sz * 16u;
            const u32 b0 = __builtin_amdgcn_readlane(sb, 0);
            bytes = b0 + __builtin_amdgcn_readlane(sb, 1); nsub = 2;
            d.x = pieces * 1024u + (s ? b0 : 0u); d.y = pieces + (((s ? bytes : b0) + 1023u) >> 10);
            d.z = dead ? (62u << 12) | (2u << 24) : sz | ((r.y >> 25) << 12); d.w = ((r.y >> 10) & 0x7fffu) | ((dest + s) << 16);
        } else {
            const u32 o = item - N_IN_ITEMS;
            soff = (u32)OFF_O + o * (u32)QSLOT;
            const u32 a = __builtin_amdgcn_ds_bpermute((lane & 3) * 8, r.y), b = __builtin_amdgcn_ds_bpermute((lane & 3) * 8 + 4, r.y);
            u32 off16 = a & 0x3fffu, n = (a >> 14) & 0xfffu, e = b & 0xfffu, b16 = (b >> 12) & 0x7fffu;
            n = n > 2048u ? 2048u : n; e = e > 2048u ? 2048u : e;
            u32 end = off16 * 16u + ((256u + ((n + 7u) >> 3) * 26u + e * 2u + 15u) & ~15u);
            if (dead || __builtin_amdgcn_ballot_w64(end > 44032u)) { dead = true; off16 = 0; end = 0; }
            bytes = __builtin_amdgcn_readlane(end, 3); nsub = 4;
            d.x = pieces * 1024u + off16 * 16u; d.y = pieces + ((end + 1023u) >> 10); d.z = n | (e << 12) | ((dead ? 3u : 1u) << 24);
            d.w = b16 | ((o * 4u + (lane & 3)) << 16);
        }
        const u32 np = (bytes + 1023u) >> 10;
        {
            const u64 t0 = now(); u32 sp = 0;
            for (;;) {
                u32 nx = lds_ld(L_FCTL + (lane < 6 ? 32 + lane * 4 : 12));
                u32 qm = __builtin_amdgcn_readlane(nx, 0);
#pragma unroll
                for (int i = 1; i < 6; ++i) qm = min(qm, (u32)__builtin_amdgcn_readlane(nx, i));
                const u32 first = qm < nq ? lds_ldu(L_DESC + (qm % NQ) * 16) >> 10 : pieces;
                if ((u32)__builtin_amdgcn_readlane(nx, 6) + AHEAD >= k && pieces + np - first <= RING / 1024 && (qm >= nq || nq + nsub - qm <= NQ)) break;
                if ((++sp & 1023u) == 0u && now() - t0 > 4 * DEADLINE) { st4_sys(tmo, call); break; }
                __builtin_amdgcn_s_sleep(1);
            }
        }
        if (lane < (int)nsub) asm volatile("ds_write_b128 %0, %1" :: "v"(L_DESC + ((nq + lane) % NQ) * 16), "v"(d) : "memory");
        nq += nsub;
        if (lane == 0) {
            const u32x4 it = {soff, bytes, 0u, k + 1};
            asm volatile("ds_write_b128 %0, %1" :: "v"(L_ITEM + (k & 3u) * 16), "v"(it) : "memory");
            lds_st(L_FCTL + 4, nq);
        }
        pieces += np; ++k;
        item = next;
    }
    if (lane == 0) {
        const u32x4 it = {0u, 0xffffffffu, 0u, k + 1};
        asm volatile("ds_write_b128 %0, %1" :: "v"(L_ITEM + (k & 3u) * 16), "v"(it) : "memory");
        lds_st(L_FCTL + 4, nq | 0x80000000u);
    }
}

__device__ __forceinline__ void loader(const Params& P, int src, int lane, u32 call) {
    const u64 base = (u64)P.ws[src];
    const u32 voff = lane * 16, vland = L_LAND + lane * 4;
    u32* tmo = (u32*)(P.me + OFF_TMO);
    int issued = 0, landed = 0;
    u32 k = 0;
    __builtin_amdgcn_s_setprio(3);
#define PIECE(sp) do { asm volatile("s_mov_b32 m0, %2\n\ts_waitcnt vmcnt(%3)\n\tglobal_load_lds_dwordx4 %0, %1 sc0 sc1" \
                                    :: "v"(voff), "s"(sp), "s"(((u32)issued * 1024u) & (RING - 1)), "n"(WIN - 1) : "memory"); } while (0)
    for (;;) {
        u32x4 it = lds_ld16(L_ITEM + (k & 3u) * 16);
        if (uni(it.w) != k + 1 || uni(it.y) == 0xffffffffu) {
            VMCNT(0);
            landed = issued;
            lds_st(vland, (u32)issued);
            const u64 t0 = now(); u32 sp = 0;
            while (uni(it.w) != k + 1) {
                if ((++sp & 1023u) == 0u && now() - t0 > 4 * DEADLINE) { st4_sys(tmo, call); return; }
                __builtin_amdgcn_s_sleep(1);
                it = lds_ld16(L_ITEM + (k & 3u) * 16);
            }
            if (uni(it.y) == 0xffffffffu) return;
        }
        const u32 soff = uni(it.x);
        u32 left = uni(it.y);
        ++k;
        if (lane == 0) lds_st(L_FCTL + 12, k);
        u64 sp = base + soff;
#pragma unroll 1
        for (; left >= 1024u; left -= 1024u, sp += 1024) {
            PIECE(sp);
            lds_st(vland, (u32)max(landed, issued - (WIN - 1)));
            ++issued;
        }
        if (left) {
            if (voff < left) PIECE(sp);
            lds_st(vland, (u32)max(landed, issued - (WIN - 1)));
            ++issued;
        }
    }
#undef PIECE
}

__device__ __forceinline__ void decoder(const Params& P, LAS char* L, int src, int dw, int lane, u32 call) {
    char* ws = P.me;
    u32* tmo = (u32*)(ws + OFF_TMO);
    LAS char* scr = L + L_SCR + dw * SCR_B;
    for (u32 q = dw;; q += 6) {
        {
            const u64 t0 = now(); u32 sp = 0;
            u32 nq = lds_ldu(L_FCTL + 4);
            while ((nq & 0x7fffffffu) <= q && !(nq >> 31)) {
                if ((++sp & 1023u) == 0u && now() - t0 > 4 * DEADLINE) { st4_sys(tmo, call); return; }
                __builtin_amdgcn_s_sleep(2);
                nq = lds_ldu(L_FCTL + 4);
            }
            if ((nq & 0x7fffffffu) <= q) return;
        }
        const u32x4 dv = lds_ld16(L_DESC + (q % NQ) * 16);
        const u32 pos = uni(dv.x), endp = uni(dv.y), z = uni(dv.z), wv = uni(dv.w);
        {
            const u64 t0 = now(); u32 sp = 0;
            while (lds_ldu(L_LAND) < endp) {
                if ((++sp & 1023u) == 0u && now() - t0 > 4 * DEADLINE) { st4_sys(tmo, call); break; }
                __builtin_amdgcn_s_sleep(1);
            }
        }
        const u32 type = z >> 24, dest = wv >> 16;
        if (type == 0u || type == 2u) {
            if (type == 0u) {
                const u32 sz = z & 0xfffu, b2 = (wv & 0xffffu) * 0x10001u, pl = pos + lane * 16u;
                char* d = ws + OFF_A + (size_t)dest * 8192;
                u32x4 c[8];
#pragma unroll
                for (int i = 0; i < 6; ++i) c[i] = *(LAS u32x4*)(L + ((pl + i * 1024u) & (RING - 1)));
                if (sz == 512u) {
                    c[6] = *(LAS u32x4*)(L + ((pl + 6144u) & (RING - 1))); c[7] = *(LAS u32x4*)(L + ((pl + 7168u) & (RING - 1)));
                    LGKM0();
#pragma unroll
                    for (int i = 0; i < 8; ++i) asm volatile("" : "+v"(c[i]));
                    if (lane == 0) lds_st(L_FCTL + 32 + dw * 4, q + 6);
#pragma unroll
                    for (int f = 0; f < 8; ++f) st16_sys(d + f * 1024 + lane * 16, c[f]);
                } else {
                    u32x2 c6 = *(LAS u32x2*)(L + ((pos + 6144u + lane * 8u) & (RING - 1)));
                    for (u32 g = lane; g + 416u < sz; g += 64u) *(LAS u32x4*)(scr + g * 16u) = *(LAS u32x4*)(L + ((pos + 6656u + g * 16u) & (RING - 1)));
                    LGKM0();
#pragma unroll
                    for (int i = 0; i < 6; ++i) asm volatile("" : "+v"(c[i]));
                    asm volatile("" : "+v"(c6));
                    if (lane == 0) lds_st(L_FCTL + 32 + dw * 4, q + 6);
                    const u32 e = min(uni(*(LAS u32*)scr), 384u);
                    u32 w[32], fl[32], ac[32], len[8], lo[8], tot = 0;
#pragma unroll
                    for (int i = 0; i < 24; ++i) w[i] = c[i >> 2][i & 3];
                    w[24] = c6.x; w[25] = c6.y;
#pragma unroll
                    for (int i = 0; i < 6; ++i) {
                        u32 t = (w[24 + i / 3] << (1 + i % 3)) & 0x80008000u;
#pragma unroll
                        for (int k = 0; k < 4; ++k) t |= (w[i * 4 + k] >> (12 - 3 * k)) & (0x00070007u << (3 * k));
                        w[26 + i] = t;
                    }
#pragma unroll
                    for (int f = 0; f < 8; ++f) {
                        u32 acc = 0;
#pragma unroll
                        for (int k = 0; k < 4; ++k) {
                            const int i = f * 4 + k;
                            fl[i] = PW((u16x2)(PU((w[i] & 0x0c000c00u) + 0x04000400u) >> (u16x2){12, 12}));
                            ac[i] = acc; acc = pk_mad3(fl[i], acc);
                        }
                        lo[f] = acc; len[f] = (acc & 0xffffu) + (acc >> 16); tot += len[f];
                    }
                    u32 bp = (e + 1u) * 32u + wave_scan(tot) - tot;
                    u32 T[8];
#pragma unroll
                    for (int f = 0; f < 8; ++f) {
                        const LAS u32* a = (const LAS u32*)(scr + (bp >> 5) * 4u);
                        T[f] = __builtin_amdgcn_alignbit(a[1], a[0], bp);
                        bp += len[f];
                    }
#pragma unroll
                    for (int f = 0; f < 8; ++f) {
                        const u32 TT = __builtin_amdgcn_perm(T[f] >> (lo[f] & 31u), T[f], 0x05040100u);
                        u32x4 o;
#pragma unroll
                        for (int k = 0; k < 4; ++k) {
                            const int i = f * 4 + k;
                            const u32 sv = PW((u16x2)(PU(TT) >> PU(ac[i]))) & (fl[i] * 7u);
                            o[k] = pk_mad(pk_mad3(pk_min1(sv), sv), 0xfc00fc00u, PW((u16x2)(PU(w[i] & 0x8fff8fffu) + PU(b2))));
                        }
                        st16_sys(d + f * 1024 + lane * 16, o);
                    }
                    for (u32 g = lane; g < e; g += 64u) {
                        const u32 rr = *(LAS u32*)(scr + 4u + g * 4u);
                        asm volatile("global_store_short_d16_hi %0, %1, off sc0 sc1" :: "v"(d + (rr & 4095u) * 2u), "v"(rr) : "memory");
                    }
                }
                VMCNT(0);
            } else if (lane == 0) lds_st(L_FCTL + 32 + dw * 4, q + 6);
            if (lane == 0) st8_sys(ws + OFF_FA + (size_t)dest * 8, (u32x2){call, (z >> 12) & 63u});
            continue;
        }
        const u32 tile = dest >> 5, wq = (dest >> 2) & 7u, su = dest & 3u;
        const u32 row0 = (tile >> 2) * 256u + (su >> 1) * 128u + (wq >> 2) * 64u + (su & 1u) * 32u + (lane >> 4) * 4u;
        const u32 col0 = 8192u + src * 1024u + (tile & 3u) * 256u + (wq & 3u) * 64u + (lane & 15) * 4u;
        GAS float* o = (GAS float*)P.out + (size_t)row0 * GN + col0;
        if (type == 3u) {
            if (lane == 0) lds_st(L_FCTL + 32 + dw * 4, q + 6);
            const float qn = __builtin_nanf("");
#pragma unroll
            for (int g = 0; g < 8; ++g) *(GAS f32x4*)(o + (size_t)((g >> 2) * 16 + (g & 3)) * GN) = (f32x4){qn, qn, qn, qn};
            continue;
        }
        const u32 n = z & 0xfffu, e = (z >> 12) & 0xfffu, b2 = (wv & 0xffffu) * 0x10001u, n8 = (n + 7u) >> 3;
        const u32x2 mk = *(LAS u32x2*)(L + ((pos + (lane & 31) * 8u) & (RING - 1)));
        const u32 o_lo = pos + 256u, o_mid = o_lo + n8 * 16u, o_hi = o_mid + n8 * 8u, o_esc = o_hi + n8 * 2u;
        for (u32 g = lane; g < n8; g += 64u) {
            const u32x4 lo = *(LAS u32x4*)(L + ((o_lo + g * 16u) & (RING - 1)));
            const u32x2 mid = *(LAS u32x2*)(L + ((o_mid + g * 8u) & (RING - 1)));
            const u32 H = *(LAS unsigned short*)(L + ((o_hi + g * 2u) & (RING - 1)));
            const u32 T = H | (H << 12), ta = T & 0x03030303u, tb = (T >> 2) & 0x03030303u;
            const u32 u0 = __builtin_amdgcn_perm(ta, mid.x, 0x05010400u) + b2, u1 = __builtin_amdgcn_perm(ta, mid.x, 0x07030602u) + b2;
            const u32 u2 = __builtin_amdgcn_perm(tb, mid.y, 0x05010400u) + b2, u3 = __builtin_amdgcn_perm(tb, mid.y, 0x07030602u) + b2;
            u32x4 a, b;
            a.x = __builtin_amdgcn_perm(u0, lo.x, 0x05040100u); a.y = __builtin_amdgcn_perm(u0, lo.x, 0x07060302u);
            a.z = __builtin_amdgcn_perm(u1, lo.y, 0x05040100u); a.w = __builtin_amdgcn_perm(u1, lo.y, 0x07060302u);
            b.x = __builtin_amdgcn_perm(u2, lo.z, 0x05040100u); b.y = __builtin_amdgcn_perm(u2, lo.z, 0x07060302u);
            b.z = __builtin_amdgcn_perm(u3, lo.w, 0x05040100u); b.w = __builtin_amdgcn_perm(u3, lo.w, 0x07060302u);
            *(LAS u32x4*)(scr + g * 32u) = a; *(LAS u32x4*)(scr + g * 32u + 16u) = b;
        }
        CFENCE();
        for (u32 g = lane; g < e; g += 64u) {
            const u32 rr = *(LAS unsigned short*)(L + ((o_esc + g * 2u) & (RING - 1)));
            LAS u32* t = (LAS u32*)(scr + (rr >> 5) * 4u);
            *t = (*t + ((rr & 31u) << 26)) & 0x7fffffffu;
        }
        u32 ml = mk.x, mh = mk.y;
        LGKM0();
        asm volatile("" : "+v"(ml), "+v"(mh));
        if (lane == 0) lds_st(L_FCTL + 32 + dw * 4, q + 6);
        u32 basep = 0;
#pragma unroll
        for (int g = 0; g < 8; ++g) {
            f32x4 v;
#pragma unroll
            for (int i = 0; i < 4; ++i) {
                const u64 m = ((u64)(u32)__builtin_amdgcn_readlane(mh, g * 4 + i) << 32) | (u32)__builtin_amdgcn_readlane(ml, g * 4 + i);
                const u32 p = min(basep + prefix(m), 2048u);
                const float t = *(LAS float*)(scr + p * 4u);
                v[i] = __builtin_amdgcn_inverse_ballot_w64(m) ? t : 0.0f;
                basep += __builtin_popcountll(m);
            }
            st_out(o + (size_t)((g >> 2) * 16 + (g & 3)) * GN, v);
        }
        CFENCE();
    }
}

__device__ __forceinline__ void gemm_wg(const Params& P, LAS char* L, int pm, int lp0, int wave, int lane, u32 call) {
    char* ws = P.me;
    const int wr = wave >> 2, wc = wave & 3;
    const u32 voffA = wave * 2048 + lane * 16, voffB = (wave >> 1) * 8192 + (wave & 1) * 2048 + lane * 16;
    const u32 boff = wc * 4096 + lane * 16, ldw = wave * 2048;
    auto lane_id = []() -> u32 { u32 l; asm volatile("v_mbcnt_lo_u32_b32 %0, -1, 0\n\tv_mbcnt_hi_u32_b32 %0, -1, %0" : "=v"(l)); return l; };
    constexpr size_t KA = 16384, KB = 32768, HA = 64 * KA, HB = 4096;
#define SA(b, h) (((b) * 2 + (h)) * 16384)
#define SB(b, h) ((4 + (b) * 2 + (h)) * 16384)
#define STAGE(bufoff, gbase, voff) asm volatile("s_mov_b32 m0, %2\n\ts_nop 0\n\tglobal_load_lds_dwordx4 %0, %1 sc0 sc1\n\tglobal_load_lds_dwordx4 %0, %1 offset:1024 sc0 sc1" \
        :: "v"(voff), "s"((u64)(gbase)), "s"((u32)(bufoff) + ldw) : "memory")
#define LDA(dst, b, h) do { _Pragma("unroll") for (int m = 0; m < 4; ++m) _Pragma("unroll") for (int k = 0; k < 2; ++k) \
        dst[m][k] = *(const LAS f16x8*)(L + SA(b, h) + ((h) ? aoff1 : aoff0) + m * 2048 + k * 1024); } while (0)
#define LDB(dst, b, h) do { _Pragma("unroll") for (int n = 0; n < 2; ++n) _Pragma("unroll") for (int k = 0; k < 2; ++k) \
        dst[n][k] = *(const LAS f16x8*)(L + SB(b, h) + boff + n * 2048 + k * 1024); } while (0)
#define MMA(ai, bj, At, Bt) do { __builtin_amdgcn_s_setprio(1); \
        _Pragma("unroll") for (int m = 0; m < 4; ++m) _Pragma("unroll") for (int n = 0; n < 2; ++n) _Pragma("unroll") for (int k = 0; k < 2; ++k) \
            acc[ai][bj][m][n] = __builtin_amdgcn_mfma_f32_16x16x32_f16(At[m][k], Bt[n][k], acc[ai][bj][m][n], 0, 0, 0); \
        __builtin_amdgcn_s_setprio(0); } while (0)
#define WAIT_L(n) asm volatile("s_waitcnt lgkmcnt(" #n ")" ::: "memory")
#define RBAR __builtin_amdgcn_s_barrier()
#define SCHED __builtin_amdgcn_sched_barrier(0)
    auto flag_ptr = [&](int lp, int j, int ent) -> const char* {
        return ws + OFF_FA + (size_t)(ent < 32 ? (call & 1u) * 4096u + ((pm * 2 + (ent >> 4)) * 8 + j) * 16 + (ent & 15) : U_B + (lp * 8 + j) * 32 + (ent - 32)) * 8; };
    auto poll = [&](int lp, int j) {
        const u32 ln = lane_id();
        const char* f = flag_ptr(lp, j, ln);
        const u64 t0 = now(); u32 sp = 0; u32x2 v;
        for (;;) {
            v = ld8_sys(f);
            if (__builtin_amdgcn_ballot_w64(v.x >= call) == ~0ull) break;
            if (give_up(sp, t0, (u32*)(ws + OFF_TMO), call)) { v.y = 62u; break; }
            __builtin_amdgcn_s_sleep(4);
        }
        asm volatile("ds_write_b64 %0, %1\n\ts_waitcnt lgkmcnt(0)" :: "v"(L_T2 + ln * 8), "v"(v) : "memory");
    };
    auto codes = [&](u32 tb) -> u32 {
        const u32 r = lane_id() & 7;
        const u32 a0 = *(const LAS u32*)(L + tb + wr * 8 + 4 + r * 16), a1 = *(const LAS u32*)(L + tb + 128 + wr * 8 + 4 + r * 16);
        const u32 b = *(const LAS u32*)(L + tb + 256 + wc * 8 + 4 + r * 32);
        return a0 | (a1 << 8) | (b << 16);
    };
    f32x4 acc[2][2][4][2];
    f16x8 At[4][2], B0[2][2], B1[2][2];
    u32 cur = 0, cmax = 0, bad = 0;
    u32 aoff0 = wr * 8192 + lane * 16, aoff1 = aoff0;
    u32 nm, nmn = 0;
    auto changes = [&](u32 nw, u32 old) -> u32 {
        const u32 prev = __builtin_amdgcn_update_dpp((int)__builtin_amdgcn_readlane(old, 7), (int)nw, 0x111, 0xf, 0xf, false);
        return (u32)__builtin_amdgcn_ballot_w64(nw != prev) & 255u; };
#define ADJUST(r, buf) do { if (__builtin_expect((nm >> (r)) & 1u, 0)) { const u32 cn_ = __builtin_amdgcn_readlane(pk, (r)), cb_ = (cn_ >> 16) & 63u; \
            const u32 l16_ = lane_id() * 16; \
            _Pragma("unroll") for (int ai = 0; ai < 2; ++ai) { \
                const u32 ca_ = (cn_ >> (8 * ai)) & 63u, c_ = ca_ + cb_, o_ = (cur >> (8 * ai)) & 255u, x_ = (cmax >> (8 * ai)) & 255u; \
                u32 ab_ = wr * 8192; int sh_ = 0; \
                if (ca_ == 62u || cb_ == 62u) bad = 1; \
                else if (ca_ != 63u && cb_ != 63u) { \
                    if (c_ + 10u < x_) { ab_ = L_ZERO - SA(buf, ai); nm = 255u; nmn |= 1u; }      \
                    else { sh_ = 8 * ((int)o_ - (int)c_); \
                        cur = (cur & ~(255u << (8 * ai))) | (c_ << (8 * ai)); \
                        if (c_ > x_) cmax = (cmax & ~(255u << (8 * ai))) | (c_ << (8 * ai)); } } \
                asm volatile("v_add_u32 %0, %1, %2" : "+v"(ai ? aoff1 : aoff0) : "s"(ab_), "v"(l16_)); \
                if (sh_ != 0) { \
                    asm volatile("s_nop 15" ::: "memory");        \
                    _Pragma("unroll") for (int bj = 0; bj < 2; ++bj) _Pragma("unroll") for (int m = 0; m < 4; ++m) _Pragma("unroll") for (int n = 0; n < 2; ++n) \
                        _Pragma("unroll") for (int e_ = 0; e_ < 4; ++e_) asm volatile("v_ldexp_f32 %0, %0, %1" : "+v"(acc[ai][bj][m][n][e_]) : "s"(sh_)); } } } } while (0)
#pragma unroll
    for (int a = 0; a < 2; ++a)
#pragma unroll
        for (int b = 0; b < 2; ++b)
#pragma unroll
            for (int m = 0; m < 4; ++m)
#pragma unroll
                for (int n = 0; n < 2; ++n) acc[a][b][m][n] = (f32x4){0.f, 0.f, 0.f, 0.f};
    const char* const cA = ws + OFF_A + (size_t)(call & 1u) * 4096 * 8192 + (size_t)pm * 2 * HA;
    const char* cB = ws + OFF_A + (size_t)U_B * 8192 + (size_t)lp0 * 64 * KB;
    *(LAS u32x4*)(L + L_ZERO + wave * 1024 + lane * 16) = (u32x4){0u, 0u, 0u, 0u};
    if (wave == 0) poll(lp0, 0);
    LGKM0();
    BAR();
    u32 pk = codes(L_T2), pkn = pk;
    nm = changes(pk, pk) | 1u;
    STAGE(SB(0, 0), cB, voffB); STAGE(SA(0, 0), cA, voffA); STAGE(SB(0, 1), cB + HB, voffB); STAGE(SA(0, 1), cA + HA, voffA);
    if (wr == 1) RBAR;
    VMCNT(4); RBAR;
    STAGE(SB(1, 0), cB + KB, voffB); STAGE(SA(1, 0), cA + KA, voffA); STAGE(SB(1, 1), cB + HB + KB, voffB);
    VMCNT(6); RBAR;
#pragma unroll 1
    for (int rd = 0; rd < 3; ++rd) {
        const int lp = lp0 + rd * 12;
        const char* nB = rd < 2 ? cB + 12 * 64 * KB : cB;
#define TRIP(r, a1, a2, b2, HOOK) do { \
            ADJUST(r, 0); \
            LDB(B0, 0, 0); SCHED; LDA(At, 0, 0); STAGE(SA(1, 1), (a1) + HA, voffA); \
            WAIT_L(8); RBAR; WAIT_L(0); MMA(0, 0, At, B0); RBAR; SCHED; \
            LDB(B1, 0, 1); STAGE(SB(0, 0), (b2), voffB); \
            RBAR; WAIT_L(0); MMA(0, 1, At, B1); RBAR; \
            LDA(At, 0, 1); STAGE(SA(0, 0), (a2), voffA); \
            RBAR; WAIT_L(0); MMA(1, 0, At, B0); RBAR; SCHED; \
            STAGE(SB(0, 1), (b2) + HB, voffB); \
            VMCNT(6); RBAR; MMA(1, 1, At, B1); RBAR; \
            ADJUST((r) + 1, 1); \
            LDB(B0, 1, 0); SCHED; LDA(At, 1, 0); STAGE(SA(0, 1), (a2) + HA, voffA); \
            WAIT_L(8); RBAR; WAIT_L(0); MMA(0, 0, At, B0); RBAR; SCHED; \
            LDB(B1, 1, 1); STAGE(SB(1, 0), (b2) + KB, voffB); \
            RBAR; WAIT_L(0); MMA(0, 1, At, B1); RBAR; \
            LDA(At, 1, 1); STAGE(SA(1, 0), (a2) + KA, voffA); \
            RBAR; WAIT_L(0); MMA(1, 0, At, B0); RBAR; SCHED; \
            STAGE(SB(1, 1), (b2) + KB + HB, voffB); \
            HOOK; \
            VMCNT(6); RBAR; MMA(1, 1, At, B1); RBAR; } while (0)
#pragma unroll 1
        for (int j = 0; j < 8; ++j) {
            const char* ga = cA + (size_t)j * 8 * KA; const char* gb = cB + (size_t)j * 8 * KB;
            const char* na = j == 7 ? cA : ga + 8 * KA; const char* nb = j == 7 ? nB : gb + 8 * KB;
            const bool more = j < 7 || rd < 2;
            const int nlp = j < 7 ? lp : lp + 12, nj = (j + 1) & 7;
            TRIP(0, ga + KA, ga + 2 * KA, gb + 2 * KB, do { } while (0));
            TRIP(2, ga + 3 * KA, ga + 4 * KA, gb + 4 * KB,
                 if (__builtin_expect(wave == 0 && more, 0)) { const u32 ln = lane_id(); const char* fp = flag_ptr(nlp, nj, ln * 2);
                     if (ln < 32) asm volatile("s_mov_b32 m0, %1\n\ts_nop 0\n\tglobal_load_lds_dwordx4 %0, off sc0 sc1" :: "v"(fp), "s"(L_T1) : "memory"); });
            TRIP(4, ga + 5 * KA, ga + 6 * KA, gb + 6 * KB, {
                 const u32 tag = *(const LAS u32*)(L + L_T1 + lane_id() * 8);
                 pkn = codes(L_T1);
                 if (__builtin_expect((__builtin_amdgcn_ballot_w64(tag >= call) != ~0ull) & more, 0)) {
                     if (wave == 0) poll(nlp, nj);
                     BAR();
                     pkn = codes(L_T2);
                 }
                 nmn |= changes(pkn, pk); });
            TRIP(6, ga + 7 * KA, na, nb, { pk = pkn; nm = nmn; nmn = 0; });
        }
        if (wave == 0) { STAMP(4 + rd * 2); STAMPV(50 + rd, __builtin_amdgcn_s_memtime()); }
        {
            const int ln = lane_id();
            const int gp = lp < 4 ? 32 + P.rank * 4 + lp : lp - 4;
            GAS float* o = (GAS float*)P.out + (size_t)(pm * 256 + wr * 64 + (ln >> 4) * 4) * GN + gp * 256 + wc * 64 + (ln & 15) * 4;
#pragma unroll
            for (int ai = 0; ai < 2; ++ai) {
                const int sh = 8 * (int)((cur >> (8 * ai)) & 255u) - 272;
#pragma unroll
                for (int m = 0; m < 4; ++m)
#pragma unroll
                    for (int r = 0; r < 4; ++r) {
                        f32x4 v;
#pragma unroll
                        for (int i = 0; i < 4; ++i) {
                            const float y = __builtin_elementwise_maximum(__builtin_ldexpf(acc[ai][i >> 1][m][i & 1][r], sh), 0.0f);
                            v[i] = __uint_as_float(bad ? 0x7fc00000u : __float_as_uint(y) & 0x7fffffffu);
                            acc[ai][i >> 1][m][i & 1][r] = 0.f;
                        }
                        st_out(o + (size_t)(ai * 128 + m * 16 + r) * GN, v);
                    }
            }
            if (lp < 4) {
                VMCNT(0);
                if (ln == 0) st4_sys(ws + OFF_OF + (size_t)((pm * 4 + lp) * 8 + wave) * 4, call);
            }
            cur = 0; cmax = 0; bad = 0; nm |= 1u;
        }
        if (wave == 0) STAMP(5 + rd * 2);
        cB = nB;
    }
    VMCNT(0);
    if (wr == 0) RBAR;
#undef SA
#undef SB
#undef STAGE
#undef LDA
#undef LDB
#undef MMA
#undef ADJUST
#undef TRIP
}

__device__ __forceinline__ void packer(const Params& P, LAS char* L, int pw, int wave, int lane, u32 call) {
    char* ws = P.me;
    LAS char* dense = L + wave * PK_B;
    LAS char* img = dense + PK_IMG;
    const auto rs = __builtin_amdgcn_make_buffer_rsrc((void*)P.out, 0, (u32)GM * GN * 4u, 0x00020000);
#pragma unroll 1
    for (int it = pw; it < N_OUT_ITEMS; it += 64) {
        bool bad = false;
        {
            const u64 t0 = now(); u32 sp = 0;
            while (uni(ld4_sys(ws + OFF_OF + it * 4)) < call) {
                if (give_up(sp, t0, (u32*)(ws + OFF_TMO), call)) { bad = true; break; }
                __builtin_amdgcn_s_sleep(8);
            }
        }
        const int tile = it >> 3, wq = it & 7;
        const u32 voff = ((u32)((tile >> 2) * 256 + (wq >> 2) * 64 + (lane >> 4) * 4) * GN + 8192 + P.rank * 1024 + (tile & 3) * 256 + (wq & 3) * 64 + (lane & 15) * 4) * 4u;
        char* og = ws + OFF_O + (size_t)it * QSLOT;
        u32 off16 = 0, ea = 0, eb = 0;
        u32x4 nx[8];
#pragma unroll
        for (int g = 0; g < 8; ++g) nx[g] = __builtin_amdgcn_raw_buffer_load_b128(rs, voff, ((g >> 2) * 16 + (g & 3)) * GN * 4, 17);
#pragma unroll 1
        for (int su = 0; su < 4; ++su) {
            u32 v[32]; u32 mx = 0;
#pragma unroll
            for (int r = 0; r < 32; ++r) { v[r] = bad ? 0x7fc00000u : nx[r >> 2][r & 3]; mx = max(mx, v[r]); }
            {
                const int sn = su < 3 ? su + 1 : 3;
#pragma unroll
                for (int g = 0; g < 8; ++g) nx[g] = __builtin_amdgcn_raw_buffer_load_b128(rs, voff, ((sn >> 1) * 128 + (sn & 1) * 32 + (g >> 2) * 16 + (g & 3)) * GN * 4, 17);
            }
            mx = wave_umax(mx) >> 16;
            const u32 b16 = mx >= 1023u ? mx - 1023u : 0u, B = b16 << 16;
            u32 n = 0, ml = 0, mh = 0;
#pragma unroll
            for (int r = 0; r < 32; ++r) {
                const u64 m = __builtin_amdgcn_ballot_w64(v[r] != 0u);
                const u32 ad = v[r] != 0u ? (n + prefix(m)) * 4u : 8240u;
                *(LAS u32*)(dense + ad) = v[r] - B;
                ml = llvm_writelane((u32)m, r, ml); mh = llvm_writelane((u32)(m >> 32), r, mh);
                n += __builtin_popcountll(m);
            }
            if (lane < 8) *(LAS u32*)(dense + (n + lane) * 4u) = 0u;
            if (lane < 32) *(LAS u32x2*)(img + lane * 8) = (u32x2){ml, mh};
            CFENCE();
            const u32 n8 = (n + 7u) >> 3, o_mid = 256u + n8 * 16u, o_hi = o_mid + n8 * 8u, o_esc = o_hi + n8 * 2u;
            u32 e = 0;
            for (u32 g0 = 0; g0 < n8; g0 += 64u) {
                const u32 g = g0 + lane;
                u64 W = 0;
                if (g < n8) {
                    const u32x4 a = *(LAS u32x4*)(dense + g * 32u), b = *(LAS u32x4*)(dense + g * 32u + 16u);
                    u32x4 lo;
                    lo.x = __builtin_amdgcn_perm(a.y, a.x, 0x05040100u); lo.y = __builtin_amdgcn_perm(a.w, a.z, 0x05040100u);
                    lo.z = __builtin_amdgcn_perm(b.y, b.x, 0x05040100u); lo.w = __builtin_amdgcn_perm(b.w, b.z, 0x05040100u);
                    const u32 u0 = __builtin_amdgcn_perm(a.y, a.x, 0x07060302u), u1 = __builtin_amdgcn_perm(a.w, a.z, 0x07060302u);
                    const u32 u2 = __builtin_amdgcn_perm(b.y, b.x, 0x07060302u), u3 = __builtin_amdgcn_perm(b.w, b.z, 0x07060302u);
                    const u32x2 mid = {__builtin_amdgcn_perm(u1, u0, 0x06040200u), __builtin_amdgcn_perm(u3, u2, 0x06040200u)};
                    const u32 ta = __builtin_amdgcn_perm(u1, u0, 0x07050301u), tb = __builtin_amdgcn_perm(u3, u2, 0x07050301u);
                    const u32 T = (ta & 0x03030303u) | ((tb & 0x03030303u) << 2);
                    *(LAS u32x4*)(img + 256u + g * 16u) = lo;
                    *(LAS u32x2*)(img + o_mid + g * 8u) = mid;
                    *(LAS unsigned short*)(img + o_hi + g * 2u) = (unsigned short)(T | (T >> 12));
                    W = ((u64)(tb & 0x7c7c7c7cu) << 32) | (ta & 0x7c7c7c7cu);
                }
                for (;;) {
                    const u64 em = __builtin_amdgcn_ballot_w64(W != 0ull);
                    if (em == 0ull) break;
                    if (W != 0ull) {
                        const u32 sh = (u32)__builtin_ctzll(W) & ~7u;
                        const u32 es = (u32)(W >> (sh + 2u)) & 31u;
                        *(LAS unsigned short*)(img + o_esc + (e + prefix(em)) * 2u) = (unsigned short)(((g * 8u + (sh >> 3)) << 5) | es);
                        W &= ~(0xffull << sh);
                    }
                    e += __builtin_popcountll(em);
                }
            }
            CFENCE();
            const u32 size16 = (o_esc + e * 2u + 15u) >> 4;
            if ((lane & 3) == su) { ea = off16 | (n << 14); eb = e | (b16 << 12); }
            for (u32 g = lane; g < size16; g += 64u) st16_sys(og + (off16 + g) * 16u, *(LAS u32x4*)(img + g * 16u));
            off16 += size16;
            LGKM0();
            CFENCE();
        }
        VMCNT(0);
        if (lane < 32 && (lane >> 2) != P.rank)
            st16_sys(peer_ws(P, lane >> 2) + OFF_REC + (call & 1u) * REC_PAR + (size_t)(P.rank * N_ITEMS + N_IN_ITEMS + it) * 64 + (lane & 3) * 16, (u32x4){call, ea, call, eb});
        if (it < 64) STAMP(16);
    }
    STAMP(17);
}

__global__ __launch_bounds__(NTHR, 2) __attribute__((amdgpu_waves_per_eu(2, 2))) void fused_kernel(Params P) {
    extern __shared__ __attribute__((aligned(16))) char smem[];
    LAS char* L = (LAS char*)smem;
    const int tid = threadIdx.x, lane = tid & 63, wave = __builtin_amdgcn_readfirstlane(tid >> 6);
    const int b = blockIdx.x, xg = b & 7, s = b >> 3;
    const bool fetch = s < FS;
    constexpr int NF = FS * 8, NG = NWG - NF;
    if (wave == 0) STAMP(0);
    GAS u32* const cnt = (GAS u32*)(P.me + OFF_CNT + (b * 8 + wave) * 8);
    constexpr bool diag = false;
    const int pidx = b - (NF + 120);
    const int q0 = ITEM0 + (wave >> 1) * NG + (b - NF);
    const bool loaded = !fetch && !diag && q0 < 112;
    f32x4 v[16];
    if (wave == 7 && (unsigned)pidx < (unsigned)NPUSH && !diag) {
        load_x(P, pidx, 0, lane, v);
        const u32 craw = __hip_atomic_load(cnt, __ATOMIC_RELAXED, __HIP_MEMORY_SCOPE_AGENT);
        __builtin_amdgcn_sched_barrier(0);
        BAR();
        __builtin_amdgcn_sched_barrier(0);
        STAMP(22);
        push_task(P, pidx, lane, craw, v);
        STAMP(23);
    } else {
        if (loaded) { if (q0 < 32) load_x(P, q0 * 2 + (wave & 1), 0, lane, v); else load_w(P, (q0 < 96 ? q0 + 32 : q0 - 96) * 2 + (wave & 1), 0, lane, v); }
        __builtin_amdgcn_sched_barrier(0);
        if (tid < 96) lds_st(L_ITEM + tid * 4, fetch && tid >= 24 && tid < 32 ? (u32)(tid - 24) : 0u);
        LGKM0();
        BAR();
    }
    if (!fetch && !diag) {
        const u64 tstart = now();
        u32 lpair = L_ITEM + (wave >> 1) * 4;
        auto fetch_task = [&](int q) {
            const u64 go = (u64)(PACE2 + (q - 112) * PACE3 / 112);
            while (now() - tstart < go) __builtin_amdgcn_s_sleep(4);
            int j, idx; bool isA;
            task_of(q, wave & 1, j, idx, isA);
            if (isA) load_x(P, idx, j, lane, v); else load_w(P, idx, j, lane, v);
        };
        if (!loaded) fetch_task(q0);
#pragma unroll 1
        for (int q = q0;; lpair += 16) {
            stage_task(P, L + wave * 4096, lpair, q, wave & 1, lane, cnt, v);
            q += 4 * NG;
            if (q >= N_PAIRS) break;
            fetch_task(q);
        }
    }
    const u32 call = uni(__hip_atomic_load(cnt, __ATOMIC_RELAXED, __HIP_MEMORY_SCOPE_AGENT)) + (diag ? 0u : 1u);
    if (lane == 0) st4_sys((void*)cnt, call);
    STAMP(fetch ? 1 : (wave == 0 ? 3 : 18));
    if (fetch) {
        const int src = (P.rank + 1 + s) & 7;
        if (wave == 0) { loader(P, src, lane, call); STAMP(2); }
        else if (wave == 1) scout(P, src, xg, lane, call);
        else { decoder(P, L, src, wave - 2, lane, call); STAMP(8 + wave); }
    } else if (s < FS + GS) {
        const int g = s - FS;
        gemm_wg(P, L, (xg >> 1) * 4 + g / 6, (xg & 1) * 6 + g % 6, wave, lane, call);
    } else {
        BAR();
        packer(P, L, wave * 8 + xg, wave, lane, call);
    }
}

}

extern "C" void kernel_launch(int rank, int nranks, ncclComm_t comm, hipStream_t stream,
                              void* const* d_in, const int* in_sizes, int n_in,
                              void* d_out, int out_size,
                              const int* in_dtypes, int out_dtype,
                              void* const* d_out_all,
                              void* const* d_ws_all, size_t ws_bytes) {
    (void)comm; (void)in_sizes; (void)n_in; (void)out_size; (void)in_dtypes; (void)out_dtype; (void)d_out_all;
    if (nranks != NR || (unsigned)rank >= (unsigned)NR || ws_bytes < WS_NEED) { fprintf(stderr, "unsupported configuration\n"); return; }
    static bool ready[NR];
    if (!ready[rank]) {
        HIP_CHECK(hipFuncSetAttribute((const void*)fused_kernel, hipFuncAttributeMaxDynamicSharedMemorySize, (int)LDS_BYTES));
        ready[rank] = true;
    }
    Params P;
    P.x = (const float*)d_in[0]; P.w = (const float*)d_in[1]; P.out = (float*)d_out;
    for (int i = 0; i < NR; ++i) P.ws[i] = (char*)d_ws_all[i];
    P.me = P.ws[rank]; P.rank = rank; P.pad = 0;
    hipLaunchKernelGGL(fused_kernel, dim3(NWG), dim3(NTHR), LDS_BYTES, stream, P);
    HIP_CHECK(hipGetLastError());
}
```

```cpp
#include <hip/hip_runtime.h>
#include <rccl/rccl.h>
#include <cstdio>

#define HIP_CHECK(expr) do { hipError_t _e = (expr); if (_e != hipSuccess) { \
    fprintf(stderr, "HIP error %s at %s:%d\n", hipGetErrorString(_e), __FILE__, __LINE__); \
    return; } } while (0)

typedef unsigned u32;
typedef unsigned long long u64;
typedef u32 u32x2 __attribute__((ext_vector_type(2)));
typedef u32 u32x4 __attribute__((ext_vector_type(4)));
typedef float f32x4 __attribute__((ext_vector_type(4)));
typedef _Float16 f16x8 __attribute__((ext_vector_type(8)));
#define LAS __attribute__((address_space(3)))
#define GAS __attribute__((address_space(1)))

namespace {

constexpr int NR = 8, GM = 4096, GN = 16384, KL = 512;
constexpr int NWG = 256, NTHR = 512;
#define WINV 8
#define NPUSH 36
#define GSTORE 0
#define GSLEEP 16
#define AHEAD 0
#define OSTORE 2
constexpr int FS = 7, GS = 24;
constexpr int LPP = 8;
constexpr int WIN = WINV;
constexpr int NLP = 36;
constexpr int N_IN_ITEMS = 832, N_OUT_ITEMS = 512, N_ITEMS = N_IN_ITEMS + N_OUT_ITEMS;
constexpr int N_PAIRS = 1280, ITEM0 = NPUSH / 2;
#define PACE2 150
#define PACE3 350
#define PACE4 800
static_assert(NPUSH % 2 == 0 && NPUSH >= 8 && NPUSH <= 64, "only x blocks of the first K group");
constexpr u64 DEADLINE = 30000000ull;

constexpr size_t OFF_CNT = 0;
constexpr size_t OFF_TMO = 16384;
constexpr size_t OFF_TICK = 16384 + 256;
constexpr size_t OFF_OF = 20480;
constexpr u32 U_B = 8192, N_UNITS = U_B + NLP * 256;
constexpr size_t OFF_FA = 32768;
constexpr size_t OFF_DBG = 262144;
constexpr size_t OFF_REC = 1ull << 20;
constexpr u32 REC_SRC = 65536, REC_OUT = 16384;
constexpr size_t REC_PAR = 1ull << 20;
constexpr size_t OFF_A = 4ull << 20;
constexpr size_t OFF_AX = OFF_A + (size_t)N_UNITS * 8192;
constexpr size_t OFF_BX = OFF_AX + (4ull << 20);
constexpr size_t X_PAR = 20ull << 20;
constexpr size_t OFF_O = OFF_AX + 2 * X_PAR;
constexpr size_t QSLOT = 32768;
constexpr size_t WS_NEED = OFF_O + N_OUT_ITEMS * QSLOT;
static_assert(NR * REC_SRC <= REC_PAR && N_IN_ITEMS * 16 <= REC_OUT && REC_OUT + N_OUT_ITEMS * 64 <= REC_SRC && OFF_FA + N_UNITS * 8 <= OFF_DBG, "workspace map");

constexpr u32 RING = 65536;
constexpr u32 L_SCR = RING, SCR_B = 8448;
constexpr u32 L_DESC = L_SCR + 6 * SCR_B, NQ = 64;
constexpr u32 L_ITEM = L_DESC + NQ * 16;
constexpr u32 L_FCTL = L_ITEM + 64;
constexpr u32 L_LAND = L_FCTL + 64;
constexpr u32 L_T1 = 131072, L_T2 = L_T1 + 512, L_ZERO = L_T2 + 512;
constexpr u32 PK_B = 19456, PK_IMG = 8320;
constexpr u32 LDS_BYTES = 8 * PK_B;
static_assert(L_ITEM + 1024 <= LDS_BYTES && L_LAND + 256 <= L_ITEM + 1024 && L_ZERO + 8192 <= LDS_BYTES && LDS_BYTES <= 163840, "LDS map");

struct Params {
    const float* x; const float* w; float* out;
    char* me;
    char* ws[NR];
    int rank; int pad;
};

#define VMCNT(n) asm volatile("s_waitcnt vmcnt(" #n ")" ::: "memory")
#define LGKM0() asm volatile("s_waitcnt lgkmcnt(0)" ::: "memory")
#define CFENCE() asm volatile("" ::: "memory")
#define BAR() do { CFENCE(); __builtin_amdgcn_s_barrier(); CFENCE(); } while (0)

__device__ __forceinline__ void st16_sys(void* p, u32x4 v) {
    asm volatile("global_store_dwordx4 %0, %1, off sc0 sc1\n\ts_nop 1" :: "v"(p), "v"(v) : "memory");
}
__device__ __forceinline__ void st_out(GAS float* p, f32x4 v) {
    asm volatile("global_store_dwordx4 %0, %1, off sc0 sc1 nt\n\ts_nop 1" :: "v"(p), "v"(v) : "memory");
}
__device__ __forceinline__ void st4_sys(void* p, u32 v) {
    asm volatile("global_store_dword %0, %1, off sc0 sc1" :: "v"(p), "v"(v) : "memory");
}
__device__ __forceinline__ void st8_sys(void* p, u32x2 v) {
    asm volatile("global_store_dwordx2 %0, %1, off sc0 sc1" :: "v"(p), "v"(v) : "memory");
}
__device__ __forceinline__ u32 ld4_sys(const void* p) {
    u32 v; asm volatile("global_load_dword %0, %1, off sc0 sc1\n\ts_waitcnt vmcnt(0)" : "=&v"(v) : "v"(p) : "memory"); return v;
}
__device__ __forceinline__ u32x2 ld8_sys(const void* p) {
    u32x2 v; asm volatile("global_load_dwordx2 %0, %1, off sc0 sc1\n\ts_waitcnt vmcnt(0)" : "=&v"(v) : "v"(p) : "memory"); return v;
}
__device__ __forceinline__ void lds_st(u32 a, u32 v) { asm volatile("ds_write_b32 %0, %1" :: "v"(a), "v"(v) : "memory"); }
__device__ __forceinline__ u32 lds_ld(u32 a) {
    u32 v; asm volatile("ds_read_b32 %0, %1\n\ts_waitcnt lgkmcnt(0)" : "=&v"(v) : "v"(a) : "memory"); return v;
}
__device__ __forceinline__ u32x4 lds_ld16(u32 a) {
    u32x4 v; asm volatile("ds_read_b128 %0, %1\n\ts_waitcnt lgkmcnt(0)" : "=&v"(v) : "v"(a) : "memory"); return v;
}
__device__ __forceinline__ u32 uni(u32 v) { return __builtin_amdgcn_readfirstlane(v); }
__device__ __forceinline__ u32 lds_ldu(u32 a) { return uni(lds_ld(a)); }
__device__ __forceinline__ u64 now() { return __builtin_amdgcn_s_memrealtime(); }
__device__ __forceinline__ u32 prefix(u64 m) { return __builtin_amdgcn_mbcnt_hi((u32)(m >> 32), __builtin_amdgcn_mbcnt_lo((u32)m, 0u)); }

__device__ __forceinline__ u32 wave_umax(u32 v) {
    v = max(v, (u32)__builtin_amdgcn_update_dpp(0, (int)v, 0x111, 0xf, 0xf, true));
    v = max(v, (u32)__builtin_amdgcn_update_dpp(0, (int)v, 0x112, 0xf, 0xf, true));
    v = max(v, (u32)__builtin_amdgcn_update_dpp(0, (int)v, 0x114, 0xf, 0xf, true));
    v = max(v, (u32)__builtin_amdgcn_update_dpp(0, (int)v, 0x118, 0xf, 0xf, true));
    v = max(v, (u32)__builtin_amdgcn_update_dpp(0, (int)v, 0x142, 0xa, 0xf, false));
    v = max(v, (u32)__builtin_amdgcn_update_dpp(0, (int)v, 0x143, 0xc, 0xf, false));
    return __builtin_amdgcn_readlane(v, 63);
}
__device__ __forceinline__ u32 wave_scan(u32 v) {
    v += (u32)__builtin_amdgcn_update_dpp(0, (int)v, 0x111, 0xf, 0xf, true);
    v += (u32)__builtin_amdgcn_update_dpp(0, (int)v, 0x112, 0xf, 0xf, true);
    v += (u32)__builtin_amdgcn_update_dpp(0, (int)v, 0x114, 0xf, 0xf, true);
    v += (u32)__builtin_amdgcn_update_dpp(0, (int)v, 0x118, 0xf, 0xf, true);
    v += (u32)__builtin_amdgcn_update_dpp(0, (int)v, 0x142, 0xa, 0xf, false);
    v += (u32)__builtin_amdgcn_update_dpp(0, (int)v, 0x143, 0xc, 0xf, false);
    return v;
}
typedef unsigned short u16x2 __attribute__((ext_vector_type(2)));
typedef short i16x2 __attribute__((ext_vector_type(2)));
#define PU(x) __builtin_bit_cast(u16x2, (u32)(x))
#define PI(x) __builtin_bit_cast(i16x2, (u32)(x))
#define PW(x) __builtin_bit_cast(u32, (x))
__device__ __forceinline__ u32 pk_offsets(const u32* m, u32* ac) {
    u32 tot;
    ac[0] = 0;
    asm("v_pk_mul_lo_u16 %0, %4, -3 op_sel_hi:[1,0]\n\tv_pk_mad_u16 %1, %5, -3, %0 op_sel_hi:[1,0,1]\n\t"
        "v_pk_mad_u16 %2, %6, -3, %1 op_sel_hi:[1,0,1]\n\tv_pk_mad_u16 %3, %7, -3, %2 op_sel_hi:[1,0,1]"
        : "=&v"(ac[1]), "=&v"(ac[2]), "=&v"(ac[3]), "=&v"(tot) : "v"(m[0]), "v"(m[1]), "v"(m[2]), "v"(m[3]));
    return tot;
}
__device__ __forceinline__ u32 pk_expand(u32 tt, u32 sh, u32 m, u32 k, u32 base) {
    const u32 sv = PW((u16x2)(PU(tt) >> PU(sh))) & m & 0x00070007u;
    u32 r;
    asm("v_pk_min_u16 %0, %1, 1 op_sel_hi:[1,0]\n\tv_pk_mad_u16 %0, %0, 3, %1 op_sel_hi:[1,0,1]\n\tv_pk_mad_u16 %0, %0, %2, %3"
        : "=&v"(r) : "v"(sv), "s"(k), "v"(base));
    return r;
}
template <int SH> __device__ __forceinline__ void pk_classify(const u32* x, u32* c, u32* g, u32& T, u32& len) {
    u32 m[4], ac[4], sv[4];
#pragma unroll
    for (int k = 0; k < 4; ++k) {
        const u32 o = PW((i16x2)(PI(x[k]) >> (i16x2){SH, SH}));
        sv[k] = PW(__builtin_elementwise_max((i16x2)(-PI(o)), (i16x2){0, 0})) & 0x00070007u;
        c[k] = PW(__builtin_elementwise_min(PU(o), (u16x2){3, 3}));
        m[k] = PW((i16x2)(PI(c[k] + 0x7ffd7ffdu) >> (i16x2){15, 15}));
        g[k] = PW(__builtin_elementwise_sub_sat((u16x2)(PU(o) + (u16x2){7, 7}), (u16x2){10, 10}));
    }
    const u32 acc = pk_offsets(m, ac);
    const u32 TT = sv[0] | PW((u16x2)(PU(sv[1]) << PU(ac[1]))) | PW((u16x2)(PU(sv[2]) << PU(ac[2]))) | PW((u16x2)(PU(sv[3]) << PU(ac[3])));
    T = (TT & 0xffffu) | ((TT >> 16) << (acc & 31u));
    len = (acc & 0xffffu) + (acc >> 16);
}
__device__ __forceinline__ void emit_escapes(const u32* g, const u32* raw, u32 idx0, LAS char* rec, u32 cap, u32& e) {
    if (__builtin_amdgcn_ballot_w64((g[0] | g[1] | g[2] | g[3]) != 0u) == 0ull) return;
    u32 gm = 0;
#pragma unroll
    for (int k = 0; k < 4; ++k) gm |= PW(__builtin_elementwise_min(PU(g[k]), (u16x2){1, 1})) << k;
#pragma unroll 1
    for (;;) {
        const u64 em = __builtin_amdgcn_ballot_w64(gm != 0u);
        if (em == 0ull) break;
        const u32 b = __builtin_ctz(gm | 0x80000000u), k = b & 3u;
        const u32 m1 = (u32)__builtin_amdgcn_sbfe((int)b, 0, 1), m2 = (u32)__builtin_amdgcn_sbfe((int)b, 1, 1);
        const u32 r01 = (raw[1] & m1) | (raw[0] & ~m1), r23 = (raw[3] & m1) | (raw[2] & ~m1), r = (r23 & m2) | (r01 & ~m2);
        if (gm != 0u && e < cap) *(LAS u32*)(rec + (e + prefix(em)) * 4u) = (idx0 + k * 2u + (b >> 4)) | ((r >> (b & 16u)) << 16);
        gm &= gm - 1u;
        e += __builtin_popcountll(em);
    }
}
__device__ __forceinline__ float amax3(float m, float a, float b) { float r; asm("v_max3_f32 %0, %1, |%2|, |%3|" : "=v"(r) : "v"(m), "v"(a), "v"(b)); return r; }
__device__ __forceinline__ char* peer_ws(const Params& P, int p) {
    u64 wp = (u64)P.ws[0];
#pragma unroll
    for (int i = 1; i < NR; ++i) { u64 q = (u64)P.ws[i]; asm volatile("" : "+s"(q)); wp = p == i ? q : wp; }
    return (char*)wp;
}

#define STAMP(k) do { } while (0)
#define STAMPV(k, v) do { } while (0)

__device__ __forceinline__ bool give_up(u32& spins, u64 t0, u32* tmo, u32 call) {
    if ((++spins & 63u) != 0u) return false;
    if (uni(ld4_sys(tmo)) == call) return true;
    if (now() - t0 > DEADLINE) { st4_sys(tmo, call); return true; }
    return false;
}

__device__ __forceinline__ float block_scale(const f32x4 (&v)[16], u32& code, u32& hm) {
    float mf = 0.f;
#pragma unroll
    for (int i = 0; i < 16; ++i) mf = amax3(amax3(mf, v[i][0], v[i][1]), v[i][2], v[i][3]);
    const u32 mb = wave_umax(__float_as_uint(mf));
    code = ((mb >> 23) + 2u) >> 3; code = code < 2u ? 2u : code;
    const float sc = __uint_as_float((263u - 8u * code) << 23);
    hm = uni(__builtin_bit_cast(unsigned short, (_Float16)(__uint_as_float(mb) * sc)));
    if (mb == 0u) code = 63u;
    return sc;
}
__device__ __forceinline__ void load_x(const Params& P, int idx, int j, int lane, f32x4 (&v)[16]) {
    const GAS float* p = (const GAS float*)P.x + (size_t)(idx * 64 + (lane & 15)) * KL + j * 64 + (lane >> 4) * 4;
#pragma unroll
    for (int i = 0; i < 16; ++i) v[i] = __builtin_nontemporal_load((const GAS f32x4*)(p + (i >> 2) * 16 * KL + (i & 3) * 16));
}
__device__ __forceinline__ void push_task(const Params& P, int idx, int lane, u32 craw, const f32x4 (&v)[16]) {
    u32 code, hm;
    const float sc = block_scale(v, code, hm);
    const u32 call = uni(craw) + 1u;
    const u32 dest = (call & 1u) * 4096u + ((idx >> 1) * 64 + P.rank) * 2 + (idx & 1);
    const u32 voff = (u32)OFF_A + dest * 8192u + lane * 16u;
#pragma unroll
    for (int f = 0; f < 8; ++f) { f16x8 h;
#pragma unroll
        for (int e = 0; e < 8; ++e) h[e] = (_Float16)(v[f * 2 + (e >> 2)][e & 3] * sc);
        const u32x4 hv = __builtin_bit_cast(u32x4, h);
        const u32 vo = voff + f * 1024;
#pragma unroll
        for (int r = 0; r < NR; ++r) asm volatile("global_store_dwordx4 %0, %1, %2 sc0 sc1\n\ts_nop 1" :: "v"(vo), "v"(hv), "s"((u64)P.ws[r]) : "memory");
    }
    VMCNT(0);
    if (lane < NR) st8_sys(peer_ws(P, lane) + OFF_FA + (size_t)dest * 8, (u32x2){call, code});
}
__device__ __forceinline__ void load_w(const Params& P, int idx, int j, int lane, f32x4 (&v)[16]) {
    const GAS float* p = (const GAS float*)P.w + (size_t)(j * 64 + (lane >> 4) * 4) * GN + idx * 64 + (lane & 15) * 4;
#pragma unroll
    for (int i = 0; i < 16; ++i) v[i] = __builtin_nontemporal_load((const GAS f32x4*)(p + (size_t)((i >> 2) * 16 + (i & 3)) * GN));
}
__device__ __forceinline__ void task_of(int q, int half, int& j, int& idx, bool& isA) {
    if (q < 896) { j = q / 112; const int u = q % 112; isA = u < 32; idx = isA ? u : (u < 96 ? u + 32 : u - 96); }
    else { const int u = q - 896, v = u % 192; j = v / 24; isA = false; idx = 16 + (u / 192) * 24 + v % 24; }
    idx = idx * 2 + half;
}
__device__ __forceinline__ void stage_task(const Params& P, LAS char* scr, u32 lquad, u32 lpair, int q, int half, int lane, GAS u32* cnt, const f32x4 (&v)[16]) {
    int j, idx; bool isA;
    task_of(q, half, j, idx, isA);
    const u32 craw = __hip_atomic_load(cnt, __ATOMIC_RELAXED, __HIP_MEMORY_SCOPE_AGENT);
    *(LAS u32x4*)(scr + lane * 16) = (u32x4){0u, 0u, 0u, 0u}; *(LAS u32x4*)(scr + 1024 + lane * 16) = (u32x4){0u, 0u, 0u, 0u};
    __builtin_amdgcn_sched_barrier(0);
    u32 code, hm;
    const float sc = block_scale(v, code, hm);
    const u32 bb = hm >= 4095u ? hm - 4095u : 0u, b2 = bb * 0x10001u;
    u32x4 p[8];
    if (isA) {
#pragma unroll
        for (int f = 0; f < 8; ++f) { f16x8 h;
#pragma unroll
            for (int e = 0; e < 8; ++e) h[e] = (_Float16)(v[f * 2 + (e >> 2)][e & 3] * sc);
            p[f] = __builtin_bit_cast(u32x4, h); }
    } else {
#pragma unroll
        for (int f = 0; f < 8; ++f) { f16x8 h;
#pragma unroll
            for (int e = 0; e < 8; ++e) h[e] = (_Float16)(v[(f & 1) * 8 + e][f >> 1] * sc);
            p[f] = __builtin_bit_cast(u32x4, h); }
    }
    char* ws = P.me;
    const u32 call = uni(craw) + 1u;
    const bool all = isA || idx < 128;
    const int owner = (idx - 128) >> 4, lp = idx < 128 ? 4 + (idx >> 2) : (idx >> 2) & 3;
    const bool local = all || owner == P.rank, exprt = all || owner != P.rank;
    const u32 dest = isA ? (call & 1u) * 4096u + (((idx >> 1) * 8 + j) * 8 + P.rank) * 2 + half : U_B + (((lp * 8 + j) * 8 + P.rank) * 4 + (idx & 3));
    if (local) {
        char* d = ws + OFF_A + (size_t)dest * 8192 + lane * 16;
#pragma unroll
        for (int f = 0; f < 8; ++f) st16_sys(d + f * 1024, p[f]);
    }
    u32 size16 = 0;
    if (exprt) {
        u32 w[32], T[8], len[8], e = 0, tot = 0;
#pragma unroll
        for (int f = 0; f < 8; ++f) {
            u32 x[4], c[4], g[4];
            const u32 h[4] = {p[f][0], p[f][1], p[f][2], p[f][3]};
#pragma unroll
            for (int k = 0; k < 4; ++k) x[k] = PW((u16x2)(PU(h[k] & 0x7fff7fffu) - PU(b2)));
            pk_classify<10>(x, c, g, T[f], len[f]);
#pragma unroll
            for (int k = 0; k < 4; ++k) w[f * 4 + k] = (h[k] & 0x80008000u) | (c[k] << 10) | (x[k] & 0x03ff03ffu);
            tot += len[f];
            emit_escapes(g, h, f * 512 + lane * 8u, scr + 4, 384u, e);
        }
        const u32 inc = wave_scan(tot);
        size16 = (6656u + 4u + e * 4u + (((u32)__builtin_amdgcn_readlane(inc, 63) + 31u) >> 5) * 4u + 15u) >> 4;
        if (size16 >= 512u) size16 = 512u;
        u32 off16 = 0;
        if (half == 0) { if (lane == 0) lds_st(lpair, size16); }
        else {
            const u64 t0 = now(); u32 sp = 0;
            while ((off16 = lds_ldu(lpair)) == 0u) {
                if ((++sp & 1023u) == 0u && now() - t0 > 4 * DEADLINE) { st4_sys(ws + OFF_TMO, call); break; }
                __builtin_amdgcn_s_sleep(1);
            }
        }
        char* d = ws + (call & 1u) * X_PAR + off16 * 16u + lane * 16 +
                  (isA ? OFF_AX + (size_t)((idx >> 1) * 8 + j) * 16384 : OFF_BX + (size_t)(((idx >> 2) * 8 + j) * 2 + ((idx >> 1) & 1)) * 16384);
        if (size16 == 512u) {
#pragma unroll
            for (int f = 0; f < 8; ++f) st16_sys(d + f * 1024, p[f]);
        } else {
            u32 bp = (e + 1u) * 32u + inc - tot;
#pragma unroll
            for (int f = 0; f < 8; ++f) {
                const u64 t = (u64)T[f] << (bp & 31u);
                LAS u32* a = (LAS u32*)(scr + (bp >> 5) * 4u);
                __hip_atomic_fetch_or(a, (u32)t, __ATOMIC_RELAXED, __HIP_MEMORY_SCOPE_WAVEFRONT);
                __hip_atomic_fetch_or(a + 1, (u32)(t >> 32), __ATOMIC_RELAXED, __HIP_MEMORY_SCOPE_WAVEFRONT);
                bp += len[f];
            }
            if (lane == 0) *(LAS u32*)scr = e;
#pragma unroll
            for (int i = 0; i < 24; ++i) w[i] |= (w[26 + (i >> 2)] << (12 - 3 * (i & 3))) & 0x70007000u;
#pragma unroll
            for (int i = 0; i < 6; ++i) w[24 + i / 3] |= (w[26 + i] >> (1 + i % 3)) & (0x40004000u >> (i % 3));
#pragma unroll
            for (int f = 0; f < 6; ++f) st16_sys(d + f * 1024, (u32x4){w[f * 4], w[f * 4 + 1], w[f * 4 + 2], w[f * 4 + 3]});
            st8_sys(d + 6144 - lane * 8, (u32x2){w[24], w[25]});
            CFENCE();
            for (u32 g = lane; g + 416u < size16; g += 64u) st16_sys(d + 6656 + (g - lane) * 16u, *(LAS u32x4*)(scr + g * 16u));
        }
    }
    VMCNT(0);
    if (local && lane == 0) st8_sys(ws + OFF_FA + (size_t)dest * 8, (u32x2){call, code});
    if (exprt) {
        const int hb = (lp % 12) * 2 + ((idx >> 1) & 1);
        const int item = isA ? j * 56 + (idx >> 1) : (lp < 12 ? j * 56 + 32 + hb : 256 + (lp / 12) * 192 + j * 24 + hb);
        const u32 info = size16 | (bb << 10) | (code << 25);
        const size_t ro = OFF_REC + (call & 1u) * REC_PAR + P.rank * REC_SRC;
        if (q < 112) {
            if (lane < NR && lane != P.rank && (all || lane == owner)) st8_sys(peer_ws(P, lane) + ro + item * 16 + half * 8, (u32x2){call, info});
        } else {
            u32 old = 0;
            if (lane == 0) asm volatile("ds_write_b32 %1, %2\n\tds_add_rtn_u32 %0, %3, %4\n\ts_waitcnt lgkmcnt(0)"
                                        : "=&v"(old) : "v"(lquad + ((item & 3) * 2 + half) * 4), "v"(info), "v"(lquad + 32), "v"(1u) : "memory");
            if (uni(old) == 7u) {
                u32x2 in2; asm volatile("ds_read_b64 %0, %1\n\ts_waitcnt lgkmcnt(0)" : "=&v"(in2) : "v"(lquad + (lane & 3) * 8) : "memory");
                const int r = lane >> 2;
                if (lane < 32 && r != P.rank && (all || r == owner)) st16_sys(peer_ws(P, r) + ro + ((item & ~3) + (lane & 3)) * 16, (u32x4){call, in2.x, call, in2.y});
            }
        }
    }
    CFENCE();
}

__device__ __forceinline__ void scout(const Params& P, int src, int li, int lane, u32 call) {
    char* ws = P.me;
    u32* tmo = (u32*)(ws + OFF_TMO);
    u32* tick = (u32*)(ws + OFF_TICK + src * 256);
    const char* rec = ws + OFF_REC + (call & 1u) * REC_PAR + src * REC_SRC;
    const u32 xpar = (call & 1u) * (u32)X_PAR;
    const u32 tbase = (call - 1u) * (u32)(N_ITEMS - ITEM0) - (u32)(LPP + ITEM0);
    u32 item = ITEM0 + li, pieces = 0, nq = 0, k = 0;
    __builtin_amdgcn_s_setprio(2);
    while (item < (u32)N_ITEMS) {
        const bool isin = item < (u32)N_IN_ITEMS;
        u32x2 r; bool dead = false;
        u32 tk;
        if (lane == 0) asm volatile("global_atomic_add %0, %1, %2, off sc0" : "=v"(tk) : "v"(tick), "v"(1u) : "memory");
        {
            const u64 t0 = now(); u32 sp = 0;
            for (;;) {
                r = ld8_sys(rec + (isin ? item * 16u + (lane & 1) * 8u : REC_OUT + (item - N_IN_ITEMS) * 64u + (lane & 7) * 8u));
                if (__builtin_amdgcn_ballot_w64(r.x >= call) == __builtin_amdgcn_ballot_w64(true)) break;
                if (give_up(sp, t0, tmo, call)) { dead = true; break; }
                __builtin_amdgcn_s_sleep(2);
            }
        }
        asm volatile("" : "+v"(tk));
        const u32 next = uni(tk) - tbase;
        u32 soff, bytes, nsub; u32x4 d;
        if (isin) {
            u32 dest;
            u32 j, u;
            if (item < 448u) { j = item / 56u; u = item % 56u; }
            else { const u32 v = item - 448u, w = v % 192u; j = w / 24u; u = 56u + (v / 192u) * 24u + w % 24u; }
            if (u >= 32u) {
                const u32 lp = (u - 32u) >> 1, h = u & 1u, gp = lp < 4u ? 32u + P.rank * 4u + lp : lp - 4u;
                soff = (u32)OFF_BX + xpar + (gp * 8 + j) * 32768u + h * 16384u;
                dest = U_B + ((lp * 8 + j) * 8 + src) * 4 + h * 2;
            } else {
                soff = (u32)OFF_AX + xpar + (u * 8 + j) * 16384u;
                dest = (call & 1u) * 4096u + ((u * 8 + j) * 8 + src) * 2;
            }
            const u32 s = lane & 1, sz = min(r.y & 1023u, 512u), sb = dead ? 0u : sz * 16u;
            const u32 b0 = __builtin_amdgcn_readlane(sb, 0);
            bytes = b0 + __builtin_amdgcn_readlane(sb, 1); nsub = 2;
            d.x = pieces * 1024u + (s ? b0 : 0u); d.y = pieces + (((s ? bytes : b0) + 1023u) >> 10);
            d.z = dead ? (62u << 12) | (2u << 24) : sz | ((r.y >> 25) << 12); d.w = ((r.y >> 10) & 0x7fffu) | ((dest + s) << 16);
        } else {
            const u32 o = item - N_IN_ITEMS;
            soff = (u32)OFF_O + o * (u32)QSLOT;
            const u32 a = __builtin_amdgcn_ds_bpermute((lane & 3) * 8, r.y), b = __builtin_amdgcn_ds_bpermute((lane & 3) * 8 + 4, r.y);
            const u32 sb = dead ? 0u : min(a & 1023u, 512u) * 16u, n = min((a >> 10) & 0xfffu, 2048u), e = b & 0xfffu, b16 = (b >> 12) & 0x7fffu;
            const u32 s0 = __builtin_amdgcn_readlane(sb, 0), s1 = s0 + __builtin_amdgcn_readlane(sb, 1), s2 = s1 + __builtin_amdgcn_readlane(sb, 2);
            const u32 off = (lane & 3) == 0 ? 0u : (lane & 3) == 1 ? s0 : (lane & 3) == 2 ? s1 : s2;
            bytes = s2 + __builtin_amdgcn_readlane(sb, 3); nsub = 4;
            d.x = pieces * 1024u + off; d.y = pieces + ((off + sb + 1023u) >> 10); d.z = n | (e << 12) | ((dead ? 3u : sb == 8192u ? 4u : 1u) << 24);
            d.w = b16 | ((o * 4u + (lane & 3)) << 16);
        }
        const u32 np = (bytes + 1023u) >> 10;
        {
            const u64 t0 = now(); u32 sp = 0;
            for (;;) {
                u32 nx = lds_ld(L_FCTL + (lane < 6 ? 32 + lane * 4 : 12));
                u32 qm = __builtin_amdgcn_readlane(nx, 0);
#pragma unroll
                for (int i = 1; i < 6; ++i) qm = min(qm, (u32)__builtin_amdgcn_readlane(nx, i));
                const u32 first = qm < nq ? lds_ldu(L_DESC + (qm % NQ) * 16) >> 10 : pieces;
                if ((u32)__builtin_amdgcn_readlane(nx, 6) + AHEAD >= k && pieces + np - first <= RING / 1024 && (qm >= nq || nq + nsub - qm <= NQ)) break;
                if ((++sp & 1023u) == 0u && now() - t0 > 4 * DEADLINE) { st4_sys(tmo, call); break; }
                __builtin_amdgcn_s_sleep(1);
            }
        }
        if (lane < (int)nsub) asm volatile("ds_write_b128 %0, %1" :: "v"(L_DESC + ((nq + lane) % NQ) * 16), "v"(d) : "memory");
        nq += nsub;
        if (lane == 0) {
            const u32x4 it = {soff, bytes, 0u, k + 1};
            asm volatile("ds_write_b128 %0, %1" :: "v"(L_ITEM + (k & 3u) * 16), "v"(it) : "memory");
            lds_st(L_FCTL + 4, nq);
        }
        pieces += np; ++k;
        item = next;
    }
    if (lane == 0) {
        const u32x4 it = {0u, 0xffffffffu, 0u, k + 1};
        asm volatile("ds_write_b128 %0, %1" :: "v"(L_ITEM + (k & 3u) * 16), "v"(it) : "memory");
        lds_st(L_FCTL + 4, nq | 0x80000000u);
    }
}

__device__ __forceinline__ void loader(const Params& P, int src, int lane, u32 call) {
    const u64 base = (u64)P.ws[src];
    const u32 voff = lane * 16, vland = L_LAND + lane * 4;
    u32* tmo = (u32*)(P.me + OFF_TMO);
    int issued = 0, landed = 0;
    u32 k = 0;
    __builtin_amdgcn_s_setprio(3);
#define PIECE(sp) do { asm volatile("s_mov_b32 m0, %2\n\ts_waitcnt vmcnt(%3)\n\tglobal_load_lds_dwordx4 %0, %1 sc0 sc1" \
                                    :: "v"(voff), "s"(sp), "s"(((u32)issued * 1024u) & (RING - 1)), "n"(WIN - 1) : "memory"); } while (0)
    for (;;) {
        u32x4 it = lds_ld16(L_ITEM + (k & 3u) * 16);
        if (uni(it.w) != k + 1 || uni(it.y) == 0xffffffffu) {
            VMCNT(0);
            landed = issued;
            lds_st(vland, (u32)issued);
            const u64 t0 = now(); u32 sp = 0;
            while (uni(it.w) != k + 1) {
                if ((++sp & 1023u) == 0u && now() - t0 > 4 * DEADLINE) { st4_sys(tmo, call); return; }
                __builtin_amdgcn_s_sleep(1);
                it = lds_ld16(L_ITEM + (k & 3u) * 16);
            }
            if (uni(it.y) == 0xffffffffu) return;
        }
        const u32 soff = uni(it.x);
        u32 left = uni(it.y);
        ++k;
        if (lane == 0) lds_st(L_FCTL + 12, k);
        u64 sp = base + soff;
#pragma unroll 1
        for (; left >= 1024u; left -= 1024u, sp += 1024) {
            PIECE(sp);
            lds_st(vland, (u32)max(landed, issued - (WIN - 1)));
            ++issued;
        }
        if (left) {
            if (voff < left) PIECE(sp);
            lds_st(vland, (u32)max(landed, issued - (WIN - 1)));
            ++issued;
        }
    }
#undef PIECE
}

template <int G> __device__ __forceinline__ void expand_dense(LAS char* L, LAS char* scr, u32 lane, u32 n8, u32 o_lo, u32 o_mid, u32 o_top, u32 o_sec, u32 b2) {
    u32x4 lo[G]; u32x2 mid[G]; u32 top[G], u[G][4], fl[G][4], ac[G][4], lw[G], t0[G], t1[G], bp[G], tot = 0;
#pragma unroll
    for (int i = 0; i < G; ++i) {
        const u32 g = i * 64 + lane;
        mid[i] = *(LAS u32x2*)(L + ((o_mid + g * 8u) & (RING - 1)));
        top[i] = *(LAS unsigned char*)(L + ((o_top + g) & (RING - 1)));
    }
#pragma unroll
    for (int i = 0; i < G; ++i) lo[i] = *(LAS u32x4*)(L + ((o_lo + (i * 64 + lane) * 16u) & (RING - 1)));
#pragma unroll
    for (int i = 0; i < G; ++i) {
        if (i == G - 1) top[i] = i * 64 + lane < n8 ? top[i] : 0u;
        const u32 tt = (top[i] | (top[i] << 12)) << 8;
#pragma unroll
        for (int k = 0; k < 4; ++k) {
            u[i][k] = __builtin_amdgcn_perm(0u, mid[i][k >> 1], (k & 1) ? 0x0c030c02u : 0x0c010c00u) | ((tt >> k) & 0x01000100u);
            fl[i][k] = PW((i16x2)(PI(u[i][k] + 0x7e807e80u) >> (i16x2){15, 15}));
        }
        const u32 acc = pk_offsets(fl[i], ac[i]);
        lw[i] = acc; bp[i] = tot; tot += (acc & 0xffffu) + (acc >> 16);
    }
    const u32 base = wave_scan(tot) - tot;
#pragma unroll
    for (int i = 0; i < G; ++i) {
        bp[i] += base;
        const u32 a = o_sec + (bp[i] >> 5) * 4u;
        t0[i] = *(LAS u32*)(L + (a & (RING - 1))); t1[i] = *(LAS u32*)(L + ((a + 4u) & (RING - 1)));
    }
#pragma unroll
    for (int i = 0; i < G; ++i) {
        const u32 T = __builtin_amdgcn_alignbit(t1[i], t0[i], bp[i]);
        const u32 TT = __builtin_amdgcn_perm(T >> (lw[i] & 31u), T, 0x05040100u);
        u32 r[8];
#pragma unroll
        for (int k = 0; k < 4; ++k) {
            const u32 h = pk_expand(TT, ac[i][k], fl[i][k], 0xff80ff80u, u[i][k] + b2);
            r[2 * k] = __builtin_amdgcn_perm(h, lo[i][k], 0x05040100u); r[2 * k + 1] = __builtin_amdgcn_perm(h, lo[i][k], 0x07060302u);
        }
        *(LAS u32x4*)(scr + (i * 64 + lane) * 32u) = (u32x4){r[0], r[1], r[2], r[3]};
        *(LAS u32x4*)(scr + (i * 64 + lane) * 32u + 16u) = (u32x4){r[4], r[5], r[6], r[7]};
    }
}

__device__ __forceinline__ void decoder(const Params& P, LAS char* L, int src, int dw, int lane, u32 call) {
    char* ws = P.me;
    u32* tmo = (u32*)(ws + OFF_TMO);
    LAS char* scr = L + L_SCR + dw * SCR_B;
    for (u32 q = dw;; q += 6) {
        {
            const u64 t0 = now(); u32 sp = 0;
            u32 nq = lds_ldu(L_FCTL + 4);
            while ((nq & 0x7fffffffu) <= q && !(nq >> 31)) {
                if ((++sp & 1023u) == 0u && now() - t0 > 4 * DEADLINE) { st4_sys(tmo, call); return; }
                __builtin_amdgcn_s_sleep(2);
                nq = lds_ldu(L_FCTL + 4);
            }
            if ((nq & 0x7fffffffu) <= q) return;
        }
        const u32x4 dv = lds_ld16(L_DESC + (q % NQ) * 16);
        const u32 pos = uni(dv.x), endp = uni(dv.y), z = uni(dv.z), wv = uni(dv.w);
        {
            const u64 t0 = now(); u32 sp = 0;
            while (lds_ldu(L_LAND) < endp) {
                if ((++sp & 1023u) == 0u && now() - t0 > 4 * DEADLINE) { st4_sys(tmo, call); break; }
                __builtin_amdgcn_s_sleep(1);
            }
        }
        const u32 type = z >> 24, dest = wv >> 16;
        if (type == 0u || type == 2u) {
            if (type == 0u) {
                const u32 sz = z & 0xfffu, b2 = (wv & 0xffffu) * 0x10001u, pl = pos + lane * 16u;
                char* d = ws + OFF_A + (size_t)dest * 8192;
                u32x4 c[8];
#pragma unroll
                for (int i = 0; i < 6; ++i) c[i] = *(LAS u32x4*)(L + ((pl + i * 1024u) & (RING - 1)));
                if (sz == 512u) {
                    c[6] = *(LAS u32x4*)(L + ((pl + 6144u) & (RING - 1))); c[7] = *(LAS u32x4*)(L + ((pl + 7168u) & (RING - 1)));
                    LGKM0();
#pragma unroll
                    for (int i = 0; i < 8; ++i) asm volatile("" : "+v"(c[i]));
                    if (lane == 0) lds_st(L_FCTL + 32 + dw * 4, q + 6);
#pragma unroll
                    for (int f = 0; f < 8; ++f) st16_sys(d + f * 1024 + lane * 16, c[f]);
                } else {
                    u32x2 c6 = *(LAS u32x2*)(L + ((pos + 6144u + lane * 8u) & (RING - 1)));
                    for (u32 g = lane; g + 416u < sz; g += 64u) *(LAS u32x4*)(scr + g * 16u) = *(LAS u32x4*)(L + ((pos + 6656u + g * 16u) & (RING - 1)));
                    LGKM0();
#pragma unroll
                    for (int i = 0; i < 6; ++i) asm volatile("" : "+v"(c[i]));
                    asm volatile("" : "+v"(c6));
                    if (lane == 0) lds_st(L_FCTL + 32 + dw * 4, q + 6);
                    const u32 e = min(uni(*(LAS u32*)scr), 384u);
                    u32 w[32], fl[32], ac[32], len[8], lo[8], tot = 0;
#pragma unroll
                    for (int i = 0; i < 24; ++i) w[i] = c[i >> 2][i & 3];
                    w[24] = c6.x; w[25] = c6.y;
#pragma unroll
                    for (int i = 0; i < 6; ++i) {
                        u32 t = (w[24 + i / 3] << (1 + i % 3)) & 0x80008000u;
#pragma unroll
                        for (int k = 0; k < 4; ++k) t |= (w[i * 4 + k] >> (12 - 3 * k)) & (0x00070007u << (3 * k));
                        w[26 + i] = t;
                    }
#pragma unroll
                    for (int f = 0; f < 8; ++f) {
#pragma unroll
                        for (int k = 0; k < 4; ++k) fl[f * 4 + k] = PW((i16x2)(PI((w[f * 4 + k] & 0x0c000c00u) + 0x74007400u) >> (i16x2){15, 15}));
                        const u32 acc = pk_offsets(fl + f * 4, ac + f * 4);
                        lo[f] = acc; len[f] = (acc & 0xffffu) + (acc >> 16); tot += len[f];
                    }
                    u32 bp = (e + 1u) * 32u + wave_scan(tot) - tot;
                    u32 T[8];
#pragma unroll
                    for (int f = 0; f < 8; ++f) {
                        const LAS u32* a = (const LAS u32*)(scr + (bp >> 5) * 4u);
                        T[f] = __builtin_amdgcn_alignbit(a[1], a[0], bp);
                        bp += len[f];
                    }
#pragma unroll
                    for (int f = 0; f < 8; ++f) {
                        const u32 TT = __builtin_amdgcn_perm(T[f] >> (lo[f] & 31u), T[f], 0x05040100u);
                        u32x4 o;
#pragma unroll
                        for (int k = 0; k < 4; ++k) {
                            const int i = f * 4 + k;
                            o[k] = pk_expand(TT, ac[i], fl[i], 0xfc00fc00u, PW((u16x2)(PU(w[i] & 0x8fff8fffu) + PU(b2))));
                        }
                        st16_sys(d + f * 1024 + lane * 16, o);
                    }
                    for (u32 g = lane; g < e; g += 64u) {
                        const u32 rr = *(LAS u32*)(scr + 4u + g * 4u);
                        asm volatile("global_store_short_d16_hi %0, %1, off sc0 sc1" :: "v"(d + (rr & 4095u) * 2u), "v"(rr) : "memory");
                    }
                }
                VMCNT(0);
            } else if (lane == 0) lds_st(L_FCTL + 32 + dw * 4, q + 6);
            if (lane == 0) st8_sys(ws + OFF_FA + (size_t)dest * 8, (u32x2){call, (z >> 12) & 63u});
            continue;
        }
        const u32 tile = dest >> 5, wq = (dest >> 2) & 7u, su = dest & 3u;
        const u32 row0 = (tile >> 2) * 256u + (su >> 1) * 128u + (wq >> 2) * 64u + (su & 1u) * 32u + (lane >> 4) * 4u;
        const u32 col0 = 8192u + src * 1024u + (tile & 3u) * 256u + (wq & 3u) * 64u + (lane & 15) * 4u;
        GAS float* o = (GAS float*)P.out + (size_t)row0 * GN + col0;
        if (type == 3u) {
            if (lane == 0) lds_st(L_FCTL + 32 + dw * 4, q + 6);
            const float qn = __builtin_nanf("");
#pragma unroll
            for (int g = 0; g < 8; ++g) *(GAS f32x4*)(o + (size_t)((g >> 2) * 16 + (g & 3)) * GN) = (f32x4){qn, qn, qn, qn};
            continue;
        }
        if (type == 4u) {
            u32x4 c[8];
#pragma unroll
            for (int g = 0; g < 8; ++g) c[g] = *(LAS u32x4*)(L + ((pos + g * 1024u + lane * 16u) & (RING - 1)));
            LGKM0();
#pragma unroll
            for (int g = 0; g < 8; ++g) asm volatile("" : "+v"(c[g]));
            if (lane == 0) lds_st(L_FCTL + 32 + dw * 4, q + 6);
#pragma unroll
            for (int g = 0; g < 8; ++g) st_out(o + (size_t)((g >> 2) * 16 + (g & 3)) * GN, __builtin_bit_cast(f32x4, c[g]));
            continue;
        }
        const u32 n = z & 0xfffu, e = (z >> 12) & 0xfffu, b2 = (wv & 0xffffu) * 0x10001u, n8 = (n + 7u) >> 3, G = (n8 + 63u) >> 6;
        u32 mk = *(LAS u32*)(L + ((pos + lane * 4u) & (RING - 1)));
        const u32 o_lo = pos + 256u, o_mid = o_lo + n8 * 16u, o_top = o_mid + n8 * 8u, o_esc = o_top + ((n8 + 3u) & ~3u), o_sec = o_esc + e * 4u;
        if (G == 3u) expand_dense<3>(L, scr, lane, n8, o_lo, o_mid, o_top, o_sec, b2);
        else if (G == 2u) expand_dense<2>(L, scr, lane, n8, o_lo, o_mid, o_top, o_sec, b2);
        else if (G == 1u) expand_dense<1>(L, scr, lane, n8, o_lo, o_mid, o_top, o_sec, b2);
        else if (G == 4u) expand_dense<4>(L, scr, lane, n8, o_lo, o_mid, o_top, o_sec, b2);
        CFENCE();
#pragma unroll 1
        for (u32 g = lane; g < e; g += 64u) {
            const u32 rr = *(LAS u32*)(L + ((o_esc + g * 4u) & (RING - 1)));
            *(LAS unsigned short*)(scr + (rr & 2047u) * 4u + 2u) = (unsigned short)(rr >> 16);
        }
        LGKM0();
        asm volatile("" : "+v"(mk));
        if (lane == 0) lds_st(L_FCTL + 32 + dw * 4, q + 6);
        {
            const u32 pc = __builtin_popcount(mk);
            int ad = (wave_scan(pc) - pc) * 4u;
            u32 r[32];
#pragma unroll
            for (int i = 0; i < 32; ++i) {
                const int m = __builtin_amdgcn_sbfe((int)mk, i, 1);
                r[i] = *(LAS u32*)(scr + ad) & (u32)m;
                ad -= m * 4;
            }
#pragma unroll
            for (int g = 0; g < 8; ++g)
                st_out(o + (size_t)((g >> 2) * 16 + (g & 3)) * GN, __builtin_bit_cast(f32x4, (u32x4){r[g * 4], r[g * 4 + 1], r[g * 4 + 2], r[g * 4 + 3]}));
        }
        CFENCE();
    }
}

__device__ __forceinline__ void gemm_wg(const Params& P, LAS char* L, int pm, int lp0, int wave, int lane, u32 call) {
    char* ws = P.me;
    const int wr = wave >> 2, wc = wave & 3;
    const u32 voffA = wave * 2048 + lane * 16, voffB = (wave >> 1) * 8192 + (wave & 1) * 2048 + lane * 16;
    const u32 boff = wc * 4096 + lane * 16, ldw = wave * 2048;
    auto lane_id = []() -> u32 { u32 l; asm volatile("v_mbcnt_lo_u32_b32 %0, -1, 0\n\tv_mbcnt_hi_u32_b32 %0, -1, %0" : "=v"(l)); return l; };
    constexpr size_t KA = 16384, KB = 32768, HA = 64 * KA, HB = 4096;
#define SA(b, h) (((b) * 2 + (h)) * 16384)
#define SB(b, h) ((4 + (b) * 2 + (h)) * 16384)
#define STAGE(bufoff, gbase, voff) asm volatile("s_mov_b32 m0, %2\n\ts_nop 0\n\tglobal_load_lds_dwordx4 %0, %1 sc0 sc1\n\tglobal_load_lds_dwordx4 %0, %1 offset:1024 sc0 sc1" \
        :: "v"(voff), "s"((u64)(gbase)), "s"((u32)(bufoff) + ldw) : "memory")
#define LDA(dst, b, h) do { _Pragma("unroll") for (int m = 0; m < 4; ++m) _Pragma("unroll") for (int k = 0; k < 2; ++k) \
        dst[m][k] = *(const LAS f16x8*)(L + SA(b, h) + ((h) ? aoff1 : aoff0) + m * 2048 + k * 1024); } while (0)
#define LDB(dst, b, h) do { _Pragma("unroll") for (int n = 0; n < 2; ++n) _Pragma("unroll") for (int k = 0; k < 2; ++k) \
        dst[n][k] = *(const LAS f16x8*)(L + SB(b, h) + boff + n * 2048 + k * 1024); } while (0)
#define MMA(ai, bj, At, Bt) do { __builtin_amdgcn_s_setprio(1); \
        _Pragma("unroll") for (int m = 0; m < 4; ++m) _Pragma("unroll") for (int n = 0; n < 2; ++n) _Pragma("unroll") for (int k = 0; k < 2; ++k) \
            acc[ai][bj][m][n] = __builtin_amdgcn_mfma_f32_16x16x32_f16(At[m][k], Bt[n][k], acc[ai][bj][m][n], 0, 0, 0); \
        __builtin_amdgcn_s_setprio(0); } while (0)
#define WAIT_L(n) asm volatile("s_waitcnt lgkmcnt(" #n ")" ::: "memory")
#define RBAR __builtin_amdgcn_s_barrier()
#define SCHED __builtin_amdgcn_sched_barrier(0)
    auto flag_ptr = [&](int lp, int j, int ent) -> const char* {
        return ws + OFF_FA + (size_t)(ent < 32 ? (call & 1u) * 4096u + ((pm * 2 + (ent >> 4)) * 8 + j) * 16 + (ent & 15) : U_B + (lp * 8 + j) * 32 + (ent - 32)) * 8; };
    auto poll = [&](int lp, int j) {
        const u32 ln = lane_id();
        const char* f = flag_ptr(lp, j, ln);
        const u64 t0 = now(); u32 sp = 0; u32x2 v;
        for (;;) {
            v = ld8_sys(f);
            if (__builtin_amdgcn_ballot_w64(v.x >= call) == ~0ull) break;
            if (give_up(sp, t0, (u32*)(ws + OFF_TMO), call)) { v.y = 62u; break; }
            __builtin_amdgcn_s_sleep(4);
        }
        asm volatile("ds_write_b64 %0, %1\n\ts_waitcnt lgkmcnt(0)" :: "v"(L_T2 + ln * 8), "v"(v) : "memory");
    };
    auto codes = [&](u32 tb) -> u32 {
        const u32 r = lane_id() & 7;
        const u32 a0 = *(const LAS u32*)(L + tb + wr * 8 + 4 + r * 16), a1 = *(const LAS u32*)(L + tb + 128 + wr * 8 + 4 + r * 16);
        const u32 b = *(const LAS u32*)(L + tb + 256 + wc * 8 + 4 + r * 32);
        return a0 | (a1 << 8) | (b << 16);
    };
    f32x4 acc[2][2][4][2];
    f16x8 At[4][2], B0[2][2], B1[2][2];
    u32 cur = 0, cmax = 0, bad = 0;
    u32 aoff0 = wr * 8192 + lane * 16, aoff1 = aoff0;
    u32 nm, nmn = 0;
    auto changes = [&](u32 nw, u32 old) -> u32 {
        const u32 prev = __builtin_amdgcn_update_dpp((int)__builtin_amdgcn_readlane(old, 7), (int)nw, 0x111, 0xf, 0xf, false);
        return (u32)__builtin_amdgcn_ballot_w64(nw != prev) & 255u; };
#define ADJUST(r, buf) do { if (__builtin_expect((nm >> (r)) & 1u, 0)) { const u32 cn_ = __builtin_amdgcn_readlane(pk, (r)), cb_ = (cn_ >> 16) & 63u; \
            const u32 l16_ = lane_id() * 16; \
            _Pragma("unroll") for (int ai = 0; ai < 2; ++ai) { \
                const u32 ca_ = (cn_ >> (8 * ai)) & 63u, c_ = ca_ + cb_, o_ = (cur >> (8 * ai)) & 255u, x_ = (cmax >> (8 * ai)) & 255u; \
                u32 ab_ = wr * 8192; int sh_ = 0; \
                if (ca_ == 62u || cb_ == 62u) bad = 1; \
                else if (ca_ != 63u && cb_ != 63u) { \
                    if (c_ + 10u < x_) { ab_ = L_ZERO - SA(buf, ai); nm = 255u; nmn |= 1u; }      \
                    else { sh_ = 8 * ((int)o_ - (int)c_); \
                        cur = (cur & ~(255u << (8 * ai))) | (c_ << (8 * ai)); \
                        if (c_ > x_) cmax = (cmax & ~(255u << (8 * ai))) | (c_ << (8 * ai)); } } \
                asm volatile("v_add_u32 %0, %1, %2" : "+v"(ai ? aoff1 : aoff0) : "s"(ab_), "v"(l16_)); \
                if (sh_ != 0) { \
                    asm volatile("s_nop 15" ::: "memory");        \
                    _Pragma("unroll") for (int bj = 0; bj < 2; ++bj) _Pragma("unroll") for (int m = 0; m < 4; ++m) _Pragma("unroll") for (int n = 0; n < 2; ++n) \
                        _Pragma("unroll") for (int e_ = 0; e_ < 4; ++e_) asm volatile("v_ldexp_f32 %0, %0, %1" : "+v"(acc[ai][bj][m][n][e_]) : "s"(sh_)); } } } } while (0)
#pragma unroll
    for (int a = 0; a < 2; ++a)
#pragma unroll
        for (int b = 0; b < 2; ++b)
#pragma unroll
            for (int m = 0; m < 4; ++m)
#pragma unroll
                for (int n = 0; n < 2; ++n) acc[a][b][m][n] = (f32x4){0.f, 0.f, 0.f, 0.f};
    const char* const cA = ws + OFF_A + (size_t)(call & 1u) * 4096 * 8192 + (size_t)pm * 2 * HA;
    const char* cB = ws + OFF_A + (size_t)U_B * 8192 + (size_t)lp0 * 64 * KB;
    *(LAS u32x4*)(L + L_ZERO + wave * 1024 + lane * 16) = (u32x4){0u, 0u, 0u, 0u};
    if (wave == 0) poll(lp0, 0);
    LGKM0();
    BAR();
    u32 pk = codes(L_T2), pkn = pk;
    nm = changes(pk, pk) | 1u;
    STAGE(SB(0, 0), cB, voffB); STAGE(SA(0, 0), cA, voffA); STAGE(SB(0, 1), cB + HB, voffB); STAGE(SA(0, 1), cA + HA, voffA);
    if (wr == 1) RBAR;
    VMCNT(4); RBAR;
    STAGE(SB(1, 0), cB + KB, voffB); STAGE(SA(1, 0), cA + KA, voffA); STAGE(SB(1, 1), cB + HB + KB, voffB);
    VMCNT(6); RBAR;
#pragma unroll 1
    for (int rd = 0; rd < 3; ++rd) {
        const int lp = lp0 + rd * 12;
        const char* nB = rd < 2 ? cB + 12 * 64 * KB : cB;
#define TRIP(r, a1, a2, b2, HOOK) do { \
            ADJUST(r, 0); \
            LDB(B0, 0, 0); SCHED; LDA(At, 0, 0); STAGE(SA(1, 1), (a1) + HA, voffA); \
            WAIT_L(8); RBAR; WAIT_L(0); MMA(0, 0, At, B0); RBAR; SCHED; \
            LDB(B1, 0, 1); STAGE(SB(0, 0), (b2), voffB); \
            RBAR; WAIT_L(0); MMA(0, 1, At, B1); RBAR; \
            LDA(At, 0, 1); STAGE(SA(0, 0), (a2), voffA); \
            RBAR; WAIT_L(0); MMA(1, 0, At, B0); RBAR; SCHED; \
            STAGE(SB(0, 1), (b2) + HB, voffB); \
            VMCNT(6); RBAR; MMA(1, 1, At, B1); RBAR; \
            ADJUST((r) + 1, 1); \
            LDB(B0, 1, 0); SCHED; LDA(At, 1, 0); STAGE(SA(0, 1), (a2) + HA, voffA); \
            WAIT_L(8); RBAR; WAIT_L(0); MMA(0, 0, At, B0); RBAR; SCHED; \
            LDB(B1, 1, 1); STAGE(SB(1, 0), (b2) + KB, voffB); \
            RBAR; WAIT_L(0); MMA(0, 1, At, B1); RBAR; \
            LDA(At, 1, 1); STAGE(SA(1, 0), (a2) + KA, voffA); \
            RBAR; WAIT_L(0); MMA(1, 0, At, B0); RBAR; SCHED; \
            STAGE(SB(1, 1), (b2) + KB + HB, voffB); \
            HOOK; \
            VMCNT(6); RBAR; MMA(1, 1, At, B1); RBAR; } while (0)
#pragma unroll 1
        for (int j = 0; j < 8; ++j) {
            const char* ga = cA + (size_t)j * 8 * KA; const char* gb = cB + (size_t)j * 8 * KB;
            const char* na = j == 7 ? cA : ga + 8 * KA; const char* nb = j == 7 ? nB : gb + 8 * KB;
            const bool more = j < 7 || rd < 2;
            const int nlp = j < 7 ? lp : lp + 12, nj = (j + 1) & 7;
            TRIP(0, ga + KA, ga + 2 * KA, gb + 2 * KB, do { } while (0));
            TRIP(2, ga + 3 * KA, ga + 4 * KA, gb + 4 * KB,
                 if (__builtin_expect(wave == 0 && more, 0)) { const u32 ln = lane_id(); const char* fp = flag_ptr(nlp, nj, ln * 2);
                     if (ln < 32) asm volatile("s_mov_b32 m0, %1\n\ts_nop 0\n\tglobal_load_lds_dwordx4 %0, off sc0 sc1" :: "v"(fp), "s"(L_T1) : "memory"); });
            TRIP(4, ga + 5 * KA, ga + 6 * KA, gb + 6 * KB, {
                 const u32 tag = *(const LAS u32*)(L + L_T1 + lane_id() * 8);
                 pkn = codes(L_T1);
                 if (__builtin_expect((__builtin_amdgcn_ballot_w64(tag >= call) != ~0ull) & more, 0)) {
                     if (wave == 0) poll(nlp, nj);
                     BAR();
                     pkn = codes(L_T2);
                 }
                 nmn |= changes(pkn, pk); });
            TRIP(6, ga + 7 * KA, na, nb, { pk = pkn; nm = nmn; nmn = 0; });
        }
        if (wave == 0) { STAMP(4 + rd * 2); STAMPV(50 + rd, __builtin_amdgcn_s_memtime()); }
        {
            const int ln = lane_id();
            const int gp = lp < 4 ? 32 + P.rank * 4 + lp : lp - 4;
            GAS float* o = (GAS float*)P.out + (size_t)(pm * 256 + wr * 64 + (ln >> 4) * 4) * GN + gp * 256 + wc * 64 + (ln & 15) * 4;
#pragma unroll
            for (int ai = 0; ai < 2; ++ai) {
                const int sh = 8 * (int)((cur >> (8 * ai)) & 255u) - 272;
#pragma unroll
                for (int m = 0; m < 4; ++m)
#pragma unroll
                    for (int r = 0; r < 4; ++r) {
                        f32x4 v;
#pragma unroll
                        for (int i = 0; i < 4; ++i) {
                            const float y = __builtin_elementwise_maximum(__builtin_ldexpf(acc[ai][i >> 1][m][i & 1][r], sh), 0.0f);
                            v[i] = __uint_as_float(bad ? 0x7fc00000u : __float_as_uint(y) & 0x7fffffffu);
                            acc[ai][i >> 1][m][i & 1][r] = 0.f;
                        }
                        st_out(o + (size_t)(ai * 128 + m * 16 + r) * GN, v);
                    }
            }
            if (lp < 4) {
                VMCNT(0);
                if (ln == 0) st4_sys(ws + OFF_OF + (size_t)((pm * 4 + lp) * 8 + wave) * 4, call);
            }
            cur = 0; cmax = 0; bad = 0; nm |= 1u;
        }
        if (wave == 0) STAMP(5 + rd * 2);
        cB = nB;
    }
    VMCNT(0);
    if (wr == 0) RBAR;
#undef SA
#undef SB
#undef STAGE
#undef LDA
#undef LDB
#undef MMA
#undef ADJUST
#undef TRIP
}

template <int G> __device__ __forceinline__ void pack_dense(LAS char* dense, LAS char* img, u32 lane, u32 n8, u32 o_mid, u32 o_top, u32 o_esc, u32 b2,
                                                            u32* T, u32* bp, u32& tot, u32& e) {
    u32x4 a[G], b[G], lo[G]; u32x2 mid[G]; u32 hi[G][4], ge[G][4], top[G];
#pragma unroll
    for (int i = 0; i < G; ++i) { a[i] = *(LAS u32x4*)(dense + (i * 64 + lane) * 32u); b[i] = *(LAS u32x4*)(dense + (i * 64 + lane) * 32u + 16u); }
#pragma unroll
    for (int i = 0; i < G; ++i) {
        const u32 dd[8] = {a[i].x, a[i].y, a[i].z, a[i].w, b[i].x, b[i].y, b[i].z, b[i].w};
        u32 x[4], c[4], f9[4], len, tp = 0;
#pragma unroll
        for (int k = 0; k < 4; ++k) {
            lo[i][k] = __builtin_amdgcn_perm(dd[2 * k + 1], dd[2 * k], 0x05040100u);
            hi[i][k] = __builtin_amdgcn_perm(dd[2 * k + 1], dd[2 * k], 0x07060302u);
            x[k] = PW((u16x2)(PU(hi[i][k]) - PU(b2)));
            if (i == G - 1) x[k] = i * 64 + lane < n8 ? x[k] : 0u;
        }
        pk_classify<7>(x, c, ge[i], T[i], len);
#pragma unroll
        for (int k = 0; k < 4; ++k) { f9[k] = (c[k] << 7) | (x[k] & 0x007f007fu); tp |= ((c[k] >> 1) & 0x00010001u) << k; }
        mid[i] = (u32x2){__builtin_amdgcn_perm(f9[1], f9[0], 0x06040200u), __builtin_amdgcn_perm(f9[3], f9[2], 0x06040200u)};
        top[i] = tp | (tp >> 12);
        bp[i] = tot; tot += len;
    }
#pragma unroll
    for (int i = 0; i < G; ++i) {
        const u32 g = i * 64 + lane;
        if (i < G - 1 || g < n8) {
            *(LAS u32x4*)(img + 256u + g * 16u) = lo[i];
            *(LAS u32x2*)(img + o_mid + g * 8u) = mid[i];
            *(LAS unsigned char*)(img + o_top + g) = (unsigned char)top[i];
        }
    }
#pragma unroll
    for (int i = 0; i < G; ++i) emit_escapes(ge[i], hi[i], (i * 64 + lane) * 8u, img + o_esc, (8192u - o_esc) >> 2, e);
}
__device__ __forceinline__ void packer(const Params& P, LAS char* L, int pw, int wave, int lane, u32 call) {
    char* ws = P.me;
    LAS char* dense = L + wave * PK_B;
    LAS char* img = dense + PK_IMG;
#define PST(x) do { } while (0)
    const auto rs = __builtin_amdgcn_make_buffer_rsrc((void*)P.out, 0, (u32)GM * GN * 4u, 0x00020000);
    auto wait_item = [&](int it) -> u32 {
        const u64 t0 = now(); u32 sp = 0;
        while (uni(ld4_sys(ws + OFF_OF + it * 4)) < call) {
            if (give_up(sp, t0, (u32*)(ws + OFF_TMO), call)) return 0x7fc00000u;
            __builtin_amdgcn_s_sleep(8);
        }
        return 0u;
    };
    u32x4 nx[8];
    auto fetch_block = [&](int it, int su) {
        const int tile = it >> 3, wq = it & 7;
        const u32 so = ((u32)((tile >> 2) * 256 + (su >> 1) * 128 + (wq >> 2) * 64 + (su & 1) * 32) * GN + 8192 + P.rank * 1024 + (tile & 3) * 256 + (wq & 3) * 64) * 4u;
#pragma unroll
        for (int g = 0; g < 8; ++g) nx[g] = __builtin_amdgcn_raw_buffer_load_b128(rs, ((lane >> 4) * 4 * GN + (lane & 15) * 4) * 4, so + ((g >> 2) * 16 + (g & 3)) * GN * 4, 17);
    };
    auto publish = [&](int it, u32 ea, u32 eb) {
        if (lane < 32 && (lane >> 2) != P.rank)
            st16_sys(peer_ws(P, lane >> 2) + OFF_REC + (call & 1u) * REC_PAR + P.rank * REC_SRC + REC_OUT + it * 64 + (lane & 3) * 16, (u32x4){call, ea, call, eb});
    };
    u32 nanm = wait_item(pw), nann = 0, off16 = 0, ea = 0, eb = 0;
    fetch_block(pw, 0);
#pragma unroll
    for (int g = 0; g < 8; ++g) asm volatile("" : "+v"(nx[g]));
#pragma unroll 1
    for (int bk = 0; bk < N_OUT_ITEMS / 16; ++bk) {
        {
            const int it = pw + (bk >> 2) * 64, su = bk & 3;
            char* og = ws + OFF_O + (size_t)it * QSLOT;
            u32 v[32]; u32 mx = 0;
#pragma unroll
            for (int r = 0; r < 32; ++r) { v[r] = nx[r >> 2][r & 3] | nanm; mx = max(mx, v[r]); }
            { const int bn = bk + 1 < N_OUT_ITEMS / 16 ? bk + 1 : bk; fetch_block(pw + (bn >> 2) * 64, bn & 3); }
            mx = wave_umax(mx) >> 16;
            PST(pa);
            const u32 b16 = mx >= 511u ? mx - 511u : 0u, b2 = b16 * 0x10001u;
            u32 f[32], mk0 = 0, mk1 = 0;
#pragma unroll
            for (int r = 0; r < 32; ++r) asm("v_min_u32 %0, 1, %1" : "=v"(f[r]) : "v"(v[r]));
#pragma unroll
            for (int r = 0; r < 16; ++r) { mk0 |= f[r] << r; mk1 |= f[16 + r] << (16 + r); }
            const u32 mk = mk0 | mk1, pc = __builtin_popcount(mk), inc = wave_scan(pc), n = __builtin_amdgcn_readlane(inc, 63);
            {
                u32 ad = wave * PK_B + (inc - pc) * 4u;
#pragma unroll
                for (int r = 0; r < 32; r += 8) {
                    u32 a[8]; u64 m[8];
#pragma unroll
                    for (int i = 0; i < 8; ++i) { a[i] = ad; ad += f[r + i] * 4u; m[i] = __builtin_amdgcn_ballot_w64(v[r + i] != 0u); }
                    asm volatile("s_mov_b64 exec, %16\n\tds_write_b32 %0, %8\n\ts_mov_b64 exec, %17\n\tds_write_b32 %1, %9\n\t"
                                 "s_mov_b64 exec, %18\n\tds_write_b32 %2, %10\n\ts_mov_b64 exec, %19\n\tds_write_b32 %3, %11\n\t"
                                 "s_mov_b64 exec, %20\n\tds_write_b32 %4, %12\n\ts_mov_b64 exec, %21\n\tds_write_b32 %5, %13\n\t"
                                 "s_mov_b64 exec, %22\n\tds_write_b32 %6, %14\n\ts_mov_b64 exec, %23\n\tds_write_b32 %7, %15\n\ts_mov_b64 exec, -1"
                                 :: "v"(a[0]), "v"(a[1]), "v"(a[2]), "v"(a[3]), "v"(a[4]), "v"(a[5]), "v"(a[6]), "v"(a[7]),
                                    "v"(v[r]), "v"(v[r + 1]), "v"(v[r + 2]), "v"(v[r + 3]), "v"(v[r + 4]), "v"(v[r + 5]), "v"(v[r + 6]), "v"(v[r + 7]),
                                    "s"(m[0]), "s"(m[1]), "s"(m[2]), "s"(m[3]), "s"(m[4]), "s"(m[5]), "s"(m[6]), "s"(m[7]) : "memory");
                }
            }
            if (lane < 8) *(LAS u32*)(dense + (n + lane) * 4u) = b16 << 16;
            *(LAS u32*)(img + lane * 4) = mk;
            const u32 n8 = (n + 7u) >> 3, G = (n8 + 63u) >> 6, o_mid = 256u + n8 * 16u, o_top = o_mid + n8 * 8u, o_esc = o_top + ((n8 + 3u) & ~3u);
            CFENCE();
            PST(pb);
            u32 e = 0, tot = 0, T[4] = {0u, 0u, 0u, 0u}, bp[4] = {0u, 0u, 0u, 0u};
            if (G == 3u) pack_dense<3>(dense, img, lane, n8, o_mid, o_top, o_esc, b2, T, bp, tot, e);
            else if (G == 2u) pack_dense<2>(dense, img, lane, n8, o_mid, o_top, o_esc, b2, T, bp, tot, e);
            else if (G == 1u) pack_dense<1>(dense, img, lane, n8, o_mid, o_top, o_esc, b2, T, bp, tot, e);
            else if (G == 4u) pack_dense<4>(dense, img, lane, n8, o_mid, o_top, o_esc, b2, T, bp, tot, e);
            PST(pc_);
#pragma unroll
            for (int g = 0; g < 8; ++g) asm volatile("" : "+v"(nx[g]));
            VMCNT(0);
            if (su == 0) { if (bk) publish(it - 64, ea, eb); off16 = 0; }
            if (su == 2 && it + 64 < N_OUT_ITEMS) nann = wait_item(it + 64);
            PST(pd);
            const u32 binc = wave_scan(tot), o_sec = o_esc + e * 4u;
            u32 size16 = (o_sec + (((u32)__builtin_amdgcn_readlane(binc, 63) + 31u) >> 5) * 4u + 15u) >> 4;
            if (size16 >= 512u) {
                size16 = 512u;
#pragma unroll
                for (int g = 0; g < 8; ++g) st16_sys(og + off16 * 16u + g * 1024 + lane * 16, (u32x4){v[g * 4], v[g * 4 + 1], v[g * 4 + 2], v[g * 4 + 3]});
            } else {
#pragma unroll
                for (int i = 0; i < 4; ++i) *(LAS u32*)(img + o_sec + i * 256 + lane * 4) = 0u;
                CFENCE();
#pragma unroll
                for (int i = 0; i < 4; ++i) if (i < (int)G) {
                    const u32 bq = bp[i] + binc - tot;
                    const u64 t = (u64)T[i] << (bq & 31u);
                    LAS u32* a = (LAS u32*)(img + o_sec + (bq >> 5) * 4u);
                    __hip_atomic_fetch_or(a, (u32)t, __ATOMIC_RELAXED, __HIP_MEMORY_SCOPE_WAVEFRONT);
                    __hip_atomic_fetch_or(a + 1, (u32)(t >> 32), __ATOMIC_RELAXED, __HIP_MEMORY_SCOPE_WAVEFRONT);
                }
                CFENCE();
#pragma unroll 1
                for (u32 g = lane; g < size16 + lane; g += 256u) {
                    u32x4 t[4];
#pragma unroll
                    for (int i = 0; i < 4; ++i) t[i] = *(LAS u32x4*)(img + (g + i * 64) * 16u);
#pragma unroll
                    for (int i = 0; i < 4; ++i) if (g + i * 64 < size16) st16_sys(og + (off16 + g + i * 64) * 16u, t[i]);
                }
            }
            if ((lane & 3) == su) { ea = size16 | (n << 10); eb = (e & 0xfffu) | (b16 << 12); }
            off16 += size16;
            if (su == 3) nanm = nann;
            PST(pe);
            LGKM0();
            CFENCE();
        }
        if (bk == 4) STAMP(16);
    }
    VMCNT(0);
    publish(pw + N_OUT_ITEMS - 64, ea, eb);
    STAMP(17);
}

__global__ __launch_bounds__(NTHR, 2) __attribute__((amdgpu_waves_per_eu(2, 2))) void fused_kernel(Params P) {
    extern __shared__ __attribute__((aligned(16))) char smem[];
    LAS char* L = (LAS char*)smem;
    const int tid = threadIdx.x, lane = tid & 63, wave = __builtin_amdgcn_readfirstlane(tid >> 6);
    const int b = blockIdx.x, xg = b & 7, s = b >> 3;
    const bool fetch = s < FS;
    constexpr int NF = FS * 8;
    if (wave == 0) STAMP(0);
    GAS u32* const cnt = (GAS u32*)(P.me + OFF_CNT + (b * 8 + wave) * 8);
    constexpr bool diag = false;
    const int pidx = b - (NF + 120);
    constexpr int NU = 112 - ITEM0;
    const int q0 = ITEM0 + (b - NF);
    const bool loaded = !fetch && !diag && q0 < 112 && wave < 2;
    f32x4 v[16];
    if (wave == 7 && (unsigned)pidx < (unsigned)NPUSH && !diag) {
        load_x(P, pidx, 0, lane, v);
        const u32 craw = __hip_atomic_load(cnt, __ATOMIC_RELAXED, __HIP_MEMORY_SCOPE_AGENT);
        __builtin_amdgcn_sched_barrier(0);
        BAR();
        __builtin_amdgcn_sched_barrier(0);
        STAMP(22);
        push_task(P, pidx, lane, craw, v);
        STAMP(23);
    } else {
        if (loaded) { if (q0 < 32) load_x(P, q0 * 2 + (wave & 1), 0, lane, v); else load_w(P, (q0 < 96 ? q0 + 32 : q0 - 96) * 2 + (wave & 1), 0, lane, v); }
        __builtin_amdgcn_sched_barrier(0);
        if (tid < 256) lds_st(L_ITEM + tid * 4, fetch && tid >= 24 && tid < 32 ? (u32)(tid - 24) : 0u);
        LGKM0();
        BAR();
    }
    if (!fetch && !diag) {
        const u64 tstart = now();
        const bool pack = s >= FS + GS;
        auto fetch_task = [&](int q) {
            const u64 go = pack ? (u64)PACE4 : (u64)(PACE2 + (q - 112) * PACE3 / 112);
            while (now() - tstart < go) __builtin_amdgcn_s_sleep(4);
            int j, idx; bool isA;
            task_of(q, wave & 1, j, idx, isA);
            if (isA) load_x(P, idx, j, lane, v); else load_w(P, idx, j, lane, v);
        };
        int qn = 112 + (pack ? GS * 8 + xg : (b - NF + GS * 8 - NU) % (GS * 8)) * 4 + (wave >> 1), q = q0;
        const int qs = pack ? 32 : N_PAIRS;
        u32 lq = L_ITEM;
        if (!loaded) { q = qn; qn += qs; lq += 64; fetch_task(q); }
#pragma unroll 1
        for (;;) {
            stage_task(P, L + wave * 4096, lq, lq + 48 + (wave >> 1) * 4, q, wave & 1, lane, cnt, v);
            if (qn >= N_PAIRS) break;
            q = qn; qn += qs; lq += 64;
            fetch_task(q);
        }
    }
    const u32 call = uni(__hip_atomic_load(cnt, __ATOMIC_RELAXED, __HIP_MEMORY_SCOPE_AGENT)) + (diag ? 0u : 1u);
    if (lane == 0) st4_sys((void*)cnt, call);
    STAMP(fetch ? 1 : (s >= FS + GS ? 19 : 18));
    if (fetch) {
        const int src = (P.rank + 1 + s) & 7;
        if (wave == 0) { loader(P, src, lane, call); STAMP(2); }
        else if (wave == 1) scout(P, src, xg, lane, call);
        else { decoder(P, L, src, wave - 2, lane, call); STAMP(8 + wave); }
    } else if (s < FS + GS) {
        const int g = s - FS;
        gemm_wg(P, L, (xg >> 1) * 4 + g / 6, (xg & 1) * 6 + g % 6, wave, lane, call);
    } else {
        BAR();
        packer(P, L, wave * 8 + xg, wave, lane, call);
    }
}

}

extern "C" void kernel_launch(int rank, int nranks, ncclComm_t comm, hipStream_t stream,
                              void* const* d_in, const int* in_sizes, int n_in,
                              void* d_out, int out_size,
                              const int* in_dtypes, int out_dtype,
                              void* const* d_out_all,
                              void* const* d_ws_all, size_t ws_bytes) {
    (void)comm; (void)in_sizes; (void)n_in; (void)out_size; (void)in_dtypes; (void)out_dtype; (void)d_out_all;
    if (nranks != NR || (unsigned)rank >= (unsigned)NR || ws_bytes < WS_NEED) { fprintf(stderr, "unsupported configuration\n"); return; }
    static bool ready[NR];
    if (!ready[rank]) {
        HIP_CHECK(hipFuncSetAttribute((const void*)fused_kernel, hipFuncAttributeMaxDynamicSharedMemorySize, (int)LDS_BYTES));
        ready[rank] = true;
    }
    Params P;
    P.x = (const float*)d_in[0]; P.w = (const float*)d_in[1]; P.out = (float*)d_out;
    for (int i = 0; i < NR; ++i) P.ws[i] = (char*)d_ws_all[i];
    P.me = P.ws[rank]; P.rank = rank; P.pad = 0;
    hipLaunchKernelGGL(fused_kernel, dim3(NWG), dim3(NTHR), LDS_BYTES, stream, P);
    HIP_CHECK(hipGetLastError());
}
```

```cpp
#include <hip/hip_runtime.h>
#include <rccl/rccl.h>
#include <cstdio>

#define HIP_CHECK(expr) do { hipError_t _e = (expr); if (_e != hipSuccess) { \
    fprintf(stderr, "HIP error %s at %s:%d\n", hipGetErrorString(_e), __FILE__, __LINE__); \
    return; } } while (0)

typedef unsigned u32;
typedef unsigned long long u64;
typedef u32 u32x2 __attribute__((ext_vector_type(2)));
typedef u32 u32x4 __attribute__((ext_vector_type(4)));
typedef float f32x4 __attribute__((ext_vector_type(4)));
typedef _Float16 f16x8 __attribute__((ext_vector_type(8)));
#define LAS __attribute__((address_space(3)))
#define GAS __attribute__((address_space(1)))

namespace {

constexpr int NR = 8, GM = 4096, GN = 16384, KL = 512;
constexpr int NWG = 256, NTHR = 512;
#define WINV 8
#define NPUSH 36
constexpr int FS = 7, GS = 24;
constexpr int LPP = 8;
constexpr int WIN = WINV;
constexpr int NLP = 36;
constexpr int N_IN_ITEMS = 832, N_OUT_ITEMS = 512, N_ITEMS = N_IN_ITEMS + N_OUT_ITEMS;
constexpr int N_PAIRS = 1280, ITEM0 = NPUSH / 2;
#define PACE2 150
#define PACE3 350
#define PACE4 800
static_assert(NPUSH % 2 == 0 && NPUSH >= 8 && NPUSH <= 64, "only x blocks of the first K group");
constexpr u64 DEADLINE = 30000000ull;

constexpr size_t OFF_CNT = 0;
constexpr size_t OFF_TMO = 16384;
constexpr size_t OFF_TICK = 16384 + 256;
constexpr size_t OFF_OF = 20480;
constexpr u32 U_B = 8192, N_UNITS = U_B + NLP * 256;
constexpr size_t OFF_FA = 32768;
constexpr size_t OFF_DBG = 262144;
constexpr size_t OFF_REC = 1ull << 20;
constexpr u32 REC_SRC = 65536, REC_OUT = 16384;
constexpr size_t REC_PAR = 1ull << 20;
constexpr size_t OFF_A = 4ull << 20;
constexpr size_t OFF_AX = OFF_A + (size_t)N_UNITS * 8192;
constexpr size_t OFF_BX = OFF_AX + (4ull << 20);
constexpr size_t X_PAR = 20ull << 20;
constexpr size_t OFF_O = OFF_AX + 2 * X_PAR;
constexpr size_t QSLOT = 32768;
constexpr size_t WS_NEED = OFF_O + N_OUT_ITEMS * QSLOT;
static_assert(NR * REC_SRC <= REC_PAR && N_IN_ITEMS * 16 <= REC_OUT && REC_OUT + N_OUT_ITEMS * 64 <= REC_SRC && OFF_FA + N_UNITS * 8 <= OFF_DBG, "workspace map");

constexpr u32 RING = 65536;
constexpr u32 L_SCR = RING, SCR_B = 8448;
constexpr u32 L_DESC = L_SCR + 6 * SCR_B, NQ = 64;
constexpr u32 L_ITEM = L_DESC + NQ * 16;
constexpr u32 L_FCTL = L_ITEM + 64;
constexpr u32 L_LAND = L_FCTL + 64;
constexpr u32 L_T1 = 131072, L_T2 = L_T1 + 512, L_ZERO = L_T2 + 512;
constexpr u32 PK_B = 19456, PK_IMG = 8320;
constexpr u32 LDS_BYTES = 8 * PK_B;
static_assert(L_ITEM + 1024 <= LDS_BYTES && L_LAND + 256 <= L_ITEM + 1024 && L_ZERO + 8192 <= LDS_BYTES && LDS_BYTES <= 163840, "LDS map");

struct Params {
    const float* x; const float* w; float* out;
    char* me;
    char* ws[NR];
    int rank; int pad;
};

#define VMCNT(n) asm volatile("s_waitcnt vmcnt(" #n ")" ::: "memory")
#define LGKM0() asm volatile("s_waitcnt lgkmcnt(0)" ::: "memory")
#define CFENCE() asm volatile("" ::: "memory")
#define BAR() do { CFENCE(); __builtin_amdgcn_s_barrier(); CFENCE(); } while (0)

__device__ __forceinline__ void st16_sys(void* p, u32x4 v) {
    asm volatile("global_store_dwordx4 %0, %1, off sc0 sc1\n\ts_nop 1" :: "v"(p), "v"(v) : "memory");
}
__device__ __forceinline__ void st_out(GAS float* p, f32x4 v) {
    asm volatile("global_store_dwordx4 %0, %1, off sc0 sc1 nt\n\ts_nop 1" :: "v"(p), "v"(v) : "memory");
}
__device__ __forceinline__ void st4_sys(void* p, u32 v) {
    asm volatile("global_store_dword %0, %1, off sc0 sc1" :: "v"(p), "v"(v) : "memory");
}
__device__ __forceinline__ void st8_sys(void* p, u32x2 v) {
    asm volatile("global_store_dwordx2 %0, %1, off sc0 sc1" :: "v"(p), "v"(v) : "memory");
}
__device__ __forceinline__ u32 ld4_sys(const void* p) {
    u32 v; asm volatile("global_load_dword %0, %1, off sc0 sc1\n\ts_waitcnt vmcnt(0)" : "=&v"(v) : "v"(p) : "memory"); return v;
}
__device__ __forceinline__ u32x2 ld8_sys(const void* p) {
    u32x2 v; asm volatile("global_load_dwordx2 %0, %1, off sc0 sc1\n\ts_waitcnt vmcnt(0)" : "=&v"(v) : "v"(p) : "memory"); return v;
}
__device__ __forceinline__ void lds_st(u32 a, u32 v) { asm volatile("ds_write_b32 %0, %1" :: "v"(a), "v"(v) : "memory"); }
__device__ __forceinline__ u32 lds_ld(u32 a) {
    u32 v; asm volatile("ds_read_b32 %0, %1\n\ts_waitcnt lgkmcnt(0)" : "=&v"(v) : "v"(a) : "memory"); return v;
}
__device__ __forceinline__ u32x4 lds_ld16(u32 a) {
    u32x4 v; asm volatile("ds_read_b128 %0, %1\n\ts_waitcnt lgkmcnt(0)" : "=&v"(v) : "v"(a) : "memory"); return v;
}
__device__ __forceinline__ u32 uni(u32 v) { return __builtin_amdgcn_readfirstlane(v); }
__device__ __forceinline__ u32 lds_ldu(u32 a) { return uni(lds_ld(a)); }
__device__ __forceinline__ u64 now() { return __builtin_amdgcn_s_memrealtime(); }
__device__ __forceinline__ u32 prefix(u64 m) { return __builtin_amdgcn_mbcnt_hi((u32)(m >> 32), __builtin_amdgcn_mbcnt_lo((u32)m, 0u)); }

__device__ __forceinline__ u32 wave_umax(u32 v) {
    v = max(v, (u32)__builtin_amdgcn_update_dpp(0, (int)v, 0x111, 0xf, 0xf, true));
    v = max(v, (u32)__builtin_amdgcn_update_dpp(0, (int)v, 0x112, 0xf, 0xf, true));
    v = max(v, (u32)__builtin_amdgcn_update_dpp(0, (int)v, 0x114, 0xf, 0xf, true));
    v = max(v, (u32)__builtin_amdgcn_update_dpp(0, (int)v, 0x118, 0xf, 0xf, true));
    v = max(v, (u32)__builtin_amdgcn_update_dpp(0, (int)v, 0x142, 0xa, 0xf, false));
    v = max(v, (u32)__builtin_amdgcn_update_dpp(0, (int)v, 0x143, 0xc, 0xf, false));
    return __builtin_amdgcn_readlane(v, 63);
}
__device__ __forceinline__ u32 wave_scan(u32 v) {
    v += (u32)__builtin_amdgcn_update_dpp(0, (int)v, 0x111, 0xf, 0xf, true);
    v += (u32)__builtin_amdgcn_update_dpp(0, (int)v, 0x112, 0xf, 0xf, true);
    v += (u32)__builtin_amdgcn_update_dpp(0, (int)v, 0x114, 0xf, 0xf, true);
    v += (u32)__builtin_amdgcn_update_dpp(0, (int)v, 0x118, 0xf, 0xf, true);
    v += (u32)__builtin_amdgcn_update_dpp(0, (int)v, 0x142, 0xa, 0xf, false);
    v += (u32)__builtin_amdgcn_update_dpp(0, (int)v, 0x143, 0xc, 0xf, false);
    return v;
}
typedef unsigned short u16x2 __attribute__((ext_vector_type(2)));
typedef short i16x2 __attribute__((ext_vector_type(2)));
#define PU(x) __builtin_bit_cast(u16x2, (u32)(x))
#define PI(x) __builtin_bit_cast(i16x2, (u32)(x))
#define PW(x) __builtin_bit_cast(u32, (x))
template <int W> __device__ __forceinline__ u32 pk_offsets(const u32* m, u32* ac) {
    u32 tot;
    ac[0] = 0;
    asm("v_pk_mul_lo_u16 %0, %4, %8 op_sel_hi:[1,0]\n\tv_pk_mad_u16 %1, %5, %8, %0 op_sel_hi:[1,0,1]\n\t"
        "v_pk_mad_u16 %2, %6, %8, %1 op_sel_hi:[1,0,1]\n\tv_pk_mad_u16 %3, %7, %8, %2 op_sel_hi:[1,0,1]"
        : "=&v"(ac[1]), "=&v"(ac[2]), "=&v"(ac[3]), "=&v"(tot) : "v"(m[0]), "v"(m[1]), "v"(m[2]), "v"(m[3]), "n"(-W));
    return tot;
}
__device__ __forceinline__ u32 pk_is3(u32 c) { return PW((i16x2)(PI(c + 0x7ffd7ffdu) >> (i16x2){15, 15})); }
__device__ __forceinline__ u32 pk_sum(u32 a) { return (a & 0xffffu) + (a >> 16); }
__device__ __forceinline__ u32 pk_join(const u32* sv, const u32* ac, u32 acc) {
    const u32 tt = sv[0] | PW((u16x2)(PU(sv[1]) << PU(ac[1]))) | PW((u16x2)(PU(sv[2]) << PU(ac[2]))) | PW((u16x2)(PU(sv[3]) << PU(ac[3])));
    return (tt & 0xffffu) | ((tt >> 16) << (acc & 31u));
}
__device__ __forceinline__ u32 pk_split(u32 t, u32 acc) { return __builtin_amdgcn_perm(t >> (acc & 31u), t, 0x05040100u); }
__device__ __forceinline__ u32 pk_field(u32 tt, u32 sh, u32 m, u32 mask) { return PW((u16x2)(PU(tt) >> PU(sh))) & m & mask; }
__device__ __forceinline__ u32 pk_expand(u32 sv, u32 k, u32 base) {
    u32 r;
    asm("v_pk_min_u16 %0, %1, 1 op_sel_hi:[1,0]\n\tv_pk_mad_u16 %0, %0, 3, %1 op_sel_hi:[1,0,1]\n\tv_pk_mad_u16 %0, %0, %2, %3"
        : "=&v"(r) : "v"(sv), "s"(k), "v"(base));
    return r;
}
template <int SH> __device__ __forceinline__ void pk_classify(const u32* x, u32* c, u32* g, u32& T2, u32& T3, u32& len) {
    u32 m2[4], m3[4], ac[4], s2[4], s3[4];
#pragma unroll
    for (int k = 0; k < 4; ++k) {
        const u32 o = PW((i16x2)(PI(x[k]) >> (i16x2){SH, SH}));
        const u32 sv = PW(__builtin_elementwise_max((i16x2)(-PI(o)), (i16x2){0, 0}));
        c[k] = PW(__builtin_elementwise_min(PU(o), (u16x2){3, 3}));
        s2[k] = PW(__builtin_elementwise_min(PU(sv), (u16x2){3, 3}));
        s3[k] = (sv - s2[k]) & 0x00070007u;
        m2[k] = pk_is3(c[k]); m3[k] = pk_is3(s2[k]);
        g[k] = PW(__builtin_elementwise_sub_sat((u16x2)(PU(o) + (u16x2){10, 10}), (u16x2){13, 13}));
    }
    const u32 a2 = pk_offsets<2>(m2, ac);
    T2 = pk_join(s2, ac, a2);
    const u32 a3 = pk_offsets<3>(m3, ac);
    T3 = pk_join(s3, ac, a3);
    len = pk_sum(a2) | (pk_sum(a3) << 16);
}
__device__ __forceinline__ void or_bits(LAS char* buf, u32 bp, u32 t) {
    const u64 v = (u64)t << (bp & 31u);
    LAS u32* a = (LAS u32*)(buf + (bp >> 5) * 4u);
    __hip_atomic_fetch_or(a, (u32)v, __ATOMIC_RELAXED, __HIP_MEMORY_SCOPE_WAVEFRONT);
    __hip_atomic_fetch_or(a + 1, (u32)(v >> 32), __ATOMIC_RELAXED, __HIP_MEMORY_SCOPE_WAVEFRONT);
}
__device__ __forceinline__ void emit_escapes(const u32* g, const u32* raw, u32 idx0, LAS char* rec, u32 cap, u32& e) {
    if (__builtin_amdgcn_ballot_w64((g[0] | g[1] | g[2] | g[3]) != 0u) == 0ull) return;
    u32 gm = 0;
#pragma unroll
    for (int k = 0; k < 4; ++k) gm |= PW(__builtin_elementwise_min(PU(g[k]), (u16x2){1, 1})) << k;
#pragma unroll 1
    for (;;) {
        const u64 em = __builtin_amdgcn_ballot_w64(gm != 0u);
        if (em == 0ull) break;
        const u32 b = __builtin_ctz(gm | 0x80000000u), k = b & 3u;
        const u32 m1 = (u32)__builtin_amdgcn_sbfe((int)b, 0, 1), m2 = (u32)__builtin_amdgcn_sbfe((int)b, 1, 1);
        const u32 r01 = (raw[1] & m1) | (raw[0] & ~m1), r23 = (raw[3] & m1) | (raw[2] & ~m1), r = (r23 & m2) | (r01 & ~m2);
        if (gm != 0u && e < cap) *(LAS u32*)(rec + (e + prefix(em)) * 4u) = (idx0 + k * 2u + (b >> 4)) | ((r >> (b & 16u)) << 16);
        gm &= gm - 1u;
        e += __builtin_popcountll(em);
    }
}
__device__ __forceinline__ float amax3(float m, float a, float b) { float r; asm("v_max3_f32 %0, %1, |%2|, |%3|" : "=v"(r) : "v"(m), "v"(a), "v"(b)); return r; }
__device__ __forceinline__ char* peer_ws(const Params& P, int p) {
    u64 wp = (u64)P.ws[0];
#pragma unroll
    for (int i = 1; i < NR; ++i) { u64 q = (u64)P.ws[i]; asm volatile("" : "+s"(q)); wp = p == i ? q : wp; }
    return (char*)wp;
}

#define STAMP(k) do { } while (0)
#define STAMPV(k, v) do { } while (0)

__device__ __forceinline__ bool give_up(u32& spins, u64 t0, u32* tmo, u32 call) {
    if ((++spins & 63u) != 0u) return false;
    if (uni(ld4_sys(tmo)) == call) return true;
    if (now() - t0 > DEADLINE) { st4_sys(tmo, call); return true; }
    return false;
}

__device__ __forceinline__ float block_scale(const f32x4 (&v)[16], u32& code, u32& hm) {
    float mf = 0.f;
#pragma unroll
    for (int i = 0; i < 16; ++i) mf = amax3(amax3(mf, v[i][0], v[i][1]), v[i][2], v[i][3]);
    const u32 mb = wave_umax(__float_as_uint(mf));
    code = ((mb >> 23) + 2u) >> 3; code = code < 2u ? 2u : code;
    const float sc = __uint_as_float((263u - 8u * code) << 23);
    hm = uni(__builtin_bit_cast(unsigned short, (_Float16)(__uint_as_float(mb) * sc)));
    if (mb == 0u) code = 63u;
    return sc;
}
__device__ __forceinline__ void load_x(const Params& P, int idx, int j, int lane, f32x4 (&v)[16]) {
    const GAS float* p = (const GAS float*)P.x + (size_t)(idx * 64 + (lane & 15)) * KL + j * 64 + (lane >> 4) * 4;
#pragma unroll
    for (int i = 0; i < 16; ++i) v[i] = __builtin_nontemporal_load((const GAS f32x4*)(p + (i >> 2) * 16 * KL + (i & 3) * 16));
}
__device__ __forceinline__ void push_task(const Params& P, int idx, int lane, u32 craw, const f32x4 (&v)[16]) {
    u32 code, hm;
    const float sc = block_scale(v, code, hm);
    const u32 call = uni(craw) + 1u;
    const u32 dest = (call & 1u) * 4096u + ((idx >> 1) * 64 + P.rank) * 2 + (idx & 1);
    const u32 voff = (u32)OFF_A + dest * 8192u + lane * 16u;
#pragma unroll
    for (int f = 0; f < 8; ++f) { f16x8 h;
#pragma unroll
        for (int e = 0; e < 8; ++e) h[e] = (_Float16)(v[f * 2 + (e >> 2)][e & 3] * sc);
        const u32x4 hv = __builtin_bit_cast(u32x4, h);
        const u32 vo = voff + f * 1024;
#pragma unroll
        for (int r = 0; r < NR; ++r) asm volatile("global_store_dwordx4 %0, %1, %2 sc0 sc1\n\ts_nop 1" :: "v"(vo), "v"(hv), "s"((u64)P.ws[r]) : "memory");
    }
    VMCNT(0);
    if (lane < NR) st8_sys(peer_ws(P, lane) + OFF_FA + (size_t)dest * 8, (u32x2){call, code});
}
__device__ __forceinline__ void load_w(const Params& P, int idx, int j, int lane, f32x4 (&v)[16]) {
    const GAS float* p = (const GAS float*)P.w + (size_t)(j * 64 + (lane >> 4) * 4) * GN + idx * 64 + (lane & 15) * 4;
#pragma unroll
    for (int i = 0; i < 16; ++i) v[i] = __builtin_nontemporal_load((const GAS f32x4*)(p + (size_t)((i >> 2) * 16 + (i & 3)) * GN));
}
__device__ __forceinline__ void task_of(int q, int half, int& j, int& idx, bool& isA) {
    if (q < 896) { j = q / 112; const int u = q % 112; isA = u < 32; idx = isA ? u : (u < 96 ? u + 32 : u - 96); }
    else { const int u = q - 896, v = u % 192; j = v / 24; isA = false; idx = 16 + (u / 192) * 24 + v % 24; }
    idx = idx * 2 + half;
}
__device__ __forceinline__ void stage_task(const Params& P, LAS char* scr, u32 lquad, u32 lpair, int q, int half, int lane, GAS u32* cnt, const f32x4 (&v)[16]) {
    int j, idx; bool isA;
    task_of(q, half, j, idx, isA);
    const u32 craw = __hip_atomic_load(cnt, __ATOMIC_RELAXED, __HIP_MEMORY_SCOPE_AGENT);
    *(LAS u32x4*)(scr + lane * 16) = (u32x4){0u, 0u, 0u, 0u}; *(LAS u32x4*)(scr + 1024 + lane * 16) = (u32x4){0u, 0u, 0u, 0u};
    __builtin_amdgcn_sched_barrier(0);
    u32 code, hm;
    const float sc = block_scale(v, code, hm);
    const u32 bb = hm >= 4095u ? hm - 4095u : 0u, b2 = bb * 0x10001u;
    u32x4 p[8];
    if (isA) {
#pragma unroll
        for (int f = 0; f < 8; ++f) { f16x8 h;
#pragma unroll
            for (int e = 0; e < 8; ++e) h[e] = (_Float16)(v[f * 2 + (e >> 2)][e & 3] * sc);
            p[f] = __builtin_bit_cast(u32x4, h); }
    } else {
#pragma unroll
        for (int f = 0; f < 8; ++f) { f16x8 h;
#pragma unroll
            for (int e = 0; e < 8; ++e) h[e] = (_Float16)(v[(f & 1) * 8 + e][f >> 1] * sc);
            p[f] = __builtin_bit_cast(u32x4, h); }
    }
    char* ws = P.me;
    const u32 call = uni(craw) + 1u;
    const bool all = isA || idx < 128;
    const int owner = (idx - 128) >> 4, lp = idx < 128 ? 4 + (idx >> 2) : (idx >> 2) & 3;
    const bool local = all || owner == P.rank, exprt = all || owner != P.rank;
    const u32 dest = isA ? (call & 1u) * 4096u + (((idx >> 1) * 8 + j) * 8 + P.rank) * 2 + half : U_B + (((lp * 8 + j) * 8 + P.rank) * 4 + (idx & 3));
    if (local) {
        char* d = ws + OFF_A + (size_t)dest * 8192 + lane * 16;
#pragma unroll
        for (int f = 0; f < 8; ++f) st16_sys(d + f * 1024, p[f]);
    }
    u32 size16 = 0;
    if (exprt) {
        u32 w[32], T2[8], T3[8], len[8], e = 0, tot = 0;
#pragma unroll
        for (int f = 0; f < 8; ++f) {
            u32 x[4], c[4], g[4];
            const u32 h[4] = {p[f][0], p[f][1], p[f][2], p[f][3]};
#pragma unroll
            for (int k = 0; k < 4; ++k) x[k] = PW((u16x2)(PU(h[k] & 0x7fff7fffu) - PU(b2)));
            pk_classify<10>(x, c, g, T2[f], T3[f], len[f]);
#pragma unroll
            for (int k = 0; k < 4; ++k) w[f * 4 + k] = (h[k] & 0x80008000u) | (c[k] << 10) | (x[k] & 0x03ff03ffu);
            tot += len[f];
            asm volatile("" : "+v"(w[f * 4]), "+v"(w[f * 4 + 1]), "+v"(w[f * 4 + 2]), "+v"(w[f * 4 + 3]), "+v"(T2[f]), "+v"(T3[f]));
            emit_escapes(g, h, f * 512 + lane * 8u, scr + 4, 384u, e);
        }
        const u32 inc = wave_scan(tot), all2 = __builtin_amdgcn_readlane(inc, 63);
        size16 = (6656u + 4u + e * 4u + ((pk_sum(all2) + 7u) >> 3) + 15u) >> 4;
        if (size16 >= 512u) size16 = 512u;
        u32 off16 = 0;
        if (half == 0) { if (lane == 0) lds_st(lpair, size16); }
        else {
            const u64 t0 = now(); u32 sp = 0;
            while ((off16 = lds_ldu(lpair)) == 0u) {
                if ((++sp & 1023u) == 0u && now() - t0 > 4 * DEADLINE) { st4_sys(ws + OFF_TMO, call); break; }
                __builtin_amdgcn_s_sleep(1);
            }
        }
        char* d = ws + (call & 1u) * X_PAR + off16 * 16u + lane * 16 +
                  (isA ? OFF_AX + (size_t)((idx >> 1) * 8 + j) * 16384 : OFF_BX + (size_t)(((idx >> 2) * 8 + j) * 2 + ((idx >> 1) & 1)) * 16384);
        if (size16 == 512u) {
#pragma unroll
            for (int f = 0; f < 8; ++f) st16_sys(d + f * 1024, p[f]);
        } else {
            u32 bp = (e + 1u) * 32u * 0x10001u + (all2 << 16) + inc - tot;
#pragma unroll
            for (int f = 0; f < 8; f += 2) {
                or_bits(scr, bp & 0xffffu, T2[f] | (T2[f + 1] << (len[f] & 31u)));
                or_bits(scr, bp >> 16, T3[f]);
                bp += len[f];
                or_bits(scr, bp >> 16, T3[f + 1]);
                bp += len[f + 1];
            }
            if (lane == 0) *(LAS u32*)scr = e;
#pragma unroll
            for (int i = 0; i < 24; ++i) w[i] |= (w[26 + (i >> 2)] << (12 - 3 * (i & 3))) & 0x70007000u;
#pragma unroll
            for (int i = 0; i < 6; ++i) w[24 + i / 3] |= (w[26 + i] >> (1 + i % 3)) & (0x40004000u >> (i % 3));
#pragma unroll
            for (int f = 0; f < 6; ++f) st16_sys(d + f * 1024, (u32x4){w[f * 4], w[f * 4 + 1], w[f * 4 + 2], w[f * 4 + 3]});
            st8_sys(d + 6144 - lane * 8, (u32x2){w[24], w[25]});
            CFENCE();
            for (u32 g = lane; g + 416u < size16; g += 64u) st16_sys(d + 6656 + (g - lane) * 16u, *(LAS u32x4*)(scr + g * 16u));
        }
    }
    VMCNT(0);
    if (local && lane == 0) st8_sys(ws + OFF_FA + (size_t)dest * 8, (u32x2){call, code});
    if (exprt) {
        const int hb = (lp % 12) * 2 + ((idx >> 1) & 1);
        const int item = isA ? j * 56 + (idx >> 1) : (lp < 12 ? j * 56 + 32 + hb : 256 + (lp / 12) * 192 + j * 24 + hb);
        const u32 info = size16 | (bb << 10) | (code << 25);
        const size_t ro = OFF_REC + (call & 1u) * REC_PAR + P.rank * REC_SRC;
        if (q < 112) {
            if (lane < NR && lane != P.rank && (all || lane == owner)) st8_sys(peer_ws(P, lane) + ro + item * 16 + half * 8, (u32x2){call, info});
        } else {
            u32 old = 0;
            if (lane == 0) asm volatile("ds_write_b32 %1, %2\n\tds_add_rtn_u32 %0, %3, %4\n\ts_waitcnt lgkmcnt(0)"
                                        : "=&v"(old) : "v"(lquad + ((item & 3) * 2 + half) * 4), "v"(info), "v"(lquad + 32), "v"(1u) : "memory");
            if (uni(old) == 7u) {
                u32x2 in2; asm volatile("ds_read_b64 %0, %1\n\ts_waitcnt lgkmcnt(0)" : "=&v"(in2) : "v"(lquad + (lane & 3) * 8) : "memory");
                const int r = lane >> 2;
                if (lane < 32 && r != P.rank && (all || r == owner)) st16_sys(peer_ws(P, r) + ro + ((item & ~3) + (lane & 3)) * 16, (u32x4){call, in2.x, call, in2.y});
            }
        }
    }
    CFENCE();
}

__device__ __forceinline__ void scout(const Params& P, int src, int li, int lane, u32 call) {
    char* ws = P.me;
    u32* tmo = (u32*)(ws + OFF_TMO);
    u32* tick = (u32*)(ws + OFF_TICK + src * 256);
    const char* rec = ws + OFF_REC + (call & 1u) * REC_PAR + src * REC_SRC;
    const u32 xpar = (call & 1u) * (u32)X_PAR;
    const u32 tbase = (call - 1u) * (u32)(N_ITEMS - ITEM0) - (u32)(LPP + ITEM0);
    u32 item = ITEM0 + li, pieces = 0, nq = 0, k = 0;
    __builtin_amdgcn_s_setprio(2);
    while (item < (u32)N_ITEMS) {
        const bool isin = item < (u32)N_IN_ITEMS;
        u32x2 r; bool dead = false;
        u32 tk;
        if (lane == 0) asm volatile("global_atomic_add %0, %1, %2, off sc0" : "=v"(tk) : "v"(tick), "v"(1u) : "memory");
        {
            const u64 t0 = now(); u32 sp = 0;
            for (;;) {
                r = ld8_sys(rec + (isin ? item * 16u + (lane & 1) * 8u : REC_OUT + (item - N_IN_ITEMS) * 64u + (lane & 7) * 8u));
                if (__builtin_amdgcn_ballot_w64(r.x >= call) == __builtin_amdgcn_ballot_w64(true)) break;
                if (give_up(sp, t0, tmo, call)) { dead = true; break; }
                __builtin_amdgcn_s_sleep(2);
            }
        }
        asm volatile("" : "+v"(tk));
        const u32 next = uni(tk) - tbase;
        u32 soff, bytes, nsub; u32x4 d;
        if (isin) {
            u32 dest;
            u32 j, u;
            if (item < 448u) { j = item / 56u; u = item % 56u; }
            else { const u32 v = item - 448u, w = v % 192u; j = w / 24u; u = 56u + (v / 192u) * 24u + w % 24u; }
            if (u >= 32u) {
                const u32 lp = (u - 32u) >> 1, h = u & 1u, gp = lp < 4u ? 32u + P.rank * 4u + lp : lp - 4u;
                soff = (u32)OFF_BX + xpar + (gp * 8 + j) * 32768u + h * 16384u;
                dest = U_B + ((lp * 8 + j) * 8 + src) * 4 + h * 2;
            } else {
                soff = (u32)OFF_AX + xpar + (u * 8 + j) * 16384u;
                dest = (call & 1u) * 4096u + ((u * 8 + j) * 8 + src) * 2;
            }
            const u32 s = lane & 1, sz = min(r.y & 1023u, 512u), sb = dead ? 0u : sz * 16u;
            const u32 b0 = __builtin_amdgcn_readlane(sb, 0);
            bytes = b0 + __builtin_amdgcn_readlane(sb, 1); nsub = 2;
            d.x = pieces * 1024u + (s ? b0 : 0u); d.y = pieces + (((s ? bytes : b0) + 1023u) >> 10);
            d.z = dead ? (62u << 12) | (2u << 24) : sz | ((r.y >> 25) << 12); d.w = ((r.y >> 10) & 0x7fffu) | ((dest + s) << 16);
        } else {
            const u32 o = item - N_IN_ITEMS;
            soff = (u32)OFF_O + o * (u32)QSLOT;
            const u32 a = __builtin_amdgcn_ds_bpermute((lane & 3) * 8, r.y), b = __builtin_amdgcn_ds_bpermute((lane & 3) * 8 + 4, r.y);
            const u32 sb = dead ? 0u : min(a & 1023u, 512u) * 16u, n = min((a >> 10) & 0xfffu, 2048u), e = b & 0xfffu, b16 = (b >> 12) & 0x7fffu;
            const u32 s0 = __builtin_amdgcn_readlane(sb, 0), s1 = s0 + __builtin_amdgcn_readlane(sb, 1), s2 = s1 + __builtin_amdgcn_readlane(sb, 2);
            const u32 off = (lane & 3) == 0 ? 0u : (lane & 3) == 1 ? s0 : (lane & 3) == 2 ? s1 : s2;
            bytes = s2 + __builtin_amdgcn_readlane(sb, 3); nsub = 4;
            d.x = pieces * 1024u + off; d.y = pieces + ((off + sb + 1023u) >> 10); d.z = n | (e << 12) | ((dead ? 3u : sb == 8192u ? 4u : 1u) << 24);
            d.w = b16 | ((o * 4u + (lane & 3)) << 16);
        }
        const u32 np = (bytes + 1023u) >> 10;
        {
            u64 t0 = 0; u32 sp = 0;
            for (;;) {
                u32 nx = lds_ld(L_FCTL + (lane < 6 ? 32 + lane * 4 : 12));
                u32 qm = __builtin_amdgcn_readlane(nx, 0);
#pragma unroll
                for (int i = 1; i < 6; ++i) qm = min(qm, (u32)__builtin_amdgcn_readlane(nx, i));
                const u32 first = qm < nq ? lds_ldu(L_DESC + (qm % NQ) * 16) >> 10 : pieces;
                if ((u32)__builtin_amdgcn_readlane(nx, 6) >= k && pieces + np - first <= RING / 1024 && (qm >= nq || nq + nsub - qm <= NQ)) break;
                if ((++sp & 1023u) == 0u) { if (!t0) t0 = now(); else if (now() - t0 > 4 * DEADLINE) { st4_sys(tmo, call); break; } }
                __builtin_amdgcn_s_sleep(4);
            }
        }
        if (lane < (int)nsub) asm volatile("ds_write_b128 %0, %1" :: "v"(L_DESC + ((nq + lane) % NQ) * 16), "v"(d) : "memory");
        nq += nsub;
        if (lane == 0) {
            const u32x4 it = {soff, bytes, 0u, k + 1};
            asm volatile("ds_write_b128 %0, %1" :: "v"(L_ITEM + (k & 3u) * 16), "v"(it) : "memory");
            lds_st(L_FCTL + 4, nq);
        }
        pieces += np; ++k;
        item = next;
    }
    if (lane == 0) {
        const u32x4 it = {0u, 0xffffffffu, 0u, k + 1};
        asm volatile("ds_write_b128 %0, %1" :: "v"(L_ITEM + (k & 3u) * 16), "v"(it) : "memory");
        lds_st(L_FCTL + 4, nq | 0x80000000u);
    }
}

__device__ __forceinline__ void loader(const Params& P, int src, int lane, u32 call) {
    const u64 base = (u64)P.ws[src];
    const u32 voff = lane * 16, vland = L_LAND + lane * 4;
    u32* tmo = (u32*)(P.me + OFF_TMO);
    int issued = 0, landed = 0;
    u32 k = 0;
    __builtin_amdgcn_s_setprio(3);
#define PIECE(sp) do { asm volatile("s_mov_b32 m0, %2\n\ts_waitcnt vmcnt(%3)\n\tglobal_load_lds_dwordx4 %0, %1 sc0 sc1" \
                                    :: "v"(voff), "s"(sp), "s"(((u32)issued * 1024u) & (RING - 1)), "n"(WIN - 1) : "memory"); } while (0)
    for (;;) {
        u32x4 it = lds_ld16(L_ITEM + (k & 3u) * 16);
        if (uni(it.w) != k + 1 || uni(it.y) == 0xffffffffu) {
            VMCNT(0);
            landed = issued;
            lds_st(vland, (u32)issued);
            const u64 t0 = now(); u32 sp = 0;
            while (uni(it.w) != k + 1) {
                if ((++sp & 1023u) == 0u && now() - t0 > 4 * DEADLINE) { st4_sys(tmo, call); return; }
                __builtin_amdgcn_s_sleep(1);
                it = lds_ld16(L_ITEM + (k & 3u) * 16);
            }
            if (uni(it.y) == 0xffffffffu) return;
        }
        const u32 soff = uni(it.x);
        u32 left = uni(it.y);
        ++k;
        if (lane == 0) lds_st(L_FCTL + 12, k);
        u64 sp = base + soff;
#pragma unroll 1
        for (; left >= 1024u; left -= 1024u, sp += 1024) {
            PIECE(sp);
            landed = max(landed, issued - (WIN - 1));
            lds_st(vland, (u32)landed);
            ++issued;
        }
        if (left) {
            if (voff < left) PIECE(sp);
            landed = max(landed, issued - (WIN - 1));
            lds_st(vland, (u32)landed);
            ++issued;
        }
    }
#undef PIECE
}

template <int G> __device__ __forceinline__ void expand_dense(LAS char* L, LAS char* scr, u32 lane, u32 n8, u32 o_lo, u32 o_mid, u32 o_top, u32 o_sec, u32 b2) {
    u32x4 lo[G]; u32x2 mid[G]; u32 top[G], u[G][4], fl[G][4], ac[G][4], lw[G], t0[G], t1[G], bp[G], tot = 0;
#pragma unroll
    for (int i = 0; i < G; ++i) {
        const u32 g = i * 64 + lane;
        mid[i] = *(LAS u32x2*)(L + ((o_mid + g * 8u) & (RING - 1)));
        top[i] = *(LAS unsigned char*)(L + ((o_top + g) & (RING - 1)));
    }
#pragma unroll
    for (int i = 0; i < G; ++i) lo[i] = *(LAS u32x4*)(L + ((o_lo + (i * 64 + lane) * 16u) & (RING - 1)));
#pragma unroll
    for (int i = 0; i < G; ++i) {
        if (i == G - 1) top[i] = i * 64 + lane < n8 ? top[i] : 0u;
        const u32 tt = (top[i] | (top[i] << 12)) << 8;
#pragma unroll
        for (int k = 0; k < 4; ++k) {
            u[i][k] = __builtin_amdgcn_perm(0u, mid[i][k >> 1], (k & 1) ? 0x0c030c02u : 0x0c010c00u) | ((tt >> k) & 0x01000100u);
            fl[i][k] = PW((i16x2)(PI(u[i][k] + 0x7e807e80u) >> (i16x2){15, 15}));
        }
        lw[i] = pk_offsets<2>(fl[i], ac[i]);
        bp[i] = tot; tot += pk_sum(lw[i]);
    }
    const u32 inc = wave_scan(tot);
#pragma unroll
    for (int i = 0; i < G; ++i) {
        bp[i] += inc - tot;
        const u32 a = o_sec + (bp[i] >> 5) * 4u;
        t0[i] = *(LAS u32*)(L + (a & (RING - 1))); t1[i] = *(LAS u32*)(L + ((a + 4u) & (RING - 1)));
    }
    u32 s2[G][4];
    tot = 0;
#pragma unroll
    for (int i = 0; i < G; ++i) {
        const u32 TT = pk_split(__builtin_amdgcn_alignbit(t1[i], t0[i], bp[i]), lw[i]);
#pragma unroll
        for (int k = 0; k < 4; ++k) { s2[i][k] = pk_field(TT, ac[i][k], fl[i][k], 0x00030003u); fl[i][k] = pk_is3(s2[i][k]); }
        lw[i] = pk_offsets<3>(fl[i], ac[i]);
        bp[i] = tot; tot += pk_sum(lw[i]);
    }
    const u32 base = wave_scan(tot) - tot + __builtin_amdgcn_readlane(inc, 63);
#pragma unroll
    for (int i = 0; i < G; ++i) {
        bp[i] += base;
        const u32 a = o_sec + (bp[i] >> 5) * 4u;
        t0[i] = *(LAS u32*)(L + (a & (RING - 1))); t1[i] = *(LAS u32*)(L + ((a + 4u) & (RING - 1)));
    }
#pragma unroll
    for (int i = 0; i < G; ++i) {
        const u32 TT = pk_split(__builtin_amdgcn_alignbit(t1[i], t0[i], bp[i]), lw[i]);
        u32 r[8];
#pragma unroll
        for (int k = 0; k < 4; ++k) {
            const u32 h = pk_expand(s2[i][k] + pk_field(TT, ac[i][k], fl[i][k], 0x00070007u), 0xff80ff80u, u[i][k] + b2);
            r[2 * k] = __builtin_amdgcn_perm(h, lo[i][k], 0x05040100u); r[2 * k + 1] = __builtin_amdgcn_perm(h, lo[i][k], 0x07060302u);
        }
        *(LAS u32x4*)(scr + (i * 64 + lane) * 32u) = (u32x4){r[0], r[1], r[2], r[3]};
        *(LAS u32x4*)(scr + (i * 64 + lane) * 32u + 16u) = (u32x4){r[4], r[5], r[6], r[7]};
    }
}

__device__ __forceinline__ void decoder(const Params& P, LAS char* L, int src, int dw, int lane, u32 call) {
    char* ws = P.me;
    u32* tmo = (u32*)(ws + OFF_TMO);
    LAS char* scr = L + L_SCR + dw * SCR_B;
    u32 qn = dw, pend = 0;
    auto flush = [&]() { if (pend) { VMCNT(0); if (lane == 0) st8_sys(ws + OFF_FA + (size_t)(pend & 0xffffu) * 8, (u32x2){call, (pend >> 16) & 63u}); pend = 0; } };
    auto release = [&]() { if (lane == 0) asm volatile("ds_add_rtn_u32 %0, %1, %2\n\ts_waitcnt lgkmcnt(0)\n\tds_write_b32 %3, %0"
                                                       : "=&v"(qn) : "v"(L_FCTL + 8), "v"(1u), "v"(L_FCTL + 32 + dw * 4) : "memory"); };
    for (;;) {
        const u32 q = uni(qn);
        u32x4 dv;
        {
            u64 t0 = 0; u32 sp = 0;
            for (;;) {
                u32 nq, ld;
                asm volatile("ds_read_b32 %0, %3\n\tds_read_b128 %1, %4\n\tds_read_b32 %2, %5\n\ts_waitcnt lgkmcnt(0)"
                             : "=&v"(nq), "=&v"(dv), "=&v"(ld) : "v"(L_FCTL + 4), "v"(L_DESC + (q % NQ) * 16), "v"(L_LAND) : "memory");
                nq = uni(nq);
                if ((nq & 0x7fffffffu) > q && uni(ld) >= uni(dv.y)) break;
                flush();
                if ((nq & 0x7fffffffu) <= q && (nq >> 31)) return;
                if ((++sp & 1023u) == 0u) { if (!t0) t0 = now(); else if (now() - t0 > 4 * DEADLINE) { st4_sys(tmo, call); return; } }
                __builtin_amdgcn_s_sleep(1);
            }
        }
        const u32 pos = uni(dv.x), z = uni(dv.z), wv = uni(dv.w);
        const u32 type = z >> 24, dest = wv >> 16;
        if (type == 0u || type == 2u) {
            if (type == 0u) {
                const u32 sz = z & 0xfffu, b2 = (wv & 0xffffu) * 0x10001u, pl = pos + lane * 16u;
                char* d = ws + OFF_A + (size_t)dest * 8192;
                u32x4 c[8];
#pragma unroll
                for (int i = 0; i < 6; ++i) c[i] = *(LAS u32x4*)(L + ((pl + i * 1024u) & (RING - 1)));
                if (sz == 512u) {
                    c[6] = *(LAS u32x4*)(L + ((pl + 6144u) & (RING - 1))); c[7] = *(LAS u32x4*)(L + ((pl + 7168u) & (RING - 1)));
                    LGKM0();
#pragma unroll
                    for (int i = 0; i < 8; ++i) asm volatile("" : "+v"(c[i]));
                    release();
                    flush();
#pragma unroll
                    for (int f = 0; f < 8; ++f) st16_sys(d + f * 1024 + lane * 16, c[f]);
                } else {
                    u32x2 c6 = *(LAS u32x2*)(L + ((pos + 6144u + lane * 8u) & (RING - 1)));
                    for (u32 g = lane; g + 416u < sz; g += 64u) *(LAS u32x4*)(scr + g * 16u) = *(LAS u32x4*)(L + ((pos + 6656u + g * 16u) & (RING - 1)));
                    LGKM0();
#pragma unroll
                    for (int i = 0; i < 6; ++i) asm volatile("" : "+v"(c[i]));
                    asm volatile("" : "+v"(c6));
                    release();
                    const u32 e = min(uni(*(LAS u32*)scr), 384u);
                    u32 w[32], fl[32], ac[32], len[8], lo[8], tot = 0;
#pragma unroll
                    for (int i = 0; i < 24; ++i) w[i] = c[i >> 2][i & 3];
                    w[24] = c6.x; w[25] = c6.y;
#pragma unroll
                    for (int i = 0; i < 6; ++i) {
                        u32 t = (w[24 + i / 3] << (1 + i % 3)) & 0x80008000u;
#pragma unroll
                        for (int k = 0; k < 4; ++k) t |= (w[i * 4 + k] >> (12 - 3 * k)) & (0x00070007u << (3 * k));
                        w[26 + i] = t;
                    }
#pragma unroll
                    for (int f = 0; f < 8; ++f) {
#pragma unroll
                        for (int k = 0; k < 4; ++k) fl[f * 4 + k] = PW((i16x2)(PI((w[f * 4 + k] & 0x0c000c00u) + 0x74007400u) >> (i16x2){15, 15}));
                        lo[f] = pk_offsets<2>(fl + f * 4, ac + f * 4);
                        len[f] = pk_sum(lo[f]); tot += len[f];
                    }
                    const u32 inc = wave_scan(tot);
                    u32 bp = (e + 1u) * 32u + inc - tot;
                    u32 T[8], s2[32];
#pragma unroll
                    for (int f = 0; f < 8; f += 2) {
                        const LAS u32* a = (const LAS u32*)(scr + (bp >> 5) * 4u);
                        T[f] = __builtin_amdgcn_alignbit(a[1], a[0], bp);
                        T[f + 1] = T[f] >> len[f];
                        bp += len[f] + len[f + 1];
                    }
                    tot = 0;
#pragma unroll
                    for (int f = 0; f < 8; ++f) {
                        const u32 TT = pk_split(T[f], lo[f]);
#pragma unroll
                        for (int k = 0; k < 4; ++k) { const int i = f * 4 + k; s2[i] = pk_field(TT, ac[i], fl[i], 0x00030003u); fl[i] = pk_is3(s2[i]); }
                        lo[f] = pk_offsets<3>(fl + f * 4, ac + f * 4);
                        len[f] = pk_sum(lo[f]); tot += len[f];
                    }
                    flush();
                    bp = (e + 1u) * 32u + __builtin_amdgcn_readlane(inc, 63) + wave_scan(tot) - tot;
#pragma unroll
                    for (int f = 0; f < 8; ++f) {
                        const LAS u32* a = (const LAS u32*)(scr + (bp >> 5) * 4u);
                        T[f] = __builtin_amdgcn_alignbit(a[1], a[0], bp);
                        bp += len[f];
                    }
#pragma unroll
                    for (int f = 0; f < 8; ++f) {
                        const u32 TT = pk_split(T[f], lo[f]);
                        u32x4 o;
#pragma unroll
                        for (int k = 0; k < 4; ++k) {
                            const int i = f * 4 + k;
                            o[k] = pk_expand(s2[i] + pk_field(TT, ac[i], fl[i], 0x00070007u), 0xfc00fc00u, PW((u16x2)(PU(w[i] & 0x8fff8fffu) + PU(b2))));
                        }
                        st16_sys(d + f * 1024 + lane * 16, o);
                    }
                    for (u32 g = lane; g < e; g += 64u) {
                        const u32 rr = *(LAS u32*)(scr + 4u + g * 4u);
                        asm volatile("global_store_short_d16_hi %0, %1, off sc0 sc1" :: "v"(d + (rr & 4095u) * 2u), "v"(rr) : "memory");
                    }
                }
            } else { release(); flush(); }
            pend = dest | (((z >> 12) & 63u) << 16) | 0x80000000u;
            continue;
        }
        flush();
        const u32 tile = dest >> 5, wq = (dest >> 2) & 7u, su = dest & 3u;
        const u32 row0 = (tile >> 2) * 256u + (su >> 1) * 128u + (wq >> 2) * 64u + (su & 1u) * 32u + (lane >> 4) * 4u;
        const u32 col0 = 8192u + src * 1024u + (tile & 3u) * 256u + (wq & 3u) * 64u + (lane & 15) * 4u;
        GAS float* o = (GAS float*)P.out + (size_t)row0 * GN + col0;
        if (type == 3u) {
            release();
            const float qn = __builtin_nanf("");
#pragma unroll
            for (int g = 0; g < 8; ++g) *(GAS f32x4*)(o + (size_t)((g >> 2) * 16 + (g & 3)) * GN) = (f32x4){qn, qn, qn, qn};
            continue;
        }
        if (type == 4u) {
            u32x4 c[8];
#pragma unroll
            for (int g = 0; g < 8; ++g) c[g] = *(LAS u32x4*)(L + ((pos + g * 1024u + lane * 16u) & (RING - 1)));
            LGKM0();
#pragma unroll
            for (int g = 0; g < 8; ++g) asm volatile("" : "+v"(c[g]));
            release();
#pragma unroll
            for (int g = 0; g < 8; ++g) st_out(o + (size_t)((g >> 2) * 16 + (g & 3)) * GN, __builtin_bit_cast(f32x4, c[g]));
            continue;
        }
        const u32 n = z & 0xfffu, e = (z >> 12) & 0xfffu, b2 = (wv & 0xffffu) * 0x10001u, n8 = (n + 7u) >> 3, G = (n8 + 63u) >> 6;
        u32 mk = *(LAS u32*)(L + ((pos + lane * 4u) & (RING - 1)));
        const u32 o_lo = pos + 256u, o_mid = o_lo + n8 * 16u, o_top = o_mid + n8 * 8u, o_esc = o_top + ((n8 + 3u) & ~3u), o_sec = o_esc + e * 4u;
        if (G == 3u) expand_dense<3>(L, scr, lane, n8, o_lo, o_mid, o_top, o_sec, b2);
        else if (G == 2u) expand_dense<2>(L, scr, lane, n8, o_lo, o_mid, o_top, o_sec, b2);
        else if (G == 1u) expand_dense<1>(L, scr, lane, n8, o_lo, o_mid, o_top, o_sec, b2);
        else if (G == 4u) expand_dense<4>(L, scr, lane, n8, o_lo, o_mid, o_top, o_sec, b2);
        CFENCE();
#pragma unroll 1
        for (u32 g = lane; g < e; g += 64u) {
            const u32 rr = *(LAS u32*)(L + ((o_esc + g * 4u) & (RING - 1)));
            *(LAS unsigned short*)(scr + (rr & 2047u) * 4u + 2u) = (unsigned short)(rr >> 16);
        }
        LGKM0();
        asm volatile("" : "+v"(mk));
        release();
        {
            u32 ad = L_SCR + dw * SCR_B + wave_scan(__builtin_popcount(mk)) * 4u;
            u32 r[32], m[32];
#pragma unroll
            for (int i = 31; i >= 0; --i) {
                m[i] = (u32)__builtin_amdgcn_sbfe((int)mk, i, 1);
                asm("v_lshl_add_u32 %0, %1, 2, %2" : "=v"(ad) : "v"(m[i]), "v"(ad));
                r[i] = *(LAS u32*)(size_t)ad;
            }
            __builtin_amdgcn_sched_barrier(0);
#pragma unroll
            for (int g = 7; g >= 0; --g)
                st_out(o + (size_t)((g >> 2) * 16 + (g & 3)) * GN,
                       __builtin_bit_cast(f32x4, (u32x4){r[g * 4] & m[g * 4], r[g * 4 + 1] & m[g * 4 + 1], r[g * 4 + 2] & m[g * 4 + 2], r[g * 4 + 3] & m[g * 4 + 3]}));
        }
        CFENCE();
    }
}

__device__ __forceinline__ void gemm_wg(const Params& P, LAS char* L, int pm, int lp0, int wave, int lane, u32 call) {
    char* ws = P.me;
    const int wr = wave >> 2, wc = wave & 3;
    const u32 voffA = wave * 2048 + lane * 16, voffB = (wave >> 1) * 8192 + (wave & 1) * 2048 + lane * 16;
    const u32 boff = wc * 4096 + lane * 16, ldw = wave * 2048;
    auto lane_id = []() -> u32 { u32 l; asm volatile("v_mbcnt_lo_u32_b32 %0, -1, 0\n\tv_mbcnt_hi_u32_b32 %0, -1, %0" : "=v"(l)); return l; };
    constexpr size_t KA = 16384, KB = 32768, HA = 64 * KA, HB = 4096;
#define SA(b, h) (((b) * 2 + (h)) * 16384)
#define SB(b, h) ((4 + (b) * 2 + (h)) * 16384)
#define STAGE(bufoff, gbase, voff) asm volatile("s_mov_b32 m0, %2\n\ts_nop 0\n\tglobal_load_lds_dwordx4 %0, %1 sc0 sc1\n\tglobal_load_lds_dwordx4 %0, %1 offset:1024 sc0 sc1" \
        :: "v"(voff), "s"((u64)(gbase)), "s"((u32)(bufoff) + ldw) : "memory")
#define LDA(dst, b, h) do { _Pragma("unroll") for (int m = 0; m < 4; ++m) _Pragma("unroll") for (int k = 0; k < 2; ++k) \
        dst[m][k] = *(const LAS f16x8*)(L + SA(b, h) + ((h) ? aoff1 : aoff0) + m * 2048 + k * 1024); } while (0)
#define LDB(dst, b, h) do { _Pragma("unroll") for (int n = 0; n < 2; ++n) _Pragma("unroll") for (int k = 0; k < 2; ++k) \
        dst[n][k] = *(const LAS f16x8*)(L + SB(b, h) + boff + n * 2048 + k * 1024); } while (0)
#define MMA(ai, bj, At, Bt) do { __builtin_amdgcn_s_setprio(1); \
        _Pragma("unroll") for (int m = 0; m < 4; ++m) _Pragma("unroll") for (int n = 0; n < 2; ++n) _Pragma("unroll") for (int k = 0; k < 2; ++k) \
            acc[ai][bj][m][n] = __builtin_amdgcn_mfma_f32_16x16x32_f16(At[m][k], Bt[n][k], acc[ai][bj][m][n], 0, 0, 0); \
        __builtin_amdgcn_s_setprio(0); } while (0)
#define WAIT_L(n) asm volatile("s_waitcnt lgkmcnt(" #n ")" ::: "memory")
#define RBAR __builtin_amdgcn_s_barrier()
#define SCHED __builtin_amdgcn_sched_barrier(0)
    auto flag_ptr = [&](int lp, int j, int ent) -> const char* {
        return ws + OFF_FA + (size_t)(ent < 32 ? (call & 1u) * 4096u + ((pm * 2 + (ent >> 4)) * 8 + j) * 16 + (ent & 15) : U_B + (lp * 8 + j) * 32 + (ent - 32)) * 8; };
    auto poll = [&](int lp, int j) {
        const u32 ln = lane_id();
        const char* f = flag_ptr(lp, j, ln);
        const u64 t0 = now(); u32 sp = 0; u32x2 v;
        for (;;) {
            v = ld8_sys(f);
            if (__builtin_amdgcn_ballot_w64(v.x >= call) == ~0ull) break;
            if (give_up(sp, t0, (u32*)(ws + OFF_TMO), call)) { v.y = 62u; break; }
            __builtin_amdgcn_s_sleep(4);
        }
        asm volatile("ds_write_b64 %0, %1\n\ts_waitcnt lgkmcnt(0)" :: "v"(L_T2 + ln * 8), "v"(v) : "memory");
    };
    auto codes = [&](u32 tb) -> u32 {
        const u32 r = lane_id() & 7;
        const u32 a0 = *(const LAS u32*)(L + tb + wr * 8 + 4 + r * 16), a1 = *(const LAS u32*)(L + tb + 128 + wr * 8 + 4 + r * 16);
        const u32 b = *(const LAS u32*)(L + tb + 256 + wc * 8 + 4 + r * 32);
        return a0 | (a1 << 8) | (b << 16);
    };
    f32x4 acc[2][2][4][2];
    f16x8 At[4][2], B0[2][2], B1[2][2];
    u32 cur = 0, cmax = 0, bad = 0;
    u32 aoff0 = wr * 8192 + lane * 16, aoff1 = aoff0;
    u32 nm, nmn = 0;
    auto changes = [&](u32 nw, u32 old) -> u32 {
        const u32 prev = __builtin_amdgcn_update_dpp((int)__builtin_amdgcn_readlane(old, 7), (int)nw, 0x111, 0xf, 0xf, false);
        return (u32)__builtin_amdgcn_ballot_w64(nw != prev) & 255u; };
#define ADJUST(r, buf) do { if (__builtin_expect((nm >> (r)) & 1u, 0)) { const u32 cn_ = __builtin_amdgcn_readlane(pk, (r)), cb_ = (cn_ >> 16) & 63u; \
            const u32 l16_ = lane_id() * 16; \
            _Pragma("unroll") for (int ai = 0; ai < 2; ++ai) { \
                const u32 ca_ = (cn_ >> (8 * ai)) & 63u, c_ = ca_ + cb_, o_ = (cur >> (8 * ai)) & 255u, x_ = (cmax >> (8 * ai)) & 255u; \
                u32 ab_ = wr * 8192; int sh_ = 0; \
                if (ca_ == 62u || cb_ == 62u) bad = 1; \
                else if (ca_ != 63u && cb_ != 63u) { \
                    if (c_ + 10u < x_) { ab_ = L_ZERO - SA(buf, ai); nm = 255u; nmn |= 1u; }      \
                    else { sh_ = 8 * ((int)o_ - (int)c_); \
                        cur = (cur & ~(255u << (8 * ai))) | (c_ << (8 * ai)); \
                        if (c_ > x_) cmax = (cmax & ~(255u << (8 * ai))) | (c_ << (8 * ai)); } } \
                asm volatile("v_add_u32 %0, %1, %2" : "+v"(ai ? aoff1 : aoff0) : "s"(ab_), "v"(l16_)); \
                if (sh_ != 0) { \
                    asm volatile("s_nop 15" ::: "memory");        \
                    _Pragma("unroll") for (int bj = 0; bj < 2; ++bj) _Pragma("unroll") for (int m = 0; m < 4; ++m) _Pragma("unroll") for (int n = 0; n < 2; ++n) \
                        _Pragma("unroll") for (int e_ = 0; e_ < 4; ++e_) asm volatile("v_ldexp_f32 %0, %0, %1" : "+v"(acc[ai][bj][m][n][e_]) : "s"(sh_)); } } } } while (0)
#pragma unroll
    for (int a = 0; a < 2; ++a)
#pragma unroll
        for (int b = 0; b < 2; ++b)
#pragma unroll
            for (int m = 0; m < 4; ++m)
#pragma unroll
                for (int n = 0; n < 2; ++n) acc[a][b][m][n] = (f32x4){0.f, 0.f, 0.f, 0.f};
    const char* const cA = ws + OFF_A + (size_t)(call & 1u) * 4096 * 8192 + (size_t)pm * 2 * HA;
    const char* cB = ws + OFF_A + (size_t)U_B * 8192 + (size_t)lp0 * 64 * KB;
    *(LAS u32x4*)(L + L_ZERO + wave * 1024 + lane * 16) = (u32x4){0u, 0u, 0u, 0u};
    if (wave == 0) poll(lp0, 0);
    LGKM0();
    BAR();
    u32 pk = codes(L_T2), pkn = pk;
    nm = changes(pk, pk) | 1u;
    STAGE(SB(0, 0), cB, voffB); STAGE(SA(0, 0), cA, voffA); STAGE(SB(0, 1), cB + HB, voffB); STAGE(SA(0, 1), cA + HA, voffA);
    if (wr == 1) RBAR;
    VMCNT(4); RBAR;
    STAGE(SB(1, 0), cB + KB, voffB); STAGE(SA(1, 0), cA + KA, voffA); STAGE(SB(1, 1), cB + HB + KB, voffB);
    VMCNT(6); RBAR;
#pragma unroll 1
    for (int rd = 0; rd < 3; ++rd) {
        const int lp = lp0 + rd * 12;
        const char* nB = rd < 2 ? cB + 12 * 64 * KB : cB;
#define TRIP(r, a1, a2, b2, HOOK) do { \
            ADJUST(r, 0); \
            LDB(B0, 0, 0); SCHED; LDA(At, 0, 0); STAGE(SA(1, 1), (a1) + HA, voffA); \
            WAIT_L(8); RBAR; WAIT_L(0); MMA(0, 0, At, B0); RBAR; SCHED; \
            LDB(B1, 0, 1); STAGE(SB(0, 0), (b2), voffB); \
            RBAR; WAIT_L(0); MMA(0, 1, At, B1); RBAR; \
            LDA(At, 0, 1); STAGE(SA(0, 0), (a2), voffA); \
            RBAR; WAIT_L(0); MMA(1, 0, At, B0); RBAR; SCHED; \
            STAGE(SB(0, 1), (b2) + HB, voffB); \
            VMCNT(6); RBAR; MMA(1, 1, At, B1); RBAR; \
            ADJUST((r) + 1, 1); \
            LDB(B0, 1, 0); SCHED; LDA(At, 1, 0); STAGE(SA(0, 1), (a2) + HA, voffA); \
            WAIT_L(8); RBAR; WAIT_L(0); MMA(0, 0, At, B0); RBAR; SCHED; \
            LDB(B1, 1, 1); STAGE(SB(1, 0), (b2) + KB, voffB); \
            RBAR; WAIT_L(0); MMA(0, 1, At, B1); RBAR; \
            LDA(At, 1, 1); STAGE(SA(1, 0), (a2) + KA, voffA); \
            RBAR; WAIT_L(0); MMA(1, 0, At, B0); RBAR; SCHED; \
            STAGE(SB(1, 1), (b2) + KB + HB, voffB); \
            HOOK; \
            VMCNT(6); RBAR; MMA(1, 1, At, B1); RBAR; } while (0)
#pragma unroll 1
        for (int j = 0; j < 8; ++j) {
            const char* ga = cA + (size_t)j * 8 * KA; const char* gb = cB + (size_t)j * 8 * KB;
            const char* na = j == 7 ? cA : ga + 8 * KA; const char* nb = j == 7 ? nB : gb + 8 * KB;
            const bool more = j < 7 || rd < 2;
            const int nlp = j < 7 ? lp : lp + 12, nj = (j + 1) & 7;
            TRIP(0, ga + KA, ga + 2 * KA, gb + 2 * KB, do { } while (0));
            TRIP(2, ga + 3 * KA, ga + 4 * KA, gb + 4 * KB,
                 if (__builtin_expect(wave == 0 && more, 0)) { const u32 ln = lane_id(); const char* fp = flag_ptr(nlp, nj, ln * 2);
                     if (ln < 32) asm volatile("s_mov_b32 m0, %1\n\ts_nop 0\n\tglobal_load_lds_dwordx4 %0, off sc0 sc1" :: "v"(fp), "s"(L_T1) : "memory"); });
            TRIP(4, ga + 5 * KA, ga + 6 * KA, gb + 6 * KB, {
                 const u32 tag = *(const LAS u32*)(L + L_T1 + lane_id() * 8);
                 pkn = codes(L_T1);
                 if (__builtin_expect((__builtin_amdgcn_ballot_w64(tag >= call) != ~0ull) & more, 0)) {
                     if (wave == 0) poll(nlp, nj);
                     BAR();
                     pkn = codes(L_T2);
                 }
                 nmn |= changes(pkn, pk); });
            TRIP(6, ga + 7 * KA, na, nb, { pk = pkn; nm = nmn; nmn = 0; });
        }
        if (wave == 0) { STAMP(4 + rd * 2); STAMPV(50 + rd, __builtin_amdgcn_s_memtime()); }
        {
            const int ln = lane_id();
            const int gp = lp < 4 ? 32 + P.rank * 4 + lp : lp - 4;
            GAS float* o = (GAS float*)P.out + (size_t)(pm * 256 + wr * 64 + (ln >> 4) * 4) * GN + gp * 256 + wc * 64 + (ln & 15) * 4;
#pragma unroll
            for (int ai = 0; ai < 2; ++ai) {
                const int sh = 8 * (int)((cur >> (8 * ai)) & 255u) - 272;
#pragma unroll
                for (int m = 0; m < 4; ++m)
#pragma unroll
                    for (int r = 0; r < 4; ++r) {
                        f32x4 v;
#pragma unroll
                        for (int i = 0; i < 4; ++i) {
                            const float y = __builtin_elementwise_maximum(__builtin_ldexpf(acc[ai][i >> 1][m][i & 1][r], sh), 0.0f);
                            v[i] = __uint_as_float(bad ? 0x7fc00000u : __float_as_uint(y) & 0x7fffffffu);
                            acc[ai][i >> 1][m][i & 1][r] = 0.f;
                        }
                        st_out(o + (size_t)(ai * 128 + m * 16 + r) * GN, v);
                    }
            }
            if (lp < 4) {
                VMCNT(0);
                if (ln == 0) st4_sys(ws + OFF_OF + (size_t)((pm * 4 + lp) * 8 + wave) * 4, call);
            }
            cur = 0; cmax = 0; bad = 0; nm |= 1u;
        }
        if (wave == 0) STAMP(5 + rd * 2);
        cB = nB;
    }
    VMCNT(0);
    if (wr == 0) RBAR;
#undef SA
#undef SB
#undef STAGE
#undef LDA
#undef LDB
#undef MMA
#undef ADJUST
#undef TRIP
}

template <int G> __device__ __forceinline__ void pack_dense(LAS char* dense, LAS char* img, u32 lane, u32 n8, u32 o_mid, u32 o_top, u32 o_esc, u32 b2,
                                                            u32* T, u32* T3, u32* bp, u32& tot, u32& e) {
    u32x4 a[G], b[G], lo[G]; u32x2 mid[G]; u32 hi[G][4], ge[G][4], top[G];
#pragma unroll
    for (int i = 0; i < G; ++i) { a[i] = *(LAS u32x4*)(dense + (i * 64 + lane) * 32u); b[i] = *(LAS u32x4*)(dense + (i * 64 + lane) * 32u + 16u); }
#pragma unroll
    for (int i = 0; i < G; ++i) {
        const u32 dd[8] = {a[i].x, a[i].y, a[i].z, a[i].w, b[i].x, b[i].y, b[i].z, b[i].w};
        u32 x[4], c[4], f9[4], len, tp = 0;
#pragma unroll
        for (int k = 0; k < 4; ++k) {
            lo[i][k] = __builtin_amdgcn_perm(dd[2 * k + 1], dd[2 * k], 0x05040100u);
            hi[i][k] = __builtin_amdgcn_perm(dd[2 * k + 1], dd[2 * k], 0x07060302u);
            x[k] = PW((u16x2)(PU(hi[i][k]) - PU(b2)));
            if (i == G - 1) x[k] = i * 64 + lane < n8 ? x[k] : 0u;
        }
        pk_classify<7>(x, c, ge[i], T[i], T3[i], len);
#pragma unroll
        for (int k = 0; k < 4; ++k) { f9[k] = (c[k] << 7) | (x[k] & 0x007f007fu); tp |= ((c[k] >> 1) & 0x00010001u) << k; }
        mid[i] = (u32x2){__builtin_amdgcn_perm(f9[1], f9[0], 0x06040200u), __builtin_amdgcn_perm(f9[3], f9[2], 0x06040200u)};
        top[i] = tp | (tp >> 12);
        bp[i] = tot; tot += len;
    }
#pragma unroll
    for (int i = 0; i < G; ++i) {
        const u32 g = i * 64 + lane;
        if (i < G - 1 || g < n8) {
            *(LAS u32x4*)(img + 256u + g * 16u) = lo[i];
            *(LAS u32x2*)(img + o_mid + g * 8u) = mid[i];
            *(LAS unsigned char*)(img + o_top + g) = (unsigned char)top[i];
        }
    }
#pragma unroll
    for (int i = 0; i < G; ++i) emit_escapes(ge[i], hi[i], (i * 64 + lane) * 8u, img + o_esc, (8192u - o_esc) >> 2, e);
}
__device__ __forceinline__ void packer(const Params& P, LAS char* L, int pw, int wave, int lane, u32 call) {
    char* ws = P.me;
    LAS char* dense = L + wave * PK_B;
    LAS char* img = dense + PK_IMG;
    const auto rs = __builtin_amdgcn_make_buffer_rsrc((void*)P.out, 0, (u32)GM * GN * 4u, 0x00020000);
    auto wait_item = [&](int it) -> u32 {
        const u64 t0 = now(); u32 sp = 0;
        while (uni(ld4_sys(ws + OFF_OF + it * 4)) < call) {
            if (give_up(sp, t0, (u32*)(ws + OFF_TMO), call)) return 0x7fc00000u;
            __builtin_amdgcn_s_sleep(8);
        }
        return 0u;
    };
    u32x4 nx[8];
    auto fetch_block = [&](int it, int su) {
        const int tile = it >> 3, wq = it & 7;
        const u32 so = ((u32)((tile >> 2) * 256 + (su >> 1) * 128 + (wq >> 2) * 64 + (su & 1) * 32) * GN + 8192 + P.rank * 1024 + (tile & 3) * 256 + (wq & 3) * 64) * 4u;
#pragma unroll
        for (int g = 0; g < 8; ++g) nx[g] = __builtin_amdgcn_raw_buffer_load_b128(rs, ((lane >> 4) * 4 * GN + (lane & 15) * 4) * 4, so + ((g >> 2) * 16 + (g & 3)) * GN * 4, 17);
    };
    auto publish = [&](int it, u32 ea, u32 eb) {
        if (lane < 32 && (lane >> 2) != P.rank)
            st16_sys(peer_ws(P, lane >> 2) + OFF_REC + (call & 1u) * REC_PAR + P.rank * REC_SRC + REC_OUT + it * 64 + (lane & 3) * 16, (u32x4){call, ea, call, eb});
    };
    u32 nanm = wait_item(pw), nann = 0, off16 = 0, ea = 0, eb = 0;
    fetch_block(pw, 0);
#pragma unroll
    for (int g = 0; g < 8; ++g) asm volatile("" : "+v"(nx[g]));
#pragma unroll 1
    for (int bk = 0; bk < N_OUT_ITEMS / 16; ++bk) {
        {
            const int it = pw + (bk >> 2) * 64, su = bk & 3;
            char* og = ws + OFF_O + (size_t)it * QSLOT;
            u32 v[32]; u32 mx = 0;
#pragma unroll
            for (int r = 0; r < 32; ++r) { v[r] = nx[r >> 2][r & 3] | nanm; mx = max(mx, v[r]); }
            { const int bn = bk + 1 < N_OUT_ITEMS / 16 ? bk + 1 : bk; fetch_block(pw + (bn >> 2) * 64, bn & 3); }
            mx = wave_umax(mx) >> 16;
            const u32 b16 = mx >= 511u ? mx - 511u : 0u, b2 = b16 * 0x10001u;
            u32 f[32], mk0 = 0, mk1 = 0;
#pragma unroll
            for (int r = 0; r < 32; ++r) asm("v_min_u32 %0, 1, %1" : "=v"(f[r]) : "v"(v[r]));
#pragma unroll
            for (int r = 0; r < 16; ++r) asm("v_lshl_or_b32 %0, %2, %4, %0\n\tv_lshl_or_b32 %1, %3, %5, %1" : "+v"(mk0), "+v"(mk1) : "v"(f[r]), "v"(f[16 + r]), "n"(r), "n"(16 + r));
            const u32 mk = mk0 | mk1, pc = __builtin_popcount(mk), inc = wave_scan(pc), n = __builtin_amdgcn_readlane(inc, 63);
            {
                u32 ad = wave * PK_B + (inc - pc) * 4u;
#pragma unroll
                for (int r = 0; r < 32; r += 8) {
                    u32 a[8]; u64 m[8];
#pragma unroll
                    for (int i = 0; i < 8; ++i) { a[i] = ad; asm("v_lshl_add_u32 %0, %1, 2, %2" : "=v"(ad) : "v"(f[r + i]), "v"(a[i])); m[i] = __builtin_amdgcn_ballot_w64(v[r + i] != 0u); }
                    asm volatile("s_mov_b64 exec, %16\n\tds_write_b32 %0, %8\n\ts_mov_b64 exec, %17\n\tds_write_b32 %1, %9\n\t"
                                 "s_mov_b64 exec, %18\n\tds_write_b32 %2, %10\n\ts_mov_b64 exec, %19\n\tds_write_b32 %3, %11\n\t"
                                 "s_mov_b64 exec, %20\n\tds_write_b32 %4, %12\n\ts_mov_b64 exec, %21\n\tds_write_b32 %5, %13\n\t"
                                 "s_mov_b64 exec, %22\n\tds_write_b32 %6, %14\n\ts_mov_b64 exec, %23\n\tds_write_b32 %7, %15\n\ts_mov_b64 exec, -1"
                                 :: "v"(a[0]), "v"(a[1]), "v"(a[2]), "v"(a[3]), "v"(a[4]), "v"(a[5]), "v"(a[6]), "v"(a[7]),
                                    "v"(v[r]), "v"(v[r + 1]), "v"(v[r + 2]), "v"(v[r + 3]), "v"(v[r + 4]), "v"(v[r + 5]), "v"(v[r + 6]), "v"(v[r + 7]),
                                    "s"(m[0]), "s"(m[1]), "s"(m[2]), "s"(m[3]), "s"(m[4]), "s"(m[5]), "s"(m[6]), "s"(m[7]) : "memory");
                }
            }
            if (lane < 8) *(LAS u32*)(dense + (n + lane) * 4u) = b16 << 16;
            *(LAS u32*)(img + lane * 4) = mk;
            const u32 n8 = (n + 7u) >> 3, G = (n8 + 63u) >> 6, o_mid = 256u + n8 * 16u, o_top = o_mid + n8 * 8u, o_esc = o_top + ((n8 + 3u) & ~3u);
            CFENCE();
            u32 e = 0, tot = 0, T[4] = {0u, 0u, 0u, 0u}, T3[4] = {0u, 0u, 0u, 0u}, bp[4] = {0u, 0u, 0u, 0u};
            if (G == 3u) pack_dense<3>(dense, img, lane, n8, o_mid, o_top, o_esc, b2, T, T3, bp, tot, e);
            else if (G == 2u) pack_dense<2>(dense, img, lane, n8, o_mid, o_top, o_esc, b2, T, T3, bp, tot, e);
            else if (G == 1u) pack_dense<1>(dense, img, lane, n8, o_mid, o_top, o_esc, b2, T, T3, bp, tot, e);
            else if (G == 4u) pack_dense<4>(dense, img, lane, n8, o_mid, o_top, o_esc, b2, T, T3, bp, tot, e);
#pragma unroll
            for (int g = 0; g < 8; ++g) asm volatile("" : "+v"(nx[g]));
            VMCNT(0);
            if (su == 0) { if (bk) publish(it - 64, ea, eb); off16 = 0; }
            if (su == 2 && it + 64 < N_OUT_ITEMS) nann = wait_item(it + 64);
            const u32 binc = wave_scan(tot), all2 = __builtin_amdgcn_readlane(binc, 63), o_sec = o_esc + e * 4u;
            u32 size16 = (o_sec + ((pk_sum(all2) + 7u) >> 3) + 15u) >> 4;
            if (size16 >= 512u) {
                size16 = 512u;
#pragma unroll
                for (int g = 0; g < 8; ++g) st16_sys(og + off16 * 16u + g * 1024 + lane * 16, (u32x4){v[g * 4], v[g * 4 + 1], v[g * 4 + 2], v[g * 4 + 3]});
            } else {
#pragma unroll
                for (int i = 0; i < 6; ++i) *(LAS u32*)(img + o_sec + i * 256 + lane * 4) = 0u;
                CFENCE();
#pragma unroll
                for (int i = 0; i < 4; ++i) if (i < (int)G) {
                    const u32 bq = bp[i] + binc - tot + (all2 << 16);
                    or_bits(img + o_sec, bq & 0xffffu, T[i]);
                    or_bits(img + o_sec, bq >> 16, T3[i]);
                }
                CFENCE();
#pragma unroll 1
                for (u32 g = lane; g < size16 + lane; g += 256u) {
                    u32x4 t[4];
#pragma unroll
                    for (int i = 0; i < 4; ++i) t[i] = *(LAS u32x4*)(img + (g + i * 64) * 16u);
#pragma unroll
                    for (int i = 0; i < 4; ++i) if (g + i * 64 < size16) st16_sys(og + (off16 + g + i * 64) * 16u, t[i]);
                }
            }
            if ((lane & 3) == su) { ea = size16 | (n << 10); eb = (e & 0xfffu) | (b16 << 12); }
            off16 += size16;
            if (su == 3) nanm = nann;
            LGKM0();
            CFENCE();
        }
        if (bk == 4) STAMP(16);
    }
    VMCNT(0);
    publish(pw + N_OUT_ITEMS - 64, ea, eb);
    STAMP(17);
}

__global__ __launch_bounds__(NTHR, 2) __attribute__((amdgpu_waves_per_eu(2, 2))) void fused_kernel(Params P) {
    extern __shared__ __attribute__((aligned(16))) char smem[];
    LAS char* L = (LAS char*)smem;
    const int tid = threadIdx.x, lane = tid & 63, wave = __builtin_amdgcn_readfirstlane(tid >> 6);
    const int b = blockIdx.x, xg = b & 7, s = b >> 3;
    const bool fetch = s < FS;
    constexpr int NF = FS * 8;
    if (wave == 0) STAMP(0);
    GAS u32* const cnt = (GAS u32*)(P.me + OFF_CNT + (b * 8 + wave) * 8);
    constexpr bool diag = false;
    const int pidx = b - (NF + 120);
    constexpr int NU = 112 - ITEM0;
    const int q0 = ITEM0 + (b - NF);
    const bool loaded = !fetch && !diag && q0 < 112 && wave < 2;
    f32x4 v[16];
    if (wave == 7 && (unsigned)pidx < (unsigned)NPUSH && !diag) {
        load_x(P, pidx, 0, lane, v);
        const u32 craw = __hip_atomic_load(cnt, __ATOMIC_RELAXED, __HIP_MEMORY_SCOPE_AGENT);
        __builtin_amdgcn_sched_barrier(0);
        BAR();
        __builtin_amdgcn_sched_barrier(0);
        STAMP(22);
        push_task(P, pidx, lane, craw, v);
        STAMP(23);
    } else {
        if (loaded) { if (q0 < 32) load_x(P, q0 * 2 + (wave & 1), 0, lane, v); else load_w(P, (q0 < 96 ? q0 + 32 : q0 - 96) * 2 + (wave & 1), 0, lane, v); }
        __builtin_amdgcn_sched_barrier(0);
        if (tid < 256) lds_st(L_ITEM + tid * 4, !fetch ? 0u : tid == 18 ? 6u : tid >= 24 && tid < 32 ? (u32)(tid - 24) : 0u);
        LGKM0();
        BAR();
    }
    if (!fetch && !diag) {
        const u64 tstart = now();
        const bool pack = s >= FS + GS;
        auto fetch_task = [&](int q) {
            const u64 go = pack ? (u64)PACE4 : (u64)(PACE2 + (q - 112) * PACE3 / 112);
            while (now() - tstart < go) __builtin_amdgcn_s_sleep(4);
            int j, idx; bool isA;
            task_of(q, wave & 1, j, idx, isA);
            if (isA) load_x(P, idx, j, lane, v); else load_w(P, idx, j, lane, v);
        };
        int qn = 112 + (pack ? GS * 8 + xg : (b - NF + GS * 8 - NU) % (GS * 8)) * 4 + (wave >> 1), q = q0;
        const int qs = pack ? 32 : N_PAIRS;
        u32 lq = L_ITEM;
        if (!loaded) { q = qn; qn += qs; lq += 64; fetch_task(q); }
#pragma unroll 1
        for (;;) {
            stage_task(P, L + wave * 4096, lq, lq + 48 + (wave >> 1) * 4, q, wave & 1, lane, cnt, v);
            if (qn >= N_PAIRS) break;
            q = qn; qn += qs; lq += 64;
            fetch_task(q);
        }
    }
    const u32 call = uni(__hip_atomic_load(cnt, __ATOMIC_RELAXED, __HIP_MEMORY_SCOPE_AGENT)) + (diag ? 0u : 1u);
    if (lane == 0) st4_sys((void*)cnt, call);
    STAMP(fetch ? 1 : (s >= FS + GS ? 19 : 18));
    if (fetch) {
        const int src = (P.rank + 1 + s) & 7;
        if (wave == 0) { loader(P, src, lane, call); STAMP(2); }
        else if (wave == 1) scout(P, src, xg, lane, call);
        else { decoder(P, L, src, wave - 2, lane, call); STAMP(8 + wave); }
    } else if (s < FS + GS) {
        const int g = s - FS;
        gemm_wg(P, L, (xg >> 1) * 4 + g / 6, (xg & 1) * 6 + g % 6, wave, lane, call);
    } else {
        BAR();
        packer(P, L, wave * 8 + xg, wave, lane, call);
    }
}

}

extern "C" void kernel_launch(int rank, int nranks, ncclComm_t comm, hipStream_t stream,
                              void* const* d_in, const int* in_sizes, int n_in,
                              void* d_out, int out_size,
                              const int* in_dtypes, int out_dtype,
                              void* const* d_out_all,
                              void* const* d_ws_all, size_t ws_bytes) {
    (void)comm; (void)in_sizes; (void)n_in; (void)out_size; (void)in_dtypes; (void)out_dtype; (void)d_out_all;
    if (nranks != NR || (unsigned)rank >= (unsigned)NR || ws_bytes < WS_NEED) { fprintf(stderr, "unsupported configuration\n"); return; }
    static bool ready[NR];
    if (!ready[rank]) {
        HIP_CHECK(hipFuncSetAttribute((const void*)fused_kernel, hipFuncAttributeMaxDynamicSharedMemorySize, (int)LDS_BYTES));
        ready[rank] = true;
    }
    Params P;
    P.x = (const float*)d_in[0]; P.w = (const float*)d_in[1]; P.out = (float*)d_out;
    for (int i = 0; i < NR; ++i) P.ws[i] = (char*)d_ws_all[i];
    P.me = P.ws[rank]; P.rank = rank; P.pad = 0;
    hipLaunchKernelGGL(fused_kernel, dim3(NWG), dim3(NTHR), LDS_BYTES, stream, P);
    HIP_CHECK(hipGetLastError());
}
```

```cpp
#include <hip/hip_runtime.h>
#include <rccl/rccl.h>
#include <cstdio>

#define HIP_CHECK(expr) do { hipError_t _e = (expr); if (_e != hipSuccess) { \
    fprintf(stderr, "HIP error %s at %s:%d\n", hipGetErrorString(_e), __FILE__, __LINE__); \
    return; } } while (0)

typedef unsigned u32;
typedef unsigned long long u64;
typedef u32 u32x2 __attribute__((ext_vector_type(2)));
typedef u32 u32x4 __attribute__((ext_vector_type(4)));
typedef float f32x4 __attribute__((ext_vector_type(4)));
typedef _Float16 f16x8 __attribute__((ext_vector_type(8)));
#define LAS __attribute__((address_space(3)))
#define GAS __attribute__((address_space(1)))

namespace {

constexpr int NR = 8, GM = 4096, GN = 16384, KL = 512;
constexpr int NWG = 256, NTHR = 512;
#define WINV 8
#define NPUSH 36
constexpr int FS = 7, GS = 24;
constexpr int LPP = 8;
constexpr int WIN = WINV;
constexpr int NLP = 36;
constexpr int N_IN_ITEMS = 832, N_OUT_ITEMS = 512, N_ITEMS = N_IN_ITEMS + N_OUT_ITEMS;
constexpr int N_PAIRS = 1280, ITEM0 = NPUSH / 2;
#define PACE2 150
#define PACE3 350
#define PACE4 800
static_assert(NPUSH % 2 == 0 && NPUSH >= 8 && NPUSH <= 64, "only x blocks of the first K group");
constexpr u64 DEADLINE = 30000000ull;

constexpr size_t OFF_CNT = 0;
constexpr size_t OFF_TMO = 16384;
constexpr size_t OFF_TICK = 16384 + 256;
constexpr size_t OFF_OF = 20480;
constexpr u32 U_B = 8192, N_UNITS = U_B + NLP * 256;
constexpr size_t OFF_FA = 32768;
constexpr size_t OFF_DBG = 262144;
constexpr size_t OFF_REC = 1ull << 20;
constexpr u32 REC_SRC = 65536, REC_OUT = 16384;
constexpr size_t REC_PAR = 1ull << 20;
constexpr size_t OFF_A = 4ull << 20;
constexpr size_t OFF_AX = OFF_A + (size_t)N_UNITS * 8192;
constexpr size_t OFF_BX = OFF_AX + (4ull << 20);
constexpr size_t X_PAR = 20ull << 20;
constexpr size_t OFF_O = OFF_AX + 2 * X_PAR;
constexpr size_t QSLOT = 32768;
constexpr size_t WS_NEED = OFF_O + N_OUT_ITEMS * QSLOT;
static_assert(NR * REC_SRC <= REC_PAR && N_IN_ITEMS * 16 <= REC_OUT && REC_OUT + N_OUT_ITEMS * 64 <= REC_SRC && OFF_FA + N_UNITS * 8 <= OFF_DBG, "workspace map");

constexpr u32 RING = 65536;
constexpr u32 L_SCR = RING, SCR_B = 8448;
constexpr u32 L_DESC = L_SCR + 6 * SCR_B, NQ = 64;
constexpr u32 L_ITEM = L_DESC + NQ * 16;
constexpr u32 L_FCTL = L_ITEM + 64;
constexpr u32 L_LAND = L_FCTL + 64;
constexpr u32 L_T1 = 131072, L_T2 = L_T1 + 512, L_ZERO = L_T2 + 512;
constexpr u32 PK_B = 19456, PK_IMG = 8320;
constexpr u32 LDS_BYTES = 8 * PK_B;
static_assert(L_ITEM + 1024 <= LDS_BYTES && L_LAND + 256 <= L_ITEM + 1024 && L_ZERO + 8192 <= LDS_BYTES && LDS_BYTES <= 163840, "LDS map");

struct Params {
    const float* x; const float* w; float* out;
    char* me;
    char* ws[NR];
    int rank; int pad;
};

#define VMCNT(n) asm volatile("s_waitcnt vmcnt(" #n ")" ::: "memory")
#define LGKM0() asm volatile("s_waitcnt lgkmcnt(0)" ::: "memory")
#define CFENCE() asm volatile("" ::: "memory")
#define BAR() do { CFENCE(); __builtin_amdgcn_s_barrier(); CFENCE(); } while (0)

__device__ __forceinline__ void st16_sys(void* p, u32x4 v) {
    asm volatile("global_store_dwordx4 %0, %1, off sc0 sc1\n\ts_nop 1" :: "v"(p), "v"(v) : "memory");
}
__device__ __forceinline__ void st_out(GAS float* p, f32x4 v) {
    asm volatile("global_store_dwordx4 %0, %1, off sc0 sc1 nt\n\ts_nop 1" :: "v"(p), "v"(v) : "memory");
}
__device__ __forceinline__ void st4_sys(void* p, u32 v) {
    asm volatile("global_store_dword %0, %1, off sc0 sc1" :: "v"(p), "v"(v) : "memory");
}
__device__ __forceinline__ void st8_sys(void* p, u32x2 v) {
    asm volatile("global_store_dwordx2 %0, %1, off sc0 sc1" :: "v"(p), "v"(v) : "memory");
}
__device__ __forceinline__ u32 ld4_sys(const void* p) {
    u32 v; asm volatile("global_load_dword %0, %1, off sc0 sc1\n\ts_waitcnt vmcnt(0)" : "=&v"(v) : "v"(p) : "memory"); return v;
}
__device__ __forceinline__ u32x2 ld8_sys(const void* p) {
    u32x2 v; asm volatile("global_load_dwordx2 %0, %1, off sc0 sc1\n\ts_waitcnt vmcnt(0)" : "=&v"(v) : "v"(p) : "memory"); return v;
}
__device__ __forceinline__ void lds_st(u32 a, u32 v) { asm volatile("ds_write_b32 %0, %1" :: "v"(a), "v"(v) : "memory"); }
__device__ __forceinline__ u32 lds_ld(u32 a) {
    u32 v; asm volatile("ds_read_b32 %0, %1\n\ts_waitcnt lgkmcnt(0)" : "=&v"(v) : "v"(a) : "memory"); return v;
}
__device__ __forceinline__ u32x4 lds_ld16(u32 a) {
    u32x4 v; asm volatile("ds_read_b128 %0, %1\n\ts_waitcnt lgkmcnt(0)" : "=&v"(v) : "v"(a) : "memory"); return v;
}
__device__ __forceinline__ u32 uni(u32 v) { return __builtin_amdgcn_readfirstlane(v); }
__device__ __forceinline__ u32 lds_ldu(u32 a) { return uni(lds_ld(a)); }
__device__ __forceinline__ u64 now() { return __builtin_amdgcn_s_memrealtime(); }
__device__ __forceinline__ u32 prefix(u64 m) { return __builtin_amdgcn_mbcnt_hi((u32)(m >> 32), __builtin_amdgcn_mbcnt_lo((u32)m, 0u)); }

__device__ __forceinline__ u32 wave_umax(u32 v) {
    v = max(v, (u32)__builtin_amdgcn_update_dpp(0, (int)v, 0x111, 0xf, 0xf, true));
    v = max(v, (u32)__builtin_amdgcn_update_dpp(0, (int)v, 0x112, 0xf, 0xf, true));
    v = max(v, (u32)__builtin_amdgcn_update_dpp(0, (int)v, 0x114, 0xf, 0xf, true));
    v = max(v, (u32)__builtin_amdgcn_update_dpp(0, (int)v, 0x118, 0xf, 0xf, true));
    v = max(v, (u32)__builtin_amdgcn_update_dpp(0, (int)v, 0x142, 0xa, 0xf, false));
    v = max(v, (u32)__builtin_amdgcn_update_dpp(0, (int)v, 0x143, 0xc, 0xf, false));
    return __builtin_amdgcn_readlane(v, 63);
}
__device__ __forceinline__ u32 wave_scan(u32 v) {
    v += (u32)__builtin_amdgcn_update_dpp(0, (int)v, 0x111, 0xf, 0xf, true);
    v += (u32)__builtin_amdgcn_update_dpp(0, (int)v, 0x112, 0xf, 0xf, true);
    v += (u32)__builtin_amdgcn_update_dpp(0, (int)v, 0x114, 0xf, 0xf, true);
    v += (u32)__builtin_amdgcn_update_dpp(0, (int)v, 0x118, 0xf, 0xf, true);
    v += (u32)__builtin_amdgcn_update_dpp(0, (int)v, 0x142, 0xa, 0xf, false);
    v += (u32)__builtin_amdgcn_update_dpp(0, (int)v, 0x143, 0xc, 0xf, false);
    return v;
}
typedef unsigned short u16x2 __attribute__((ext_vector_type(2)));
typedef short i16x2 __attribute__((ext_vector_type(2)));
#define PU(x) __builtin_bit_cast(u16x2, (u32)(x))
#define PI(x) __builtin_bit_cast(i16x2, (u32)(x))
#define PW(x) __builtin_bit_cast(u32, (x))
template <int W> __device__ __forceinline__ u32 pk_offsets(const u32* m, u32* ac) {
    u32 tot;
    ac[0] = 0;
    asm("v_pk_mul_lo_u16 %0, %4, %8 op_sel_hi:[1,0]\n\tv_pk_mad_u16 %1, %5, %8, %0 op_sel_hi:[1,0,1]\n\t"
        "v_pk_mad_u16 %2, %6, %8, %1 op_sel_hi:[1,0,1]\n\tv_pk_mad_u16 %3, %7, %8, %2 op_sel_hi:[1,0,1]"
        : "=&v"(ac[1]), "=&v"(ac[2]), "=&v"(ac[3]), "=&v"(tot) : "v"(m[0]), "v"(m[1]), "v"(m[2]), "v"(m[3]), "n"(-W));
    return tot;
}
__device__ __forceinline__ u32 pk_is3(u32 c) { return PW((i16x2)(PI(c + 0x7ffd7ffdu) >> (i16x2){15, 15})); }
__device__ __forceinline__ u32 pk_sum(u32 a) { return (a & 0xffffu) + (a >> 16); }
__device__ __forceinline__ u32 pk_join(const u32* sv, const u32* ac, u32 acc) {
    const u32 tt = sv[0] | PW((u16x2)(PU(sv[1]) << PU(ac[1]))) | PW((u16x2)(PU(sv[2]) << PU(ac[2]))) | PW((u16x2)(PU(sv[3]) << PU(ac[3])));
    return (tt & 0xffffu) | ((tt >> 16) << (acc & 31u));
}
__device__ __forceinline__ u32 pk_split(u32 t, u32 acc) { return __builtin_amdgcn_perm(t >> (acc & 31u), t, 0x05040100u); }
__device__ __forceinline__ u32 pk_field(u32 tt, u32 sh, u32 m, u32 mask) { return PW((u16x2)(PU(tt) >> PU(sh))) & m & mask; }
__device__ __forceinline__ u32 pk_expand(u32 sv, u32 k, u32 base) {
    u32 r;
    asm("v_pk_min_u16 %0, %1, 1 op_sel_hi:[1,0]\n\tv_pk_mad_u16 %0, %0, 3, %1 op_sel_hi:[1,0,1]\n\tv_pk_mad_u16 %0, %0, %2, %3"
        : "=&v"(r) : "v"(sv), "s"(k), "v"(base));
    return r;
}
template <int SH> __device__ __forceinline__ void pk_classify(const u32* x, u32* c, u32* g, u32& T2, u32& T3, u32& len) {
    u32 m2[4], m3[4], ac[4], s2[4], s3[4];
#pragma unroll
    for (int k = 0; k < 4; ++k) {
        const u32 o = PW((i16x2)(PI(x[k]) >> (i16x2){SH, SH}));
        const u32 sv = PW(__builtin_elementwise_max((i16x2)(-PI(o)), (i16x2){0, 0}));
        c[k] = PW(__builtin_elementwise_min(PU(o), (u16x2){3, 3}));
        s2[k] = PW(__builtin_elementwise_min(PU(sv), (u16x2){3, 3}));
        s3[k] = (sv - s2[k]) & 0x00070007u;
        m2[k] = pk_is3(c[k]); m3[k] = pk_is3(s2[k]);
        g[k] = PW(__builtin_elementwise_sub_sat((u16x2)(PU(o) + (u16x2){10, 10}), (u16x2){13, 13}));
    }
    const u32 a2 = pk_offsets<2>(m2, ac);
    T2 = pk_join(s2, ac, a2);
    const u32 a3 = pk_offsets<3>(m3, ac);
    T3 = pk_join(s3, ac, a3);
    len = pk_sum(a2) | (pk_sum(a3) << 16);
}
__device__ __forceinline__ void or_bits(LAS char* buf, u32 bp, u32 t) {
    const u64 v = (u64)t << (bp & 31u);
    LAS u32* a = (LAS u32*)(buf + (bp >> 5) * 4u);
    __hip_atomic_fetch_or(a, (u32)v, __ATOMIC_RELAXED, __HIP_MEMORY_SCOPE_WAVEFRONT);
    __hip_atomic_fetch_or(a + 1, (u32)(v >> 32), __ATOMIC_RELAXED, __HIP_MEMORY_SCOPE_WAVEFRONT);
}
__device__ __forceinline__ void emit_escapes(const u32* g, const u32* raw, u32 idx0, LAS char* rec, u32 cap, u32& e) {
    if (__builtin_amdgcn_ballot_w64((g[0] | g[1] | g[2] | g[3]) != 0u) == 0ull) return;
    u32 gm = 0;
#pragma unroll
    for (int k = 0; k < 4; ++k) gm |= PW(__builtin_elementwise_min(PU(g[k]), (u16x2){1, 1})) << k;
#pragma unroll 1
    for (;;) {
        const u64 em = __builtin_amdgcn_ballot_w64(gm != 0u);
        if (em == 0ull) break;
        const u32 b = __builtin_ctz(gm | 0x80000000u), k = b & 3u;
        const u32 m1 = (u32)__builtin_amdgcn_sbfe((int)b, 0, 1), m2 = (u32)__builtin_amdgcn_sbfe((int)b, 1, 1);
        const u32 r01 = (raw[1] & m1) | (raw[0] & ~m1), r23 = (raw[3] & m1) | (raw[2] & ~m1), r = (r23 & m2) | (r01 & ~m2);
        if (gm != 0u && e < cap) *(LAS u32*)(rec + (e + prefix(em)) * 4u) = (idx0 + k * 2u + (b >> 4)) | ((r >> (b & 16u)) << 16);
        gm &= gm - 1u;
        e += __builtin_popcountll(em);
    }
}
typedef float f32x2 __attribute__((ext_vector_type(2)));
typedef _Float16 f16x2 __attribute__((ext_vector_type(2)));
__device__ __forceinline__ u32 cvt2(float a, float b, float sc) { return __builtin_bit_cast(u32, __builtin_convertvector((f32x2){a, b} * sc, f16x2)); }
__device__ __forceinline__ char* peer_ws(const Params& P, int p) {
    u64 wp = (u64)P.ws[0];
#pragma unroll
    for (int i = 1; i < NR; ++i) { u64 q = (u64)P.ws[i]; asm volatile("" : "+s"(q)); wp = p == i ? q : wp; }
    return (char*)wp;
}

#define STAMP(k) do { } while (0)
#define STAMPV(k, v) do { } while (0)

__device__ __forceinline__ bool give_up(u32& spins, u64 t0, u32* tmo, u32 call) {
    if ((++spins & 63u) != 0u) return false;
    if (uni(ld4_sys(tmo)) == call) return true;
    if (now() - t0 > DEADLINE) { st4_sys(tmo, call); return true; }
    return false;
}

__device__ __forceinline__ float block_scale(const f32x4 (&v)[16], u32& code, u32& hm) {
    float m0 = 0.f, m1 = 0.f, m2 = 0.f, m3 = 0.f;
#pragma unroll
    for (int i = 0; i < 16; i += 2)
        asm("v_max3_f32 %0, %0, |%4|, |%5|\n\tv_max3_f32 %1, %1, |%6|, |%7|\n\tv_max3_f32 %2, %2, |%8|, |%9|\n\tv_max3_f32 %3, %3, |%10|, |%11|"
            : "+v"(m0), "+v"(m1), "+v"(m2), "+v"(m3)
            : "v"(v[i][0]), "v"(v[i][1]), "v"(v[i][2]), "v"(v[i][3]), "v"(v[i + 1][0]), "v"(v[i + 1][1]), "v"(v[i + 1][2]), "v"(v[i + 1][3]));
    const u32 mb = wave_umax(max(max(__float_as_uint(m0), __float_as_uint(m1)), max(__float_as_uint(m2), __float_as_uint(m3))));
    code = ((mb >> 23) + 2u) >> 3; code = code < 2u ? 2u : code;
    const float sc = __uint_as_float((263u - 8u * code) << 23);
    hm = uni(__builtin_bit_cast(unsigned short, (_Float16)(__uint_as_float(mb) * sc)));
    if (mb == 0u) code = 63u;
    return sc;
}
__device__ __forceinline__ void load_x(const Params& P, int idx, int j, int lane, f32x4 (&v)[16]) {
    const GAS float* p = (const GAS float*)P.x + (size_t)(idx * 64 + (lane & 15)) * KL + j * 64 + (lane >> 4) * 4;
#pragma unroll
    for (int i = 0; i < 16; ++i) v[i] = __builtin_nontemporal_load((const GAS f32x4*)(p + (i >> 2) * 16 * KL + (i & 3) * 16));
}
__device__ __forceinline__ void push_task(const Params& P, int idx, int lane, u32 craw, const f32x4 (&v)[16]) {
    asm volatile("" :: "s"(P.ws[0]), "s"(P.ws[1]), "s"(P.ws[2]), "s"(P.ws[3]), "s"(P.ws[4]), "s"(P.ws[5]), "s"(P.ws[6]), "s"(P.ws[7]));
    __builtin_amdgcn_sched_barrier(0);
    u32 code, hm;
    const float sc = block_scale(v, code, hm);
    const u32 call = uni(craw) + 1u;
    const u32 dest = (call & 1u) * 4096u + ((idx >> 1) * 64 + P.rank) * 2 + (idx & 1);
    const u32 voff = (u32)OFF_A + dest * 8192u + lane * 16u;
#pragma unroll
    for (int f = 0; f < 8; ++f) {
        const u32x4 hv = {cvt2(v[2 * f][0], v[2 * f][1], sc), cvt2(v[2 * f][2], v[2 * f][3], sc), cvt2(v[2 * f + 1][0], v[2 * f + 1][1], sc), cvt2(v[2 * f + 1][2], v[2 * f + 1][3], sc)};
        VMCNT(8);
        asm volatile("global_store_dwordx4 %0, %1, %2 sc0 sc1\n\tglobal_store_dwordx4 %0, %1, %3 sc0 sc1\n\tglobal_store_dwordx4 %0, %1, %4 sc0 sc1\n\t"
                     "global_store_dwordx4 %0, %1, %5 sc0 sc1\n\tglobal_store_dwordx4 %0, %1, %6 sc0 sc1\n\tglobal_store_dwordx4 %0, %1, %7 sc0 sc1\n\t"
                     "global_store_dwordx4 %0, %1, %8 sc0 sc1\n\tglobal_store_dwordx4 %0, %1, %9 sc0 sc1\n\ts_nop 1"
                     :: "v"(voff + f * 1024), "v"(hv), "s"((u64)P.ws[0]), "s"((u64)P.ws[1]), "s"((u64)P.ws[2]), "s"((u64)P.ws[3]),
                        "s"((u64)P.ws[4]), "s"((u64)P.ws[5]), "s"((u64)P.ws[6]), "s"((u64)P.ws[7]) : "memory");
        __builtin_amdgcn_sched_barrier(0);
    }
    VMCNT(0);
    if (lane < NR) st8_sys(peer_ws(P, lane) + OFF_FA + (size_t)dest * 8, (u32x2){call, code});
}
__device__ __forceinline__ void load_w(const Params& P, int idx, int j, int lane, f32x4 (&v)[16]) {
    const GAS float* p = (const GAS float*)P.w + (size_t)(j * 64 + (lane >> 4) * 4) * GN + idx * 64 + (lane & 15) * 4;
#pragma unroll
    for (int i = 0; i < 16; ++i) v[i] = __builtin_nontemporal_load((const GAS f32x4*)(p + (size_t)((i >> 2) * 16 + (i & 3)) * GN));
}
__device__ __forceinline__ void task_of(int q, int half, int& j, int& idx, bool& isA) {
    if (q < 896) { j = q / 112; const int u = q % 112; isA = u < 32; idx = isA ? u : (u < 96 ? u + 32 : u - 96); }
    else { const int u = q - 896, v = u % 192; j = v / 24; isA = false; idx = 16 + (u / 192) * 24 + v % 24; }
    idx = idx * 2 + half;
}
__device__ __forceinline__ void stage_task(const Params& P, LAS char* scr, u32 lquad, u32 lpair, int q, int half, int lane, GAS u32* cnt, const f32x4 (&v)[16]) {
    int j, idx; bool isA;
    task_of(q, half, j, idx, isA);
    const u32 craw = __hip_atomic_load(cnt, __ATOMIC_RELAXED, __HIP_MEMORY_SCOPE_AGENT);
    *(LAS u32x4*)(scr + lane * 16) = (u32x4){0u, 0u, 0u, 0u}; *(LAS u32x4*)(scr + 1024 + lane * 16) = (u32x4){0u, 0u, 0u, 0u};
    __builtin_amdgcn_sched_barrier(0);
    u32 code, hm;
    const float sc = block_scale(v, code, hm);
    const u32 bb = hm >= 4095u ? hm - 4095u : 0u, b2 = bb * 0x10001u;
    u32x4 p[8];
    if (isA) {
#pragma unroll
        for (int f = 0; f < 8; ++f)
#pragma unroll
            for (int k = 0; k < 4; ++k) p[f][k] = cvt2(v[f * 2 + (k >> 1)][(k & 1) * 2], v[f * 2 + (k >> 1)][(k & 1) * 2 + 1], sc);
        CFENCE();
    } else {
#pragma unroll
        for (int f = 0; f < 8; ++f)
#pragma unroll
            for (int k = 0; k < 4; ++k) p[f][k] = cvt2(v[(f & 1) * 8 + k * 2][f >> 1], v[(f & 1) * 8 + k * 2 + 1][f >> 1], sc);
        asm volatile("s_nop 0" ::: "memory");
    }
    char* ws = P.me;
    const u32 call = uni(craw) + 1u;
    const bool all = isA || idx < 128;
    const int owner = (idx - 128) >> 4, lp = idx < 128 ? 4 + (idx >> 2) : (idx >> 2) & 3;
    const bool local = all || owner == P.rank, exprt = all || owner != P.rank;
    const u32 dest = isA ? (call & 1u) * 4096u + (((idx >> 1) * 8 + j) * 8 + P.rank) * 2 + half : U_B + (((lp * 8 + j) * 8 + P.rank) * 4 + (idx & 3));
    u32 size16 = 0;
    if (exprt) {
        u32 w[32], T2[8], T3[8], len[8], e = 0, tot = 0;
#pragma unroll
        for (int f = 0; f < 8; ++f) {
            u32 x[4], c[4], g[4];
            const u32 h[4] = {p[f][0], p[f][1], p[f][2], p[f][3]};
#pragma unroll
            for (int k = 0; k < 4; ++k) x[k] = PW((u16x2)(PU(h[k] & 0x7fff7fffu) - PU(b2)));
            pk_classify<10>(x, c, g, T2[f], T3[f], len[f]);
#pragma unroll
            for (int k = 0; k < 4; ++k) w[f * 4 + k] = (h[k] & 0x80008000u) | (c[k] << 10) | (x[k] & 0x03ff03ffu);
            tot += len[f];
            asm volatile("" : "+v"(w[f * 4]), "+v"(w[f * 4 + 1]), "+v"(w[f * 4 + 2]), "+v"(w[f * 4 + 3]), "+v"(T2[f]), "+v"(T3[f]));
            emit_escapes(g, h, f * 512 + lane * 8u, scr + 4, 384u, e);
        }
        const u32 inc = wave_scan(tot), all2 = __builtin_amdgcn_readlane(inc, 63);
        size16 = (6656u + 4u + e * 4u + ((pk_sum(all2) + 7u) >> 3) + 15u) >> 4;
        if (size16 >= 512u) size16 = 512u;
        u32 off16 = 0;
        if (half == 0) { if (lane == 0) lds_st(lpair, size16); }
        else {
            const u64 t0 = now(); u32 sp = 0;
            while ((off16 = lds_ldu(lpair)) == 0u) {
                if ((++sp & 1023u) == 0u && now() - t0 > 4 * DEADLINE) { st4_sys(ws + OFF_TMO, call); break; }
                __builtin_amdgcn_s_sleep(1);
            }
        }
        char* d = ws + (call & 1u) * X_PAR + off16 * 16u + lane * 16 +
                  (isA ? OFF_AX + (size_t)((idx >> 1) * 8 + j) * 16384 : OFF_BX + (size_t)(((idx >> 2) * 8 + j) * 2 + ((idx >> 1) & 1)) * 16384);
        if (size16 == 512u) {
#pragma unroll
            for (int f = 0; f < 8; ++f) st16_sys(d + f * 1024, p[f]);
        } else {
            u32 bp = (e + 1u) * 32u * 0x10001u + (all2 << 16) + inc - tot;
#pragma unroll
            for (int f = 0; f < 8; f += 2) {
                or_bits(scr, bp & 0xffffu, T2[f] | (T2[f + 1] << (len[f] & 31u)));
                or_bits(scr, bp >> 16, T3[f]);
                bp += len[f];
                or_bits(scr, bp >> 16, T3[f + 1]);
                bp += len[f + 1];
            }
            if (lane == 0) *(LAS u32*)scr = e;
#pragma unroll
            for (int i = 0; i < 24; ++i) w[i] |= (w[26 + (i >> 2)] << (12 - 3 * (i & 3))) & 0x70007000u;
#pragma unroll
            for (int i = 0; i < 6; ++i) w[24 + i / 3] |= (w[26 + i] >> (1 + i % 3)) & (0x40004000u >> (i % 3));
#pragma unroll
            for (int f = 0; f < 6; ++f) st16_sys(d + f * 1024, (u32x4){w[f * 4], w[f * 4 + 1], w[f * 4 + 2], w[f * 4 + 3]});
            st8_sys(d + 6144 - lane * 8, (u32x2){w[24], w[25]});
            CFENCE();
            for (u32 g = lane; g + 416u < size16; g += 64u) st16_sys(d + 6656 + (g - lane) * 16u, *(LAS u32x4*)(scr + g * 16u));
        }
    }
    if (local) {
        char* d = ws + OFF_A + (size_t)dest * 8192 + lane * 16;
#pragma unroll
        for (int f = 0; f < 8; ++f) st16_sys(d + f * 1024, p[f]);
    }
    VMCNT(0);
    if (local && lane == 0) st8_sys(ws + OFF_FA + (size_t)dest * 8, (u32x2){call, code});
    if (exprt) {
        const int hb = (lp % 12) * 2 + ((idx >> 1) & 1);
        const int item = isA ? j * 56 + (idx >> 1) : (lp < 12 ? j * 56 + 32 + hb : 256 + (lp / 12) * 192 + j * 24 + hb);
        const u32 info = size16 | (bb << 10) | (code << 25);
        const size_t ro = OFF_REC + (call & 1u) * REC_PAR + P.rank * REC_SRC;
        if (q < 112) {
            if (lane < NR && lane != P.rank && (all || lane == owner)) st8_sys(peer_ws(P, lane) + ro + item * 16 + half * 8, (u32x2){call, info});
        } else {
            u32 old = 0;
            if (lane == 0) asm volatile("ds_write_b32 %1, %2\n\tds_add_rtn_u32 %0, %3, %4\n\ts_waitcnt lgkmcnt(0)"
                                        : "=&v"(old) : "v"(lquad + ((item & 3) * 2 + half) * 4), "v"(info), "v"(lquad + 32), "v"(1u) : "memory");
            if (uni(old) == 7u) {
                u32x2 in2; asm volatile("ds_read_b64 %0, %1\n\ts_waitcnt lgkmcnt(0)" : "=&v"(in2) : "v"(lquad + (lane & 3) * 8) : "memory");
                const int r = lane >> 2;
                if (lane < 32 && r != P.rank && (all || r == owner)) st16_sys(peer_ws(P, r) + ro + ((item & ~3) + (lane & 3)) * 16, (u32x4){call, in2.x, call, in2.y});
            }
        }
    }
    CFENCE();
}

__device__ __forceinline__ void scout(const Params& P, int src, int li, int lane, u32 call) {
    char* ws = P.me;
    u32* tmo = (u32*)(ws + OFF_TMO);
    u32* tick = (u32*)(ws + OFF_TICK + src * 256);
    const char* rec = ws + OFF_REC + (call & 1u) * REC_PAR + src * REC_SRC;
    const u32 xpar = (call & 1u) * (u32)X_PAR;
    const u32 tbase = (call - 1u) * (u32)(N_ITEMS - ITEM0) - (u32)(LPP + ITEM0);
    u32 item = 32 + li,
        pieces = 0, nq = 0, k = 0;
    __builtin_amdgcn_s_setprio(2);
    while (item < (u32)N_ITEMS) {
        const bool isin = item < (u32)N_IN_ITEMS;
        u32x2 r; bool dead = false;
        u32 tk;
        if (lane == 0) asm volatile("global_atomic_add %0, %1, %2, off sc0" : "=v"(tk) : "v"(tick), "v"(1u) : "memory");
        {
            const u64 t0 = now(); u32 sp = 0;
            for (;;) {
                r = ld8_sys(rec + (isin ? item * 16u + (lane & 1) * 8u : REC_OUT + (item - N_IN_ITEMS) * 64u + (lane & 7) * 8u));
                if (__builtin_amdgcn_ballot_w64(r.x >= call) == __builtin_amdgcn_ballot_w64(true)) break;
                if (give_up(sp, t0, tmo, call)) { dead = true; break; }
                __builtin_amdgcn_s_sleep(2);
            }
        }
        asm volatile("" : "+v"(tk));
        u32 next = uni(tk) - tbase;
        if (next < 32u + LPP) next -= LPP;
        u32 soff, bytes, nsub; u32x4 d;
        if (isin) {
            u32 dest;
            u32 j, u;
            if (item < 448u) { j = item / 56u; u = item % 56u; }
            else { const u32 v = item - 448u, w = v % 192u; j = w / 24u; u = 56u + (v / 192u) * 24u + w % 24u; }
            if (u >= 32u) {
                const u32 lp = (u - 32u) >> 1, h = u & 1u, gp = lp < 4u ? 32u + P.rank * 4u + lp : lp - 4u;
                soff = (u32)OFF_BX + xpar + (gp * 8 + j) * 32768u + h * 16384u;
                dest = U_B + ((lp * 8 + j) * 8 + src) * 4 + h * 2;
            } else {
                soff = (u32)OFF_AX + xpar + (u * 8 + j) * 16384u;
                dest = (call & 1u) * 4096u + ((u * 8 + j) * 8 + src) * 2;
            }
            const u32 s = lane & 1, sz = min(r.y & 1023u, 512u), sb = dead ? 0u : sz * 16u;
            const u32 b0 = __builtin_amdgcn_readlane(sb, 0);
            bytes = b0 + __builtin_amdgcn_readlane(sb, 1); nsub = 2;
            d.x = pieces * 1024u + (s ? b0 : 0u); d.y = pieces + (((s ? bytes : b0) + 1023u) >> 10);
            d.z = dead ? (62u << 12) | (2u << 24) : sz | ((r.y >> 25) << 12); d.w = ((r.y >> 10) & 0x7fffu) | ((dest + s) << 16);
        } else {
            const u32 o = item - N_IN_ITEMS;
            soff = (u32)OFF_O + o * (u32)QSLOT;
            const u32 a = __builtin_amdgcn_ds_bpermute((lane & 3) * 8, r.y), b = __builtin_amdgcn_ds_bpermute((lane & 3) * 8 + 4, r.y);
            const u32 sb = dead ? 0u : min(a & 1023u, 512u) * 16u, n = min((a >> 10) & 0xfffu, 2048u), e = b & 0xfffu, b16 = (b >> 12) & 0x7fffu;
            const u32 s0 = __builtin_amdgcn_readlane(sb, 0), s1 = s0 + __builtin_amdgcn_readlane(sb, 1), s2 = s1 + __builtin_amdgcn_readlane(sb, 2);
            const u32 off = (lane & 3) == 0 ? 0u : (lane & 3) == 1 ? s0 : (lane & 3) == 2 ? s1 : s2;
            bytes = s2 + __builtin_amdgcn_readlane(sb, 3); nsub = 4;
            d.x = pieces * 1024u + off; d.y = pieces + ((off + sb + 1023u) >> 10); d.z = n | (e << 12) | ((dead ? 3u : sb == 8192u ? 4u : 1u) << 24);
            d.w = b16 | ((o * 4u + (lane & 3)) << 16);
        }
        const u32 np = (bytes + 1023u) >> 10;
        {
            u64 t0 = 0; u32 sp = 0;
            for (;;) {
                u32 nx = lds_ld(L_FCTL + (lane < 6 ? 32 + lane * 4 : 12));
                u32 qm = __builtin_amdgcn_readlane(nx, 0);
#pragma unroll
                for (int i = 1; i < 6; ++i) qm = min(qm, (u32)__builtin_amdgcn_readlane(nx, i));
                const u32 first = qm < nq ? lds_ldu(L_DESC + (qm % NQ) * 16) >> 10 : pieces;
                if ((u32)__builtin_amdgcn_readlane(nx, 6) >= k && pieces + np - first <= RING / 1024 && (qm >= nq || nq + nsub - qm <= NQ)) break;
                if ((++sp & 1023u) == 0u) { if (!t0) t0 = now(); else if (now() - t0 > 4 * DEADLINE) { st4_sys(tmo, call); break; } }
                __builtin_amdgcn_s_sleep(4);
            }
        }
        if (lane < (int)nsub) asm volatile("ds_write_b128 %0, %1" :: "v"(L_DESC + ((nq + lane) % NQ) * 16), "v"(d) : "memory");
        nq += nsub;
        if (lane == 0) {
            const u32x4 it = {soff, bytes, 0u, k + 1};
            asm volatile("ds_write_b128 %0, %1" :: "v"(L_ITEM + (k & 3u) * 16), "v"(it) : "memory");
            lds_st(L_FCTL + 4, nq);
        }
        pieces += np; ++k;
        item = next;
    }
    if (lane == 0) {
        const u32x4 it = {0u, 0xffffffffu, 0u, k + 1};
        asm volatile("ds_write_b128 %0, %1" :: "v"(L_ITEM + (k & 3u) * 16), "v"(it) : "memory");
        lds_st(L_FCTL + 4, nq | 0x80000000u);
    }
}

__device__ __forceinline__ void loader(const Params& P, int src, int lane, u32 call) {
    const u64 base = (u64)P.ws[src];
    const u32 voff = lane * 16, vland = L_LAND + lane * 4;
    u32* tmo = (u32*)(P.me + OFF_TMO);
    int issued = 0, landed = 0;
    u32 k = 0;
    __builtin_amdgcn_s_setprio(3);
#define PIECE(sp) do { asm volatile("s_mov_b32 m0, %2\n\ts_waitcnt vmcnt(%3)\n\tglobal_load_lds_dwordx4 %0, %1 sc0 sc1" \
                                    :: "v"(voff), "s"(sp), "s"(((u32)issued * 1024u) & (RING - 1)), "n"(WIN - 1) : "memory"); } while (0)
    for (;;) {
        u32x4 it = lds_ld16(L_ITEM + (k & 3u) * 16);
        if (uni(it.w) != k + 1 || uni(it.y) == 0xffffffffu) {
            VMCNT(0);
            landed = issued;
            lds_st(vland, (u32)issued);
            const u64 t0 = now(); u32 sp = 0;
            while (uni(it.w) != k + 1) {
                if ((++sp & 1023u) == 0u && now() - t0 > 4 * DEADLINE) { st4_sys(tmo, call); return; }
                __builtin_amdgcn_s_sleep(1);
                it = lds_ld16(L_ITEM + (k & 3u) * 16);
            }
            if (uni(it.y) == 0xffffffffu) return;
        }
        const u32 soff = uni(it.x);
        u32 left = uni(it.y);
        ++k;
        if (lane == 0) lds_st(L_FCTL + 12, k);
        u64 sp = base + soff;
#pragma unroll 1
        for (; left >= 1024u; left -= 1024u, sp += 1024) {
            PIECE(sp);
            landed = max(landed, issued - (WIN - 1));
            lds_st(vland, (u32)landed);
            ++issued;
        }
        if (left) {
            if (voff < left) PIECE(sp);
            landed = max(landed, issued - (WIN - 1));
            lds_st(vland, (u32)landed);
            ++issued;
        }
    }
#undef PIECE
}

template <int G> __device__ __forceinline__ void expand_dense(LAS char* L, LAS char* scr, u32 lane, u32 n8, u32 o_lo, u32 o_mid, u32 o_top, u32 o_sec, u32 b2) {
    u32x4 lo[G]; u32x2 mid[G]; u32 top[G], u[G][4], fl[G][4], ac[G][4], lw[G], t0[G], t1[G], bp[G], tot = 0;
#pragma unroll
    for (int i = 0; i < G; ++i) {
        const u32 g = i * 64 + lane;
        mid[i] = *(LAS u32x2*)(L + ((o_mid + g * 8u) & (RING - 1)));
        top[i] = *(LAS unsigned char*)(L + ((o_top + g) & (RING - 1)));
    }
#pragma unroll
    for (int i = 0; i < G; ++i) lo[i] = *(LAS u32x4*)(L + ((o_lo + (i * 64 + lane) * 16u) & (RING - 1)));
#pragma unroll
    for (int i = 0; i < G; ++i) {
        if (i == G - 1) top[i] = i * 64 + lane < n8 ? top[i] : 0u;
        const u32 tt = (top[i] | (top[i] << 12)) << 8;
#pragma unroll
        for (int k = 0; k < 4; ++k) {
            u[i][k] = __builtin_amdgcn_perm(0u, mid[i][k >> 1], (k & 1) ? 0x0c030c02u : 0x0c010c00u) | ((tt >> k) & 0x01000100u);
            fl[i][k] = PW((i16x2)(PI(u[i][k] + 0x7e807e80u) >> (i16x2){15, 15}));
        }
        lw[i] = pk_offsets<2>(fl[i], ac[i]);
        bp[i] = tot; tot += pk_sum(lw[i]);
    }
    const u32 inc = wave_scan(tot);
#pragma unroll
    for (int i = 0; i < G; ++i) {
        bp[i] += inc - tot;
        const u32 a = o_sec + (bp[i] >> 5) * 4u;
        t0[i] = *(LAS u32*)(L + (a & (RING - 1))); t1[i] = *(LAS u32*)(L + ((a + 4u) & (RING - 1)));
    }
    u32 s2[G][4];
    tot = 0;
#pragma unroll
    for (int i = 0; i < G; ++i) {
        const u32 TT = pk_split(__builtin_amdgcn_alignbit(t1[i], t0[i], bp[i]), lw[i]);
#pragma unroll
        for (int k = 0; k < 4; ++k) { s2[i][k] = pk_field(TT, ac[i][k], fl[i][k], 0x00030003u); fl[i][k] = pk_is3(s2[i][k]); }
        lw[i] = pk_offsets<3>(fl[i], ac[i]);
        bp[i] = tot; tot += pk_sum(lw[i]);
    }
    const u32 base = wave_scan(tot) - tot + __builtin_amdgcn_readlane(inc, 63);
#pragma unroll
    for (int i = 0; i < G; ++i) {
        bp[i] += base;
        const u32 a = o_sec + (bp[i] >> 5) * 4u;
        t0[i] = *(LAS u32*)(L + (a & (RING - 1))); t1[i] = *(LAS u32*)(L + ((a + 4u) & (RING - 1)));
    }
#pragma unroll
    for (int i = 0; i < G; ++i) {
        const u32 TT = pk_split(__builtin_amdgcn_alignbit(t1[i], t0[i], bp[i]), lw[i]);
        u32 r[8];
#pragma unroll
        for (int k = 0; k < 4; ++k) {
            const u32 h = pk_expand(s2[i][k] + pk_field(TT, ac[i][k], fl[i][k], 0x00070007u), 0xff80ff80u, u[i][k] + b2);
            r[2 * k] = __builtin_amdgcn_perm(h, lo[i][k], 0x05040100u); r[2 * k + 1] = __builtin_amdgcn_perm(h, lo[i][k], 0x07060302u);
        }
        *(LAS u32x4*)(scr + (i * 64 + lane) * 32u) = (u32x4){r[0], r[1], r[2], r[3]};
        *(LAS u32x4*)(scr + (i * 64 + lane) * 32u + 16u) = (u32x4){r[4], r[5], r[6], r[7]};
    }
}

__device__ __forceinline__ void decoder(const Params& P, LAS char* L, int src, int dw, int lane, u32 call) {
    char* ws = P.me;
    u32* tmo = (u32*)(ws + OFF_TMO);
    LAS char* scr = L + L_SCR + dw * SCR_B;
    u32 qn = dw, pend = 0;
    auto flush = [&]() { if (pend) { VMCNT(0); if (lane == 0) st8_sys(ws + OFF_FA + (size_t)(pend & 0xffffu) * 8, (u32x2){call, (pend >> 16) & 63u}); pend = 0; } };
    auto release = [&]() { if (lane == 0) asm volatile("ds_add_rtn_u32 %0, %1, %2\n\ts_waitcnt lgkmcnt(0)\n\tds_write_b32 %3, %0"
                                                       : "=&v"(qn) : "v"(L_FCTL + 8), "v"(1u), "v"(L_FCTL + 32 + dw * 4) : "memory"); };
    for (;;) {
        const u32 q = uni(qn);
        u32x4 dv;
        {
            u64 t0 = 0; u32 sp = 0;
            for (;;) {
                u32 nq, ld;
                asm volatile("ds_read_b32 %0, %3\n\tds_read_b128 %1, %4\n\tds_read_b32 %2, %5\n\ts_waitcnt lgkmcnt(0)"
                             : "=&v"(nq), "=&v"(dv), "=&v"(ld) : "v"(L_FCTL + 4), "v"(L_DESC + (q % NQ) * 16), "v"(L_LAND) : "memory");
                nq = uni(nq);
                if ((nq & 0x7fffffffu) > q && uni(ld) >= uni(dv.y)) break;
                flush();
                if ((nq & 0x7fffffffu) <= q && (nq >> 31)) return;
                if ((++sp & 1023u) == 0u) { if (!t0) t0 = now(); else if (now() - t0 > 4 * DEADLINE) { st4_sys(tmo, call); return; } }
                __builtin_amdgcn_s_sleep(1);
            }
        }
        const u32 pos = uni(dv.x), z = uni(dv.z), wv = uni(dv.w);
        const u32 type = z >> 24, dest = wv >> 16;
        if (type == 0u || type == 2u) {
            if (type == 0u) {
                const u32 sz = z & 0xfffu, b2 = (wv & 0xffffu) * 0x10001u, pl = pos + lane * 16u;
                char* d = ws + OFF_A + (size_t)dest * 8192;
                u32x4 c[8];
#pragma unroll
                for (int i = 0; i < 6; ++i) c[i] = *(LAS u32x4*)(L + ((pl + i * 1024u) & (RING - 1)));
                if (sz == 512u) {
                    c[6] = *(LAS u32x4*)(L + ((pl + 6144u) & (RING - 1))); c[7] = *(LAS u32x4*)(L + ((pl + 7168u) & (RING - 1)));
                    LGKM0();
#pragma unroll
                    for (int i = 0; i < 8; ++i) asm volatile("" : "+v"(c[i]));
                    release();
                    flush();
#pragma unroll
                    for (int f = 0; f < 8; ++f) st16_sys(d + f * 1024 + lane * 16, c[f]);
                } else {
                    u32x2 c6 = *(LAS u32x2*)(L + ((pos + 6144u + lane * 8u) & (RING - 1)));
                    for (u32 g = lane; g + 416u < sz; g += 64u) *(LAS u32x4*)(scr + g * 16u) = *(LAS u32x4*)(L + ((pos + 6656u + g * 16u) & (RING - 1)));
                    LGKM0();
#pragma unroll
                    for (int i = 0; i < 6; ++i) asm volatile("" : "+v"(c[i]));
                    asm volatile("" : "+v"(c6));
                    release();
                    const u32 e = min(uni(*(LAS u32*)scr), 384u);
                    u32 w[32], fl[32], ac[32], len[8], lo[8], tot = 0;
#pragma unroll
                    for (int i = 0; i < 24; ++i) w[i] = c[i >> 2][i & 3];
                    w[24] = c6.x; w[25] = c6.y;
#pragma unroll
                    for (int i = 0; i < 6; ++i) {
                        u32 t = (w[24 + i / 3] << (1 + i % 3)) & 0x80008000u;
#pragma unroll
                        for (int k = 0; k < 4; ++k) t |= (w[i * 4 + k] >> (12 - 3 * k)) & (0x00070007u << (3 * k));
                        w[26 + i] = t;
                    }
#pragma unroll
                    for (int f = 0; f < 8; ++f) {
#pragma unroll
                        for (int k = 0; k < 4; ++k) fl[f * 4 + k] = PW((i16x2)(PI((w[f * 4 + k] & 0x0c000c00u) + 0x74007400u) >> (i16x2){15, 15}));
                        lo[f] = pk_offsets<2>(fl + f * 4, ac + f * 4);
                        len[f] = pk_sum(lo[f]); tot += len[f];
                    }
                    const u32 inc = wave_scan(tot);
                    u32 bp = (e + 1u) * 32u + inc - tot;
                    u32 T[8], s2[32];
#pragma unroll
                    for (int f = 0; f < 8; f += 2) {
                        const LAS u32* a = (const LAS u32*)(scr + (bp >> 5) * 4u);
                        T[f] = __builtin_amdgcn_alignbit(a[1], a[0], bp);
                        T[f + 1] = T[f] >> len[f];
                        bp += len[f] + len[f + 1];
                    }
                    tot = 0;
#pragma unroll
                    for (int f = 0; f < 8; ++f) {
                        const u32 TT = pk_split(T[f], lo[f]);
#pragma unroll
                        for (int k = 0; k < 4; ++k) { const int i = f * 4 + k; s2[i] = pk_field(TT, ac[i], fl[i], 0x00030003u); fl[i] = pk_is3(s2[i]); }
                        lo[f] = pk_offsets<3>(fl + f * 4, ac + f * 4);
                        len[f] = pk_sum(lo[f]); tot += len[f];
                    }
                    flush();
                    bp = (e + 1u) * 32u + __builtin_amdgcn_readlane(inc, 63) + wave_scan(tot) - tot;
#pragma unroll
                    for (int f = 0; f < 8; ++f) {
                        const LAS u32* a = (const LAS u32*)(scr + (bp >> 5) * 4u);
                        T[f] = __builtin_amdgcn_alignbit(a[1], a[0], bp);
                        bp += len[f];
                    }
#pragma unroll
                    for (int f = 0; f < 8; ++f) {
                        const u32 TT = pk_split(T[f], lo[f]);
                        u32x4 o;
#pragma unroll
                        for (int k = 0; k < 4; ++k) {
                            const int i = f * 4 + k;
                            o[k] = pk_expand(s2[i] + pk_field(TT, ac[i], fl[i], 0x00070007u), 0xfc00fc00u, PW((u16x2)(PU(w[i] & 0x8fff8fffu) + PU(b2))));
                        }
                        st16_sys(d + f * 1024 + lane * 16, o);
                    }
                    for (u32 g = lane; g < e; g += 64u) {
                        const u32 rr = *(LAS u32*)(scr + 4u + g * 4u);
                        asm volatile("global_store_short_d16_hi %0, %1, off sc0 sc1" :: "v"(d + (rr & 4095u) * 2u), "v"(rr) : "memory");
                    }
                }
            } else { release(); flush(); }
            pend = dest | (((z >> 12) & 63u) << 16) | 0x80000000u;
            continue;
        }
        flush();
        const u32 tile = dest >> 5, wq = (dest >> 2) & 7u, su = dest & 3u;
        const u32 row0 = (tile >> 2) * 256u + (su >> 1) * 128u + (wq >> 2) * 64u + (su & 1u) * 32u + (lane >> 4) * 4u;
        const u32 col0 = 8192u + src * 1024u + (tile & 3u) * 256u + (wq & 3u) * 64u + (lane & 15) * 4u;
        GAS float* o = (GAS float*)P.out + (size_t)row0 * GN + col0;
        if (type == 3u) {
            release();
            const float qn = __builtin_nanf("");
#pragma unroll
            for (int g = 0; g < 8; ++g) *(GAS f32x4*)(o + (size_t)((g >> 2) * 16 + (g & 3)) * GN) = (f32x4){qn, qn, qn, qn};
            continue;
        }
        if (type == 4u) {
            u32x4 c[8];
#pragma unroll
            for (int g = 0; g < 8; ++g) c[g] = *(LAS u32x4*)(L + ((pos + g * 1024u + lane * 16u) & (RING - 1)));
            LGKM0();
#pragma unroll
            for (int g = 0; g < 8; ++g) asm volatile("" : "+v"(c[g]));
            release();
#pragma unroll
            for (int g = 0; g < 8; ++g) st_out(o + (size_t)((g >> 2) * 16 + (g & 3)) * GN, __builtin_bit_cast(f32x4, c[g]));
            continue;
        }
        const u32 n = z & 0xfffu, e = (z >> 12) & 0xfffu, b2 = (wv & 0xffffu) * 0x10001u;
        const u32 nr = (n & 511u) <= 64u ? n & 511u : 0u, n8 = (n - nr + 7u) >> 3, G = (n8 + 63u) >> 6;
        u32 mk = *(LAS u32*)(L + ((pos + lane * 4u) & (RING - 1)));
        const u32 o_lo = pos + 256u, o_mid = o_lo + n8 * 16u, o_top = o_mid + n8 * 8u, o_raw = o_top + ((n8 + 3u) & ~3u);
        const u32 o_esc = o_raw + nr * 4u, o_sec = o_esc + e * 4u;
        if (nr) *(LAS u32*)(scr + n8 * 32u + lane * 4u) = *(LAS u32*)(L + ((o_raw + lane * 4u) & (RING - 1)));
        if (G == 3u) expand_dense<3>(L, scr, lane, n8, o_lo, o_mid, o_top, o_sec, b2);
        else if (G == 2u) expand_dense<2>(L, scr, lane, n8, o_lo, o_mid, o_top, o_sec, b2);
        else if (G == 1u) expand_dense<1>(L, scr, lane, n8, o_lo, o_mid, o_top, o_sec, b2);
        else if (G == 4u) expand_dense<4>(L, scr, lane, n8, o_lo, o_mid, o_top, o_sec, b2);
        CFENCE();
#pragma unroll 1
        for (u32 g = lane; g < e; g += 64u) {
            const u32 rr = *(LAS u32*)(L + ((o_esc + g * 4u) & (RING - 1)));
            *(LAS unsigned short*)(scr + (rr & 2047u) * 4u + 2u) = (unsigned short)(rr >> 16);
        }
        LGKM0();
        asm volatile("" : "+v"(mk));
        release();
        {
            u32 ad = L_SCR + dw * SCR_B + wave_scan(__builtin_popcount(mk)) * 4u;
            u32 r[32], m[32];
#pragma unroll
            for (int i = 31; i >= 0; --i) {
                m[i] = (u32)__builtin_amdgcn_sbfe((int)mk, i, 1);
                asm("v_lshl_add_u32 %0, %1, 2, %2" : "=v"(ad) : "v"(m[i]), "v"(ad));
                r[i] = *(LAS u32*)(size_t)ad;
            }
            __builtin_amdgcn_sched_barrier(0);
#pragma unroll
            for (int g = 7; g >= 0; --g)
                st_out(o + (size_t)((g >> 2) * 16 + (g & 3)) * GN,
                       __builtin_bit_cast(f32x4, (u32x4){r[g * 4] & m[g * 4], r[g * 4 + 1] & m[g * 4 + 1], r[g * 4 + 2] & m[g * 4 + 2], r[g * 4 + 3] & m[g * 4 + 3]}));
        }
        CFENCE();
    }
}

__device__ __forceinline__ void gemm_wg(const Params& P, LAS char* L, int pm, int lp0, int wave, int lane, u32 call) {
    char* ws = P.me;
    const int wr = wave >> 2, wc = wave & 3;
    const u32 voffA = wave * 2048 + lane * 16, voffB = (wave >> 1) * 8192 + (wave & 1) * 2048 + lane * 16;
    const u32 boff = wc * 4096 + lane * 16, ldw = wave * 2048;
    auto lane_id = []() -> u32 { u32 l; asm volatile("v_mbcnt_lo_u32_b32 %0, -1, 0\n\tv_mbcnt_hi_u32_b32 %0, -1, %0" : "=v"(l)); return l; };
    constexpr size_t KA = 16384, KB = 32768, HA = 64 * KA, HB = 4096;
#define SA(b, h) (((b) * 2 + (h)) * 16384)
#define SB(b, h) ((4 + (b) * 2 + (h)) * 16384)
#define STAGE(bufoff, gbase, voff) asm volatile("s_mov_b32 m0, %2\n\ts_nop 0\n\tglobal_load_lds_dwordx4 %0, %1 sc0 sc1\n\tglobal_load_lds_dwordx4 %0, %1 offset:1024 sc0 sc1" \
        :: "v"(voff), "s"((u64)(gbase)), "s"((u32)(bufoff) + ldw) : "memory")
#define LDA(dst, b, h) do { _Pragma("unroll") for (int m = 0; m < 4; ++m) _Pragma("unroll") for (int k = 0; k < 2; ++k) \
        dst[m][k] = *(const LAS f16x8*)(L + SA(b, h) + ((h) ? aoff1 : aoff0) + m * 2048 + k * 1024); } while (0)
#define LDB(dst, b, h) do { _Pragma("unroll") for (int n = 0; n < 2; ++n) _Pragma("unroll") for (int k = 0; k < 2; ++k) \
        dst[n][k] = *(const LAS f16x8*)(L + SB(b, h) + boff + n * 2048 + k * 1024); } while (0)
#define MMA(ai, bj, At, Bt) do { __builtin_amdgcn_sched_barrier(0);         \
        _Pragma("unroll") for (int k = 0; k < 2; ++k) _Pragma("unroll") for (int mm = 0; mm < 4; ++mm) _Pragma("unroll") for (int nn = 0; nn < 2; ++nn) { \
            const int m = k ? 3 - mm : mm, n = (mm & 1) ? 1 - nn : nn; \
            acc[ai][bj][m][n] = __builtin_amdgcn_mfma_f32_16x16x32_f16(At[m][k], Bt[n][k], acc[ai][bj][m][n], 0, 0, 0); \
            __builtin_amdgcn_sched_barrier(0); } } while (0)
#define WAIT_L(n) asm volatile("s_waitcnt lgkmcnt(" #n ")" ::: "memory")
#define WAIT_L0 __builtin_amdgcn_s_waitcnt(0xC07F)
#define RBAR __builtin_amdgcn_s_barrier()
#define SCHED __builtin_amdgcn_sched_barrier(0)
    auto flag_ptr = [&](int lp, int j, int ent) -> const char* {
        return ws + OFF_FA + (size_t)(ent < 32 ? (call & 1u) * 4096u + ((pm * 2 + (ent >> 4)) * 8 + j) * 16 + (ent & 15) : U_B + (lp * 8 + j) * 32 + (ent - 32)) * 8; };
    auto poll = [&](int lp, int j) {
        const u32 ln = lane_id();
        const char* f = flag_ptr(lp, j, ln);
        const u64 t0 = now(); u32 sp = 0; u32x2 v;
        for (;;) {
            v = ld8_sys(f);
            if (__builtin_amdgcn_ballot_w64(v.x >= call) == ~0ull) break;
            if (give_up(sp, t0, (u32*)(ws + OFF_TMO), call)) { v.y = 62u; break; }
            __builtin_amdgcn_s_sleep(4);
        }
        asm volatile("ds_write_b64 %0, %1\n\ts_waitcnt lgkmcnt(0)" :: "v"(L_T2 + ln * 8), "v"(v) : "memory");
    };
    auto codes = [&](u32 tb) -> u32 {
        const u32 r = lane_id() & 7;
        const u32 a0 = *(const LAS u32*)(L + tb + wr * 8 + 4 + r * 16), a1 = *(const LAS u32*)(L + tb + 128 + wr * 8 + 4 + r * 16);
        const u32 b = *(const LAS u32*)(L + tb + 256 + wc * 8 + 4 + r * 32);
        return a0 | (a1 << 8) | (b << 16);
    };
    f32x4 acc[2][2][4][2];
    f16x8 At[4][2], B0[2][2], B1[2][2];
    u32 cur = 0, cmax = 0, bad = 0;
    u32 aoff0 = wr * 8192 + lane * 16, aoff1 = aoff0;
    u32 nm, nmn = 0;
    auto changes = [&](u32 nw, u32 old) -> u32 {
        const u32 prev = __builtin_amdgcn_update_dpp((int)__builtin_amdgcn_readlane(old, 7), (int)nw, 0x111, 0xf, 0xf, false);
        return (u32)__builtin_amdgcn_ballot_w64(nw != prev) & 255u; };
#define ADJUST(r, buf) do { if (__builtin_expect((nm >> (r)) & 1u, 0)) { const u32 cn_ = __builtin_amdgcn_readlane(pk, (r)), cb_ = (cn_ >> 16) & 63u; \
            const u32 l16_ = lane_id() * 16; \
            _Pragma("unroll") for (int ai = 0; ai < 2; ++ai) { \
                const u32 ca_ = (cn_ >> (8 * ai)) & 63u, c_ = ca_ + cb_, o_ = (cur >> (8 * ai)) & 255u, x_ = (cmax >> (8 * ai)) & 255u; \
                u32 ab_ = wr * 8192; int sh_ = 0; \
                if (ca_ == 62u || cb_ == 62u) bad = 1; \
                else if (ca_ != 63u && cb_ != 63u) { \
                    if (c_ + 10u < x_) { ab_ = L_ZERO - SA(buf, ai); nm = 255u; nmn |= 1u; }      \
                    else { sh_ = 8 * ((int)o_ - (int)c_); \
                        cur = (cur & ~(255u << (8 * ai))) | (c_ << (8 * ai)); \
                        if (c_ > x_) cmax = (cmax & ~(255u << (8 * ai))) | (c_ << (8 * ai)); } } \
                asm volatile("v_add_u32 %0, %1, %2" : "+v"(ai ? aoff1 : aoff0) : "s"(ab_), "v"(l16_)); \
                if (sh_ != 0) { \
                    asm volatile("s_nop 15" ::: "memory");        \
                    _Pragma("unroll") for (int bj = 0; bj < 2; ++bj) _Pragma("unroll") for (int m = 0; m < 4; ++m) _Pragma("unroll") for (int n = 0; n < 2; ++n) \
                        _Pragma("unroll") for (int e_ = 0; e_ < 4; ++e_) asm volatile("v_ldexp_f32 %0, %0, %1" : "+v"(acc[ai][bj][m][n][e_]) : "s"(sh_)); } } } } while (0)
#pragma unroll
    for (int a = 0; a < 2; ++a)
#pragma unroll
        for (int b = 0; b < 2; ++b)
#pragma unroll
            for (int m = 0; m < 4; ++m)
#pragma unroll
                for (int n = 0; n < 2; ++n) acc[a][b][m][n] = (f32x4){0.f, 0.f, 0.f, 0.f};
    const char* const cA = ws + OFF_A + (size_t)(call & 1u) * 4096 * 8192 + (size_t)pm * 2 * HA;
    const char* cB = ws + OFF_A + (size_t)U_B * 8192 + (size_t)lp0 * 64 * KB;
    *(LAS u32x4*)(L + L_ZERO + wave * 1024 + lane * 16) = (u32x4){0u, 0u, 0u, 0u};
    if (wave == 0) poll(lp0, 0);
    LGKM0();
    BAR();
    u32 pk = codes(L_T2), pkn = pk;
    nm = changes(pk, pk) | 1u;
    STAGE(SB(0, 0), cB, voffB); STAGE(SA(0, 0), cA, voffA); STAGE(SB(0, 1), cB + HB, voffB); STAGE(SA(0, 1), cA + HA, voffA);
    if (wr == 1) RBAR;
    VMCNT(4); RBAR;
    STAGE(SB(1, 0), cB + KB, voffB); STAGE(SA(1, 0), cA + KA, voffA); STAGE(SB(1, 1), cB + HB + KB, voffB);
    VMCNT(6); RBAR;
#pragma unroll 1
    for (int rd = 0; rd < 3; ++rd) {
        const int lp = lp0 + rd * 12;
        const char* nB = rd < 2 ? cB + 12 * 64 * KB : cB;
#define TRIP(r, a1, a2, b2, HOOK) do { \
            ADJUST(r, 0); \
            LDB(B0, 0, 0); SCHED; LDA(At, 0, 0); STAGE(SA(1, 1), (a1) + HA, voffA); \
            WAIT_L(8); RBAR; WAIT_L0; MMA(0, 0, At, B0); RBAR; SCHED; \
            LDB(B1, 0, 1); STAGE(SB(0, 0), (b2), voffB); \
            RBAR; WAIT_L0; MMA(0, 1, At, B1); RBAR; \
            LDA(At, 0, 1); STAGE(SA(0, 0), (a2), voffA); \
            RBAR; WAIT_L0; MMA(1, 0, At, B0); RBAR; SCHED; \
            STAGE(SB(0, 1), (b2) + HB, voffB); \
            VMCNT(6); RBAR; MMA(1, 1, At, B1); RBAR; \
            ADJUST((r) + 1, 1); \
            LDB(B0, 1, 0); SCHED; LDA(At, 1, 0); STAGE(SA(0, 1), (a2) + HA, voffA); \
            WAIT_L(8); RBAR; WAIT_L0; MMA(0, 0, At, B0); RBAR; SCHED; \
            LDB(B1, 1, 1); STAGE(SB(1, 0), (b2) + KB, voffB); \
            RBAR; WAIT_L0; MMA(0, 1, At, B1); RBAR; \
            LDA(At, 1, 1); STAGE(SA(1, 0), (a2) + KA, voffA); \
            RBAR; WAIT_L0; MMA(1, 0, At, B0); RBAR; SCHED; \
            STAGE(SB(1, 1), (b2) + KB + HB, voffB); \
            HOOK; \
            VMCNT(6); RBAR; MMA(1, 1, At, B1); RBAR; } while (0)
#pragma unroll 1
        for (int j = 0; j < 8; ++j) {
            const char* ga = cA + (size_t)j * 8 * KA; const char* gb = cB + (size_t)j * 8 * KB;
            const char* na = j == 7 ? cA : ga + 8 * KA; const char* nb = j == 7 ? nB : gb + 8 * KB;
            const bool more = j < 7 || rd < 2;
            const int nlp = j < 7 ? lp : lp + 12, nj = (j + 1) & 7;
            TRIP(0, ga + KA, ga + 2 * KA, gb + 2 * KB, do { } while (0));
            TRIP(2, ga + 3 * KA, ga + 4 * KA, gb + 4 * KB,
                 if (__builtin_expect(wave == 0 && more, 0)) { const u32 ln = lane_id(); const char* fp = flag_ptr(nlp, nj, ln * 2);
                     if (ln < 32) asm volatile("s_mov_b32 m0, %1\n\ts_nop 0\n\tglobal_load_lds_dwordx4 %0, off sc0 sc1" :: "v"(fp), "s"(L_T1) : "memory"); });
            TRIP(4, ga + 5 * KA, ga + 6 * KA, gb + 6 * KB, {
                 const u32 tag = *(const LAS u32*)(L + L_T1 + lane_id() * 8);
                 pkn = codes(L_T1);
                 if (__builtin_expect((__builtin_amdgcn_ballot_w64(tag >= call) != ~0ull) & more, 0)) {
                     if (wave == 0) poll(nlp, nj);
                     BAR();
                     pkn = codes(L_T2);
                 }
                 nmn |= changes(pkn, pk); });
            TRIP(6, ga + 7 * KA, na, nb, { pk = pkn; nm = nmn; nmn = 0; });
        }
        if (wave == 0) { STAMP(4 + rd * 2); STAMPV(50 + rd, __builtin_amdgcn_s_memtime()); }
        {
            const int ln = lane_id();
            const int gp = lp < 4 ? 32 + P.rank * 4 + lp : lp - 4;
            GAS float* o = (GAS float*)P.out + (size_t)(pm * 256 + wr * 64 + (ln >> 4) * 4) * GN + gp * 256 + wc * 64 + (ln & 15) * 4;
#pragma unroll
            for (int ai = 0; ai < 2; ++ai) {
                const int sh = 8 * (int)((cur >> (8 * ai)) & 255u) - 272;
#pragma unroll
                for (int m = 0; m < 4; ++m)
#pragma unroll
                    for (int r = 0; r < 4; ++r) {
                        f32x4 v;
#pragma unroll
                        for (int i = 0; i < 4; ++i) {
                            const float y = __builtin_elementwise_maximum(__builtin_ldexpf(acc[ai][i >> 1][m][i & 1][r], sh), 0.0f);
                            v[i] = __uint_as_float(bad ? 0x7fc00000u : __float_as_uint(y) & 0x7fffffffu);
                            acc[ai][i >> 1][m][i & 1][r] = 0.f;
                        }
                        st_out(o + (size_t)(ai * 128 + m * 16 + r) * GN, v);
                    }
            }
            if (lp < 4) {
                VMCNT(0);
                if (ln == 0) st4_sys(ws + OFF_OF + (size_t)((pm * 4 + lp) * 8 + wave) * 4, call);
            }
            cur = 0; cmax = 0; bad = 0; nm |= 1u;
        }
        if (wave == 0) STAMP(5 + rd * 2);
        cB = nB;
    }
    VMCNT(0);
    if (wr == 0) RBAR;
#undef SA
#undef SB
#undef STAGE
#undef LDA
#undef LDB
#undef MMA
#undef ADJUST
#undef TRIP
}

template <int G> __device__ __forceinline__ void pack_dense(LAS char* dense, LAS char* img, u32 lane, u32 n8, u32 o_mid, u32 o_top, u32 o_esc, u32 b2,
                                                            u32* T, u32* T3, u32* bp, u32& tot, u32& e) {
    u32x4 a[G], b[G], lo[G]; u32x2 mid[G]; u32 hi[G][4], ge[G][4], top[G];
#pragma unroll
    for (int i = 0; i < G; ++i) { a[i] = *(LAS u32x4*)(dense + (i * 64 + lane) * 32u); b[i] = *(LAS u32x4*)(dense + (i * 64 + lane) * 32u + 16u); }
#pragma unroll
    for (int i = 0; i < G; ++i) {
        const u32 dd[8] = {a[i].x, a[i].y, a[i].z, a[i].w, b[i].x, b[i].y, b[i].z, b[i].w};
        u32 x[4], c[4], f9[4], len, tp = 0;
#pragma unroll
        for (int k = 0; k < 4; ++k) {
            lo[i][k] = __builtin_amdgcn_perm(dd[2 * k + 1], dd[2 * k], 0x05040100u);
            hi[i][k] = __builtin_amdgcn_perm(dd[2 * k + 1], dd[2 * k], 0x07060302u);
            x[k] = PW((u16x2)(PU(hi[i][k]) - PU(b2)));
            if (i == G - 1) x[k] = i * 64 + lane < n8 ? x[k] : 0u;
        }
        pk_classify<7>(x, c, ge[i], T[i], T3[i], len);
#pragma unroll
        for (int k = 0; k < 4; ++k) { f9[k] = (c[k] << 7) | (x[k] & 0x007f007fu); tp |= ((c[k] >> 1) & 0x00010001u) << k; }
        mid[i] = (u32x2){__builtin_amdgcn_perm(f9[1], f9[0], 0x06040200u), __builtin_amdgcn_perm(f9[3], f9[2], 0x06040200u)};
        top[i] = tp | (tp >> 12);
        bp[i] = tot; tot += len;
    }
#pragma unroll
    for (int i = 0; i < G; ++i) {
        const u32 g = i * 64 + lane;
        if (i < G - 1 || g < n8) {
            *(LAS u32x4*)(img + 256u + g * 16u) = lo[i];
            *(LAS u32x2*)(img + o_mid + g * 8u) = mid[i];
            *(LAS unsigned char*)(img + o_top + g) = (unsigned char)top[i];
        }
    }
#pragma unroll
    for (int i = 0; i < G; ++i) emit_escapes(ge[i], hi[i], (i * 64 + lane) * 8u, img + o_esc, (8192u - o_esc) >> 2, e);
}
__device__ __forceinline__ void packer(const Params& P, LAS char* L, int pw, int wave, int lane, u32 call) {
    char* ws = P.me;
    LAS char* dense = L + wave * PK_B;
    LAS char* img = dense + PK_IMG;
    const auto rs = __builtin_amdgcn_make_buffer_rsrc((void*)P.out, 0, (u32)GM * GN * 4u, 0x00020000);
    auto wait_item = [&](int it) -> u32 {
        const u64 t0 = now(); u32 sp = 0;
        while (uni(ld4_sys(ws + OFF_OF + it * 4)) < call) {
            if (give_up(sp, t0, (u32*)(ws + OFF_TMO), call)) return 0x7fc00000u;
            __builtin_amdgcn_s_sleep(8);
        }
        return 0u;
    };
    u32x4 nx[8];
    auto fetch_block = [&](int it, int su) {
        const int tile = it >> 3, wq = it & 7;
        const u32 so = ((u32)((tile >> 2) * 256 + (su >> 1) * 128 + (wq >> 2) * 64 + (su & 1) * 32) * GN + 8192 + P.rank * 1024 + (tile & 3) * 256 + (wq & 3) * 64) * 4u;
#pragma unroll
        for (int g = 0; g < 8; ++g) nx[g] = __builtin_amdgcn_raw_buffer_load_b128(rs, ((lane >> 4) * 4 * GN + (lane & 15) * 4) * 4, so + ((g >> 2) * 16 + (g & 3)) * GN * 4, 17);
    };
    auto publish = [&](int it, u32 ea, u32 eb) {
        if (lane < 32 && (lane >> 2) != P.rank)
            st16_sys(peer_ws(P, lane >> 2) + OFF_REC + (call & 1u) * REC_PAR + P.rank * REC_SRC + REC_OUT + it * 64 + (lane & 3) * 16, (u32x4){call, ea, call, eb});
    };
    {
        const char* bx = ws + OFF_BX + (call & 1u) * X_PAR + (size_t)8 * 8 * 32768 + lane * 16;
        u32 acc = 0;
#pragma unroll 1
        for (int i = 0; i < 6; ++i) {
            const int sl = pw * 3 + (i >> 1);
            u32x4 t[7];
#pragma unroll
            for (int g = 0; g < 7; ++g) t[g] = *(const GAS u32x4*)(bx + (size_t)sl * 16384 + ((i & 1) * 7 + g) * 1024);
#pragma unroll
            for (int g = 0; g < 7; ++g) acc ^= t[g].x;
        }
        asm volatile("" :: "v"(acc));
    }
    u32 nanm = wait_item(pw), nann = 0, off16 = 0, ea = 0, eb = 0;
    fetch_block(pw, 0);
#pragma unroll
    for (int g = 0; g < 8; ++g) asm volatile("" : "+v"(nx[g]));
#pragma unroll 1
    for (int bk = 0; bk < N_OUT_ITEMS / 16; ++bk) {
        {
            const int it = pw + (bk >> 2) * 64, su = bk & 3;
            char* og = ws + OFF_O + (size_t)it * QSLOT;
            u32 v[32]; u32 mx = 0;
#pragma unroll
            for (int r = 0; r < 32; ++r) { v[r] = nx[r >> 2][r & 3] | nanm; mx = max(mx, v[r]); }
            { const int bn = bk + 1 < N_OUT_ITEMS / 16 ? bk + 1 : bk; fetch_block(pw + (bn >> 2) * 64, bn & 3); }
            mx = wave_umax(mx) >> 16;
            const u32 b16 = mx >= 511u ? mx - 511u : 0u, b2 = b16 * 0x10001u;
            u32 f[32], mk0 = 0, mk1 = 0;
#pragma unroll
            for (int r = 0; r < 32; ++r) asm("v_min_u32 %0, 1, %1" : "=v"(f[r]) : "v"(v[r]));
#pragma unroll
            for (int r = 0; r < 16; ++r) asm("v_lshl_or_b32 %0, %2, %4, %0\n\tv_lshl_or_b32 %1, %3, %5, %1" : "+v"(mk0), "+v"(mk1) : "v"(f[r]), "v"(f[16 + r]), "n"(r), "n"(16 + r));
            const u32 mk = mk0 | mk1, pc = __builtin_popcount(mk), inc = wave_scan(pc), n = __builtin_amdgcn_readlane(inc, 63);
            {
                u32 ad = wave * PK_B + (inc - pc) * 4u;
#pragma unroll
                for (int r = 0; r < 32; r += 8) {
                    u32 a[8]; u64 m[8];
#pragma unroll
                    for (int i = 0; i < 8; ++i) { a[i] = ad; asm("v_lshl_add_u32 %0, %1, 2, %2" : "=v"(ad) : "v"(f[r + i]), "v"(a[i])); m[i] = __builtin_amdgcn_ballot_w64(v[r + i] != 0u); }
                    asm volatile("s_mov_b64 exec, %16\n\tds_write_b32 %0, %8\n\ts_mov_b64 exec, %17\n\tds_write_b32 %1, %9\n\t"
                                 "s_mov_b64 exec, %18\n\tds_write_b32 %2, %10\n\ts_mov_b64 exec, %19\n\tds_write_b32 %3, %11\n\t"
                                 "s_mov_b64 exec, %20\n\tds_write_b32 %4, %12\n\ts_mov_b64 exec, %21\n\tds_write_b32 %5, %13\n\t"
                                 "s_mov_b64 exec, %22\n\tds_write_b32 %6, %14\n\ts_mov_b64 exec, %23\n\tds_write_b32 %7, %15\n\ts_mov_b64 exec, -1"
                                 :: "v"(a[0]), "v"(a[1]), "v"(a[2]), "v"(a[3]), "v"(a[4]), "v"(a[5]), "v"(a[6]), "v"(a[7]),
                                    "v"(v[r]), "v"(v[r + 1]), "v"(v[r + 2]), "v"(v[r + 3]), "v"(v[r + 4]), "v"(v[r + 5]), "v"(v[r + 6]), "v"(v[r + 7]),
                                    "s"(m[0]), "s"(m[1]), "s"(m[2]), "s"(m[3]), "s"(m[4]), "s"(m[5]), "s"(m[6]), "s"(m[7]) : "memory");
                }
            }
            if (lane < 8) *(LAS u32*)(dense + (n + lane) * 4u) = b16 << 16;
            *(LAS u32*)(img + lane * 4) = mk;
            const u32 nr = (n & 511u) <= 64u ? n & 511u : 0u, n8 = (n - nr + 7u) >> 3, G = (n8 + 63u) >> 6;
            const u32 o_mid = 256u + n8 * 16u, o_top = o_mid + n8 * 8u, o_raw = o_top + ((n8 + 3u) & ~3u), o_esc = o_raw + nr * 4u;
            CFENCE();
            if (nr) *(LAS u32*)(img + o_raw + lane * 4u) = *(LAS u32*)(dense + n8 * 32u + lane * 4u);
            CFENCE();
            u32 e = 0, tot = 0, T[4] = {0u, 0u, 0u, 0u}, T3[4] = {0u, 0u, 0u, 0u}, bp[4] = {0u, 0u, 0u, 0u};
            if (G == 3u) pack_dense<3>(dense, img, lane, n8, o_mid, o_top, o_esc, b2, T, T3, bp, tot, e);
            else if (G == 2u) pack_dense<2>(dense, img, lane, n8, o_mid, o_top, o_esc, b2, T, T3, bp, tot, e);
            else if (G == 1u) pack_dense<1>(dense, img, lane, n8, o_mid, o_top, o_esc, b2, T, T3, bp, tot, e);
            else if (G == 4u) pack_dense<4>(dense, img, lane, n8, o_mid, o_top, o_esc, b2, T, T3, bp, tot, e);
#pragma unroll
            for (int g = 0; g < 8; ++g) asm volatile("" : "+v"(nx[g]));
            VMCNT(0);
            if (su == 0) { if (bk) publish(it - 64, ea, eb); off16 = 0; }
            if (su == 2 && it + 64 < N_OUT_ITEMS) nann = wait_item(it + 64);
            const u32 binc = wave_scan(tot), all2 = __builtin_amdgcn_readlane(binc, 63), o_sec = o_esc + e * 4u;
            u32 size16 = (o_sec + ((pk_sum(all2) + 7u) >> 3) + 15u) >> 4;
            if (size16 >= 512u) {
                size16 = 512u;
#pragma unroll
                for (int g = 0; g < 8; ++g) st16_sys(og + off16 * 16u + g * 1024 + lane * 16, (u32x4){v[g * 4], v[g * 4 + 1], v[g * 4 + 2], v[g * 4 + 3]});
            } else {
#pragma unroll
                for (int i = 0; i < 6; ++i) *(LAS u32*)(img + o_sec + i * 256 + lane * 4) = 0u;
                CFENCE();
#pragma unroll
                for (int i = 0; i < 4; ++i) if (i < (int)G) {
                    const u32 bq = bp[i] + binc - tot + (all2 << 16);
                    or_bits(img + o_sec, bq & 0xffffu, T[i]);
                    or_bits(img + o_sec, bq >> 16, T3[i]);
                }
                CFENCE();
#pragma unroll 1
                for (u32 g = lane; g < size16 + lane; g += 256u) {
                    u32x4 t[4];
#pragma unroll
                    for (int i = 0; i < 4; ++i) t[i] = *(LAS u32x4*)(img + (g + i * 64) * 16u);
#pragma unroll
                    for (int i = 0; i < 4; ++i) if (g + i * 64 < size16) st16_sys(og + (off16 + g + i * 64) * 16u, t[i]);
                }
            }
            if ((lane & 3) == su) { ea = size16 | (n << 10); eb = (e & 0xfffu) | (b16 << 12); }
            off16 += size16;
            if (su == 3) nanm = nann;
            LGKM0();
            CFENCE();
        }
        if (bk == 4) STAMP(16);
    }
    VMCNT(0);
    publish(pw + N_OUT_ITEMS - 64, ea, eb);
    STAMP(17);
}

__global__ __launch_bounds__(NTHR, 2) __attribute__((amdgpu_waves_per_eu(2, 2))) void fused_kernel(Params P) {
    extern __shared__ __attribute__((aligned(16))) char smem[];
    LAS char* L = (LAS char*)smem;
    asm volatile("" :: "s"(P.x), "s"(P.w), "s"(P.me), "s"(P.rank));
    const int tid = threadIdx.x, lane = tid & 63, wave = __builtin_amdgcn_readfirstlane(tid >> 6);
    const int b = blockIdx.x, xg = b & 7, s = b >> 3;
    const bool fetch = s < FS;
    constexpr int NF = FS * 8;
    if (wave == 0) STAMP(0);
    GAS u32* const cnt = (GAS u32*)(P.me + OFF_CNT + (b * 8 + wave) * 8);
    constexpr bool diag = false;
    const int pidx = b - (NF + 120);
    constexpr int NU = 112 - ITEM0;
    const int q0 = ITEM0 + (b - NF);
    const bool loaded = !fetch && !diag && q0 < 112 && wave < 2;
    f32x4 v[16];
    if (wave == 7 && (unsigned)pidx < (unsigned)NPUSH && !diag) {
        load_x(P, pidx, 0, lane, v);
        const u32 craw = __hip_atomic_load(cnt, __ATOMIC_RELAXED, __HIP_MEMORY_SCOPE_AGENT);
        __builtin_amdgcn_sched_barrier(0);
        BAR();
        __builtin_amdgcn_sched_barrier(0);
        STAMP(22);
        push_task(P, pidx, lane, craw, v);
        STAMP(23);
    } else {
        if (loaded) { if (q0 < 32) load_x(P, q0 * 2 + (wave & 1), 0, lane, v); else load_w(P, (q0 < 96 ? q0 + 32 : q0 - 96) * 2 + (wave & 1), 0, lane, v); }
        __builtin_amdgcn_sched_barrier(0);
        if (tid < 256) lds_st(L_ITEM + tid * 4, !fetch ? 0u : tid == 18 ? 6u : tid >= 24 && tid < 32 ? (u32)(tid - 24) : 0u);
        LGKM0();
        BAR();
    }
    if (!fetch && !diag) {
        const u64 tstart = now();
        const bool pack = s >= FS + GS;
        auto fetch_task = [&](int q) {
            const u64 go = pack ? (u64)PACE4 : (u64)(PACE2 + (q - 112) * PACE3 / 112);
            while (now() - tstart < go) __builtin_amdgcn_s_sleep(4);
            int j, idx; bool isA;
            task_of(q, wave & 1, j, idx, isA);
            if (isA) load_x(P, idx, j, lane, v); else load_w(P, idx, j, lane, v);
        };
        int qn = 112 + (pack ? GS * 8 + xg : (b - NF + GS * 8 - NU) % (GS * 8)) * 4 + (wave >> 1), q = q0;
        const int qs = pack ? 32 : N_PAIRS;
        u32 lq = L_ITEM;
        if (!loaded) { q = qn; qn += qs; lq += 64; fetch_task(q); }
#pragma unroll 1
        for (;;) {
            stage_task(P, L + wave * 4096, lq, lq + 48 + (wave >> 1) * 4, q, wave & 1, lane, cnt, v);
            if (qn >= N_PAIRS) break;
            q = qn; qn += qs; lq += 64;
            fetch_task(q);
        }
    }
    const u32 call = uni(__hip_atomic_load(cnt, __ATOMIC_RELAXED, __HIP_MEMORY_SCOPE_AGENT)) + (diag ? 0u : 1u);
    if (lane == 0) st4_sys((void*)cnt, call);
    STAMP(fetch ? 1 : (s >= FS + GS ? 19 : 18));
    if (fetch) {
        const int src = (P.rank + 1 + s) & 7;
        if (wave == 0) { loader(P, src, lane, call); STAMP(2); }
        else if (wave == 1) scout(P, src, xg, lane, call);
        else { decoder(P, L, src, wave - 2, lane, call); STAMP(8 + wave); }
    } else if (s < FS + GS) {
        const int g = s - FS;
        gemm_wg(P, L, (xg >> 1) * 4 + g / 6, (xg & 1) * 6 + g % 6, wave, lane, call);
    } else {
        BAR();
        packer(P, L, wave * 8 + xg, wave, lane, call);
    }
}

}

extern "C" void kernel_launch(int rank, int nranks, ncclComm_t comm, hipStream_t stream,
                              void* const* d_in, const int* in_sizes, int n_in,
                              void* d_out, int out_size,
                              const int* in_dtypes, int out_dtype,
                              void* const* d_out_all,
                              void* const* d_ws_all, size_t ws_bytes) {
    (void)comm; (void)in_sizes; (void)n_in; (void)out_size; (void)in_dtypes; (void)out_dtype; (void)d_out_all;
    if (nranks != NR || (unsigned)rank >= (unsigned)NR || ws_bytes < WS_NEED) { fprintf(stderr, "unsupported configuration\n"); return; }
    static bool ready[NR];
    if (!ready[rank]) {
        HIP_CHECK(hipFuncSetAttribute((const void*)fused_kernel, hipFuncAttributeMaxDynamicSharedMemorySize, (int)LDS_BYTES));
        ready[rank] = true;
    }
    Params P;
    P.x = (const float*)d_in[0]; P.w = (const float*)d_in[1]; P.out = (float*)d_out;
    for (int i = 0; i < NR; ++i) P.ws[i] = (char*)d_ws_all[i];
    P.me = P.ws[rank]; P.rank = rank; P.pad = 0;
    hipLaunchKernelGGL(fused_kernel, dim3(NWG), dim3(NTHR), LDS_BYTES, stream, P);
    HIP_CHECK(hipGetLastError());
}
```

```cpp
#include <hip/hip_runtime.h>
#include <rccl/rccl.h>
#include <cstdio>

#define HIP_CHECK(expr) do { hipError_t _e = (expr); if (_e != hipSuccess) { \
    fprintf(stderr, "HIP error %s at %s:%d\n", hipGetErrorString(_e), __FILE__, __LINE__); \
    return; } } while (0)

typedef unsigned u32;
typedef unsigned long long u64;
typedef u32 u32x2 __attribute__((ext_vector_type(2)));
typedef u32 u32x4 __attribute__((ext_vector_type(4)));
typedef float f32x4 __attribute__((ext_vector_type(4)));
typedef _Float16 f16x8 __attribute__((ext_vector_type(8)));
#define LAS __attribute__((address_space(3)))
#define GAS __attribute__((address_space(1)))

namespace {

constexpr int NR = 8, GM = 4096, GN = 16384, KL = 512;
constexpr int NWG = 256, NTHR = 512;
#define WINV 8
#define NPUSH 36
constexpr int FS = 7, GS = 24;
constexpr int LPP = 8;
constexpr int WIN = WINV;
constexpr int NLP = 36;
constexpr int N_IN_ITEMS = 832, N_OUT_ITEMS = 512, N_ITEMS = N_IN_ITEMS + N_OUT_ITEMS;
constexpr u32 R2_ITEM = 688, ROT = 20;
#define AHEAD 1
#define SSL 4
#define HELPM 7
#define HELP_LAG 5
#define HELP_STEP 0
constexpr int N_PAIRS = 1280, ITEM0 = NPUSH / 2;
#define PACE2 150
#define PACE3 350
#define PACE4 800
static_assert(NPUSH % 2 == 0 && NPUSH >= 8 && NPUSH <= 64, "only x blocks of the first K group");
constexpr u64 DEADLINE = 30000000ull;

constexpr size_t OFF_CNT = 0;
constexpr size_t OFF_TMO = 16384;
constexpr size_t OFF_TICK = 16384 + 256;
constexpr size_t OFF_OF = 20480;
constexpr u32 U_B = 4096, U_PAR = U_B + NLP * 256, N_UNITS = 2 * U_PAR;
constexpr size_t OFF_FA = 32768;
constexpr size_t OFF_DBG = 262144;
constexpr size_t OFF_REC = 1ull << 20;
constexpr u32 REC_SRC = 65536, REC_OUT = 16384;
constexpr size_t REC_PAR = 1ull << 20;
constexpr size_t OFF_A = 4ull << 20;
constexpr size_t OFF_AX = OFF_A + (size_t)N_UNITS * 8192;
constexpr size_t OFF_BX = OFF_AX + (4ull << 20);
constexpr size_t X_PAR = 20ull << 20;
constexpr size_t OFF_O = OFF_AX + 2 * X_PAR;
constexpr size_t QSLOT = 32768;
constexpr size_t WS_NEED = OFF_O + N_OUT_ITEMS * QSLOT;
static_assert(NR * REC_SRC <= REC_PAR && N_IN_ITEMS * 16 <= REC_OUT && REC_OUT + N_OUT_ITEMS * 64 <= REC_SRC && OFF_FA + N_UNITS * 8 <= OFF_DBG, "workspace map");

constexpr u32 RING = 65536;
constexpr u32 L_SCR = RING, SCR_B = 8448;
constexpr u32 L_DESC = L_SCR + 6 * SCR_B, NQ = 64;
constexpr u32 L_ITEM = L_DESC + NQ * 16;
constexpr u32 L_FCTL = L_ITEM + 64;
constexpr u32 L_LAND = L_FCTL + 64;
constexpr u32 L_T1 = 131072, L_T2 = L_T1 + 512, L_ZERO = L_T2 + 512;
constexpr u32 PK_B = 19456, PK_IMG = 8320;
constexpr u32 LDS_BYTES = 8 * PK_B;
static_assert(L_ITEM + 1024 <= LDS_BYTES && L_LAND + 256 <= L_ITEM + 1024 && L_ZERO + 8192 <= LDS_BYTES && LDS_BYTES <= 163840, "LDS map");

struct Params {
    const float* x; const float* w; float* out;
    char* me;
    char* ws[NR];
    int rank; int pad;
};

#define VMCNT(n) asm volatile("s_waitcnt vmcnt(" #n ")" ::: "memory")
#define LGKM0() asm volatile("s_waitcnt lgkmcnt(0)" ::: "memory")
#define CFENCE() asm volatile("" ::: "memory")
#define BAR() do { CFENCE(); __builtin_amdgcn_s_barrier(); CFENCE(); } while (0)

__device__ __forceinline__ void st16_sys(void* p, u32x4 v) {
    asm volatile("global_store_dwordx4 %0, %1, off sc0 sc1\n\ts_nop 1" :: "v"(p), "v"(v) : "memory");
}
__device__ __forceinline__ void st_out(GAS float* p, f32x4 v) {
    asm volatile("global_store_dwordx4 %0, %1, off sc0 sc1 nt\n\ts_nop 1" :: "v"(p), "v"(v) : "memory");
}
__device__ __forceinline__ void st4_sys(void* p, u32 v) {
    asm volatile("global_store_dword %0, %1, off sc0 sc1" :: "v"(p), "v"(v) : "memory");
}
__device__ __forceinline__ void st8_sys(void* p, u32x2 v) {
    asm volatile("global_store_dwordx2 %0, %1, off sc0 sc1" :: "v"(p), "v"(v) : "memory");
}
__device__ __forceinline__ u32 ld4_sys(const void* p) {
    u32 v; asm volatile("global_load_dword %0, %1, off sc0 sc1\n\ts_waitcnt vmcnt(0)" : "=&v"(v) : "v"(p) : "memory"); return v;
}
__device__ __forceinline__ u32x2 ld8_sys(const void* p) {
    u32x2 v; asm volatile("global_load_dwordx2 %0, %1, off sc0 sc1\n\ts_waitcnt vmcnt(0)" : "=&v"(v) : "v"(p) : "memory"); return v;
}
__device__ __forceinline__ void lds_st(u32 a, u32 v) { asm volatile("ds_write_b32 %0, %1" :: "v"(a), "v"(v) : "memory"); }
__device__ __forceinline__ u32 lds_ld(u32 a) {
    u32 v; asm volatile("ds_read_b32 %0, %1\n\ts_waitcnt lgkmcnt(0)" : "=&v"(v) : "v"(a) : "memory"); return v;
}
__device__ __forceinline__ u32x4 lds_ld16(u32 a) {
    u32x4 v; asm volatile("ds_read_b128 %0, %1\n\ts_waitcnt lgkmcnt(0)" : "=&v"(v) : "v"(a) : "memory"); return v;
}
__device__ __forceinline__ u32 uni(u32 v) { return __builtin_amdgcn_readfirstlane(v); }
__device__ __forceinline__ u32 lds_ldu(u32 a) { return uni(lds_ld(a)); }
__device__ __forceinline__ u64 now() { return __builtin_amdgcn_s_memrealtime(); }
__device__ __forceinline__ u32 prefix(u64 m) { return __builtin_amdgcn_mbcnt_hi((u32)(m >> 32), __builtin_amdgcn_mbcnt_lo((u32)m, 0u)); }

__device__ __forceinline__ u32 wave_umax(u32 v) {
    v = max(v, (u32)__builtin_amdgcn_update_dpp(0, (int)v, 0x111, 0xf, 0xf, true));
    v = max(v, (u32)__builtin_amdgcn_update_dpp(0, (int)v, 0x112, 0xf, 0xf, true));
    v = max(v, (u32)__builtin_amdgcn_update_dpp(0, (int)v, 0x114, 0xf, 0xf, true));
    v = max(v, (u32)__builtin_amdgcn_update_dpp(0, (int)v, 0x118, 0xf, 0xf, true));
    v = max(v, (u32)__builtin_amdgcn_update_dpp(0, (int)v, 0x142, 0xa, 0xf, false));
    v = max(v, (u32)__builtin_amdgcn_update_dpp(0, (int)v, 0x143, 0xc, 0xf, false));
    return __builtin_amdgcn_readlane(v, 63);
}
__device__ __forceinline__ u32 wave_scan(u32 v) {
    v += (u32)__builtin_amdgcn_update_dpp(0, (int)v, 0x111, 0xf, 0xf, true);
    v += (u32)__builtin_amdgcn_update_dpp(0, (int)v, 0x112, 0xf, 0xf, true);
    v += (u32)__builtin_amdgcn_update_dpp(0, (int)v, 0x114, 0xf, 0xf, true);
    v += (u32)__builtin_amdgcn_update_dpp(0, (int)v, 0x118, 0xf, 0xf, true);
    v += (u32)__builtin_amdgcn_update_dpp(0, (int)v, 0x142, 0xa, 0xf, false);
    v += (u32)__builtin_amdgcn_update_dpp(0, (int)v, 0x143, 0xc, 0xf, false);
    return v;
}
typedef unsigned short u16x2 __attribute__((ext_vector_type(2)));
typedef short i16x2 __attribute__((ext_vector_type(2)));
#define PU(x) __builtin_bit_cast(u16x2, (u32)(x))
#define PI(x) __builtin_bit_cast(i16x2, (u32)(x))
#define PW(x) __builtin_bit_cast(u32, (x))
template <int W> __device__ __forceinline__ u32 pk_offsets(const u32* m, u32* ac) {
    u32 tot;
    ac[0] = 0;
    asm("v_pk_mul_lo_u16 %0, %4, %8 op_sel_hi:[1,0]\n\tv_pk_mad_u16 %1, %5, %8, %0 op_sel_hi:[1,0,1]\n\t"
        "v_pk_mad_u16 %2, %6, %8, %1 op_sel_hi:[1,0,1]\n\tv_pk_mad_u16 %3, %7, %8, %2 op_sel_hi:[1,0,1]"
        : "=&v"(ac[1]), "=&v"(ac[2]), "=&v"(ac[3]), "=&v"(tot) : "v"(m[0]), "v"(m[1]), "v"(m[2]), "v"(m[3]), "n"(-W));
    return tot;
}
__device__ __forceinline__ u32 pk_is3(u32 c) { return PW((i16x2)(PI(c + 0x7ffd7ffdu) >> (i16x2){15, 15})); }
__device__ __forceinline__ u32 pk_sum(u32 a) { return (a & 0xffffu) + (a >> 16); }
__device__ __forceinline__ u32 pk_join(const u32* sv, const u32* ac, u32 acc) {
    const u32 tt = sv[0] | PW((u16x2)(PU(sv[1]) << PU(ac[1]))) | PW((u16x2)(PU(sv[2]) << PU(ac[2]))) | PW((u16x2)(PU(sv[3]) << PU(ac[3])));
    return (tt & 0xffffu) | ((tt >> 16) << (acc & 31u));
}
__device__ __forceinline__ u32 pk_split(u32 t, u32 acc) { return __builtin_amdgcn_perm(t >> (acc & 31u), t, 0x05040100u); }
__device__ __forceinline__ u32 pk_field(u32 tt, u32 sh, u32 m, u32 mask) { return PW((u16x2)(PU(tt) >> PU(sh))) & m & mask; }
__device__ __forceinline__ u32 pk_expand(u32 sv, u32 k, u32 base) {
    u32 r;
    asm("v_pk_min_u16 %0, %1, 1 op_sel_hi:[1,0]\n\tv_pk_mad_u16 %0, %0, 3, %1 op_sel_hi:[1,0,1]\n\tv_pk_mad_u16 %0, %0, %2, %3"
        : "=&v"(r) : "v"(sv), "s"(k), "v"(base));
    return r;
}
template <int SH> __device__ __forceinline__ void pk_classify(const u32* x, u32* c, u32* g, u32& T2, u32& T3, u32& len) {
    u32 m2[4], m3[4], ac[4], s2[4], s3[4];
#pragma unroll
    for (int k = 0; k < 4; ++k) {
        const u32 o = PW((i16x2)(PI(x[k]) >> (i16x2){SH, SH}));
        const u32 sv = PW(__builtin_elementwise_max((i16x2)(-PI(o)), (i16x2){0, 0}));
        c[k] = PW(__builtin_elementwise_min(PU(o), (u16x2){3, 3}));
        s2[k] = PW(__builtin_elementwise_min(PU(sv), (u16x2){3, 3}));
        s3[k] = (sv - s2[k]) & 0x00070007u;
        m2[k] = pk_is3(c[k]); m3[k] = pk_is3(s2[k]);
        g[k] = PW(__builtin_elementwise_sub_sat((u16x2)(PU(o) + (u16x2){10, 10}), (u16x2){13, 13}));
    }
    const u32 a2 = pk_offsets<2>(m2, ac);
    T2 = pk_join(s2, ac, a2);
    const u32 a3 = pk_offsets<3>(m3, ac);
    T3 = pk_join(s3, ac, a3);
    len = pk_sum(a2) | (pk_sum(a3) << 16);
}
__device__ __forceinline__ void or_bits(LAS char* buf, u32 bp, u32 t) {
    const u64 v = (u64)t << (bp & 31u);
    LAS u32* a = (LAS u32*)(buf + (bp >> 5) * 4u);
    __hip_atomic_fetch_or(a, (u32)v, __ATOMIC_RELAXED, __HIP_MEMORY_SCOPE_WAVEFRONT);
    __hip_atomic_fetch_or(a + 1, (u32)(v >> 32), __ATOMIC_RELAXED, __HIP_MEMORY_SCOPE_WAVEFRONT);
}
__device__ __forceinline__ void emit_escapes(const u32* g, const u32* raw, u32 idx0, LAS char* rec, u32 cap, u32& e) {
    if (__builtin_amdgcn_ballot_w64((g[0] | g[1] | g[2] | g[3]) != 0u) == 0ull) return;
    u32 gm = 0;
#pragma unroll
    for (int k = 0; k < 4; ++k) gm |= PW(__builtin_elementwise_min(PU(g[k]), (u16x2){1, 1})) << k;
#pragma unroll 1
    for (;;) {
        const u64 em = __builtin_amdgcn_ballot_w64(gm != 0u);
        if (em == 0ull) break;
        const u32 b = __builtin_ctz(gm | 0x80000000u), k = b & 3u;
        const u32 m1 = (u32)__builtin_amdgcn_sbfe((int)b, 0, 1), m2 = (u32)__builtin_amdgcn_sbfe((int)b, 1, 1);
        const u32 r01 = (raw[1] & m1) | (raw[0] & ~m1), r23 = (raw[3] & m1) | (raw[2] & ~m1), r = (r23 & m2) | (r01 & ~m2);
        if (gm != 0u && e < cap) *(LAS u32*)(rec + (e + prefix(em)) * 4u) = (idx0 + k * 2u + (b >> 4)) | ((r >> (b & 16u)) << 16);
        gm &= gm - 1u;
        e += __builtin_popcountll(em);
    }
}
typedef float f32x2 __attribute__((ext_vector_type(2)));
typedef _Float16 f16x2 __attribute__((ext_vector_type(2)));
__device__ __forceinline__ u32 cvt2(float a, float b, float sc) { return __builtin_bit_cast(u32, __builtin_convertvector((f32x2){a, b} * sc, f16x2)); }
__device__ __forceinline__ char* peer_ws(const Params& P, int p) {
    u64 wp = (u64)P.ws[0];
#pragma unroll
    for (int i = 1; i < NR; ++i) { u64 q = (u64)P.ws[i]; asm volatile("" : "+s"(q)); wp = p == i ? q : wp; }
    return (char*)wp;
}

#define STAMP(k) do { } while (0)
#define STAMPV(k, v) do { } while (0)

__device__ __forceinline__ bool give_up(u32& spins, u64 t0, u32* tmo, u32 call) {
    if ((++spins & 63u) != 0u) return false;
    if (uni(ld4_sys(tmo)) == call) return true;
    if (now() - t0 > DEADLINE) { st4_sys(tmo, call); return true; }
    return false;
}

__device__ __forceinline__ float block_scale(const f32x4 (&v)[16], u32& code, u32& hm) {
    float m0 = 0.f, m1 = 0.f, m2 = 0.f, m3 = 0.f;
#pragma unroll
    for (int i = 0; i < 16; i += 2)
        asm("v_max3_f32 %0, %0, |%4|, |%5|\n\tv_max3_f32 %1, %1, |%6|, |%7|\n\tv_max3_f32 %2, %2, |%8|, |%9|\n\tv_max3_f32 %3, %3, |%10|, |%11|"
            : "+v"(m0), "+v"(m1), "+v"(m2), "+v"(m3)
            : "v"(v[i][0]), "v"(v[i][1]), "v"(v[i][2]), "v"(v[i][3]), "v"(v[i + 1][0]), "v"(v[i + 1][1]), "v"(v[i + 1][2]), "v"(v[i + 1][3]));
    const u32 mb = wave_umax(max(max(__float_as_uint(m0), __float_as_uint(m1)), max(__float_as_uint(m2), __float_as_uint(m3))));
    code = ((mb >> 23) + 2u) >> 3; code = code < 2u ? 2u : code;
    const float sc = __uint_as_float((263u - 8u * code) << 23);
    hm = uni(__builtin_bit_cast(unsigned short, (_Float16)(__uint_as_float(mb) * sc)));
    if (mb == 0u) code = 63u;
    return sc;
}
__device__ __forceinline__ void load_x(const Params& P, int idx, int j, int lane, f32x4 (&v)[16]) {
    const GAS float* p = (const GAS float*)P.x + (size_t)(idx * 64 + (lane & 15)) * KL + j * 64 + (lane >> 4) * 4;
#pragma unroll
    for (int i = 0; i < 16; ++i) v[i] = __builtin_nontemporal_load((const GAS f32x4*)(p + (i >> 2) * 16 * KL + (i & 3) * 16));
}
__device__ __forceinline__ void push_task(const Params& P, int idx, int lane, u32 craw, const f32x4 (&v)[16]) {
    asm volatile("" :: "s"(P.ws[0]), "s"(P.ws[1]), "s"(P.ws[2]), "s"(P.ws[3]), "s"(P.ws[4]), "s"(P.ws[5]), "s"(P.ws[6]), "s"(P.ws[7]));
    __builtin_amdgcn_sched_barrier(0);
    u32 code, hm;
    const float sc = block_scale(v, code, hm);
    const u32 call = uni(craw) + 1u;
    const u32 dest = (call & 1u) * U_PAR + ((idx >> 1) * 64 + P.rank) * 2 + (idx & 1);
    const u32 voff = (u32)OFF_A + dest * 8192u + lane * 16u;
#pragma unroll
    for (int f = 0; f < 8; ++f) {
        const u32x4 hv = {cvt2(v[2 * f][0], v[2 * f][1], sc), cvt2(v[2 * f][2], v[2 * f][3], sc), cvt2(v[2 * f + 1][0], v[2 * f + 1][1], sc), cvt2(v[2 * f + 1][2], v[2 * f + 1][3], sc)};
        VMCNT(8);
        asm volatile("global_store_dwordx4 %0, %1, %2 sc0 sc1\n\tglobal_store_dwordx4 %0, %1, %3 sc0 sc1\n\tglobal_store_dwordx4 %0, %1, %4 sc0 sc1\n\t"
                     "global_store_dwordx4 %0, %1, %5 sc0 sc1\n\tglobal_store_dwordx4 %0, %1, %6 sc0 sc1\n\tglobal_store_dwordx4 %0, %1, %7 sc0 sc1\n\t"
                     "global_store_dwordx4 %0, %1, %8 sc0 sc1\n\tglobal_store_dwordx4 %0, %1, %9 sc0 sc1\n\ts_nop 1"
                     :: "v"(voff + f * 1024), "v"(hv), "s"((u64)P.ws[0]), "s"((u64)P.ws[1]), "s"((u64)P.ws[2]), "s"((u64)P.ws[3]),
                        "s"((u64)P.ws[4]), "s"((u64)P.ws[5]), "s"((u64)P.ws[6]), "s"((u64)P.ws[7]) : "memory");
        __builtin_amdgcn_sched_barrier(0);
    }
    VMCNT(0);
    if (lane < NR) st8_sys(peer_ws(P, lane) + OFF_FA + (size_t)dest * 8, (u32x2){call, code});
}
__device__ __forceinline__ void load_w(const Params& P, int idx, int j, int lane, f32x4 (&v)[16]) {
    const GAS float* p = (const GAS float*)P.w + (size_t)(j * 64 + (lane >> 4) * 4) * GN + idx * 64 + (lane & 15) * 4;
#pragma unroll
    for (int i = 0; i < 16; ++i) v[i] = __builtin_nontemporal_load((const GAS f32x4*)(p + (size_t)((i >> 2) * 16 + (i & 3)) * GN));
}
__device__ __forceinline__ void task_of(int q, int half, int& j, int& idx, bool& isA) {
    if (q < 896) { j = q / 112; const int u = q % 112; isA = u < 32; idx = isA ? u : (u < 96 ? u + 32 : u - 96); }
    else { const int u = q - 896, v = u % 192; j = v / 24; isA = false; idx = 16 + (u / 192) * 24 + v % 24; }
    idx = idx * 2 + half;
}
__device__ __forceinline__ void stage_task(const Params& P, LAS char* scr, u32 lquad, u32 lpair, int q, int half, int lane, GAS u32* cnt, const f32x4 (&v)[16]) {
    int j, idx; bool isA;
    task_of(q, half, j, idx, isA);
    const u32 craw = __hip_atomic_load(cnt, __ATOMIC_RELAXED, __HIP_MEMORY_SCOPE_AGENT);
    *(LAS u32x4*)(scr + lane * 16) = (u32x4){0u, 0u, 0u, 0u}; *(LAS u32x4*)(scr + 1024 + lane * 16) = (u32x4){0u, 0u, 0u, 0u};
    __builtin_amdgcn_sched_barrier(0);
    u32 code, hm;
    const float sc = block_scale(v, code, hm);
    const u32 bb = hm >= 4095u ? hm - 4095u : 0u, b2 = bb * 0x10001u;
    u32x4 p[8];
    if (isA) {
#pragma unroll
        for (int f = 0; f < 8; ++f)
#pragma unroll
            for (int k = 0; k < 4; ++k) p[f][k] = cvt2(v[f * 2 + (k >> 1)][(k & 1) * 2], v[f * 2 + (k >> 1)][(k & 1) * 2 + 1], sc);
        CFENCE();
    } else {
#pragma unroll
        for (int f = 0; f < 8; ++f)
#pragma unroll
            for (int k = 0; k < 4; ++k) p[f][k] = cvt2(v[(f & 1) * 8 + k * 2][f >> 1], v[(f & 1) * 8 + k * 2 + 1][f >> 1], sc);
        asm volatile("s_nop 0" ::: "memory");
    }
    char* ws = P.me;
    const u32 call = uni(craw) + 1u;
    const bool all = isA || idx < 128;
    const int owner = (idx - 128) >> 4, lp = idx < 128 ? 4 + (idx >> 2) : (idx >> 2) & 3;
    const bool local = all || owner == P.rank, exprt = all || owner != P.rank;
    const u32 dest = (call & 1u) * U_PAR + (isA ? (((idx >> 1) * 8 + j) * 8 + P.rank) * 2 + half : U_B + (((lp * 8 + j) * 8 + P.rank) * 4 + (idx & 3)));
    u32 size16 = 0;
    if (exprt) {
        u32 w[32], T2[8], T3[8], len[8], e = 0, tot = 0;
#pragma unroll
        for (int f = 0; f < 8; ++f) {
            u32 x[4], c[4], g[4];
            const u32 h[4] = {p[f][0], p[f][1], p[f][2], p[f][3]};
#pragma unroll
            for (int k = 0; k < 4; ++k) x[k] = PW((u16x2)(PU(h[k] & 0x7fff7fffu) - PU(b2)));
            pk_classify<10>(x, c, g, T2[f], T3[f], len[f]);
#pragma unroll
            for (int k = 0; k < 4; ++k) w[f * 4 + k] = (h[k] & 0x80008000u) | (c[k] << 10) | (x[k] & 0x03ff03ffu);
            tot += len[f];
            asm volatile("" : "+v"(w[f * 4]), "+v"(w[f * 4 + 1]), "+v"(w[f * 4 + 2]), "+v"(w[f * 4 + 3]), "+v"(T2[f]), "+v"(T3[f]));
            emit_escapes(g, h, f * 512 + lane * 8u, scr + 4, 384u, e);
        }
        const u32 inc = wave_scan(tot), all2 = __builtin_amdgcn_readlane(inc, 63);
        size16 = (6656u + 4u + e * 4u + ((pk_sum(all2) + 7u) >> 3) + 15u) >> 4;
        if (size16 >= 512u) size16 = 512u;
        u32 off16 = 0;
        if (half == 0) { if (lane == 0) lds_st(lpair, size16); }
        else {
            const u64 t0 = now(); u32 sp = 0;
            while ((off16 = lds_ldu(lpair)) == 0u) {
                if ((++sp & 1023u) == 0u && now() - t0 > 4 * DEADLINE) { st4_sys(ws + OFF_TMO, call); break; }
                __builtin_amdgcn_s_sleep(1);
            }
        }
        char* d = ws + (call & 1u) * X_PAR + off16 * 16u + lane * 16 +
                  (isA ? OFF_AX + (size_t)((idx >> 1) * 8 + j) * 16384 : OFF_BX + (size_t)(((idx >> 2) * 8 + j) * 2 + ((idx >> 1) & 1)) * 16384);
        if (size16 == 512u) {
#pragma unroll
            for (int f = 0; f < 8; ++f) st16_sys(d + f * 1024, p[f]);
        } else {
            u32 bp = (e + 1u) * 32u * 0x10001u + (all2 << 16) + inc - tot;
#pragma unroll
            for (int f = 0; f < 8; f += 2) {
                or_bits(scr, bp & 0xffffu, T2[f] | (T2[f + 1] << (len[f] & 31u)));
                or_bits(scr, bp >> 16, T3[f]);
                bp += len[f];
                or_bits(scr, bp >> 16, T3[f + 1]);
                bp += len[f + 1];
            }
            if (lane == 0) *(LAS u32*)scr = e;
#pragma unroll
            for (int i = 0; i < 24; ++i) w[i] |= (w[26 + (i >> 2)] << (12 - 3 * (i & 3))) & 0x70007000u;
#pragma unroll
            for (int i = 0; i < 6; ++i) w[24 + i / 3] |= (w[26 + i] >> (1 + i % 3)) & (0x40004000u >> (i % 3));
#pragma unroll
            for (int f = 0; f < 6; ++f) st16_sys(d + f * 1024, (u32x4){w[f * 4], w[f * 4 + 1], w[f * 4 + 2], w[f * 4 + 3]});
            st8_sys(d + 6144 - lane * 8, (u32x2){w[24], w[25]});
            CFENCE();
            for (u32 g = lane; g + 416u < size16; g += 64u) st16_sys(d + 6656 + (g - lane) * 16u, *(LAS u32x4*)(scr + g * 16u));
        }
    }
    if (local) {
        char* d = ws + OFF_A + (size_t)dest * 8192 + lane * 16;
#pragma unroll
        for (int f = 0; f < 8; ++f) st16_sys(d + f * 1024, p[f]);
    }
    VMCNT(0);
    if (local && lane == 0) st8_sys(ws + OFF_FA + (size_t)dest * 8, (u32x2){call, code});
    if (exprt) {
        const int hb = (lp % 12) * 2 + ((idx >> 1) & 1);
        const int item = isA ? j * 56 + (idx >> 1) : (lp < 12 ? j * 56 + 32 + hb : 256 + (lp / 12) * 192 + j * 24 + hb);
        const u32 info = size16 | (bb << 10) | (code << 25);
        const size_t ro = OFF_REC + (call & 1u) * REC_PAR + P.rank * REC_SRC;
        if (q < 112) {
            if (lane < NR && lane != P.rank && (all || lane == owner)) st8_sys(peer_ws(P, lane) + ro + item * 16 + half * 8, (u32x2){call, info});
        } else {
            u32 old = 0;
            if (lane == 0) asm volatile("ds_write_b32 %1, %2\n\tds_add_rtn_u32 %0, %3, %4\n\ts_waitcnt lgkmcnt(0)"
                                        : "=&v"(old) : "v"(lquad + ((item & 3) * 2 + half) * 4), "v"(info), "v"(lquad + 32), "v"(1u) : "memory");
            if (uni(old) == 7u) {
                u32x2 in2; asm volatile("ds_read_b64 %0, %1\n\ts_waitcnt lgkmcnt(0)" : "=&v"(in2) : "v"(lquad + (lane & 3) * 8) : "memory");
                const int r = lane >> 2;
                if (lane < 32 && r != P.rank && (all || r == owner)) st16_sys(peer_ws(P, r) + ro + ((item & ~3) + (lane & 3)) * 16, (u32x4){call, in2.x, call, in2.y});
            }
        }
    }
    CFENCE();
}

__device__ __forceinline__ void scout(const Params& P, int src, int li, int lane, u32 call) {
    char* ws = P.me;
    u32* tmo = (u32*)(ws + OFF_TMO);
    const char* recs = ws + OFF_REC + (call & 1u) * REC_PAR;
    const u32 xpar = (call & 1u) * (u32)X_PAR;
    const u64 base = (u64)P.ws[src];
    const u64 tbase = (u64)call << 12;
    u32 seq = 32 + li,
        isrc = src, cand = src, pieces = 0, nq = 0, k = 0;
    __builtin_amdgcn_s_setprio(2);
    if (lane == 0) asm volatile("global_atomic_umax_x2 %0, %1, off" :: "v"(ws + OFF_TICK + src * 256), "v"(tbase + (ITEM0 + LPP)) : "memory");
    const u32 s8 = lane & 7, ahead = ((((src - s8 - 1) & 7) - ((P.rank - s8 - 1) & 7) + 7) % 7);
    const u32 hfrom = (s8 != (u32)P.rank && s8 != (u32)src && ahead >= 2u) ? R2_ITEM + ahead * ROT + 5u : ~0u, hlag = HELP_LAG + (ahead - 2u) * HELP_STEP;
    while (seq < (u32)N_ITEMS || isrc != (u32)src) {
        const bool isin = seq < (u32)N_IN_ITEMS;
        u32 item = seq;
        if (isin && seq >= R2_ITEM) { item += ((P.rank - isrc - 1) & 7) * ROT; if (item >= (u32)N_IN_ITEMS) item -= N_IN_ITEMS - R2_ITEM; }
        const char* rp = recs + isrc * REC_SRC + (isin ? item * 16u + (lane & 1) * 8u : REC_OUT + ((item - N_IN_ITEMS) & 511u) * 64u + s8 * 8u);
        u32x2 r; bool dead = false, relay = false;
        u64 tk; u32 prog = 0;
        asm volatile("global_load_dwordx2 %0, %1, off sc0 sc1" : "=v"(r) : "v"(rp) : "memory");
        if (lane == 0) asm volatile("global_atomic_add_x2 %0, %1, %2, off sc0" : "=v"(tk) : "v"(ws + OFF_TICK + cand * 256), "v"(1ull) : "memory");
        const bool look = ((li ^ src) & HELPM) == 0 && seq - (R2_ITEM + 2 * ROT) < (u32)N_IN_ITEMS + 80u - (R2_ITEM + 2 * ROT);
        if (look && lane < 8) asm volatile("global_atomic_add %0, %1, %2, off sc0" : "=v"(prog) : "v"(ws + OFF_TICK + s8 * 256), "v"(0u) : "memory");
        u32 j = 0, u = 0;
        if (item < 448u) { j = item / 56u; u = item % 56u; }
        else if (isin) { const u32 v = item - 448u, w = v % 192u; j = w / 24u; u = 56u + (v / 192u) * 24u + w % 24u; }
        const u32 dest = (call & 1u) * U_PAR + (u >= 32u ? U_B + ((((u - 32u) >> 1) * 8 + j) * 8 + isrc) * 4 + (u & 1u) * 2 : ((u * 8 + j) * 8 + isrc) * 2);
        asm volatile("s_waitcnt vmcnt(0)" : "+v"(r), "+v"(tk), "+v"(prog) :: "memory");
        if (isrc != (u32)src && isin && seq >= R2_ITEM) {
            const u32x2 f = ld8_sys((const char*)base + OFF_FA + (size_t)(dest + (lane & 1)) * 8);
            relay = __builtin_amdgcn_ballot_w64(f.x == call) == ~0ull;
            if (relay) r = (u32x2){call, 512u | (f.y << 25)};
        }
        if (seq < (u32)N_ITEMS && __builtin_amdgcn_ballot_w64(r.x >= call) != ~0ull) {
            const u64 t0 = now(); u32 sp = 0;
            do {
                if (give_up(sp, t0, tmo, call)) { dead = true; break; }
                __builtin_amdgcn_s_sleep(2);
                r = ld8_sys(rp);
            } while (__builtin_amdgcn_ballot_w64(r.x >= call) != ~0ull);
        }
        u32 next = uni((u32)tk) - (u32)tbase;
        if (next < 32u + LPP) next -= LPP;
        const u32 nsrc = cand;
        cand = src;
        if (look) {
            const u32 ps = prog - (u32)tbase;
            const u64 m = __builtin_amdgcn_ballot_w64(lane < 8 && ps >= hfrom && ps + 12u <= (u32)N_IN_ITEMS && ps + hlag <= (u32)__builtin_amdgcn_readlane(ps, src));
            if (m) cand = __builtin_ctzll(m);
        }
        if (seq >= (u32)N_ITEMS) { seq = next; isrc = nsrc; continue; }
        u32 soff, bytes, nsub; u32x4 d;
        if (isin) {
            if (relay) soff = (u32)OFF_A + dest * 8192u;
            else if (u >= 32u) { const u32 lp = (u - 32u) >> 1, gp = lp < 4u ? 32u + P.rank * 4u + lp : lp - 4u; soff = (u32)OFF_BX + xpar + (gp * 8 + j) * 32768u + (u & 1u) * 16384u; }
            else soff = (u32)OFF_AX + xpar + (u * 8 + j) * 16384u;
            const u32 s = lane & 1, sz = min(r.y & 1023u, 512u), sb = dead ? 0u : sz * 16u;
            const u32 b0 = __builtin_amdgcn_readlane(sb, 0);
            bytes = b0 + __builtin_amdgcn_readlane(sb, 1); nsub = 2;
            d.x = pieces * 1024u + (s ? b0 : 0u); d.y = pieces + (((s ? bytes : b0) + 1023u) >> 10);
            d.z = dead ? (62u << 12) | (2u << 24) : sz | ((r.y >> 25) << 12); d.w = ((r.y >> 10) & 0x7fffu) | ((dest + s) << 16);
        } else {
            const u32 o = item - N_IN_ITEMS;
            soff = (u32)OFF_O + o * (u32)QSLOT;
            const u32 a = __builtin_amdgcn_ds_bpermute((lane & 3) * 8, r.y), b = __builtin_amdgcn_ds_bpermute((lane & 3) * 8 + 4, r.y);
            const u32 sb = dead ? 0u : min(a & 1023u, 512u) * 16u, n = min((a >> 10) & 0xfffu, 2048u), e = b & 0xfffu, b16 = (b >> 12) & 0x7fffu;
            const u32 s0 = __builtin_amdgcn_readlane(sb, 0), s1 = s0 + __builtin_amdgcn_readlane(sb, 1), s2 = s1 + __builtin_amdgcn_readlane(sb, 2);
            const u32 off = (lane & 3) == 0 ? 0u : (lane & 3) == 1 ? s0 : (lane & 3) == 2 ? s1 : s2;
            bytes = s2 + __builtin_amdgcn_readlane(sb, 3); nsub = 4;
            d.x = pieces * 1024u + off; d.y = pieces + ((off + sb + 1023u) >> 10); d.z = n | (e << 12) | ((dead ? 3u : sb == 8192u ? 4u : 1u) << 24);
            d.w = b16 | ((o * 4u + (lane & 3)) << 16) | (isrc << 29);
        }
        const u32 np = (bytes + 1023u) >> 10;
        u64 ibase = base;
        if (__builtin_expect(isrc != (u32)src && !relay, 0)) ibase = (u64)P.ws[isrc & 7];
        {
            u64 t0 = 0; u32 sp = 0;
            for (;;) {
                u32 nx = lds_ld(L_FCTL + (lane < 6 ? 32 + lane * 4 : 12));
                u32 qm = __builtin_amdgcn_readlane(nx, 0);
#pragma unroll
                for (int i = 1; i < 6; ++i) qm = min(qm, (u32)__builtin_amdgcn_readlane(nx, i));
                const u32 first = qm < nq ? lds_ldu(L_DESC + (qm % NQ) * 16) >> 10 : pieces;
                if ((u32)__builtin_amdgcn_readlane(nx, 6) + AHEAD >= k && pieces + np - first <= RING / 1024 && (qm >= nq || nq + nsub - qm <= NQ)) break;
                if ((++sp & 1023u) == 0u) { if (!t0) t0 = now(); else if (now() - t0 > 4 * DEADLINE) { st4_sys(tmo, call); break; } }
                __builtin_amdgcn_s_sleep(SSL);
            }
        }
        if (lane < (int)nsub) asm volatile("ds_write_b128 %0, %1" :: "v"(L_DESC + ((nq + lane) % NQ) * 16), "v"(d) : "memory");
        nq += nsub;
        if (lane == 0) {
            const u64 ad = ibase + soff;
            const u32x4 it = {(u32)ad, bytes, (u32)(ad >> 32), k + 1};
            asm volatile("ds_write_b128 %0, %1" :: "v"(L_ITEM + (k & 3u) * 16), "v"(it) : "memory");
            lds_st(L_FCTL + 4, nq);
        }
        pieces += np; ++k;
        seq = next; isrc = nsrc;
    }
    if (lane == 0) {
        const u32x4 it = {0u, 0xffffffffu, 0u, k + 1};
        asm volatile("ds_write_b128 %0, %1" :: "v"(L_ITEM + (k & 3u) * 16), "v"(it) : "memory");
        lds_st(L_FCTL + 4, nq | 0x80000000u);
    }
}

__device__ __forceinline__ void loader(const Params& P, int src, int lane, u32 call) {
    const u32 voff = lane * 16, vland = L_LAND + lane * 4;
    u32* tmo = (u32*)(P.me + OFF_TMO);
    int issued = 0, landed = 0;
    u32 k = 0;
    __builtin_amdgcn_s_setprio(3);
#define PIECE(sp) do { asm volatile("s_mov_b32 m0, %2\n\ts_waitcnt vmcnt(%3)\n\tglobal_load_lds_dwordx4 %0, %1 sc0 sc1" \
                                    :: "v"(voff), "s"(sp), "s"(((u32)issued * 1024u) & (RING - 1)), "n"(WIN - 1) : "memory"); } while (0)
    for (;;) {
        u32x4 it = lds_ld16(L_ITEM + (k & 3u) * 16);
        if (uni(it.w) != k + 1 || uni(it.y) == 0xffffffffu) {
            VMCNT(0);
            landed = issued;
            lds_st(vland, (u32)issued);
            const u64 t0 = now(); u32 sp = 0;
            while (uni(it.w) != k + 1) {
                if ((++sp & 1023u) == 0u && now() - t0 > 4 * DEADLINE) { st4_sys(tmo, call); return; }
                __builtin_amdgcn_s_sleep(1);
                it = lds_ld16(L_ITEM + (k & 3u) * 16);
            }
            if (uni(it.y) == 0xffffffffu) return;
        }
        u64 sp = ((u64)uni(it.z) << 32) | uni(it.x);
        u32 left = uni(it.y);
        asm volatile("s_nop 4" : "+s"(sp));
        ++k;
        if (lane == 0) lds_st(L_FCTL + 12, k);
#pragma unroll 1
        for (; left >= 1024u; left -= 1024u, sp += 1024) {
            PIECE(sp);
            landed = max(landed, issued - (WIN - 1));
            lds_st(vland, (u32)landed);
            ++issued;
        }
        if (left) {
            if (voff < left) PIECE(sp);
            landed = max(landed, issued - (WIN - 1));
            lds_st(vland, (u32)landed);
            ++issued;
        }
    }
#undef PIECE
}

template <int G> __device__ __forceinline__ void expand_dense(LAS char* L, LAS char* scr, u32 lane, u32 n8, u32 o_lo, u32 o_mid, u32 o_top, u32 o_sec, u32 b2) {
    u32x4 lo[G]; u32x2 mid[G]; u32 top[G], u[G][4], fl[G][4], ac[G][4], lw[G], t0[G], t1[G], bp[G], tot = 0;
#pragma unroll
    for (int i = 0; i < G; ++i) {
        const u32 g = i * 64 + lane;
        mid[i] = *(LAS u32x2*)(L + ((o_mid + g * 8u) & (RING - 1)));
        top[i] = *(LAS unsigned char*)(L + ((o_top + g) & (RING - 1)));
    }
#pragma unroll
    for (int i = 0; i < G; ++i) lo[i] = *(LAS u32x4*)(L + ((o_lo + (i * 64 + lane) * 16u) & (RING - 1)));
#pragma unroll
    for (int i = 0; i < G; ++i) {
        if (i == G - 1) top[i] = i * 64 + lane < n8 ? top[i] : 0u;
        const u32 tt = (top[i] | (top[i] << 12)) << 8;
#pragma unroll
        for (int k = 0; k < 4; ++k) {
            u[i][k] = __builtin_amdgcn_perm(0u, mid[i][k >> 1], (k & 1) ? 0x0c030c02u : 0x0c010c00u) | ((tt >> k) & 0x01000100u);
            fl[i][k] = PW((i16x2)(PI(u[i][k] + 0x7e807e80u) >> (i16x2){15, 15}));
        }
        lw[i] = pk_offsets<2>(fl[i], ac[i]);
        bp[i] = tot; tot += pk_sum(lw[i]);
    }
    const u32 inc = wave_scan(tot);
#pragma unroll
    for (int i = 0; i < G; ++i) {
        bp[i] += inc - tot;
        const u32 a = o_sec + (bp[i] >> 5) * 4u;
        t0[i] = *(LAS u32*)(L + (a & (RING - 1))); t1[i] = *(LAS u32*)(L + ((a + 4u) & (RING - 1)));
    }
    u32 s2[G][4];
    tot = 0;
#pragma unroll
    for (int i = 0; i < G; ++i) {
        const u32 TT = pk_split(__builtin_amdgcn_alignbit(t1[i], t0[i], bp[i]), lw[i]);
#pragma unroll
        for (int k = 0; k < 4; ++k) { s2[i][k] = pk_field(TT, ac[i][k], fl[i][k], 0x00030003u); fl[i][k] = pk_is3(s2[i][k]); }
        lw[i] = pk_offsets<3>(fl[i], ac[i]);
        bp[i] = tot; tot += pk_sum(lw[i]);
    }
    const u32 base = wave_scan(tot) - tot + __builtin_amdgcn_readlane(inc, 63);
#pragma unroll
    for (int i = 0; i < G; ++i) {
        bp[i] += base;
        const u32 a = o_sec + (bp[i] >> 5) * 4u;
        t0[i] = *(LAS u32*)(L + (a & (RING - 1))); t1[i] = *(LAS u32*)(L + ((a + 4u) & (RING - 1)));
    }
#pragma unroll
    for (int i = 0; i < G; ++i) {
        const u32 TT = pk_split(__builtin_amdgcn_alignbit(t1[i], t0[i], bp[i]), lw[i]);
        u32 r[8];
#pragma unroll
        for (int k = 0; k < 4; ++k) {
            const u32 h = pk_expand(s2[i][k] + pk_field(TT, ac[i][k], fl[i][k], 0x00070007u), 0xff80ff80u, u[i][k] + b2);
            r[2 * k] = __builtin_amdgcn_perm(h, lo[i][k], 0x05040100u); r[2 * k + 1] = __builtin_amdgcn_perm(h, lo[i][k], 0x07060302u);
        }
        *(LAS u32x4*)(scr + (i * 64 + lane) * 32u) = (u32x4){r[0], r[1], r[2], r[3]};
        *(LAS u32x4*)(scr + (i * 64 + lane) * 32u + 16u) = (u32x4){r[4], r[5], r[6], r[7]};
    }
}

__device__ __forceinline__ void decoder(const Params& P, LAS char* L, int dw, int lane, u32 call) {
    char* ws = P.me;
    u32* tmo = (u32*)(ws + OFF_TMO);
    LAS char* scr = L + L_SCR + dw * SCR_B;
    u32 qn = dw, pend = 0;
    auto flush = [&]() { if (pend) { VMCNT(0); if (lane == 0) st8_sys(ws + OFF_FA + (size_t)(pend & 0xffffu) * 8, (u32x2){call, (pend >> 16) & 63u}); pend = 0; } };
    auto release = [&]() { if (lane == 0) asm volatile("ds_add_rtn_u32 %0, %1, %2\n\ts_waitcnt lgkmcnt(0)\n\tds_write_b32 %3, %0"
                                                       : "=&v"(qn) : "v"(L_FCTL + 8), "v"(1u), "v"(L_FCTL + 32 + dw * 4) : "memory"); };
    for (;;) {
        const u32 q = uni(qn);
        u32x4 dv;
        {
            u64 t0 = 0; u32 sp = 0;
            for (;;) {
                u32 nq, ld;
                asm volatile("ds_read_b32 %0, %3\n\tds_read_b128 %1, %4\n\tds_read_b32 %2, %5\n\ts_waitcnt lgkmcnt(0)"
                             : "=&v"(nq), "=&v"(dv), "=&v"(ld) : "v"(L_FCTL + 4), "v"(L_DESC + (q % NQ) * 16), "v"(L_LAND) : "memory");
                nq = uni(nq);
                if ((nq & 0x7fffffffu) > q && uni(ld) >= uni(dv.y)) break;
                flush();
                if ((nq & 0x7fffffffu) <= q && (nq >> 31)) return;
                if ((++sp & 1023u) == 0u) { if (!t0) t0 = now(); else if (now() - t0 > 4 * DEADLINE) { st4_sys(tmo, call); return; } }
                __builtin_amdgcn_s_sleep(1);
            }
        }
        const u32 pos = uni(dv.x), z = uni(dv.z), wv = uni(dv.w);
        const u32 type = z >> 24, dest = wv >> 16;
        if (type == 0u || type == 2u) {
            if (type == 0u) {
                const u32 sz = z & 0xfffu, b2 = (wv & 0xffffu) * 0x10001u, pl = pos + lane * 16u;
                char* d = ws + OFF_A + (size_t)dest * 8192;
                u32x4 c[8];
#pragma unroll
                for (int i = 0; i < 6; ++i) c[i] = *(LAS u32x4*)(L + ((pl + i * 1024u) & (RING - 1)));
                if (sz == 512u) {
                    c[6] = *(LAS u32x4*)(L + ((pl + 6144u) & (RING - 1))); c[7] = *(LAS u32x4*)(L + ((pl + 7168u) & (RING - 1)));
                    LGKM0();
#pragma unroll
                    for (int i = 0; i < 8; ++i) asm volatile("" : "+v"(c[i]));
                    release();
                    flush();
#pragma unroll
                    for (int f = 0; f < 8; ++f) st16_sys(d + f * 1024 + lane * 16, c[f]);
                } else {
                    u32x2 c6 = *(LAS u32x2*)(L + ((pos + 6144u + lane * 8u) & (RING - 1)));
                    for (u32 g = lane; g + 416u < sz; g += 64u) *(LAS u32x4*)(scr + g * 16u) = *(LAS u32x4*)(L + ((pos + 6656u + g * 16u) & (RING - 1)));
                    LGKM0();
#pragma unroll
                    for (int i = 0; i < 6; ++i) asm volatile("" : "+v"(c[i]));
                    asm volatile("" : "+v"(c6));
                    release();
                    const u32 e = min(uni(*(LAS u32*)scr), 384u);
                    u32 w[32], fl[32], ac[32], len[8], lo[8], tot = 0;
#pragma unroll
                    for (int i = 0; i < 24; ++i) w[i] = c[i >> 2][i & 3];
                    w[24] = c6.x; w[25] = c6.y;
#pragma unroll
                    for (int i = 0; i < 6; ++i) {
                        u32 t = (w[24 + i / 3] << (1 + i % 3)) & 0x80008000u;
#pragma unroll
                        for (int k = 0; k < 4; ++k) t |= (w[i * 4 + k] >> (12 - 3 * k)) & (0x00070007u << (3 * k));
                        w[26 + i] = t;
                    }
#pragma unroll
                    for (int f = 0; f < 8; ++f) {
#pragma unroll
                        for (int k = 0; k < 4; ++k) fl[f * 4 + k] = PW((i16x2)(PI((w[f * 4 + k] & 0x0c000c00u) + 0x74007400u) >> (i16x2){15, 15}));
                        lo[f] = pk_offsets<2>(fl + f * 4, ac + f * 4);
                        len[f] = pk_sum(lo[f]); tot += len[f];
                    }
                    const u32 inc = wave_scan(tot);
                    u32 bp = (e + 1u) * 32u + inc - tot;
                    u32 T[8], s2[32];
#pragma unroll
                    for (int f = 0; f < 8; f += 2) {
                        const LAS u32* a = (const LAS u32*)(scr + (bp >> 5) * 4u);
                        T[f] = __builtin_amdgcn_alignbit(a[1], a[0], bp);
                        T[f + 1] = T[f] >> len[f];
                        bp += len[f] + len[f + 1];
                    }
                    tot = 0;
#pragma unroll
                    for (int f = 0; f < 8; ++f) {
                        const u32 TT = pk_split(T[f], lo[f]);
#pragma unroll
                        for (int k = 0; k < 4; ++k) { const int i = f * 4 + k; s2[i] = pk_field(TT, ac[i], fl[i], 0x00030003u); fl[i] = pk_is3(s2[i]); }
                        lo[f] = pk_offsets<3>(fl + f * 4, ac + f * 4);
                        len[f] = pk_sum(lo[f]); tot += len[f];
                    }
                    flush();
                    bp = (e + 1u) * 32u + __builtin_amdgcn_readlane(inc, 63) + wave_scan(tot) - tot;
#pragma unroll
                    for (int f = 0; f < 8; ++f) {
                        const LAS u32* a = (const LAS u32*)(scr + (bp >> 5) * 4u);
                        T[f] = __builtin_amdgcn_alignbit(a[1], a[0], bp);
                        bp += len[f];
                    }
#pragma unroll
                    for (int f = 0; f < 8; ++f) {
                        const u32 TT = pk_split(T[f], lo[f]);
                        u32x4 o;
#pragma unroll
                        for (int k = 0; k < 4; ++k) {
                            const int i = f * 4 + k;
                            o[k] = pk_expand(s2[i] + pk_field(TT, ac[i], fl[i], 0x00070007u), 0xfc00fc00u, PW((u16x2)(PU(w[i] & 0x8fff8fffu) + PU(b2))));
                        }
                        st16_sys(d + f * 1024 + lane * 16, o);
                    }
                    for (u32 g = lane; g < e; g += 64u) {
                        const u32 rr = *(LAS u32*)(scr + 4u + g * 4u);
                        asm volatile("global_store_short_d16_hi %0, %1, off sc0 sc1" :: "v"(d + (rr & 4095u) * 2u), "v"(rr) : "memory");
                    }
                }
            } else { release(); flush(); }
            pend = dest | (((z >> 12) & 63u) << 16) | 0x80000000u;
            continue;
        }
        flush();
        const u32 tile = (dest >> 5) & 63u, wq = (dest >> 2) & 7u, su = dest & 3u;
        const u32 row0 = (tile >> 2) * 256u + (su >> 1) * 128u + (wq >> 2) * 64u + (su & 1u) * 32u + (lane >> 4) * 4u;
        const u32 col0 = 8192u + (dest >> 13) * 1024u + (tile & 3u) * 256u + (wq & 3u) * 64u + (lane & 15) * 4u;
        GAS float* o = (GAS float*)P.out + (size_t)row0 * GN + col0;
        if (type == 3u) {
            release();
            const float qn = __builtin_nanf("");
#pragma unroll
            for (int g = 0; g < 8; ++g) *(GAS f32x4*)(o + (size_t)((g >> 2) * 16 + (g & 3)) * GN) = (f32x4){qn, qn, qn, qn};
            continue;
        }
        if (type == 4u) {
            u32x4 c[8];
#pragma unroll
            for (int g = 0; g < 8; ++g) c[g] = *(LAS u32x4*)(L + ((pos + g * 1024u + lane * 16u) & (RING - 1)));
            LGKM0();
#pragma unroll
            for (int g = 0; g < 8; ++g) asm volatile("" : "+v"(c[g]));
            release();
#pragma unroll
            for (int g = 0; g < 8; ++g) st_out(o + (size_t)((g >> 2) * 16 + (g & 3)) * GN, __builtin_bit_cast(f32x4, c[g]));
            continue;
        }
        const u32 n = z & 0xfffu, e = (z >> 12) & 0xfffu, b2 = (wv & 0xffffu) * 0x10001u;
        const u32 nr = (n & 511u) <= 64u ? n & 511u : 0u, n8 = (n - nr + 7u) >> 3, G = (n8 + 63u) >> 6;
        u32 mk = *(LAS u32*)(L + ((pos + lane * 4u) & (RING - 1)));
        const u32 o_lo = pos + 256u, o_mid = o_lo + n8 * 16u, o_top = o_mid + n8 * 8u, o_raw = o_top + ((n8 + 3u) & ~3u);
        const u32 o_esc = o_raw + nr * 4u, o_sec = o_esc + e * 4u;
        if (nr) *(LAS u32*)(scr + n8 * 32u + lane * 4u) = *(LAS u32*)(L + ((o_raw + lane * 4u) & (RING - 1)));
        if (G == 3u) expand_dense<3>(L, scr, lane, n8, o_lo, o_mid, o_top, o_sec, b2);
        else if (G == 2u) expand_dense<2>(L, scr, lane, n8, o_lo, o_mid, o_top, o_sec, b2);
        else if (G == 1u) expand_dense<1>(L, scr, lane, n8, o_lo, o_mid, o_top, o_sec, b2);
        else if (G == 4u) expand_dense<4>(L, scr, lane, n8, o_lo, o_mid, o_top, o_sec, b2);
        CFENCE();
#pragma unroll 1
        for (u32 g = lane; g < e; g += 64u) {
            const u32 rr = *(LAS u32*)(L + ((o_esc + g * 4u) & (RING - 1)));
            *(LAS unsigned short*)(scr + (rr & 2047u) * 4u + 2u) = (unsigned short)(rr >> 16);
        }
        LGKM0();
        asm volatile("" : "+v"(mk));
        release();
        {
            u32 ad = L_SCR + dw * SCR_B + wave_scan(__builtin_popcount(mk)) * 4u;
            u32 r[32], m[32];
#pragma unroll
            for (int i = 31; i >= 0; --i) {
                m[i] = (u32)__builtin_amdgcn_sbfe((int)mk, i, 1);
                asm("v_lshl_add_u32 %0, %1, 2, %2" : "=v"(ad) : "v"(m[i]), "v"(ad));
                r[i] = *(LAS u32*)(size_t)ad;
            }
            __builtin_amdgcn_sched_barrier(0);
#pragma unroll
            for (int g = 7; g >= 0; --g)
                st_out(o + (size_t)((g >> 2) * 16 + (g & 3)) * GN,
                       __builtin_bit_cast(f32x4, (u32x4){r[g * 4] & m[g * 4], r[g * 4 + 1] & m[g * 4 + 1], r[g * 4 + 2] & m[g * 4 + 2], r[g * 4 + 3] & m[g * 4 + 3]}));
        }
        CFENCE();
    }
}

__device__ __forceinline__ void gemm_wg(const Params& P, LAS char* L, int pm, int lp0, int wave, int lane, u32 call) {
    char* ws = P.me;
    const int wr = wave >> 2, wc = wave & 3;
    const u32 voffA = wave * 2048 + lane * 16, voffB = (wave >> 1) * 8192 + (wave & 1) * 2048 + lane * 16;
    const u32 boff = wc * 4096 + lane * 16, ldw = wave * 2048;
    auto lane_id = []() -> u32 { u32 l; asm volatile("v_mbcnt_lo_u32_b32 %0, -1, 0\n\tv_mbcnt_hi_u32_b32 %0, -1, %0" : "=v"(l)); return l; };
    constexpr size_t KA = 16384, KB = 32768, HA = 64 * KA, HB = 4096;
#define SA(b, h) (((b) * 2 + (h)) * 16384)
#define SB(b, h) ((4 + (b) * 2 + (h)) * 16384)
#define STAGE(bufoff, gbase, voff) asm volatile("s_mov_b32 m0, %2\n\ts_nop 0\n\tglobal_load_lds_dwordx4 %0, %1 sc0 sc1\n\tglobal_load_lds_dwordx4 %0, %1 offset:1024 sc0 sc1" \
        :: "v"(voff), "s"((u64)(gbase)), "s"((u32)(bufoff) + ldw) : "memory")
#define LDA(dst, b, h) do { _Pragma("unroll") for (int m = 0; m < 4; ++m) _Pragma("unroll") for (int k = 0; k < 2; ++k) \
        dst[m][k] = *(const LAS f16x8*)(L + SA(b, h) + ((h) ? aoff1 : aoff0) + m * 2048 + k * 1024); } while (0)
#define LDB(dst, b, h) do { _Pragma("unroll") for (int n = 0; n < 2; ++n) _Pragma("unroll") for (int k = 0; k < 2; ++k) \
        dst[n][k] = *(const LAS f16x8*)(L + SB(b, h) + boff + n * 2048 + k * 1024); } while (0)
#define MMA(ai, bj, At, Bt) do { __builtin_amdgcn_sched_barrier(0);         \
        _Pragma("unroll") for (int k = 0; k < 2; ++k) _Pragma("unroll") for (int mm = 0; mm < 4; ++mm) _Pragma("unroll") for (int nn = 0; nn < 2; ++nn) { \
            const int m = k ? 3 - mm : mm, n = (mm & 1) ? 1 - nn : nn; \
            acc[ai][bj][m][n] = __builtin_amdgcn_mfma_f32_16x16x32_f16(At[m][k], Bt[n][k], acc[ai][bj][m][n], 0, 0, 0); \
            __builtin_amdgcn_sched_barrier(0); } } while (0)
#define WAIT_L(n) asm volatile("s_waitcnt lgkmcnt(" #n ")" ::: "memory")
#define WAIT_L0 __builtin_amdgcn_s_waitcnt(0xC07F)
#define RBAR __builtin_amdgcn_s_barrier()
#define SCHED __builtin_amdgcn_sched_barrier(0)
    auto flag_ptr = [&](int lp, int j, int ent) -> const char* {
        return ws + OFF_FA + (size_t)((call & 1u) * U_PAR + (ent < 32 ? ((pm * 2 + (ent >> 4)) * 8 + j) * 16 + (ent & 15) : U_B + (lp * 8 + j) * 32 + (ent - 32))) * 8; };
    auto poll = [&](int lp, int j) {
        const u32 ln = lane_id();
        const char* f = flag_ptr(lp, j, ln);
        const u64 t0 = now(); u32 sp = 0; u32x2 v;
        for (;;) {
            v = ld8_sys(f);
            if (__builtin_amdgcn_ballot_w64(v.x >= call) == ~0ull) break;
            if (give_up(sp, t0, (u32*)(ws + OFF_TMO), call)) { v.y = 62u; break; }
            __builtin_amdgcn_s_sleep(4);
        }
        asm volatile("ds_write_b64 %0, %1\n\ts_waitcnt lgkmcnt(0)" :: "v"(L_T2 + ln * 8), "v"(v) : "memory");
    };
    auto codes = [&](u32 tb) -> u32 {
        const u32 r = lane_id() & 7;
        const u32 a0 = *(const LAS u32*)(L + tb + wr * 8 + 4 + r * 16), a1 = *(const LAS u32*)(L + tb + 128 + wr * 8 + 4 + r * 16);
        const u32 b = *(const LAS u32*)(L + tb + 256 + wc * 8 + 4 + r * 32);
        return a0 | (a1 << 8) | (b << 16);
    };
    f32x4 acc[2][2][4][2];
    f16x8 At[4][2], B0[2][2], B1[2][2];
    u32 cur = 0, cmax = 0, bad = 0;
    u32 aoff0 = wr * 8192 + lane * 16, aoff1 = aoff0;
    u32 nm, nmn = 0;
    auto changes = [&](u32 nw, u32 old) -> u32 {
        const u32 prev = __builtin_amdgcn_update_dpp((int)__builtin_amdgcn_readlane(old, 7), (int)nw, 0x111, 0xf, 0xf, false);
        return (u32)__builtin_amdgcn_ballot_w64(nw != prev) & 255u; };
#define ADJUST(r, buf) do { if (__builtin_expect((nm >> (r)) & 1u, 0)) { const u32 cn_ = __builtin_amdgcn_readlane(pk, (r)), cb_ = (cn_ >> 16) & 63u; \
            const u32 l16_ = lane_id() * 16; \
            _Pragma("unroll") for (int ai = 0; ai < 2; ++ai) { \
                const u32 ca_ = (cn_ >> (8 * ai)) & 63u, c_ = ca_ + cb_, o_ = (cur >> (8 * ai)) & 255u, x_ = (cmax >> (8 * ai)) & 255u; \
                u32 ab_ = wr * 8192; int sh_ = 0; \
                if (ca_ == 62u || cb_ == 62u) bad = 1; \
                else if (ca_ != 63u && cb_ != 63u) { \
                    if (c_ + 10u < x_) { ab_ = L_ZERO - SA(buf, ai); nm = 255u; nmn |= 1u; }      \
                    else { sh_ = 8 * ((int)o_ - (int)c_); \
                        cur = (cur & ~(255u << (8 * ai))) | (c_ << (8 * ai)); \
                        if (c_ > x_) cmax = (cmax & ~(255u << (8 * ai))) | (c_ << (8 * ai)); } } \
                asm volatile("v_add_u32 %0, %1, %2" : "+v"(ai ? aoff1 : aoff0) : "s"(ab_), "v"(l16_)); \
                if (sh_ != 0) { \
                    asm volatile("s_nop 15" ::: "memory");        \
                    _Pragma("unroll") for (int bj = 0; bj < 2; ++bj) _Pragma("unroll") for (int m = 0; m < 4; ++m) _Pragma("unroll") for (int n = 0; n < 2; ++n) \
                        _Pragma("unroll") for (int e_ = 0; e_ < 4; ++e_) asm volatile("v_ldexp_f32 %0, %0, %1" : "+v"(acc[ai][bj][m][n][e_]) : "s"(sh_)); } } } } while (0)
#pragma unroll
    for (int a = 0; a < 2; ++a)
#pragma unroll
        for (int b = 0; b < 2; ++b)
#pragma unroll
            for (int m = 0; m < 4; ++m)
#pragma unroll
                for (int n = 0; n < 2; ++n) acc[a][b][m][n] = (f32x4){0.f, 0.f, 0.f, 0.f};
    const char* const cA = ws + OFF_A + (size_t)(call & 1u) * U_PAR * 8192 + (size_t)pm * 2 * HA;
    const char* cB = ws + OFF_A + (size_t)((call & 1u) * U_PAR + U_B) * 8192 + (size_t)lp0 * 64 * KB;
    *(LAS u32x4*)(L + L_ZERO + wave * 1024 + lane * 16) = (u32x4){0u, 0u, 0u, 0u};
    if (wave == 0) poll(lp0, 0);
    LGKM0();
    BAR();
    u32 pk = codes(L_T2), pkn = pk;
    nm = changes(pk, pk) | 1u;
    STAGE(SB(0, 0), cB, voffB); STAGE(SA(0, 0), cA, voffA); STAGE(SB(0, 1), cB + HB, voffB); STAGE(SA(0, 1), cA + HA, voffA);
    if (wr == 1) RBAR;
    VMCNT(4); RBAR;
    STAGE(SB(1, 0), cB + KB, voffB); STAGE(SA(1, 0), cA + KA, voffA); STAGE(SB(1, 1), cB + HB + KB, voffB);
    VMCNT(6); RBAR;
#pragma unroll 1
    for (int rd = 0; rd < 3; ++rd) {
        const int lp = lp0 + rd * 12;
        const char* nB = rd < 2 ? cB + 12 * 64 * KB : cB;
#define TRIP(r, a1, a2, b2, HOOK) do { \
            ADJUST(r, 0); \
            LDB(B0, 0, 0); SCHED; LDA(At, 0, 0); STAGE(SA(1, 1), (a1) + HA, voffA); \
            WAIT_L(8); RBAR; WAIT_L0; MMA(0, 0, At, B0); RBAR; SCHED; \
            LDB(B1, 0, 1); STAGE(SB(0, 0), (b2), voffB); \
            RBAR; WAIT_L0; MMA(0, 1, At, B1); RBAR; \
            LDA(At, 0, 1); STAGE(SA(0, 0), (a2), voffA); \
            RBAR; WAIT_L0; MMA(1, 0, At, B0); RBAR; SCHED; \
            STAGE(SB(0, 1), (b2) + HB, voffB); \
            VMCNT(6); RBAR; MMA(1, 1, At, B1); RBAR; \
            ADJUST((r) + 1, 1); \
            LDB(B0, 1, 0); SCHED; LDA(At, 1, 0); STAGE(SA(0, 1), (a2) + HA, voffA); \
            WAIT_L(8); RBAR; WAIT_L0; MMA(0, 0, At, B0); RBAR; SCHED; \
            LDB(B1, 1, 1); STAGE(SB(1, 0), (b2) + KB, voffB); \
            RBAR; WAIT_L0; MMA(0, 1, At, B1); RBAR; \
            LDA(At, 1, 1); STAGE(SA(1, 0), (a2) + KA, voffA); \
            RBAR; WAIT_L0; MMA(1, 0, At, B0); RBAR; SCHED; \
            STAGE(SB(1, 1), (b2) + KB + HB, voffB); \
            HOOK; \
            VMCNT(6); RBAR; MMA(1, 1, At, B1); RBAR; } while (0)
#pragma unroll 1
        for (int j = 0; j < 8; ++j) {
            const char* ga = cA + (size_t)j * 8 * KA; const char* gb = cB + (size_t)j * 8 * KB;
            const char* na = j == 7 ? cA : ga + 8 * KA; const char* nb = j == 7 ? nB : gb + 8 * KB;
            const bool more = j < 7 || rd < 2;
            const int nlp = j < 7 ? lp : lp + 12, nj = (j + 1) & 7;
            TRIP(0, ga + KA, ga + 2 * KA, gb + 2 * KB, do { } while (0));
            TRIP(2, ga + 3 * KA, ga + 4 * KA, gb + 4 * KB,
                 if (__builtin_expect(wave == 0 && more, 0)) { const u32 ln = lane_id(); const char* fp = flag_ptr(nlp, nj, ln * 2);
                     if (ln < 32) asm volatile("s_mov_b32 m0, %1\n\ts_nop 0\n\tglobal_load_lds_dwordx4 %0, off sc0 sc1" :: "v"(fp), "s"(L_T1) : "memory"); });
            TRIP(4, ga + 5 * KA, ga + 6 * KA, gb + 6 * KB, {
                 const u32 tag = *(const LAS u32*)(L + L_T1 + lane_id() * 8);
                 pkn = codes(L_T1);
                 if (__builtin_expect((__builtin_amdgcn_ballot_w64(tag >= call) != ~0ull) & more, 0)) {
                     if (wave == 0) poll(nlp, nj);
                     BAR();
                     pkn = codes(L_T2);
                 }
                 nmn |= changes(pkn, pk); });
            TRIP(6, ga + 7 * KA, na, nb, { pk = pkn; nm = nmn; nmn = 0; });
        }
        if (wave == 0) { STAMP(4 + rd * 2); STAMPV(50 + rd, __builtin_amdgcn_s_memtime()); }
        {
            const int ln = lane_id();
            const int gp = lp < 4 ? 32 + P.rank * 4 + lp : lp - 4;
            GAS float* o = (GAS float*)P.out + (size_t)(pm * 256 + wr * 64 + (ln >> 4) * 4) * GN + gp * 256 + wc * 64 + (ln & 15) * 4;
#pragma unroll
            for (int ai = 0; ai < 2; ++ai) {
                const int sh = 8 * (int)((cur >> (8 * ai)) & 255u) - 272;
#pragma unroll
                for (int m = 0; m < 4; ++m)
#pragma unroll
                    for (int r = 0; r < 4; ++r) {
                        f32x4 v;
#pragma unroll
                        for (int i = 0; i < 4; ++i) {
                            const float y = __builtin_elementwise_maximum(__builtin_ldexpf(acc[ai][i >> 1][m][i & 1][r], sh), 0.0f);
                            v[i] = __uint_as_float(bad ? 0x7fc00000u : __float_as_uint(y) & 0x7fffffffu);
                            acc[ai][i >> 1][m][i & 1][r] = 0.f;
                        }
                        st_out(o + (size_t)(ai * 128 + m * 16 + r) * GN, v);
                    }
            }
            if (lp < 4) {
                VMCNT(0);
                if (ln == 0) st4_sys(ws + OFF_OF + (size_t)((pm * 4 + lp) * 8 + wave) * 4, call);
            }
            cur = 0; cmax = 0; bad = 0; nm |= 1u;
        }
        if (wave == 0) STAMP(5 + rd * 2);
        cB = nB;
    }
    VMCNT(0);
    if (wr == 0) RBAR;
#undef SA
#undef SB
#undef STAGE
#undef LDA
#undef LDB
#undef MMA
#undef ADJUST
#undef TRIP
}

template <int G> __device__ __forceinline__ void pack_dense(LAS char* dense, LAS char* img, u32 lane, u32 n8, u32 o_mid, u32 o_top, u32 o_esc, u32 b2,
                                                            u32* T, u32* T3, u32* bp, u32& tot, u32& e) {
    u32x4 a[G], b[G], lo[G]; u32x2 mid[G]; u32 hi[G][4], ge[G][4], top[G];
#pragma unroll
    for (int i = 0; i < G; ++i) { a[i] = *(LAS u32x4*)(dense + (i * 64 + lane) * 32u); b[i] = *(LAS u32x4*)(dense + (i * 64 + lane) * 32u + 16u); }
#pragma unroll
    for (int i = 0; i < G; ++i) {
        const u32 dd[8] = {a[i].x, a[i].y, a[i].z, a[i].w, b[i].x, b[i].y, b[i].z, b[i].w};
        u32 x[4], c[4], f9[4], len, tp = 0;
#pragma unroll
        for (int k = 0; k < 4; ++k) {
            lo[i][k] = __builtin_amdgcn_perm(dd[2 * k + 1], dd[2 * k], 0x05040100u);
            hi[i][k] = __builtin_amdgcn_perm(dd[2 * k + 1], dd[2 * k], 0x07060302u);
            x[k] = PW((u16x2)(PU(hi[i][k]) - PU(b2)));
            if (i == G - 1) x[k] = i * 64 + lane < n8 ? x[k] : 0u;
        }
        pk_classify<7>(x, c, ge[i], T[i], T3[i], len);
#pragma unroll
        for (int k = 0; k < 4; ++k) { f9[k] = (c[k] << 7) | (x[k] & 0x007f007fu); tp |= ((c[k] >> 1) & 0x00010001u) << k; }
        mid[i] = (u32x2){__builtin_amdgcn_perm(f9[1], f9[0], 0x06040200u), __builtin_amdgcn_perm(f9[3], f9[2], 0x06040200u)};
        top[i] = tp | (tp >> 12);
        bp[i] = tot; tot += len;
    }
#pragma unroll
    for (int i = 0; i < G; ++i) {
        const u32 g = i * 64 + lane;
        if (i < G - 1 || g < n8) {
            *(LAS u32x4*)(img + 256u + g * 16u) = lo[i];
            *(LAS u32x2*)(img + o_mid + g * 8u) = mid[i];
            *(LAS unsigned char*)(img + o_top + g) = (unsigned char)top[i];
        }
    }
#pragma unroll
    for (int i = 0; i < G; ++i) emit_escapes(ge[i], hi[i], (i * 64 + lane) * 8u, img + o_esc, (8192u - o_esc) >> 2, e);
}
__device__ __forceinline__ void packer(const Params& P, LAS char* L, int pw, int wave, int lane, u32 call) {
    char* ws = P.me;
    LAS char* dense = L + wave * PK_B;
    LAS char* img = dense + PK_IMG;
    const auto rs = __builtin_amdgcn_make_buffer_rsrc((void*)P.out, 0, (u32)GM * GN * 4u, 0x00020000);
    auto wait_item = [&](int it) -> u32 {
        const u64 t0 = now(); u32 sp = 0;
        while (uni(ld4_sys(ws + OFF_OF + it * 4)) < call) {
            if (give_up(sp, t0, (u32*)(ws + OFF_TMO), call)) return 0x7fc00000u;
            __builtin_amdgcn_s_sleep(8);
        }
        return 0u;
    };
    u32x4 nx[8];
    auto fetch_block = [&](int it, int su) {
        const int tile = it >> 3, wq = it & 7;
        const u32 so = ((u32)((tile >> 2) * 256 + (su >> 1) * 128 + (wq >> 2) * 64 + (su & 1) * 32) * GN + 8192 + P.rank * 1024 + (tile & 3) * 256 + (wq & 3) * 64) * 4u;
#pragma unroll
        for (int g = 0; g < 8; ++g) nx[g] = __builtin_amdgcn_raw_buffer_load_b128(rs, ((lane >> 4) * 4 * GN + (lane & 15) * 4) * 4, so + ((g >> 2) * 16 + (g & 3)) * GN * 4, 17);
    };
    auto publish = [&](int it, u32 ea, u32 eb) {
        if (lane < 32 && (lane >> 2) != P.rank)
            st16_sys(peer_ws(P, lane >> 2) + OFF_REC + (call & 1u) * REC_PAR + P.rank * REC_SRC + REC_OUT + it * 64 + (lane & 3) * 16, (u32x4){call, ea, call, eb});
    };
    {
        const char* bx = ws + OFF_BX + (call & 1u) * X_PAR + (size_t)8 * 8 * 32768 + lane * 16;
        u32 acc = 0;
#pragma unroll 1
        for (int i = 0; i < 6; ++i) {
            const int sl = pw * 3 + (i >> 1);
            u32x4 t[7];
#pragma unroll
            for (int g = 0; g < 7; ++g) t[g] = *(const GAS u32x4*)(bx + (size_t)sl * 16384 + ((i & 1) * 7 + g) * 1024);
#pragma unroll
            for (int g = 0; g < 7; ++g) acc ^= t[g].x;
        }
        asm volatile("" :: "v"(acc));
    }
    u32 nanm = wait_item(pw), nann = 0, off16 = 0, ea = 0, eb = 0;
    fetch_block(pw, 0);
#pragma unroll
    for (int g = 0; g < 8; ++g) asm volatile("" : "+v"(nx[g]));
#pragma unroll 1
    for (int bk = 0; bk < N_OUT_ITEMS / 16; ++bk) {
        {
            const int it = pw + (bk >> 2) * 64, su = bk & 3;
            char* og = ws + OFF_O + (size_t)it * QSLOT;
            u32 v[32]; u32 mx = 0;
#pragma unroll
            for (int r = 0; r < 32; ++r) { v[r] = nx[r >> 2][r & 3] | nanm; mx = max(mx, v[r]); }
            { const int bn = bk + 1 < N_OUT_ITEMS / 16 ? bk + 1 : bk; fetch_block(pw + (bn >> 2) * 64, bn & 3); }
            mx = wave_umax(mx) >> 16;
            const u32 b16 = mx >= 511u ? mx - 511u : 0u, b2 = b16 * 0x10001u;
            u32 f[32], mk0 = 0, mk1 = 0;
#pragma unroll
            for (int r = 0; r < 32; ++r) asm("v_min_u32 %0, 1, %1" : "=v"(f[r]) : "v"(v[r]));
#pragma unroll
            for (int r = 0; r < 16; ++r) asm("v_lshl_or_b32 %0, %2, %4, %0\n\tv_lshl_or_b32 %1, %3, %5, %1" : "+v"(mk0), "+v"(mk1) : "v"(f[r]), "v"(f[16 + r]), "n"(r), "n"(16 + r));
            const u32 mk = mk0 | mk1, pc = __builtin_popcount(mk), inc = wave_scan(pc), n = __builtin_amdgcn_readlane(inc, 63);
            {
                u32 ad = wave * PK_B + (inc - pc) * 4u;
#pragma unroll
                for (int r = 0; r < 32; r += 8) {
                    u32 a[8]; u64 m[8];
#pragma unroll
                    for (int i = 0; i < 8; ++i) { a[i] = ad; asm("v_lshl_add_u32 %0, %1, 2, %2" : "=v"(ad) : "v"(f[r + i]), "v"(a[i])); m[i] = __builtin_amdgcn_ballot_w64(v[r + i] != 0u); }
                    asm volatile("s_mov_b64 exec, %16\n\tds_write_b32 %0, %8\n\ts_mov_b64 exec, %17\n\tds_write_b32 %1, %9\n\t"
                                 "s_mov_b64 exec, %18\n\tds_write_b32 %2, %10\n\ts_mov_b64 exec, %19\n\tds_write_b32 %3, %11\n\t"
                                 "s_mov_b64 exec, %20\n\tds_write_b32 %4, %12\n\ts_mov_b64 exec, %21\n\tds_write_b32 %5, %13\n\t"
                                 "s_mov_b64 exec, %22\n\tds_write_b32 %6, %14\n\ts_mov_b64 exec, %23\n\tds_write_b32 %7, %15\n\ts_mov_b64 exec, -1"
                                 :: "v"(a[0]), "v"(a[1]), "v"(a[2]), "v"(a[3]), "v"(a[4]), "v"(a[5]), "v"(a[6]), "v"(a[7]),
                                    "v"(v[r]), "v"(v[r + 1]), "v"(v[r + 2]), "v"(v[r + 3]), "v"(v[r + 4]), "v"(v[r + 5]), "v"(v[r + 6]), "v"(v[r + 7]),
                                    "s"(m[0]), "s"(m[1]), "s"(m[2]), "s"(m[3]), "s"(m[4]), "s"(m[5]), "s"(m[6]), "s"(m[7]) : "memory");
                }
            }
            if (lane < 8) *(LAS u32*)(dense + (n + lane) * 4u) = b16 << 16;
            *(LAS u32*)(img + lane * 4) = mk;
            const u32 nr = (n & 511u) <= 64u ? n & 511u : 0u, n8 = (n - nr + 7u) >> 3, G = (n8 + 63u) >> 6;
            const u32 o_mid = 256u + n8 * 16u, o_top = o_mid + n8 * 8u, o_raw = o_top + ((n8 + 3u) & ~3u), o_esc = o_raw + nr * 4u;
            CFENCE();
            if (nr) *(LAS u32*)(img + o_raw + lane * 4u) = *(LAS u32*)(dense + n8 * 32u + lane * 4u);
            CFENCE();
            u32 e = 0, tot = 0, T[4] = {0u, 0u, 0u, 0u}, T3[4] = {0u, 0u, 0u, 0u}, bp[4] = {0u, 0u, 0u, 0u};
            if (G == 3u) pack_dense<3>(dense, img, lane, n8, o_mid, o_top, o_esc, b2, T, T3, bp, tot, e);
            else if (G == 2u) pack_dense<2>(dense, img, lane, n8, o_mid, o_top, o_esc, b2, T, T3, bp, tot, e);
            else if (G == 1u) pack_dense<1>(dense, img, lane, n8, o_mid, o_top, o_esc, b2, T, T3, bp, tot, e);
            else if (G == 4u) pack_dense<4>(dense, img, lane, n8, o_mid, o_top, o_esc, b2, T, T3, bp, tot, e);
#pragma unroll
            for (int g = 0; g < 8; ++g) asm volatile("" : "+v"(nx[g]));
            VMCNT(0);
            if (su == 0) { if (bk) publish(it - 64, ea, eb); off16 = 0; }
            if (su == 2 && it + 64 < N_OUT_ITEMS) nann = wait_item(it + 64);
            const u32 binc = wave_scan(tot), all2 = __builtin_amdgcn_readlane(binc, 63), o_sec = o_esc + e * 4u;
            u32 size16 = (o_sec + ((pk_sum(all2) + 7u) >> 3) + 15u) >> 4;
            if (size16 >= 512u) {
                size16 = 512u;
#pragma unroll
                for (int g = 0; g < 8; ++g) st16_sys(og + off16 * 16u + g * 1024 + lane * 16, (u32x4){v[g * 4], v[g * 4 + 1], v[g * 4 + 2], v[g * 4 + 3]});
            } else {
#pragma unroll
                for (int i = 0; i < 6; ++i) *(LAS u32*)(img + o_sec + i * 256 + lane * 4) = 0u;
                CFENCE();
#pragma unroll
                for (int i = 0; i < 4; ++i) if (i < (int)G) {
                    const u32 bq = bp[i] + binc - tot + (all2 << 16);
                    or_bits(img + o_sec, bq & 0xffffu, T[i]);
                    or_bits(img + o_sec, bq >> 16, T3[i]);
                }
                CFENCE();
#pragma unroll 1
                for (u32 g = lane; g < size16 + lane; g += 256u) {
                    u32x4 t[4];
#pragma unroll
                    for (int i = 0; i < 4; ++i) t[i] = *(LAS u32x4*)(img + (g + i * 64) * 16u);
#pragma unroll
                    for (int i = 0; i < 4; ++i) if (g + i * 64 < size16) st16_sys(og + (off16 + g + i * 64) * 16u, t[i]);
                }
            }
            if ((lane & 3) == su) { ea = size16 | (n << 10); eb = (e & 0xfffu) | (b16 << 12); }
            off16 += size16;
            if (su == 3) nanm = nann;
            LGKM0();
            CFENCE();
        }
        if (bk == 4) STAMP(16);
    }
    VMCNT(0);
    publish(pw + N_OUT_ITEMS - 64, ea, eb);
    STAMP(17);
}

__global__ __launch_bounds__(NTHR, 2) __attribute__((amdgpu_waves_per_eu(2, 2))) void fused_kernel(Params P) {
    extern __shared__ __attribute__((aligned(16))) char smem[];
    LAS char* L = (LAS char*)smem;
    asm volatile("" :: "s"(P.x), "s"(P.w), "s"(P.me), "s"(P.rank));
    const int tid = threadIdx.x, lane = tid & 63, wave = __builtin_amdgcn_readfirstlane(tid >> 6);
    const int b = blockIdx.x, xg = b & 7, s = b >> 3;
    const bool fetch = s < FS;
    constexpr int NF = FS * 8;
    if (wave == 0) STAMP(0);
    GAS u32* const cnt = (GAS u32*)(P.me + OFF_CNT + (b * 8 + wave) * 8);
    constexpr bool diag = false;
    const int pidx = b - (NF + 120);
    constexpr int NU = 112 - ITEM0;
    const int q0 = ITEM0 + (b - NF);
    const bool loaded = !fetch && !diag && q0 < 112 && wave < 2;
    f32x4 v[16];
    if (wave == 7 && (unsigned)pidx < (unsigned)NPUSH && !diag) {
        load_x(P, pidx, 0, lane, v);
        const u32 craw = __hip_atomic_load(cnt, __ATOMIC_RELAXED, __HIP_MEMORY_SCOPE_AGENT);
        __builtin_amdgcn_sched_barrier(0);
        BAR();
        __builtin_amdgcn_sched_barrier(0);
        STAMP(22);
        push_task(P, pidx, lane, craw, v);
        STAMP(23);
    } else {
        if (loaded) { if (q0 < 32) load_x(P, q0 * 2 + (wave & 1), 0, lane, v); else load_w(P, (q0 < 96 ? q0 + 32 : q0 - 96) * 2 + (wave & 1), 0, lane, v); }
        __builtin_amdgcn_sched_barrier(0);
        if (tid < 256) lds_st(L_ITEM + tid * 4, !fetch ? 0u : tid == 18 ? 6u : tid >= 24 && tid < 32 ? (u32)(tid - 24) : 0u);
        LGKM0();
        BAR();
    }
    if (!fetch && !diag) {
        const u64 tstart = now();
        const bool pack = s >= FS + GS;
        auto fetch_task = [&](int q) {
            const u64 go = pack ? (u64)PACE4 : (u64)(PACE2 + (q - 112) * PACE3 / 112);
            while (now() - tstart < go) __builtin_amdgcn_s_sleep(4);
            int j, idx; bool isA;
            task_of(q, wave & 1, j, idx, isA);
            if (isA) load_x(P, idx, j, lane, v); else load_w(P, idx, j, lane, v);
        };
        int qn = 112 + (pack ? GS * 8 + xg : (b - NF + GS * 8 - NU) % (GS * 8)) * 4 + (wave >> 1), q = q0;
        const int qs = pack ? 32 : N_PAIRS;
        u32 lq = L_ITEM;
        if (!loaded) { q = qn; qn += qs; lq += 64; fetch_task(q); }
#pragma unroll 1
        for (;;) {
            stage_task(P, L + wave * 4096, lq, lq + 48 + (wave >> 1) * 4, q, wave & 1, lane, cnt, v);
            if (qn >= N_PAIRS) break;
            q = qn; qn += qs; lq += 64;
            fetch_task(q);
        }
    }
    const u32 call = uni(__hip_atomic_load(cnt, __ATOMIC_RELAXED, __HIP_MEMORY_SCOPE_AGENT)) + (diag ? 0u : 1u);
    if (lane == 0) st4_sys((void*)cnt, call);
    STAMP(fetch ? 1 : (s >= FS + GS ? 19 : 18));
    if (fetch) {
        const int src = (P.rank + 1 + s) & 7;
        if (wave == 0) { loader(P, src, lane, call); STAMP(2); }
        else if (wave == 1) scout(P, src, xg, lane, call);
        else { decoder(P, L, wave - 2, lane, call); STAMP(8 + wave); }
    } else if (s < FS + GS) {
        const int g = s - FS;
        gemm_wg(P, L, (xg >> 1) * 4 + g / 6, (xg & 1) * 6 + g % 6, wave, lane, call);
    } else {
        BAR();
        packer(P, L, wave * 8 + xg, wave, lane, call);
    }
}

}

extern "C" void kernel_launch(int rank, int nranks, ncclComm_t comm, hipStream_t stream,
                              void* const* d_in, const int* in_sizes, int n_in,
                              void* d_out, int out_size,
                              const int* in_dtypes, int out_dtype,
                              void* const* d_out_all,
                              void* const* d_ws_all, size_t ws_bytes) {
    (void)comm; (void)in_sizes; (void)n_in; (void)out_size; (void)in_dtypes; (void)out_dtype; (void)d_out_all;
    if (nranks != NR || (unsigned)rank >= (unsigned)NR || ws_bytes < WS_NEED) { fprintf(stderr, "unsupported configuration\n"); return; }
    static bool ready[NR];
    if (!ready[rank]) {
        HIP_CHECK(hipFuncSetAttribute((const void*)fused_kernel, hipFuncAttributeMaxDynamicSharedMemorySize, (int)LDS_BYTES));
        ready[rank] = true;
    }
    Params P;
    P.x = (const float*)d_in[0]; P.w = (const float*)d_in[1]; P.out = (float*)d_out;
    for (int i = 0; i < NR; ++i) P.ws[i] = (char*)d_ws_all[i];
    P.me = P.ws[rank]; P.rank = rank; P.pad = 0;
    hipLaunchKernelGGL(fused_kernel, dim3(NWG), dim3(NTHR), LDS_BYTES, stream, P);
    HIP_CHECK(hipGetLastError());
}
```

```cpp
#include <hip/hip_runtime.h>
#include <rccl/rccl.h>
#include <cstdio>

#define HIP_CHECK(expr) do { hipError_t _e = (expr); if (_e != hipSuccess) { \
    fprintf(stderr, "HIP error %s at %s:%d\n", hipGetErrorString(_e), __FILE__, __LINE__); \
    return; } } while (0)

typedef unsigned u32;
typedef unsigned long long u64;
typedef u32 u32x2 __attribute__((ext_vector_type(2)));
typedef u32 u32x4 __attribute__((ext_vector_type(4)));
typedef float f32x4 __attribute__((ext_vector_type(4)));
typedef _Float16 f16x8 __attribute__((ext_vector_type(8)));
#define LAS __attribute__((address_space(3)))
#define GAS __attribute__((address_space(1)))

namespace {

constexpr int NR = 8, GM = 4096, GN = 16384, KL = 512;
constexpr int NWG = 256, NTHR = 512;
constexpr int FS = 7, GS = 24;
constexpr int LPP = 8;
constexpr int WIN = 8;
constexpr int NLP = 36;
constexpr int N_IN_ITEMS = 832, N_OUT_ITEMS = 512, N_ITEMS = N_IN_ITEMS + N_OUT_ITEMS;
constexpr u32 R2_ITEM = 688, ROT = 20;
constexpr u32 HELP_LAG = 5;
constexpr int NPUSH = 36;
constexpr int N_PAIRS = 1280, ITEM0 = NPUSH / 2;
constexpr int PACE2 = 150, PACE3 = 350, PACE4 = 800;
static_assert(NPUSH % 2 == 0 && NPUSH >= 8 && NPUSH <= 64, "only x blocks of the first K group");
constexpr u64 DEADLINE = 30000000ull;

constexpr size_t OFF_CNT = 0;
constexpr size_t OFF_TMO = 16384;
constexpr size_t OFF_TICK = 16384 + 256;
constexpr size_t OFF_OF = 20480;
constexpr u32 U_B = 4096, U_PAR = U_B + NLP * 256, N_UNITS = 2 * U_PAR;
constexpr size_t OFF_FA = 32768;
constexpr size_t OFF_REC = 1ull << 20;
constexpr u32 REC_SRC = 65536, REC_OUT = 16384;
constexpr size_t REC_PAR = 1ull << 20;
constexpr size_t OFF_A = 4ull << 20;
constexpr size_t OFF_AX = OFF_A + (size_t)N_UNITS * 8192;
constexpr size_t OFF_BX = OFF_AX + (4ull << 20);
constexpr size_t X_PAR = 20ull << 20;
constexpr size_t OFF_O = OFF_AX + 2 * X_PAR;
constexpr size_t QSLOT = 32768;
constexpr size_t WS_NEED = OFF_O + N_OUT_ITEMS * QSLOT;
static_assert(NR * REC_SRC <= REC_PAR && N_IN_ITEMS * 16 <= REC_OUT && REC_OUT + N_OUT_ITEMS * 64 <= REC_SRC && OFF_FA + N_UNITS * 8 <= OFF_REC, "workspace map");

constexpr u32 RING = 65536;
constexpr u32 L_SCR = RING, SCR_B = 8448;
constexpr u32 L_DESC = L_SCR + 6 * SCR_B, NQ = 64;
constexpr u32 L_ITEM = L_DESC + NQ * 16;
constexpr u32 L_FCTL = L_ITEM + 64;
constexpr u32 L_LAND = L_FCTL + 64, L_START = L_LAND + 256;
constexpr u32 L_T1 = 131072, L_T2 = L_T1 + 512, L_ZERO = L_T2 + 512;
constexpr u32 PK_B = 19456, PK_IMG = 8320;
constexpr u32 LDS_BYTES = 8 * PK_B;
static_assert(L_ITEM + 1024 <= LDS_BYTES && L_START + 256 <= L_ITEM + 1024 && L_ZERO + 8192 <= LDS_BYTES && LDS_BYTES <= 163840, "LDS map");

struct Params {
    const float* x; const float* w; float* out;
    char* me;
    char* ws[NR];
    int rank; int pad;
};

#define VMCNT(n) asm volatile("s_waitcnt vmcnt(" #n ")" ::: "memory")
#define LGKM0() asm volatile("s_waitcnt lgkmcnt(0)" ::: "memory")
#define CFENCE() asm volatile("" ::: "memory")
#define BAR() do { CFENCE(); __builtin_amdgcn_s_barrier(); CFENCE(); } while (0)

__device__ __forceinline__ void st16_sys(void* p, u32x4 v) {
    asm volatile("global_store_dwordx4 %0, %1, off sc0 sc1\n\ts_nop 1" :: "v"(p), "v"(v) : "memory");
}
__device__ __forceinline__ void st_out(GAS float* p, f32x4 v) {
    asm volatile("global_store_dwordx4 %0, %1, off sc0 sc1 nt\n\ts_nop 1" :: "v"(p), "v"(v) : "memory");
}
__device__ __forceinline__ void st4_sys(void* p, u32 v) {
    asm volatile("global_store_dword %0, %1, off sc0 sc1" :: "v"(p), "v"(v) : "memory");
}
__device__ __forceinline__ void st8_sys(void* p, u32x2 v) {
    asm volatile("global_store_dwordx2 %0, %1, off sc0 sc1" :: "v"(p), "v"(v) : "memory");
}
__device__ __forceinline__ u32 ld4_sys(const void* p) {
    u32 v; asm volatile("global_load_dword %0, %1, off sc0 sc1\n\ts_waitcnt vmcnt(0)" : "=&v"(v) : "v"(p) : "memory"); return v;
}
__device__ __forceinline__ u32x2 ld8_sys(const void* p) {
    u32x2 v; asm volatile("global_load_dwordx2 %0, %1, off sc0 sc1\n\ts_waitcnt vmcnt(0)" : "=&v"(v) : "v"(p) : "memory"); return v;
}
__device__ __forceinline__ void lds_st(u32 a, u32 v) { asm volatile("ds_write_b32 %0, %1" :: "v"(a), "v"(v) : "memory"); }
__device__ __forceinline__ u32 lds_ld(u32 a) {
    u32 v; asm volatile("ds_read_b32 %0, %1\n\ts_waitcnt lgkmcnt(0)" : "=&v"(v) : "v"(a) : "memory"); return v;
}
__device__ __forceinline__ u32x4 lds_ld16(u32 a) {
    u32x4 v; asm volatile("ds_read_b128 %0, %1\n\ts_waitcnt lgkmcnt(0)" : "=&v"(v) : "v"(a) : "memory"); return v;
}
__device__ __forceinline__ u32 uni(u32 v) { return __builtin_amdgcn_readfirstlane(v); }
__device__ __forceinline__ u32 lds_ldu(u32 a) { return uni(lds_ld(a)); }
__device__ __forceinline__ u64 now() { return __builtin_amdgcn_s_memrealtime(); }
__device__ __forceinline__ u32 prefix(u64 m) { return __builtin_amdgcn_mbcnt_hi((u32)(m >> 32), __builtin_amdgcn_mbcnt_lo((u32)m, 0u)); }

__device__ __forceinline__ u32 wave_umax(u32 v) {
    v = max(v, (u32)__builtin_amdgcn_update_dpp(0, (int)v, 0x111, 0xf, 0xf, true));
    v = max(v, (u32)__builtin_amdgcn_update_dpp(0, (int)v, 0x112, 0xf, 0xf, true));
    v = max(v, (u32)__builtin_amdgcn_update_dpp(0, (int)v, 0x114, 0xf, 0xf, true));
    v = max(v, (u32)__builtin_amdgcn_update_dpp(0, (int)v, 0x118, 0xf, 0xf, true));
    v = max(v, (u32)__builtin_amdgcn_update_dpp(0, (int)v, 0x142, 0xa, 0xf, false));
    v = max(v, (u32)__builtin_amdgcn_update_dpp(0, (int)v, 0x143, 0xc, 0xf, false));
    return __builtin_amdgcn_readlane(v, 63);
}
__device__ __forceinline__ u32 wave_scan(u32 v) {
    v += (u32)__builtin_amdgcn_update_dpp(0, (int)v, 0x111, 0xf, 0xf, true);
    v += (u32)__builtin_amdgcn_update_dpp(0, (int)v, 0x112, 0xf, 0xf, true);
    v += (u32)__builtin_amdgcn_update_dpp(0, (int)v, 0x114, 0xf, 0xf, true);
    v += (u32)__builtin_amdgcn_update_dpp(0, (int)v, 0x118, 0xf, 0xf, true);
    v += (u32)__builtin_amdgcn_update_dpp(0, (int)v, 0x142, 0xa, 0xf, false);
    v += (u32)__builtin_amdgcn_update_dpp(0, (int)v, 0x143, 0xc, 0xf, false);
    return v;
}
typedef unsigned short u16x2 __attribute__((ext_vector_type(2)));
typedef short i16x2 __attribute__((ext_vector_type(2)));
#define PU(x) __builtin_bit_cast(u16x2, (u32)(x))
#define PI(x) __builtin_bit_cast(i16x2, (u32)(x))
#define PW(x) __builtin_bit_cast(u32, (x))
template <int W> __device__ __forceinline__ u32 pk_offsets(const u32* m, u32* ac) {
    u32 tot;
    ac[0] = 0;
    asm("v_pk_mul_lo_u16 %0, %4, %8 op_sel_hi:[1,0]\n\tv_pk_mad_u16 %1, %5, %8, %0 op_sel_hi:[1,0,1]\n\t"
        "v_pk_mad_u16 %2, %6, %8, %1 op_sel_hi:[1,0,1]\n\tv_pk_mad_u16 %3, %7, %8, %2 op_sel_hi:[1,0,1]"
        : "=&v"(ac[1]), "=&v"(ac[2]), "=&v"(ac[3]), "=&v"(tot) : "v"(m[0]), "v"(m[1]), "v"(m[2]), "v"(m[3]), "n"(-W));
    return tot;
}
__device__ __forceinline__ u32 pk_is3(u32 c) { return PW((i16x2)(PI(c + 0x7ffd7ffdu) >> (i16x2){15, 15})); }
__device__ __forceinline__ u32 pk_sum(u32 a) { return (a & 0xffffu) + (a >> 16); }
__device__ __forceinline__ u32 pk_join(const u32* sv, const u32* ac, u32 acc) {
    const u32 tt = sv[0] | PW((u16x2)(PU(sv[1]) << PU(ac[1]))) | PW((u16x2)(PU(sv[2]) << PU(ac[2]))) | PW((u16x2)(PU(sv[3]) << PU(ac[3])));
    return (tt & 0xffffu) | ((tt >> 16) << (acc & 31u));
}
__device__ __forceinline__ u32 pk_split(u32 t, u32 acc) { return __builtin_amdgcn_perm(t >> (acc & 31u), t, 0x05040100u); }
__device__ __forceinline__ u32 pk_field(u32 tt, u32 sh, u32 m, u32 mask) { return PW((u16x2)(PU(tt) >> PU(sh))) & m & mask; }
__device__ __forceinline__ u32 pk_expand(u32 sv, u32 k, u32 base) {
    u32 r;
    asm("v_pk_min_u16 %0, %1, 1 op_sel_hi:[1,0]\n\tv_pk_mad_u16 %0, %0, 3, %1 op_sel_hi:[1,0,1]\n\tv_pk_mad_u16 %0, %0, %2, %3"
        : "=&v"(r) : "v"(sv), "s"(k), "v"(base));
    return r;
}
template <int SH> __device__ __forceinline__ void pk_classify(const u32* x, u32* c, u32* g, u32& T2, u32& T3, u32& len) {
    u32 m2[4], m3[4], ac[4], s2[4], s3[4];
#pragma unroll
    for (int k = 0; k < 4; ++k) {
        const u32 o = PW((i16x2)(PI(x[k]) >> (i16x2){SH, SH}));
        const u32 sv = PW(__builtin_elementwise_max((i16x2)(-PI(o)), (i16x2){0, 0}));
        c[k] = PW(__builtin_elementwise_min(PU(o), (u16x2){3, 3}));
        s2[k] = PW(__builtin_elementwise_min(PU(sv), (u16x2){3, 3}));
        s3[k] = (sv - s2[k]) & 0x00070007u;
        m2[k] = pk_is3(c[k]); m3[k] = pk_is3(s2[k]);
        g[k] = PW(__builtin_elementwise_sub_sat((u16x2)(PU(o) + (u16x2){10, 10}), (u16x2){13, 13}));
    }
    const u32 a2 = pk_offsets<2>(m2, ac);
    T2 = pk_join(s2, ac, a2);
    const u32 a3 = pk_offsets<3>(m3, ac);
    T3 = pk_join(s3, ac, a3);
    len = pk_sum(a2) | (pk_sum(a3) << 16);
}
__device__ __forceinline__ void or_bits(LAS char* buf, u32 bp, u32 t) {
    const u64 v = (u64)t << (bp & 31u);
    LAS u32* a = (LAS u32*)(buf + (bp >> 5) * 4u);
    __hip_atomic_fetch_or(a, (u32)v, __ATOMIC_RELAXED, __HIP_MEMORY_SCOPE_WAVEFRONT);
    __hip_atomic_fetch_or(a + 1, (u32)(v >> 32), __ATOMIC_RELAXED, __HIP_MEMORY_SCOPE_WAVEFRONT);
}
__device__ __forceinline__ void emit_escapes(const u32* g, const u32* raw, u32 idx0, LAS char* rec, u32 cap, u32& e) {
    if (__builtin_amdgcn_ballot_w64((g[0] | g[1] | g[2] | g[3]) != 0u) == 0ull) return;
    u32 gm = 0;
#pragma unroll
    for (int k = 0; k < 4; ++k) gm |= PW(__builtin_elementwise_min(PU(g[k]), (u16x2){1, 1})) << k;
#pragma unroll 1
    for (;;) {
        const u64 em = __builtin_amdgcn_ballot_w64(gm != 0u);
        if (em == 0ull) break;
        const u32 b = __builtin_ctz(gm | 0x80000000u), k = b & 3u;
        const u32 m1 = (u32)__builtin_amdgcn_sbfe((int)b, 0, 1), m2 = (u32)__builtin_amdgcn_sbfe((int)b, 1, 1);
        const u32 r01 = (raw[1] & m1) | (raw[0] & ~m1), r23 = (raw[3] & m1) | (raw[2] & ~m1), r = (r23 & m2) | (r01 & ~m2);
        if (gm != 0u && e < cap) *(LAS u32*)(rec + (e + prefix(em)) * 4u) = (idx0 + k * 2u + (b >> 4)) | ((r >> (b & 16u)) << 16);
        gm &= gm - 1u;
        e += __builtin_popcountll(em);
    }
}
typedef float f32x2 __attribute__((ext_vector_type(2)));
typedef _Float16 f16x2 __attribute__((ext_vector_type(2)));
__device__ __forceinline__ u32 cvt2(float a, float b, float sc) { return __builtin_bit_cast(u32, __builtin_convertvector((f32x2){a, b} * sc, f16x2)); }
__device__ __forceinline__ char* peer_ws(const Params& P, int p) {
    u64 wp = (u64)P.ws[0];
#pragma unroll
    for (int i = 1; i < NR; ++i) { u64 q = (u64)P.ws[i]; asm volatile("" : "+s"(q)); wp = p == i ? q : wp; }
    return (char*)wp;
}


__device__ __forceinline__ bool give_up(u32& spins, u64 t0, u32* tmo, u32 call) {
    if ((++spins & 63u) != 0u) return false;
    if (uni(ld4_sys(tmo)) == call) return true;
    if (now() - t0 > DEADLINE) { st4_sys(tmo, call); return true; }
    return false;
}

__device__ __forceinline__ float block_scale(const f32x4 (&v)[16], u32& code, u32& hm) {
    float m0 = 0.f, m1 = 0.f, m2 = 0.f, m3 = 0.f;
#pragma unroll
    for (int i = 0; i < 16; i += 2)
        asm("v_max3_f32 %0, %0, |%4|, |%5|\n\tv_max3_f32 %1, %1, |%6|, |%7|\n\tv_max3_f32 %2, %2, |%8|, |%9|\n\tv_max3_f32 %3, %3, |%10|, |%11|"
            : "+v"(m0), "+v"(m1), "+v"(m2), "+v"(m3)
            : "v"(v[i][0]), "v"(v[i][1]), "v"(v[i][2]), "v"(v[i][3]), "v"(v[i + 1][0]), "v"(v[i + 1][1]), "v"(v[i + 1][2]), "v"(v[i + 1][3]));
    const u32 mb = wave_umax(max(max(__float_as_uint(m0), __float_as_uint(m1)), max(__float_as_uint(m2), __float_as_uint(m3))));
    code = ((mb >> 23) + 2u) >> 3; code = code < 2u ? 2u : code;
    const float sc = __uint_as_float((263u - 8u * code) << 23);
    hm = uni(__builtin_bit_cast(unsigned short, (_Float16)(__uint_as_float(mb) * sc)));
    if (mb == 0u) code = 63u;
    return sc;
}
__device__ __forceinline__ void load_x(const Params& P, int idx, int j, int lane, f32x4 (&v)[16]) {
    const GAS float* p = (const GAS float*)P.x + (size_t)(idx * 64 + (lane & 15)) * KL + j * 64 + (lane >> 4) * 4;
#pragma unroll
    for (int i = 0; i < 16; ++i) v[i] = __builtin_nontemporal_load((const GAS f32x4*)(p + (i >> 2) * 16 * KL + (i & 3) * 16));
}
__device__ __forceinline__ void push_task(const Params& P, int idx, int lane, u32 craw, const f32x4 (&v)[16]) {
    asm volatile("" :: "s"(P.ws[0]), "s"(P.ws[1]), "s"(P.ws[2]), "s"(P.ws[3]), "s"(P.ws[4]), "s"(P.ws[5]), "s"(P.ws[6]), "s"(P.ws[7]));
    __builtin_amdgcn_sched_barrier(0);
    u32 code, hm;
    const float sc = block_scale(v, code, hm);
    const u32 call = uni(craw) + 1u;
    const u32 dest = (call & 1u) * U_PAR + ((idx >> 1) * 64 + P.rank) * 2 + (idx & 1);
    const u32 voff = (u32)OFF_A + dest * 8192u + lane * 16u;
#pragma unroll
    for (int f = 0; f < 8; ++f) {
        const u32x4 hv = {cvt2(v[2 * f][0], v[2 * f][1], sc), cvt2(v[2 * f][2], v[2 * f][3], sc), cvt2(v[2 * f + 1][0], v[2 * f + 1][1], sc), cvt2(v[2 * f + 1][2], v[2 * f + 1][3], sc)};
        VMCNT(8);
        asm volatile("global_store_dwordx4 %0, %1, %2 sc0 sc1\n\tglobal_store_dwordx4 %0, %1, %3 sc0 sc1\n\tglobal_store_dwordx4 %0, %1, %4 sc0 sc1\n\t"
                     "global_store_dwordx4 %0, %1, %5 sc0 sc1\n\tglobal_store_dwordx4 %0, %1, %6 sc0 sc1\n\tglobal_store_dwordx4 %0, %1, %7 sc0 sc1\n\t"
                     "global_store_dwordx4 %0, %1, %8 sc0 sc1\n\tglobal_store_dwordx4 %0, %1, %9 sc0 sc1\n\ts_nop 1"
                     :: "v"(voff + f * 1024), "v"(hv), "s"((u64)P.ws[0]), "s"((u64)P.ws[1]), "s"((u64)P.ws[2]), "s"((u64)P.ws[3]),
                        "s"((u64)P.ws[4]), "s"((u64)P.ws[5]), "s"((u64)P.ws[6]), "s"((u64)P.ws[7]) : "memory");
        __builtin_amdgcn_sched_barrier(0);
    }
    VMCNT(0);
    if (lane < NR) st8_sys(peer_ws(P, lane) + OFF_FA + (size_t)dest * 8, (u32x2){call, code});
}
__device__ __forceinline__ void load_w(const Params& P, int idx, int j, int lane, f32x4 (&v)[16]) {
    const GAS float* p = (const GAS float*)P.w + (size_t)(j * 64 + (lane >> 4) * 4) * GN + idx * 64 + (lane & 15) * 4;
#pragma unroll
    for (int i = 0; i < 16; ++i) v[i] = __builtin_nontemporal_load((const GAS f32x4*)(p + (size_t)((i >> 2) * 16 + (i & 3)) * GN));
}
__device__ __forceinline__ void task_of(int q, int half, int& j, int& idx, bool& isA) {
    if (q < 896) { j = q / 112; const int u = q % 112; isA = u < 32; idx = isA ? u : (u < 96 ? u + 32 : u - 96); }
    else { const int u = q - 896, v = u % 192; j = v / 24; isA = false; idx = 16 + (u / 192) * 24 + v % 24; }
    idx = idx * 2 + half;
}
__device__ __forceinline__ void stage_task(const Params& P, LAS char* scr, u32 lquad, u32 lpair, int q, int half, int lane, GAS u32* cnt, const f32x4 (&v)[16]) {
    int j, idx; bool isA;
    task_of(q, half, j, idx, isA);
    const u32 craw = __hip_atomic_load(cnt, __ATOMIC_RELAXED, __HIP_MEMORY_SCOPE_AGENT);
    *(LAS u32x4*)(scr + lane * 16) = (u32x4){0u, 0u, 0u, 0u}; *(LAS u32x4*)(scr + 1024 + lane * 16) = (u32x4){0u, 0u, 0u, 0u};
    __builtin_amdgcn_sched_barrier(0);
    u32 code, hm;
    const float sc = block_scale(v, code, hm);
    const u32 bb = hm >= 4095u ? hm - 4095u : 0u, b2 = bb * 0x10001u;
    u32x4 p[8];
    if (isA) {
#pragma unroll
        for (int f = 0; f < 8; ++f)
#pragma unroll
            for (int k = 0; k < 4; ++k) p[f][k] = cvt2(v[f * 2 + (k >> 1)][(k & 1) * 2], v[f * 2 + (k >> 1)][(k & 1) * 2 + 1], sc);
        CFENCE();
    } else {
#pragma unroll
        for (int f = 0; f < 8; ++f)
#pragma unroll
            for (int k = 0; k < 4; ++k) p[f][k] = cvt2(v[(f & 1) * 8 + k * 2][f >> 1], v[(f & 1) * 8 + k * 2 + 1][f >> 1], sc);
        asm volatile("s_nop 0" ::: "memory");
    }
    char* ws = P.me;
    const u32 call = uni(craw) + 1u;
    const bool all = isA || idx < 128;
    const int owner = (idx - 128) >> 4, lp = idx < 128 ? 4 + (idx >> 2) : (idx >> 2) & 3;
    const bool local = all || owner == P.rank, exprt = all || owner != P.rank;
    const u32 dest = (call & 1u) * U_PAR + (isA ? (((idx >> 1) * 8 + j) * 8 + P.rank) * 2 + half : U_B + (((lp * 8 + j) * 8 + P.rank) * 4 + (idx & 3)));
    u32 size16 = 0;
    if (exprt) {
        u32 w[32], T2[8], T3[8], len[8], e = 0, tot = 0;
#pragma unroll
        for (int f = 0; f < 8; ++f) {
            u32 x[4], c[4], g[4];
            const u32 h[4] = {p[f][0], p[f][1], p[f][2], p[f][3]};
#pragma unroll
            for (int k = 0; k < 4; ++k) x[k] = PW((u16x2)(PU(h[k] & 0x7fff7fffu) - PU(b2)));
            pk_classify<10>(x, c, g, T2[f], T3[f], len[f]);
#pragma unroll
            for (int k = 0; k < 4; ++k) w[f * 4 + k] = (h[k] & 0x80008000u) | (c[k] << 10) | (x[k] & 0x03ff03ffu);
            tot += len[f];
            asm volatile("" : "+v"(w[f * 4]), "+v"(w[f * 4 + 1]), "+v"(w[f * 4 + 2]), "+v"(w[f * 4 + 3]), "+v"(T2[f]), "+v"(T3[f]));
            emit_escapes(g, h, f * 512 + lane * 8u, scr + 4, 384u, e);
        }
        const u32 inc = wave_scan(tot), all2 = __builtin_amdgcn_readlane(inc, 63);
        size16 = (6656u + 4u + e * 4u + ((pk_sum(all2) + 7u) >> 3) + 15u) >> 4;
        if (size16 >= 512u) size16 = 512u;
        u32 off16 = 0;
        if (half == 0) { if (lane == 0) lds_st(lpair, size16); }
        else {
            const u64 t0 = now(); u32 sp = 0;
            while ((off16 = lds_ldu(lpair)) == 0u) {
                if ((++sp & 1023u) == 0u && now() - t0 > 4 * DEADLINE) { st4_sys(ws + OFF_TMO, call); break; }
                __builtin_amdgcn_s_sleep(1);
            }
        }
        char* d = ws + (call & 1u) * X_PAR + off16 * 16u + lane * 16 +
                  (isA ? OFF_AX + (size_t)((idx >> 1) * 8 + j) * 16384 : OFF_BX + (size_t)(((idx >> 2) * 8 + j) * 2 + ((idx >> 1) & 1)) * 16384);
        if (size16 == 512u) {
#pragma unroll
            for (int f = 0; f < 8; ++f) st16_sys(d + f * 1024, p[f]);
        } else {
            u32 bp = (e + 1u) * 32u * 0x10001u + (all2 << 16) + inc - tot;
#pragma unroll
            for (int f = 0; f < 8; f += 2) {
                or_bits(scr, bp & 0xffffu, T2[f] | (T2[f + 1] << (len[f] & 31u)));
                or_bits(scr, bp >> 16, T3[f]);
                bp += len[f];
                or_bits(scr, bp >> 16, T3[f + 1]);
                bp += len[f + 1];
            }
            if (lane == 0) *(LAS u32*)scr = e;
#pragma unroll
            for (int i = 0; i < 24; ++i) w[i] |= (w[26 + (i >> 2)] << (12 - 3 * (i & 3))) & 0x70007000u;
#pragma unroll
            for (int i = 0; i < 6; ++i) w[24 + i / 3] |= (w[26 + i] >> (1 + i % 3)) & (0x40004000u >> (i % 3));
#pragma unroll
            for (int f = 0; f < 6; ++f) st16_sys(d + f * 1024, (u32x4){w[f * 4], w[f * 4 + 1], w[f * 4 + 2], w[f * 4 + 3]});
            st8_sys(d + 6144 - lane * 8, (u32x2){w[24], w[25]});
            CFENCE();
            for (u32 g = lane; g + 416u < size16; g += 64u) st16_sys(d + 6656 + (g - lane) * 16u, *(LAS u32x4*)(scr + g * 16u));
        }
    }
    if (local) {
        char* d = ws + OFF_A + (size_t)dest * 8192 + lane * 16;
#pragma unroll
        for (int f = 0; f < 8; ++f) st16_sys(d + f * 1024, p[f]);
    }
    VMCNT(0);
    if (local && lane == 0) st8_sys(ws + OFF_FA + (size_t)dest * 8, (u32x2){call, code});
    if (exprt) {
        const int hb = (lp % 12) * 2 + ((idx >> 1) & 1);
        const int item = isA ? j * 56 + (idx >> 1) : (lp < 12 ? j * 56 + 32 + hb : 256 + (lp / 12) * 192 + j * 24 + hb);
        const u32 info = size16 | (bb << 10) | (code << 25);
        const size_t ro = OFF_REC + (call & 1u) * REC_PAR + P.rank * REC_SRC;
        if (q < 112) {
            if (lane < NR && lane != P.rank && (all || lane == owner)) st8_sys(peer_ws(P, lane) + ro + item * 16 + half * 8, (u32x2){call, info});
        } else {
            u32 old = 0;
            if (lane == 0) asm volatile("ds_write_b32 %1, %2\n\tds_add_rtn_u32 %0, %3, %4\n\ts_waitcnt lgkmcnt(0)"
                                        : "=&v"(old) : "v"(lquad + ((item & 3) * 2 + half) * 4), "v"(info), "v"(lquad + 32), "v"(1u) : "memory");
            if (uni(old) == 7u) {
                u32x2 in2; asm volatile("ds_read_b64 %0, %1\n\ts_waitcnt lgkmcnt(0)" : "=&v"(in2) : "v"(lquad + (lane & 3) * 8) : "memory");
                const int r = lane >> 2;
                if (lane < 32 && r != P.rank && (all || r == owner)) st16_sys(peer_ws(P, r) + ro + ((item & ~3) + (lane & 3)) * 16, (u32x4){call, in2.x, call, in2.y});
            }
        }
    }
    CFENCE();
}

__device__ __forceinline__ void scout(const Params& P, int src, int li, int lane, u32 call) {
    char* ws = P.me;
    u32* tmo = (u32*)(ws + OFF_TMO);
    const char* recs = ws + OFF_REC + (call & 1u) * REC_PAR;
    const u32 xpar = (call & 1u) * (u32)X_PAR;
    const u64 base = (u64)P.ws[src];
    const u64 tbase = (u64)call << 12;
    u32 seq = 32 + li,
        isrc = src, cand = src, pieces = 0, nq = 0, k = 0;
    __builtin_amdgcn_s_setprio(2);
    if (lane == 0) asm volatile("global_atomic_umax_x2 %0, %1, off\n\ts_waitcnt vmcnt(0)" :: "v"(ws + OFF_TICK + src * 256), "v"(tbase + (ITEM0 + LPP)) : "memory");
    const u32 s8 = lane & 7, ahead = ((((src - s8 - 1) & 7) - ((P.rank - s8 - 1) & 7) + 7) % 7);
    const u32 hfrom = (s8 != (u32)P.rank && s8 != (u32)src && ahead >= 2u) ? R2_ITEM + ahead * ROT + 5u : ~0u;
    while (seq < (u32)N_ITEMS || isrc != (u32)src) {
        const bool isin = seq < (u32)N_IN_ITEMS;
        u32 item = seq;
        if (isin && seq >= R2_ITEM) { item += ((P.rank - isrc - 1) & 7) * ROT; if (item >= (u32)N_IN_ITEMS) item -= N_IN_ITEMS - R2_ITEM; }
        const char* rp = recs + isrc * REC_SRC + (isin ? item * 16u + (lane & 1) * 8u : REC_OUT + ((item - N_IN_ITEMS) & 511u) * 64u + s8 * 8u);
        u32x2 r; bool dead = false, relay = false;
        u64 tk; u32 prog = 0;
        asm volatile("global_load_dwordx2 %0, %1, off sc0 sc1" : "=v"(r) : "v"(rp) : "memory");
        if (lane == 0) asm volatile("global_atomic_add_x2 %0, %1, %2, off sc0" : "=v"(tk) : "v"(ws + OFF_TICK + cand * 256), "v"(1ull) : "memory");
        const bool look = li == src && seq - (R2_ITEM + 2 * ROT) < (u32)N_IN_ITEMS + 80u - (R2_ITEM + 2 * ROT);
        if (look && lane < 8) asm volatile("global_atomic_add %0, %1, %2, off sc0" : "=v"(prog) : "v"(ws + OFF_TICK + s8 * 256), "v"(0u) : "memory");
        u32 j = 0, u = 0;
        if (item < 448u) { j = item / 56u; u = item % 56u; }
        else if (isin) { const u32 v = item - 448u, w = v % 192u; j = w / 24u; u = 56u + (v / 192u) * 24u + w % 24u; }
        const u32 dest = (call & 1u) * U_PAR + (u >= 32u ? U_B + ((((u - 32u) >> 1) * 8 + j) * 8 + isrc) * 4 + (u & 1u) * 2 : ((u * 8 + j) * 8 + isrc) * 2);
        asm volatile("s_waitcnt vmcnt(0)" : "+v"(r), "+v"(tk), "+v"(prog) :: "memory");
        if (isrc != (u32)src && isin && seq >= R2_ITEM) {
            const u32x2 f = ld8_sys((const char*)base + OFF_FA + (size_t)(dest + (lane & 1)) * 8);
            relay = __builtin_amdgcn_ballot_w64(f.x == call) == ~0ull;
            if (relay) r = (u32x2){call, 512u | (f.y << 25)};
        }
        if (seq < (u32)N_ITEMS && __builtin_amdgcn_ballot_w64(r.x >= call) != ~0ull) {
            const u64 t0 = now(); u32 sp = 0;
            do {
                if (give_up(sp, t0, tmo, call)) { dead = true; break; }
                __builtin_amdgcn_s_sleep(2);
                r = ld8_sys(rp);
            } while (__builtin_amdgcn_ballot_w64(r.x >= call) != ~0ull);
        }
        u32 next = uni((u32)tk) - (u32)tbase;
        if (next < 32u + LPP) next -= LPP;
        const u32 nsrc = cand;
        cand = src;
        if (look) {
            const u32 ps = prog - (u32)tbase;
            const u64 m = __builtin_amdgcn_ballot_w64(lane < 8 && ps >= hfrom && ps + 12u <= (u32)N_IN_ITEMS && ps + HELP_LAG <= (u32)__builtin_amdgcn_readlane(ps, src));
            if (m) cand = __builtin_ctzll(m);
        }
        if (seq >= (u32)N_ITEMS) { seq = next; isrc = nsrc; continue; }
        u32 soff, bytes, nsub; u32x4 d;
        if (isin) {
            if (relay) soff = (u32)OFF_A + dest * 8192u;
            else if (u >= 32u) { const u32 lp = (u - 32u) >> 1, gp = lp < 4u ? 32u + P.rank * 4u + lp : lp - 4u; soff = (u32)OFF_BX + xpar + (gp * 8 + j) * 32768u + (u & 1u) * 16384u; }
            else soff = (u32)OFF_AX + xpar + (u * 8 + j) * 16384u;
            const u32 s = lane & 1, sz = min(r.y & 1023u, 512u), sb = dead ? 0u : sz * 16u;
            const u32 b0 = __builtin_amdgcn_readlane(sb, 0);
            bytes = b0 + __builtin_amdgcn_readlane(sb, 1); nsub = 2;
            d.x = pieces * 1024u + (s ? b0 : 0u); d.y = pieces + (((s ? bytes : b0) + 1023u) >> 10);
            d.z = dead ? (62u << 12) | (2u << 24) : sz | ((r.y >> 25) << 12); d.w = ((r.y >> 10) & 0x7fffu) | ((dest + s) << 16);
        } else {
            const u32 o = item - N_IN_ITEMS;
            soff = (u32)OFF_O + o * (u32)QSLOT;
            const u32 a = __builtin_amdgcn_ds_bpermute((lane & 3) * 8, r.y), b = __builtin_amdgcn_ds_bpermute((lane & 3) * 8 + 4, r.y);
            const u32 sb = dead ? 0u : min(a & 1023u, 512u) * 16u, n = min((a >> 10) & 0xfffu, 2048u), e = b & 0xfffu, b16 = (b >> 12) & 0x7fffu;
            const u32 s0 = __builtin_amdgcn_readlane(sb, 0), s1 = s0 + __builtin_amdgcn_readlane(sb, 1), s2 = s1 + __builtin_amdgcn_readlane(sb, 2);
            const u32 off = (lane & 3) == 0 ? 0u : (lane & 3) == 1 ? s0 : (lane & 3) == 2 ? s1 : s2;
            bytes = s2 + __builtin_amdgcn_readlane(sb, 3); nsub = 4;
            d.x = pieces * 1024u + off; d.y = pieces + ((off + sb + 1023u) >> 10); d.z = n | (e << 12) | ((dead ? 3u : sb == 8192u ? 4u : 1u) << 24);
            d.w = b16 | ((o * 4u + (lane & 3)) << 16) | (isrc << 29);
        }
        const u32 np = (bytes + 1023u) >> 10;
        u64 ibase = base;
        if (__builtin_expect(isrc != (u32)src && !relay, 0)) ibase = (u64)P.ws[isrc & 7];
        {
            u64 t0 = 0; u32 sp = 0;
            for (;;) {
                u32 nx = lds_ld(lane < 6 ? L_FCTL + 32 + lane * 4 : L_START);
                u32 qm = __builtin_amdgcn_readlane(nx, 0);
#pragma unroll
                for (int i = 1; i < 6; ++i) qm = min(qm, (u32)__builtin_amdgcn_readlane(nx, i));
                const u32 first = qm < nq ? lds_ldu(L_DESC + (qm % NQ) * 16) >> 10 : pieces;
                if ((u32)__builtin_amdgcn_readlane(nx, 6) + 1u >= k && pieces + np - first <= RING / 1024 && (qm >= nq || nq + nsub - qm <= NQ)) break;
                if ((++sp & 1023u) == 0u) { if (!t0) t0 = now(); else if (now() - t0 > 4 * DEADLINE) { st4_sys(tmo, call); break; } }
                __builtin_amdgcn_s_sleep(4);
            }
        }
        if (lane < (int)nsub) asm volatile("ds_write_b128 %0, %1" :: "v"(L_DESC + ((nq + lane) % NQ) * 16), "v"(d) : "memory");
        nq += nsub;
        if (lane == 0) {
            const u64 ad = ibase + soff;
            const u32x4 it = {(u32)ad, bytes, (u32)(ad >> 32), k + 1};
            asm volatile("ds_write_b128 %0, %1" :: "v"(L_ITEM + (k & 3u) * 16), "v"(it) : "memory");
            lds_st(L_FCTL + 4, nq);
        }
        pieces += np; ++k;
        seq = next; isrc = nsrc;
    }
    if (lane == 0) {
        const u32x4 it = {0u, 0u, 0u, (k + 1) | 0x80000000u};
        asm volatile("ds_write_b128 %0, %1" :: "v"(L_ITEM + (k & 3u) * 16), "v"(it) : "memory");
        lds_st(L_FCTL + 4, nq | 0x80000000u);
    }
}

__device__ __forceinline__ void loader(const Params& P, int lane, u32 call) {
    const u32 voff = lane * 16, vland = L_LAND + lane * 4, vstart = L_START + lane * 4;
    u32* tmo = (u32*)(P.me + OFF_TMO);
    int issued = 0, landed = 0;
    u32 k = 0;
    __builtin_amdgcn_s_setprio(3);
#define PIECE(sp) do { asm volatile("s_mov_b32 m0, %2\n\ts_waitcnt vmcnt(%3)\n\tglobal_load_lds_dwordx4 %0, %1 sc0 sc1" \
                                    :: "v"(voff), "s"(sp), "s"(((u32)issued * 1024u) & (RING - 1)), "n"(WIN - 1) : "memory"); } while (0)
    u32x4 it = lds_ld16(L_ITEM);
    for (;;) {
        ++k;
        if (__builtin_expect(uni(it.w) != k, 0)) {
            const u32 ia = L_ITEM + ((k - 1) & 3u) * 16;
#define STEP(n) if (uni(it.w) != k) { asm volatile("s_waitcnt vmcnt(%0)" :: "n"(n) : "memory"); landed = max(landed, issued - (n)); lds_st(vland, (u32)landed); it = lds_ld16(ia); }
            STEP(WIN - 1) STEP(WIN - 2) STEP(WIN - 3) STEP(WIN - 4) STEP(WIN - 5) STEP(WIN - 6) STEP(0)
#undef STEP
            const u64 t0 = now(); u32 sp = 0;
            while ((uni(it.w) & 0x7fffffffu) != k) {
                if ((++sp & 1023u) == 0u && now() - t0 > 4 * DEADLINE) { st4_sys(tmo, call); return; }
                __builtin_amdgcn_s_sleep(1);
                it = lds_ld16(ia);
            }
            if (uni(it.w) != k) return;
        }
        u64 sp = ((u64)uni(it.z) << 32) | uni(it.x);
        u32 left = uni(it.y);
        asm volatile("s_nop 4" : "+s"(sp));
        lds_st(vstart, k);
#pragma unroll 1
        for (; left > 1024u; left -= 1024u, sp += 1024) {
            PIECE(sp);
            landed = max(landed, issued - (WIN - 1));
            lds_st(vland, (u32)landed);
            ++issued;
        }
        asm volatile("ds_read_b128 %0, %1" : "=v"(it) : "v"(L_ITEM + (k & 3u) * 16) : "memory");
        if (left) {
            if (voff < left) PIECE(sp);
            landed = max(landed, issued - (WIN - 1));
            lds_st(vland, (u32)landed);
            ++issued;
        }
        asm volatile("s_waitcnt lgkmcnt(0)" : "+v"(it) :: "memory");
    }
#undef PIECE
}

template <int G> __device__ __forceinline__ void expand_dense(LAS char* L, LAS char* scr, u32 lane, u32 n8, u32 o_lo, u32 o_mid, u32 o_top, u32 o_sec, u32 b2) {
    u32x4 lo[G]; u32x2 mid[G]; u32 top[G], u[G][4], fl[G][4], ac[G][4], lw[G], t0[G], t1[G], bp[G], tot = 0;
#pragma unroll
    for (int i = 0; i < G; ++i) {
        const u32 g = i * 64 + lane;
        mid[i] = *(LAS u32x2*)(L + ((o_mid + g * 8u) & (RING - 1)));
        top[i] = *(LAS unsigned char*)(L + ((o_top + g) & (RING - 1)));
    }
#pragma unroll
    for (int i = 0; i < G; ++i) lo[i] = *(LAS u32x4*)(L + ((o_lo + (i * 64 + lane) * 16u) & (RING - 1)));
#pragma unroll
    for (int i = 0; i < G; ++i) {
        if (i == G - 1) top[i] = i * 64 + lane < n8 ? top[i] : 0u;
        const u32 tt = (top[i] | (top[i] << 12)) << 8;
#pragma unroll
        for (int k = 0; k < 4; ++k) {
            u[i][k] = __builtin_amdgcn_perm(0u, mid[i][k >> 1], (k & 1) ? 0x0c030c02u : 0x0c010c00u) | ((tt >> k) & 0x01000100u);
            fl[i][k] = PW((i16x2)(PI(u[i][k] + 0x7e807e80u) >> (i16x2){15, 15}));
        }
        lw[i] = pk_offsets<2>(fl[i], ac[i]);
        bp[i] = tot; tot += pk_sum(lw[i]);
    }
    const u32 inc = wave_scan(tot);
#pragma unroll
    for (int i = 0; i < G; ++i) {
        bp[i] += inc - tot;
        const u32 a = o_sec + (bp[i] >> 5) * 4u;
        t0[i] = *(LAS u32*)(L + (a & (RING - 1))); t1[i] = *(LAS u32*)(L + ((a + 4u) & (RING - 1)));
    }
    u32 s2[G][4];
    tot = 0;
#pragma unroll
    for (int i = 0; i < G; ++i) {
        const u32 TT = pk_split(__builtin_amdgcn_alignbit(t1[i], t0[i], bp[i]), lw[i]);
#pragma unroll
        for (int k = 0; k < 4; ++k) { s2[i][k] = pk_field(TT, ac[i][k], fl[i][k], 0x00030003u); fl[i][k] = pk_is3(s2[i][k]); }
        lw[i] = pk_offsets<3>(fl[i], ac[i]);
        bp[i] = tot; tot += pk_sum(lw[i]);
    }
    const u32 base = wave_scan(tot) - tot + __builtin_amdgcn_readlane(inc, 63);
#pragma unroll
    for (int i = 0; i < G; ++i) {
        bp[i] += base;
        const u32 a = o_sec + (bp[i] >> 5) * 4u;
        t0[i] = *(LAS u32*)(L + (a & (RING - 1))); t1[i] = *(LAS u32*)(L + ((a + 4u) & (RING - 1)));
    }
#pragma unroll
    for (int i = 0; i < G; ++i) {
        const u32 TT = pk_split(__builtin_amdgcn_alignbit(t1[i], t0[i], bp[i]), lw[i]);
        u32 r[8];
#pragma unroll
        for (int k = 0; k < 4; ++k) {
            const u32 h = pk_expand(s2[i][k] + pk_field(TT, ac[i][k], fl[i][k], 0x00070007u), 0xff80ff80u, u[i][k] + b2);
            r[2 * k] = __builtin_amdgcn_perm(h, lo[i][k], 0x05040100u); r[2 * k + 1] = __builtin_amdgcn_perm(h, lo[i][k], 0x07060302u);
        }
        *(LAS u32x4*)(scr + (i * 64 + lane) * 32u) = (u32x4){r[0], r[1], r[2], r[3]};
        *(LAS u32x4*)(scr + (i * 64 + lane) * 32u + 16u) = (u32x4){r[4], r[5], r[6], r[7]};
    }
}

__device__ __forceinline__ void decoder(const Params& P, LAS char* L, int dw, int lane, u32 call) {
    char* ws = P.me;
    u32* tmo = (u32*)(ws + OFF_TMO);
    LAS char* scr = L + L_SCR + dw * SCR_B;
    u32 qn = dw, pend = 0;
    auto flush = [&]() { if (pend) { VMCNT(0); if (lane == 0) st8_sys(ws + OFF_FA + (size_t)(pend & 0xffffu) * 8, (u32x2){call, (pend >> 16) & 63u}); pend = 0; } };
    auto release = [&]() { if (lane == 0) asm volatile("ds_add_rtn_u32 %0, %1, %2\n\ts_waitcnt lgkmcnt(0)\n\tds_write_b32 %3, %0"
                                                       : "=&v"(qn) : "v"(L_FCTL + 8), "v"(1u), "v"(L_FCTL + 32 + dw * 4) : "memory"); };
    for (;;) {
        const u32 q = uni(qn);
        u32x4 dv;
        {
            u64 t0 = 0; u32 sp = 0;
            for (;;) {
                u32 nq, ld;
                asm volatile("ds_read_b32 %0, %3\n\tds_read_b128 %1, %4\n\tds_read_b32 %2, %5\n\ts_waitcnt lgkmcnt(0)"
                             : "=&v"(nq), "=&v"(dv), "=&v"(ld) : "v"(L_FCTL + 4), "v"(L_DESC + (q % NQ) * 16), "v"(L_LAND) : "memory");
                nq = uni(nq);
                if ((nq & 0x7fffffffu) > q && uni(ld) >= uni(dv.y)) break;
                flush();
                if ((nq & 0x7fffffffu) <= q && (nq >> 31)) return;
                if ((++sp & 1023u) == 0u) { if (!t0) t0 = now(); else if (now() - t0 > 4 * DEADLINE) { st4_sys(tmo, call); return; } }
                __builtin_amdgcn_s_sleep(1);
            }
        }
        const u32 pos = uni(dv.x), z = uni(dv.z), wv = uni(dv.w);
        const u32 type = z >> 24, dest = wv >> 16;
        if (type == 0u || type == 2u) {
            if (type == 0u) {
                const u32 sz = z & 0xfffu, b2 = (wv & 0xffffu) * 0x10001u, pl = pos + lane * 16u;
                char* d = ws + OFF_A + (size_t)dest * 8192;
                u32x4 c[8];
#pragma unroll
                for (int i = 0; i < 6; ++i) c[i] = *(LAS u32x4*)(L + ((pl + i * 1024u) & (RING - 1)));
                if (sz == 512u) {
                    c[6] = *(LAS u32x4*)(L + ((pl + 6144u) & (RING - 1))); c[7] = *(LAS u32x4*)(L + ((pl + 7168u) & (RING - 1)));
                    LGKM0();
#pragma unroll
                    for (int i = 0; i < 8; ++i) asm volatile("" : "+v"(c[i]));
                    release();
                    flush();
#pragma unroll
                    for (int f = 0; f < 8; ++f) st16_sys(d + f * 1024 + lane * 16, c[f]);
                } else {
                    u32x2 c6 = *(LAS u32x2*)(L + ((pos + 6144u + lane * 8u) & (RING - 1)));
                    for (u32 g = lane; g + 416u < sz; g += 64u) *(LAS u32x4*)(scr + g * 16u) = *(LAS u32x4*)(L + ((pos + 6656u + g * 16u) & (RING - 1)));
                    LGKM0();
#pragma unroll
                    for (int i = 0; i < 6; ++i) asm volatile("" : "+v"(c[i]));
                    asm volatile("" : "+v"(c6));
                    release();
                    const u32 e = min(uni(*(LAS u32*)scr), 384u);
                    u32 w[32], fl[32], ac[32], len[8], lo[8], tot = 0;
#pragma unroll
                    for (int i = 0; i < 24; ++i) w[i] = c[i >> 2][i & 3];
                    w[24] = c6.x; w[25] = c6.y;
#pragma unroll
                    for (int i = 0; i < 6; ++i) {
                        u32 t = (w[24 + i / 3] << (1 + i % 3)) & 0x80008000u;
#pragma unroll
                        for (int k = 0; k < 4; ++k) t |= (w[i * 4 + k] >> (12 - 3 * k)) & (0x00070007u << (3 * k));
                        w[26 + i] = t;
                    }
#pragma unroll
                    for (int f = 0; f < 8; ++f) {
#pragma unroll
                        for (int k = 0; k < 4; ++k) fl[f * 4 + k] = PW((i16x2)(PI((w[f * 4 + k] & 0x0c000c00u) + 0x74007400u) >> (i16x2){15, 15}));
                        lo[f] = pk_offsets<2>(fl + f * 4, ac + f * 4);
                        len[f] = pk_sum(lo[f]); tot += len[f];
                    }
                    const u32 inc = wave_scan(tot);
                    u32 bp = (e + 1u) * 32u + inc - tot;
                    u32 T[8], s2[32];
#pragma unroll
                    for (int f = 0; f < 8; f += 2) {
                        const LAS u32* a = (const LAS u32*)(scr + (bp >> 5) * 4u);
                        T[f] = __builtin_amdgcn_alignbit(a[1], a[0], bp);
                        T[f + 1] = T[f] >> len[f];
                        bp += len[f] + len[f + 1];
                    }
                    tot = 0;
#pragma unroll
                    for (int f = 0; f < 8; ++f) {
                        const u32 TT = pk_split(T[f], lo[f]);
#pragma unroll
                        for (int k = 0; k < 4; ++k) { const int i = f * 4 + k; s2[i] = pk_field(TT, ac[i], fl[i], 0x00030003u); fl[i] = pk_is3(s2[i]); }
                        lo[f] = pk_offsets<3>(fl + f * 4, ac + f * 4);
                        len[f] = pk_sum(lo[f]); tot += len[f];
                    }
                    flush();
                    bp = (e + 1u) * 32u + __builtin_amdgcn_readlane(inc, 63) + wave_scan(tot) - tot;
#pragma unroll
                    for (int f = 0; f < 8; ++f) {
                        const LAS u32* a = (const LAS u32*)(scr + (bp >> 5) * 4u);
                        T[f] = __builtin_amdgcn_alignbit(a[1], a[0], bp);
                        bp += len[f];
                    }
#pragma unroll
                    for (int f = 0; f < 8; ++f) {
                        const u32 TT = pk_split(T[f], lo[f]);
                        u32x4 o;
#pragma unroll
                        for (int k = 0; k < 4; ++k) {
                            const int i = f * 4 + k;
                            o[k] = pk_expand(s2[i] + pk_field(TT, ac[i], fl[i], 0x00070007u), 0xfc00fc00u, PW((u16x2)(PU(w[i] & 0x8fff8fffu) + PU(b2))));
                        }
                        st16_sys(d + f * 1024 + lane * 16, o);
                    }
                    for (u32 g = lane; g < e; g += 64u) {
                        const u32 rr = *(LAS u32*)(scr + 4u + g * 4u);
                        asm volatile("global_store_short_d16_hi %0, %1, off sc0 sc1" :: "v"(d + (rr & 4095u) * 2u), "v"(rr) : "memory");
                    }
                }
            } else { release(); flush(); }
            pend = dest | (((z >> 12) & 63u) << 16) | 0x80000000u;
            continue;
        }
        flush();
        const u32 tile = (dest >> 5) & 63u, wq = (dest >> 2) & 7u, su = dest & 3u;
        const u32 row0 = (tile >> 2) * 256u + (su >> 1) * 128u + (wq >> 2) * 64u + (su & 1u) * 32u + (lane >> 4) * 4u;
        const u32 col0 = 8192u + (dest >> 13) * 1024u + (tile & 3u) * 256u + (wq & 3u) * 64u + (lane & 15) * 4u;
        GAS float* o = (GAS float*)P.out + (size_t)row0 * GN + col0;
        if (type == 3u) {
            release();
            const float qn = __builtin_nanf("");
#pragma unroll
            for (int g = 0; g < 8; ++g) *(GAS f32x4*)(o + (size_t)((g >> 2) * 16 + (g & 3)) * GN) = (f32x4){qn, qn, qn, qn};
            continue;
        }
        if (type == 4u) {
            u32x4 c[8];
#pragma unroll
            for (int g = 0; g < 8; ++g) c[g] = *(LAS u32x4*)(L + ((pos + g * 1024u + lane * 16u) & (RING - 1)));
            LGKM0();
#pragma unroll
            for (int g = 0; g < 8; ++g) asm volatile("" : "+v"(c[g]));
            release();
#pragma unroll
            for (int g = 0; g < 8; ++g) st_out(o + (size_t)((g >> 2) * 16 + (g & 3)) * GN, __builtin_bit_cast(f32x4, c[g]));
            continue;
        }
        const u32 n = z & 0xfffu, e = (z >> 12) & 0xfffu, b2 = (wv & 0xffffu) * 0x10001u;
        const u32 nr = (n & 511u) <= 64u ? n & 511u : 0u, n8 = (n - nr + 7u) >> 3, G = (n8 + 63u) >> 6;
        u32 mk = *(LAS u32*)(L + ((pos + lane * 4u) & (RING - 1)));
        const u32 o_lo = pos + 256u, o_mid = o_lo + n8 * 16u, o_top = o_mid + n8 * 8u, o_raw = o_top + ((n8 + 3u) & ~3u);
        const u32 o_esc = o_raw + nr * 4u, o_sec = o_esc + e * 4u;
        if (nr) *(LAS u32*)(scr + n8 * 32u + lane * 4u) = *(LAS u32*)(L + ((o_raw + lane * 4u) & (RING - 1)));
        if (G == 3u) expand_dense<3>(L, scr, lane, n8, o_lo, o_mid, o_top, o_sec, b2);
        else if (G == 2u) expand_dense<2>(L, scr, lane, n8, o_lo, o_mid, o_top, o_sec, b2);
        else if (G == 1u) expand_dense<1>(L, scr, lane, n8, o_lo, o_mid, o_top, o_sec, b2);
        else if (G == 4u) expand_dense<4>(L, scr, lane, n8, o_lo, o_mid, o_top, o_sec, b2);
        CFENCE();
#pragma unroll 1
        for (u32 g = lane; g < e; g += 64u) {
            const u32 rr = *(LAS u32*)(L + ((o_esc + g * 4u) & (RING - 1)));
            *(LAS unsigned short*)(scr + (rr & 2047u) * 4u + 2u) = (unsigned short)(rr >> 16);
        }
        LGKM0();
        asm volatile("" : "+v"(mk));
        release();
        {
            u32 ad = L_SCR + dw * SCR_B + wave_scan(__builtin_popcount(mk)) * 4u;
            u32 r[32], m[32];
#pragma unroll
            for (int i = 31; i >= 0; --i) {
                m[i] = (u32)__builtin_amdgcn_sbfe((int)mk, i, 1);
                asm("v_lshl_add_u32 %0, %1, 2, %2" : "=v"(ad) : "v"(m[i]), "v"(ad));
                r[i] = *(LAS u32*)(size_t)ad;
            }
            __builtin_amdgcn_sched_barrier(0);
#pragma unroll
            for (int g = 7; g >= 0; --g)
                st_out(o + (size_t)((g >> 2) * 16 + (g & 3)) * GN,
                       __builtin_bit_cast(f32x4, (u32x4){r[g * 4] & m[g * 4], r[g * 4 + 1] & m[g * 4 + 1], r[g * 4 + 2] & m[g * 4 + 2], r[g * 4 + 3] & m[g * 4 + 3]}));
        }
        CFENCE();
    }
}

__device__ __forceinline__ void gemm_wg(const Params& P, LAS char* L, int pm, int lp0, int wave, int lane, u32 call) {
    char* ws = P.me;
    const int wr = wave >> 2, wc = wave & 3;
    const u32 voffA = wave * 2048 + lane * 16, voffB = (wave >> 1) * 8192 + (wave & 1) * 2048 + lane * 16;
    const u32 boff = wc * 4096 + lane * 16, ldw = wave * 2048;
    auto lane_id = []() -> u32 { u32 l; asm volatile("v_mbcnt_lo_u32_b32 %0, -1, 0\n\tv_mbcnt_hi_u32_b32 %0, -1, %0" : "=v"(l)); return l; };
    constexpr size_t KA = 16384, KB = 32768, HA = 64 * KA, HB = 4096;
#define SA(b, h) (((b) * 2 + (h)) * 16384)
#define SB(b, h) ((4 + (b) * 2 + (h)) * 16384)
#define STAGE(bufoff, gbase, voff) asm volatile("s_mov_b32 m0, %2\n\ts_nop 0\n\tglobal_load_lds_dwordx4 %0, %1 sc0 sc1\n\tglobal_load_lds_dwordx4 %0, %1 offset:1024 sc0 sc1" \
        :: "v"(voff), "s"((u64)(gbase)), "s"((u32)(bufoff) + ldw) : "memory")
#define LDA(dst, b, h) do { _Pragma("unroll") for (int m = 0; m < 4; ++m) _Pragma("unroll") for (int k = 0; k < 2; ++k) \
        dst[m][k] = *(const LAS f16x8*)(L + SA(b, h) + ((h) ? aoff1 : aoff0) + m * 2048 + k * 1024); } while (0)
#define LDB(dst, b, h) do { _Pragma("unroll") for (int n = 0; n < 2; ++n) _Pragma("unroll") for (int k = 0; k < 2; ++k) \
        dst[n][k] = *(const LAS f16x8*)(L + SB(b, h) + boff + n * 2048 + k * 1024); } while (0)
#define MMA(ai, bj, At, Bt) do { __builtin_amdgcn_sched_barrier(0);         \
        _Pragma("unroll") for (int k = 0; k < 2; ++k) _Pragma("unroll") for (int mm = 0; mm < 4; ++mm) _Pragma("unroll") for (int nn = 0; nn < 2; ++nn) { \
            const int m = k ? 3 - mm : mm, n = (mm & 1) ? 1 - nn : nn; \
            acc[ai][bj][m][n] = __builtin_amdgcn_mfma_f32_16x16x32_f16(At[m][k], Bt[n][k], acc[ai][bj][m][n], 0, 0, 0); \
            __builtin_amdgcn_sched_barrier(0); } } while (0)
#define WAIT_L(n) asm volatile("s_waitcnt lgkmcnt(" #n ")" ::: "memory")
#define WAIT_L0 __builtin_amdgcn_s_waitcnt(0xC07F)
#define RBAR __builtin_amdgcn_s_barrier()
#define SCHED __builtin_amdgcn_sched_barrier(0)
    auto flag_ptr = [&](int lp, int j, int ent) -> const char* {
        return ws + OFF_FA + (size_t)((call & 1u) * U_PAR + (ent < 32 ? ((pm * 2 + (ent >> 4)) * 8 + j) * 16 + (ent & 15) : U_B + (lp * 8 + j) * 32 + (ent - 32))) * 8; };
    auto poll = [&](int lp, int j) {
        const u32 ln = lane_id();
        const char* f = flag_ptr(lp, j, ln);
        const u64 t0 = now(); u32 sp = 0; u32x2 v;
        for (;;) {
            v = ld8_sys(f);
            if (__builtin_amdgcn_ballot_w64(v.x >= call) == ~0ull) break;
            if (give_up(sp, t0, (u32*)(ws + OFF_TMO), call)) { v.y = 62u; break; }
            __builtin_amdgcn_s_sleep(4);
        }
        asm volatile("ds_write_b64 %0, %1\n\ts_waitcnt lgkmcnt(0)" :: "v"(L_T2 + ln * 8), "v"(v) : "memory");
    };
    auto codes = [&](u32 tb) -> u32 {
        const u32 r = lane_id() & 7;
        const u32 a0 = *(const LAS u32*)(L + tb + wr * 8 + 4 + r * 16), a1 = *(const LAS u32*)(L + tb + 128 + wr * 8 + 4 + r * 16);
        const u32 b = *(const LAS u32*)(L + tb + 256 + wc * 8 + 4 + r * 32);
        return a0 | (a1 << 8) | (b << 16);
    };
    f32x4 acc[2][2][4][2];
    f16x8 At[4][2], B0[2][2], B1[2][2];
    u32 cur = 0, cmax = 0, bad = 0;
    u32 aoff0 = wr * 8192 + lane * 16, aoff1 = aoff0;
    u32 nm, nmn = 0;
    auto changes = [&](u32 nw, u32 old) -> u32 {
        const u32 prev = __builtin_amdgcn_update_dpp((int)__builtin_amdgcn_readlane(old, 7), (int)nw, 0x111, 0xf, 0xf, false);
        return (u32)__builtin_amdgcn_ballot_w64(nw != prev) & 255u; };
#define ADJUST(r, buf) do { if (__builtin_expect((nm >> (r)) & 1u, 0)) { const u32 cn_ = __builtin_amdgcn_readlane(pk, (r)), cb_ = (cn_ >> 16) & 63u; \
            const u32 l16_ = lane_id() * 16; \
            _Pragma("unroll") for (int ai = 0; ai < 2; ++ai) { \
                const u32 ca_ = (cn_ >> (8 * ai)) & 63u, c_ = ca_ + cb_, o_ = (cur >> (8 * ai)) & 255u, x_ = (cmax >> (8 * ai)) & 255u; \
                u32 ab_ = wr * 8192; int sh_ = 0; \
                if (ca_ == 62u || cb_ == 62u) bad = 1; \
                else if (ca_ != 63u && cb_ != 63u) { \
                    if (c_ + 10u < x_) { ab_ = L_ZERO - SA(buf, ai); nm = 255u; nmn |= 1u; }      \
                    else { sh_ = 8 * ((int)o_ - (int)c_); \
                        cur = (cur & ~(255u << (8 * ai))) | (c_ << (8 * ai)); \
                        if (c_ > x_) cmax = (cmax & ~(255u << (8 * ai))) | (c_ << (8 * ai)); } } \
                asm volatile("v_add_u32 %0, %1, %2" : "+v"(ai ? aoff1 : aoff0) : "s"(ab_), "v"(l16_)); \
                if (sh_ != 0) { \
                    asm volatile("s_nop 15" ::: "memory");        \
                    _Pragma("unroll") for (int bj = 0; bj < 2; ++bj) _Pragma("unroll") for (int m = 0; m < 4; ++m) _Pragma("unroll") for (int n = 0; n < 2; ++n) \
                        _Pragma("unroll") for (int e_ = 0; e_ < 4; ++e_) asm volatile("v_ldexp_f32 %0, %0, %1" : "+v"(acc[ai][bj][m][n][e_]) : "s"(sh_)); } } } } while (0)
#pragma unroll
    for (int a = 0; a < 2; ++a)
#pragma unroll
        for (int b = 0; b < 2; ++b)
#pragma unroll
            for (int m = 0; m < 4; ++m)
#pragma unroll
                for (int n = 0; n < 2; ++n) acc[a][b][m][n] = (f32x4){0.f, 0.f, 0.f, 0.f};
    const char* const cA = ws + OFF_A + (size_t)(call & 1u) * U_PAR * 8192 + (size_t)pm * 2 * HA;
    const char* cB = ws + OFF_A + (size_t)((call & 1u) * U_PAR + U_B) * 8192 + (size_t)lp0 * 64 * KB;
    *(LAS u32x4*)(L + L_ZERO + wave * 1024 + lane * 16) = (u32x4){0u, 0u, 0u, 0u};
    if (wave == 0) poll(lp0, 0);
    LGKM0();
    BAR();
    u32 pk = codes(L_T2), pkn = pk;
    nm = changes(pk, pk) | 1u;
    STAGE(SB(0, 0), cB, voffB); STAGE(SA(0, 0), cA, voffA); STAGE(SB(0, 1), cB + HB, voffB); STAGE(SA(0, 1), cA + HA, voffA);
    if (wr == 1) RBAR;
    VMCNT(4); RBAR;
    STAGE(SB(1, 0), cB + KB, voffB); STAGE(SA(1, 0), cA + KA, voffA); STAGE(SB(1, 1), cB + HB + KB, voffB);
    VMCNT(6); RBAR;
#pragma unroll 1
    for (int rd = 0; rd < 3; ++rd) {
        const int lp = lp0 + rd * 12;
        const char* nB = rd < 2 ? cB + 12 * 64 * KB : cB;
#define TRIP(r, a1, a2, b2, HOOK) do { \
            ADJUST(r, 0); \
            LDB(B0, 0, 0); SCHED; LDA(At, 0, 0); STAGE(SA(1, 1), (a1) + HA, voffA); \
            WAIT_L(8); RBAR; WAIT_L0; MMA(0, 0, At, B0); RBAR; SCHED; \
            LDB(B1, 0, 1); STAGE(SB(0, 0), (b2), voffB); \
            RBAR; WAIT_L0; MMA(0, 1, At, B1); RBAR; \
            LDA(At, 0, 1); STAGE(SA(0, 0), (a2), voffA); \
            RBAR; WAIT_L0; MMA(1, 0, At, B0); RBAR; SCHED; \
            STAGE(SB(0, 1), (b2) + HB, voffB); \
            VMCNT(6); RBAR; MMA(1, 1, At, B1); RBAR; \
            ADJUST((r) + 1, 1); \
            LDB(B0, 1, 0); SCHED; LDA(At, 1, 0); STAGE(SA(0, 1), (a2) + HA, voffA); \
            WAIT_L(8); RBAR; WAIT_L0; MMA(0, 0, At, B0); RBAR; SCHED; \
            LDB(B1, 1, 1); STAGE(SB(1, 0), (b2) + KB, voffB); \
            RBAR; WAIT_L0; MMA(0, 1, At, B1); RBAR; \
            LDA(At, 1, 1); STAGE(SA(1, 0), (a2) + KA, voffA); \
            RBAR; WAIT_L0; MMA(1, 0, At, B0); RBAR; SCHED; \
            STAGE(SB(1, 1), (b2) + KB + HB, voffB); \
            HOOK; \
            VMCNT(6); RBAR; MMA(1, 1, At, B1); RBAR; } while (0)
#pragma unroll 1
        for (int j = 0; j < 8; ++j) {
            const char* ga = cA + (size_t)j * 8 * KA; const char* gb = cB + (size_t)j * 8 * KB;
            const char* na = j == 7 ? cA : ga + 8 * KA; const char* nb = j == 7 ? nB : gb + 8 * KB;
            const bool more = j < 7 || rd < 2;
            const int nlp = j < 7 ? lp : lp + 12, nj = (j + 1) & 7;
            TRIP(0, ga + KA, ga + 2 * KA, gb + 2 * KB, do { } while (0));
            TRIP(2, ga + 3 * KA, ga + 4 * KA, gb + 4 * KB,
                 if (__builtin_expect(wave == 0 && more, 0)) { const u32 ln = lane_id(); const char* fp = flag_ptr(nlp, nj, ln * 2);
                     if (ln < 32) asm volatile("s_mov_b32 m0, %1\n\ts_nop 0\n\tglobal_load_lds_dwordx4 %0, off sc0 sc1" :: "v"(fp), "s"(L_T1) : "memory"); });
            TRIP(4, ga + 5 * KA, ga + 6 * KA, gb + 6 * KB, {
                 const u32 tag = *(const LAS u32*)(L + L_T1 + lane_id() * 8);
                 pkn = codes(L_T1);
                 if (__builtin_expect((__builtin_amdgcn_ballot_w64(tag >= call) != ~0ull) & more, 0)) {
                     if (wave == 0) poll(nlp, nj);
                     BAR();
                     pkn = codes(L_T2);
                 }
                 nmn |= changes(pkn, pk); });
            TRIP(6, ga + 7 * KA, na, nb, { pk = pkn; nm = nmn; nmn = 0; });
        }
        {
            const int ln = lane_id();
            const int gp = lp < 4 ? 32 + P.rank * 4 + lp : lp - 4;
            GAS float* o = (GAS float*)P.out + (size_t)(pm * 256 + wr * 64 + (ln >> 4) * 4) * GN + gp * 256 + wc * 64 + (ln & 15) * 4;
#pragma unroll
            for (int ai = 0; ai < 2; ++ai) {
                const int sh = 8 * (int)((cur >> (8 * ai)) & 255u) - 272;
#pragma unroll
                for (int m = 0; m < 4; ++m)
#pragma unroll
                    for (int r = 0; r < 4; ++r) {
                        f32x4 v;
#pragma unroll
                        for (int i = 0; i < 4; ++i) {
                            const float y = __builtin_elementwise_maximum(__builtin_ldexpf(acc[ai][i >> 1][m][i & 1][r], sh), 0.0f);
                            v[i] = __uint_as_float(bad ? 0x7fc00000u : __float_as_uint(y) & 0x7fffffffu);
                            acc[ai][i >> 1][m][i & 1][r] = 0.f;
                        }
                        st_out(o + (size_t)(ai * 128 + m * 16 + r) * GN, v);
                    }
            }
            if (lp < 4) {
                VMCNT(0);
                if (ln == 0) st4_sys(ws + OFF_OF + (size_t)((pm * 4 + lp) * 8 + wave) * 4, call);
            }
            cur = 0; cmax = 0; bad = 0; nm |= 1u;
        }
        cB = nB;
    }
    VMCNT(0);
    if (wr == 0) RBAR;
#undef SA
#undef SB
#undef STAGE
#undef LDA
#undef LDB
#undef MMA
#undef ADJUST
#undef TRIP
}

template <int G> __device__ __forceinline__ void pack_dense(LAS char* dense, LAS char* img, u32 lane, u32 n8, u32 o_mid, u32 o_top, u32 o_esc, u32 b2,
                                                            u32* T, u32* T3, u32* bp, u32& tot, u32& e) {
    u32x4 a[G], b[G], lo[G]; u32x2 mid[G]; u32 hi[G][4], ge[G][4], top[G];
#pragma unroll
    for (int i = 0; i < G; ++i) { a[i] = *(LAS u32x4*)(dense + (i * 64 + lane) * 32u); b[i] = *(LAS u32x4*)(dense + (i * 64 + lane) * 32u + 16u); }
#pragma unroll
    for (int i = 0; i < G; ++i) {
        const u32 dd[8] = {a[i].x, a[i].y, a[i].z, a[i].w, b[i].x, b[i].y, b[i].z, b[i].w};
        u32 x[4], c[4], f9[4], len, tp = 0;
#pragma unroll
        for (int k = 0; k < 4; ++k) {
            lo[i][k] = __builtin_amdgcn_perm(dd[2 * k + 1], dd[2 * k], 0x05040100u);
            hi[i][k] = __builtin_amdgcn_perm(dd[2 * k + 1], dd[2 * k], 0x07060302u);
            x[k] = PW((u16x2)(PU(hi[i][k]) - PU(b2)));
            if (i == G - 1) x[k] = i * 64 + lane < n8 ? x[k] : 0u;
        }
        pk_classify<7>(x, c, ge[i], T[i], T3[i], len);
#pragma unroll
        for (int k = 0; k < 4; ++k) { f9[k] = (c[k] << 7) | (x[k] & 0x007f007fu); tp |= ((c[k] >> 1) & 0x00010001u) << k; }
        mid[i] = (u32x2){__builtin_amdgcn_perm(f9[1], f9[0], 0x06040200u), __builtin_amdgcn_perm(f9[3], f9[2], 0x06040200u)};
        top[i] = tp | (tp >> 12);
        bp[i] = tot; tot += len;
    }
#pragma unroll
    for (int i = 0; i < G; ++i) {
        const u32 g = i * 64 + lane;
        if (i < G - 1 || g < n8) {
            *(LAS u32x4*)(img + 256u + g * 16u) = lo[i];
            *(LAS u32x2*)(img + o_mid + g * 8u) = mid[i];
            *(LAS unsigned char*)(img + o_top + g) = (unsigned char)top[i];
        }
    }
#pragma unroll
    for (int i = 0; i < G; ++i) emit_escapes(ge[i], hi[i], (i * 64 + lane) * 8u, img + o_esc, (8192u - o_esc) >> 2, e);
}
__device__ __forceinline__ void packer(const Params& P, LAS char* L, int pw, int wave, int lane, u32 call) {
    char* ws = P.me;
    LAS char* dense = L + wave * PK_B;
    LAS char* img = dense + PK_IMG;
    const auto rs = __builtin_amdgcn_make_buffer_rsrc((void*)P.out, 0, (u32)GM * GN * 4u, 0x00020000);
    auto wait_item = [&](int it) -> u32 {
        const u64 t0 = now(); u32 sp = 0;
        while (uni(ld4_sys(ws + OFF_OF + it * 4)) < call) {
            if (give_up(sp, t0, (u32*)(ws + OFF_TMO), call)) return 0x7fc00000u;
            __builtin_amdgcn_s_sleep(8);
        }
        return 0u;
    };
    u32x4 nx[8];
    auto fetch_block = [&](int it, int su) {
        const int tile = it >> 3, wq = it & 7;
        const u32 so = ((u32)((tile >> 2) * 256 + (su >> 1) * 128 + (wq >> 2) * 64 + (su & 1) * 32) * GN + 8192 + P.rank * 1024 + (tile & 3) * 256 + (wq & 3) * 64) * 4u;
#pragma unroll
        for (int g = 0; g < 8; ++g) nx[g] = __builtin_amdgcn_raw_buffer_load_b128(rs, ((lane >> 4) * 4 * GN + (lane & 15) * 4) * 4, so + ((g >> 2) * 16 + (g & 3)) * GN * 4, 17);
    };
    auto publish = [&](int it, u32 ea, u32 eb) {
        if (lane < 32 && (lane >> 2) != P.rank)
            st16_sys(peer_ws(P, lane >> 2) + OFF_REC + (call & 1u) * REC_PAR + P.rank * REC_SRC + REC_OUT + it * 64 + (lane & 3) * 16, (u32x4){call, ea, call, eb});
    };
    {
        const char* bx = ws + OFF_BX + (call & 1u) * X_PAR + (size_t)8 * 8 * 32768 + lane * 16;
        u32 acc = 0;
#pragma unroll 1
        for (int i = 0; i < 6; ++i) {
            const int sl = pw * 3 + (i >> 1);
            u32x4 t[7];
#pragma unroll
            for (int g = 0; g < 7; ++g) t[g] = *(const GAS u32x4*)(bx + (size_t)sl * 16384 + ((i & 1) * 7 + g) * 1024);
#pragma unroll
            for (int g = 0; g < 7; ++g) acc ^= t[g].x;
        }
        asm volatile("" :: "v"(acc));
    }
    u32 nanm = wait_item(pw), nann = 0, off16 = 0, ea = 0, eb = 0;
    fetch_block(pw, 0);
#pragma unroll
    for (int g = 0; g < 8; ++g) asm volatile("" : "+v"(nx[g]));
#pragma unroll 1
    for (int bk = 0; bk < N_OUT_ITEMS / 16; ++bk) {
        {
            const int it = pw + (bk >> 2) * 64, su = bk & 3;
            char* og = ws + OFF_O + (size_t)it * QSLOT;
            u32 v[32]; u32 mx = 0;
#pragma unroll
            for (int r = 0; r < 32; ++r) { v[r] = nx[r >> 2][r & 3] | nanm; mx = max(mx, v[r]); }
            { const int bn = bk + 1 < N_OUT_ITEMS / 16 ? bk + 1 : bk; fetch_block(pw + (bn >> 2) * 64, bn & 3); }
            mx = wave_umax(mx) >> 16;
            const u32 b16 = mx >= 511u ? mx - 511u : 0u, b2 = b16 * 0x10001u;
            u32 f[32], mk0 = 0, mk1 = 0;
#pragma unroll
            for (int r = 0; r < 32; ++r) asm("v_min_u32 %0, 1, %1" : "=v"(f[r]) : "v"(v[r]));
#pragma unroll
            for (int r = 0; r < 16; ++r) asm("v_lshl_or_b32 %0, %2, %4, %0\n\tv_lshl_or_b32 %1, %3, %5, %1" : "+v"(mk0), "+v"(mk1) : "v"(f[r]), "v"(f[16 + r]), "n"(r), "n"(16 + r));
            const u32 mk = mk0 | mk1, pc = __builtin_popcount(mk), inc = wave_scan(pc), n = __builtin_amdgcn_readlane(inc, 63);
            {
                u32 ad = wave * PK_B + (inc - pc) * 4u;
#pragma unroll
                for (int r = 0; r < 32; r += 8) {
                    u32 a[8]; u64 m[8];
#pragma unroll
                    for (int i = 0; i < 8; ++i) { a[i] = ad; asm("v_lshl_add_u32 %0, %1, 2, %2" : "=v"(ad) : "v"(f[r + i]), "v"(a[i])); m[i] = __builtin_amdgcn_ballot_w64(v[r + i] != 0u); }
                    asm volatile("s_mov_b64 exec, %16\n\tds_write_b32 %0, %8\n\ts_mov_b64 exec, %17\n\tds_write_b32 %1, %9\n\t"
                                 "s_mov_b64 exec, %18\n\tds_write_b32 %2, %10\n\ts_mov_b64 exec, %19\n\tds_write_b32 %3, %11\n\t"
                                 "s_mov_b64 exec, %20\n\tds_write_b32 %4, %12\n\ts_mov_b64 exec, %21\n\tds_write_b32 %5, %13\n\t"
                                 "s_mov_b64 exec, %22\n\tds_write_b32 %6, %14\n\ts_mov_b64 exec, %23\n\tds_write_b32 %7, %15\n\ts_mov_b64 exec, -1"
                                 :: "v"(a[0]), "v"(a[1]), "v"(a[2]), "v"(a[3]), "v"(a[4]), "v"(a[5]), "v"(a[6]), "v"(a[7]),
                                    "v"(v[r]), "v"(v[r + 1]), "v"(v[r + 2]), "v"(v[r + 3]), "v"(v[r + 4]), "v"(v[r + 5]), "v"(v[r + 6]), "v"(v[r + 7]),
                                    "s"(m[0]), "s"(m[1]), "s"(m[2]), "s"(m[3]), "s"(m[4]), "s"(m[5]), "s"(m[6]), "s"(m[7]) : "memory");
                }
            }
            if (lane < 8) *(LAS u32*)(dense + (n + lane) * 4u) = b16 << 16;
            *(LAS u32*)(img + lane * 4) = mk;
            const u32 nr = (n & 511u) <= 64u ? n & 511u : 0u, n8 = (n - nr + 7u) >> 3, G = (n8 + 63u) >> 6;
            const u32 o_mid = 256u + n8 * 16u, o_top = o_mid + n8 * 8u, o_raw = o_top + ((n8 + 3u) & ~3u), o_esc = o_raw + nr * 4u;
            CFENCE();
            if (nr) *(LAS u32*)(img + o_raw + lane * 4u) = *(LAS u32*)(dense + n8 * 32u + lane * 4u);
            CFENCE();
            u32 e = 0, tot = 0, T[4] = {0u, 0u, 0u, 0u}, T3[4] = {0u, 0u, 0u, 0u}, bp[4] = {0u, 0u, 0u, 0u};
            if (G == 3u) pack_dense<3>(dense, img, lane, n8, o_mid, o_top, o_esc, b2, T, T3, bp, tot, e);
            else if (G == 2u) pack_dense<2>(dense, img, lane, n8, o_mid, o_top, o_esc, b2, T, T3, bp, tot, e);
            else if (G == 1u) pack_dense<1>(dense, img, lane, n8, o_mid, o_top, o_esc, b2, T, T3, bp, tot, e);
            else if (G == 4u) pack_dense<4>(dense, img, lane, n8, o_mid, o_top, o_esc, b2, T, T3, bp, tot, e);
#pragma unroll
            for (int g = 0; g < 8; ++g) asm volatile("" : "+v"(nx[g]));
            VMCNT(0);
            if (su == 0) { if (bk) publish(it - 64, ea, eb); off16 = 0; }
            if (su == 2 && it + 64 < N_OUT_ITEMS) nann = wait_item(it + 64);
            const u32 binc = wave_scan(tot), all2 = __builtin_amdgcn_readlane(binc, 63), o_sec = o_esc + e * 4u;
            u32 size16 = (o_sec + ((pk_sum(all2) + 7u) >> 3) + 15u) >> 4;
            if (size16 >= 512u) {
                size16 = 512u;
#pragma unroll
                for (int g = 0; g < 8; ++g) st16_sys(og + off16 * 16u + g * 1024 + lane * 16, (u32x4){v[g * 4], v[g * 4 + 1], v[g * 4 + 2], v[g * 4 + 3]});
            } else {
#pragma unroll
                for (int i = 0; i < 6; ++i) *(LAS u32*)(img + o_sec + i * 256 + lane * 4) = 0u;
                CFENCE();
#pragma unroll
                for (int i = 0; i < 4; ++i) if (i < (int)G) {
                    const u32 bq = bp[i] + binc - tot + (all2 << 16);
                    or_bits(img + o_sec, bq & 0xffffu, T[i]);
                    or_bits(img + o_sec, bq >> 16, T3[i]);
                }
                CFENCE();
#pragma unroll 1
                for (u32 g = lane; g < size16 + lane; g += 256u) {
                    u32x4 t[4];
#pragma unroll
                    for (int i = 0; i < 4; ++i) t[i] = *(LAS u32x4*)(img + (g + i * 64) * 16u);
#pragma unroll
                    for (int i = 0; i < 4; ++i) if (g + i * 64 < size16) st16_sys(og + (off16 + g + i * 64) * 16u, t[i]);
                }
            }
            if ((lane & 3) == su) { ea = size16 | (n << 10); eb = (e & 0xfffu) | (b16 << 12); }
            off16 += size16;
            if (su == 3) nanm = nann;
            LGKM0();
            CFENCE();
        }
    }
    VMCNT(0);
    publish(pw + N_OUT_ITEMS - 64, ea, eb);
}

__global__ __launch_bounds__(NTHR, 2) __attribute__((amdgpu_waves_per_eu(2, 2))) void fused_kernel(Params P) {
    extern __shared__ __attribute__((aligned(16))) char smem[];
    LAS char* L = (LAS char*)smem;
    asm volatile("" :: "s"(P.x), "s"(P.w), "s"(P.me), "s"(P.rank));
    const int tid = threadIdx.x, lane = tid & 63, wave = __builtin_amdgcn_readfirstlane(tid >> 6);
    const int b = blockIdx.x, xg = b & 7, s = b >> 3;
    const bool fetch = s < FS;
    constexpr int NF = FS * 8;
    GAS u32* const cnt = (GAS u32*)(P.me + OFF_CNT + (b * 8 + wave) * 8);
    const int pidx = b - (NF + 120);
    constexpr int NU = 112 - ITEM0;
    const int q0 = ITEM0 + (b - NF);
    const bool loaded = !fetch && q0 < 112 && wave < 2;
    f32x4 v[16];
    if (wave == 7 && (unsigned)pidx < (unsigned)NPUSH) {
        load_x(P, pidx, 0, lane, v);
        const u32 craw = __hip_atomic_load(cnt, __ATOMIC_RELAXED, __HIP_MEMORY_SCOPE_AGENT);
        __builtin_amdgcn_sched_barrier(0);
        BAR();
        __builtin_amdgcn_sched_barrier(0);
        push_task(P, pidx, lane, craw, v);
    } else {
        if (loaded) { if (q0 < 32) load_x(P, q0 * 2 + (wave & 1), 0, lane, v); else load_w(P, (q0 < 96 ? q0 + 32 : q0 - 96) * 2 + (wave & 1), 0, lane, v); }
        __builtin_amdgcn_sched_barrier(0);
        if (tid < 256) lds_st(L_ITEM + tid * 4, !fetch ? 0u : tid == 18 ? 6u : tid >= 24 && tid < 32 ? (u32)(tid - 24) : 0u);
        LGKM0();
        BAR();
    }
    if (!fetch) {
        const u64 tstart = now();
        const bool pack = s >= FS + GS;
        auto fetch_task = [&](int q) {
            const u64 go = pack ? (u64)PACE4 : (u64)(PACE2 + (q - 112) * PACE3 / 112);
            while (now() - tstart < go) __builtin_amdgcn_s_sleep(4);
            int j, idx; bool isA;
            task_of(q, wave & 1, j, idx, isA);
            if (isA) load_x(P, idx, j, lane, v); else load_w(P, idx, j, lane, v);
        };
        int qn = 112 + (pack ? GS * 8 + xg : (b - NF + GS * 8 - NU) % (GS * 8)) * 4 + (wave >> 1), q = q0;
        const int qs = pack ? 32 : N_PAIRS;
        u32 lq = L_ITEM;
        if (!loaded) { q = qn; qn += qs; lq += 64; fetch_task(q); }
#pragma unroll 1
        for (;;) {
            stage_task(P, L + wave * 4096, lq, lq + 48 + (wave >> 1) * 4, q, wave & 1, lane, cnt, v);
            if (qn >= N_PAIRS) break;
            q = qn; qn += qs; lq += 64;
            fetch_task(q);
        }
    }
    const u32 call = uni(__hip_atomic_load(cnt, __ATOMIC_RELAXED, __HIP_MEMORY_SCOPE_AGENT)) + 1u;
    if (lane == 0) st4_sys((void*)cnt, call);
    if (fetch) {
        const int src = (P.rank + 1 + s) & 7;
        if (wave == 0) { loader(P, lane, call); }
        else if (wave == 1) scout(P, src, xg, lane, call);
        else { decoder(P, L, wave - 2, lane, call); }
    } else if (s < FS + GS) {
        const int g = s - FS;
        gemm_wg(P, L, (xg >> 1) * 4 + g / 6, (xg & 1) * 6 + g % 6, wave, lane, call);
    } else {
        BAR();
        packer(P, L, wave * 8 + xg, wave, lane, call);
    }
}

}

extern "C" void kernel_launch(int rank, int nranks, ncclComm_t comm, hipStream_t stream,
                              void* const* d_in, const int* in_sizes, int n_in,
                              void* d_out, int out_size,
                              const int* in_dtypes, int out_dtype,
                              void* const* d_out_all,
                              void* const* d_ws_all, size_t ws_bytes) {
    (void)comm; (void)in_sizes; (void)n_in; (void)out_size; (void)in_dtypes; (void)out_dtype; (void)d_out_all;
    if (nranks != NR || (unsigned)rank >= (unsigned)NR || ws_bytes < WS_NEED) { fprintf(stderr, "unsupported configuration\n"); return; }
    static bool ready[NR];
    if (!ready[rank]) {
        HIP_CHECK(hipFuncSetAttribute((const void*)fused_kernel, hipFuncAttributeMaxDynamicSharedMemorySize, (int)LDS_BYTES));
        ready[rank] = true;
    }
    Params P;
    P.x = (const float*)d_in[0]; P.w = (const float*)d_in[1]; P.out = (float*)d_out;
    for (int i = 0; i < NR; ++i) P.ws[i] = (char*)d_ws_all[i];
    P.me = P.ws[rank]; P.rank = rank; P.pad = 0;
    hipLaunchKernelGGL(fused_kernel, dim3(NWG), dim3(NTHR), LDS_BYTES, stream, P);
    HIP_CHECK(hipGetLastError());
}
```

```cpp
#include <hip/hip_runtime.h>
#include <rccl/rccl.h>
#include <cstdio>

#define HIP_CHECK(expr) do { hipError_t _e = (expr); if (_e != hipSuccess) { \
    fprintf(stderr, "HIP error %s at %s:%d\n", hipGetErrorString(_e), __FILE__, __LINE__); \
    return; } } while (0)

typedef unsigned u32;
typedef unsigned long long u64;
typedef u32 u32x2 __attribute__((ext_vector_type(2)));
typedef u32 u32x4 __attribute__((ext_vector_type(4)));
typedef float f32x4 __attribute__((ext_vector_type(4)));
typedef _Float16 f16x8 __attribute__((ext_vector_type(8)));
#define LAS __attribute__((address_space(3)))
#define GAS __attribute__((address_space(1)))

namespace {

constexpr int NR = 8, GM = 4096, GN = 16384, KL = 512;
constexpr int NWG = 256, NTHR = 512;
constexpr int FS = 7, GS = 24;
constexpr int LPP = 8;
constexpr int WIN = 8;
constexpr int NLP = 36;
constexpr int N_IN_ITEMS = 832, N_OUT_ITEMS = 512, N_ITEMS = N_IN_ITEMS + N_OUT_ITEMS;
constexpr u32 R2_ITEM = 640, ROT = 27;
constexpr u32 HELP_LAG = 4, HELP_LAG2 = 12;
constexpr int NPUSH = 36;
constexpr int N_PAIRS = 1280, ITEM0 = NPUSH / 2;
constexpr int PACE2 = 150, PACE3 = 350, PACE4 = 800;
static_assert(NPUSH % 2 == 0 && NPUSH >= 8 && NPUSH <= 64, "only x blocks of the first K group");
constexpr u64 DEADLINE = 30000000ull;

constexpr size_t OFF_CNT = 0;
constexpr size_t OFF_TMO = 16384;
constexpr size_t OFF_TICK = 16384 + 256;
constexpr size_t OFF_OF = 20480;
constexpr u32 U_B = 4096, U_PAR = U_B + NLP * 256, N_UNITS = 2 * U_PAR;
constexpr size_t OFF_FA = 32768;
constexpr size_t OFF_REC = 1ull << 20;
constexpr u32 REC_SRC = 65536, REC_OUT = 16384;
constexpr size_t REC_PAR = 1ull << 20;
constexpr size_t OFF_A = 4ull << 20;
constexpr size_t OFF_AX = OFF_A + (size_t)N_UNITS * 8192;
constexpr size_t OFF_BX = OFF_AX + (4ull << 20);
constexpr size_t X_PAR = 20ull << 20;
constexpr size_t OFF_O = OFF_AX + 2 * X_PAR;
constexpr size_t QSLOT = 32768;
constexpr size_t WS_NEED = OFF_O + N_OUT_ITEMS * QSLOT;
static_assert(NR * REC_SRC <= REC_PAR && N_IN_ITEMS * 16 <= REC_OUT && REC_OUT + N_OUT_ITEMS * 64 <= REC_SRC && OFF_FA + N_UNITS * 8 <= OFF_REC, "workspace map");

constexpr u32 RING = 65536;
constexpr u32 L_SCR = RING, SCR_B = 8448;
constexpr u32 L_DESC = L_SCR + 6 * SCR_B, NQ = 64;
constexpr u32 L_ITEM = L_DESC + NQ * 16;
constexpr u32 L_FCTL = L_ITEM + 64;
constexpr u32 L_LAND = L_FCTL + 64, L_START = L_LAND + 256;
constexpr u32 L_T1 = 131072, L_T2 = L_T1 + 512, L_ZERO = L_T2 + 512;
constexpr u32 PK_B = 19456, PK_IMG = 8320;
constexpr u32 LDS_BYTES = 8 * PK_B;
static_assert(L_ITEM + 1024 <= LDS_BYTES && L_START + 256 <= L_ITEM + 1024 && L_ZERO + 8192 <= LDS_BYTES && LDS_BYTES <= 163840, "LDS map");

struct Params {
    const float* x; const float* w; float* out;
    char* me;
    char* ws[NR];
    int rank; int pad;
};

#define VMCNT(n) asm volatile("s_waitcnt vmcnt(" #n ")" ::: "memory")
#define LGKM0() asm volatile("s_waitcnt lgkmcnt(0)" ::: "memory")
#define CFENCE() asm volatile("" ::: "memory")
#define BAR() do { CFENCE(); __builtin_amdgcn_s_barrier(); CFENCE(); } while (0)

__device__ __forceinline__ void st16_sys(void* p, u32x4 v) {
    asm volatile("global_store_dwordx4 %0, %1, off sc0 sc1\n\ts_nop 1" :: "v"(p), "v"(v) : "memory");
}
__device__ __forceinline__ void st_out(GAS float* p, f32x4 v) {
    asm volatile("global_store_dwordx4 %0, %1, off sc0 sc1 nt\n\ts_nop 1" :: "v"(p), "v"(v) : "memory");
}
__device__ __forceinline__ void st4_sys(void* p, u32 v) {
    asm volatile("global_store_dword %0, %1, off sc0 sc1" :: "v"(p), "v"(v) : "memory");
}
__device__ __forceinline__ void st8_sys(void* p, u32x2 v) {
    asm volatile("global_store_dwordx2 %0, %1, off sc0 sc1" :: "v"(p), "v"(v) : "memory");
}
__device__ __forceinline__ u32 ld4_sys(const void* p) {
    u32 v; asm volatile("global_load_dword %0, %1, off sc0 sc1\n\ts_waitcnt vmcnt(0)" : "=&v"(v) : "v"(p) : "memory"); return v;
}
__device__ __forceinline__ u32x2 ld8_sys(const void* p) {
    u32x2 v; asm volatile("global_load_dwordx2 %0, %1, off sc0 sc1\n\ts_waitcnt vmcnt(0)" : "=&v"(v) : "v"(p) : "memory"); return v;
}
__device__ __forceinline__ void lds_st(u32 a, u32 v) { asm volatile("ds_write_b32 %0, %1" :: "v"(a), "v"(v) : "memory"); }
__device__ __forceinline__ u32 lds_ld(u32 a) {
    u32 v; asm volatile("ds_read_b32 %0, %1\n\ts_waitcnt lgkmcnt(0)" : "=&v"(v) : "v"(a) : "memory"); return v;
}
__device__ __forceinline__ u32x4 lds_ld16(u32 a) {
    u32x4 v; asm volatile("ds_read_b128 %0, %1\n\ts_waitcnt lgkmcnt(0)" : "=&v"(v) : "v"(a) : "memory"); return v;
}
__device__ __forceinline__ u32 uni(u32 v) { return __builtin_amdgcn_readfirstlane(v); }
__device__ __forceinline__ u32 lds_ldu(u32 a) { return uni(lds_ld(a)); }
__device__ __forceinline__ u64 now() { return __builtin_amdgcn_s_memrealtime(); }
__device__ __forceinline__ u32 prefix(u64 m) { return __builtin_amdgcn_mbcnt_hi((u32)(m >> 32), __builtin_amdgcn_mbcnt_lo((u32)m, 0u)); }

__device__ __forceinline__ u32 wave_umax(u32 v) {
    v = max(v, (u32)__builtin_amdgcn_update_dpp(0, (int)v, 0x111, 0xf, 0xf, true));
    v = max(v, (u32)__builtin_amdgcn_update_dpp(0, (int)v, 0x112, 0xf, 0xf, true));
    v = max(v, (u32)__builtin_amdgcn_update_dpp(0, (int)v, 0x114, 0xf, 0xf, true));
    v = max(v, (u32)__builtin_amdgcn_update_dpp(0, (int)v, 0x118, 0xf, 0xf, true));
    v = max(v, (u32)__builtin_amdgcn_update_dpp(0, (int)v, 0x142, 0xa, 0xf, false));
    v = max(v, (u32)__builtin_amdgcn_update_dpp(0, (int)v, 0x143, 0xc, 0xf, false));
    return __builtin_amdgcn_readlane(v, 63);
}
__device__ __forceinline__ u32 wave_scan(u32 v) {
    v += (u32)__builtin_amdgcn_update_dpp(0, (int)v, 0x111, 0xf, 0xf, true);
    v += (u32)__builtin_amdgcn_update_dpp(0, (int)v, 0x112, 0xf, 0xf, true);
    v += (u32)__builtin_amdgcn_update_dpp(0, (int)v, 0x114, 0xf, 0xf, true);
    v += (u32)__builtin_amdgcn_update_dpp(0, (int)v, 0x118, 0xf, 0xf, true);
    v += (u32)__builtin_amdgcn_update_dpp(0, (int)v, 0x142, 0xa, 0xf, false);
    v += (u32)__builtin_amdgcn_update_dpp(0, (int)v, 0x143, 0xc, 0xf, false);
    return v;
}
typedef unsigned short u16x2 __attribute__((ext_vector_type(2)));
typedef short i16x2 __attribute__((ext_vector_type(2)));
#define PU(x) __builtin_bit_cast(u16x2, (u32)(x))
#define PI(x) __builtin_bit_cast(i16x2, (u32)(x))
#define PW(x) __builtin_bit_cast(u32, (x))
template <int W> __device__ __forceinline__ u32 pk_offsets(const u32* m, u32* ac) {
    u32 tot;
    ac[0] = 0;
    asm("v_pk_mul_lo_u16 %0, %4, %8 op_sel_hi:[1,0]\n\tv_pk_mad_u16 %1, %5, %8, %0 op_sel_hi:[1,0,1]\n\t"
        "v_pk_mad_u16 %2, %6, %8, %1 op_sel_hi:[1,0,1]\n\tv_pk_mad_u16 %3, %7, %8, %2 op_sel_hi:[1,0,1]"
        : "=&v"(ac[1]), "=&v"(ac[2]), "=&v"(ac[3]), "=&v"(tot) : "v"(m[0]), "v"(m[1]), "v"(m[2]), "v"(m[3]), "n"(-W));
    return tot;
}
__device__ __forceinline__ u32 pk_is3(u32 c) { return PW((i16x2)(PI(c + 0x7ffd7ffdu) >> (i16x2){15, 15})); }
__device__ __forceinline__ u32 pk_sum(u32 a) { return (a & 0xffffu) + (a >> 16); }
__device__ __forceinline__ u32 pk_join(const u32* sv, const u32* ac, u32 acc) {
    const u32 tt = sv[0] | PW((u16x2)(PU(sv[1]) << PU(ac[1]))) | PW((u16x2)(PU(sv[2]) << PU(ac[2]))) | PW((u16x2)(PU(sv[3]) << PU(ac[3])));
    return (tt & 0xffffu) | ((tt >> 16) << (acc & 31u));
}
__device__ __forceinline__ u32 pk_split(u32 t, u32 acc) { return __builtin_amdgcn_perm(t >> (acc & 31u), t, 0x05040100u); }
__device__ __forceinline__ u32 pk_field(u32 tt, u32 sh, u32 m, u32 mask) { return PW((u16x2)(PU(tt) >> PU(sh))) & m & mask; }
__device__ __forceinline__ u32 pk_expand(u32 sv, u32 k, u32 base) {
    u32 r;
    asm("v_pk_min_u16 %0, %1, 1 op_sel_hi:[1,0]\n\tv_pk_mad_u16 %0, %0, 3, %1 op_sel_hi:[1,0,1]\n\tv_pk_mad_u16 %0, %0, %2, %3"
        : "=&v"(r) : "v"(sv), "s"(k), "v"(base));
    return r;
}
template <int SH> __device__ __forceinline__ void pk_classify(const u32* x, u32* c, u32* g, u32& T2, u32& T3, u32& len) {
    u32 m2[4], m3[4], ac[4], s2[4], s3[4];
#pragma unroll
    for (int k = 0; k < 4; ++k) {
        const u32 o = PW((i16x2)(PI(x[k]) >> (i16x2){SH, SH}));
        const u32 sv = PW(__builtin_elementwise_max((i16x2)(-PI(o)), (i16x2){0, 0}));
        c[k] = PW(__builtin_elementwise_min(PU(o), (u16x2){3, 3}));
        s2[k] = PW(__builtin_elementwise_min(PU(sv), (u16x2){3, 3}));
        s3[k] = (sv - s2[k]) & 0x00070007u;
        m2[k] = pk_is3(c[k]); m3[k] = pk_is3(s2[k]);
        g[k] = PW(__builtin_elementwise_sub_sat((u16x2)(PU(o) + (u16x2){10, 10}), (u16x2){13, 13}));
    }
    const u32 a2 = pk_offsets<2>(m2, ac);
    T2 = pk_join(s2, ac, a2);
    const u32 a3 = pk_offsets<3>(m3, ac);
    T3 = pk_join(s3, ac, a3);
    len = pk_sum(a2) | (pk_sum(a3) << 16);
}
__device__ __forceinline__ void or_bits(LAS char* buf, u32 bp, u32 t) {
    const u64 v = (u64)t << (bp & 31u);
    LAS u32* a = (LAS u32*)(buf + (bp >> 5) * 4u);
    __hip_atomic_fetch_or(a, (u32)v, __ATOMIC_RELAXED, __HIP_MEMORY_SCOPE_WAVEFRONT);
    __hip_atomic_fetch_or(a + 1, (u32)(v >> 32), __ATOMIC_RELAXED, __HIP_MEMORY_SCOPE_WAVEFRONT);
}
__device__ __forceinline__ void emit_escapes(const u32* g, const u32* raw, u32 idx0, LAS char* rec, u32 cap, u32& e) {
    if (__builtin_amdgcn_ballot_w64((g[0] | g[1] | g[2] | g[3]) != 0u) == 0ull) return;
    u32 gm = 0;
#pragma unroll
    for (int k = 0; k < 4; ++k) gm |= PW(__builtin_elementwise_min(PU(g[k]), (u16x2){1, 1})) << k;
#pragma unroll 1
    for (;;) {
        const u64 em = __builtin_amdgcn_ballot_w64(gm != 0u);
        if (em == 0ull) break;
        const u32 b = __builtin_ctz(gm | 0x80000000u), k = b & 3u;
        const u32 m1 = (u32)__builtin_amdgcn_sbfe((int)b, 0, 1), m2 = (u32)__builtin_amdgcn_sbfe((int)b, 1, 1);
        const u32 r01 = (raw[1] & m1) | (raw[0] & ~m1), r23 = (raw[3] & m1) | (raw[2] & ~m1), r = (r23 & m2) | (r01 & ~m2);
        if (gm != 0u && e < cap) *(LAS u32*)(rec + (e + prefix(em)) * 4u) = (idx0 + k * 2u + (b >> 4)) | ((r >> (b & 16u)) << 16);
        gm &= gm - 1u;
        e += __builtin_popcountll(em);
    }
}
typedef float f32x2 __attribute__((ext_vector_type(2)));
typedef _Float16 f16x2 __attribute__((ext_vector_type(2)));
__device__ __forceinline__ u32 cvt2(float a, float b, float sc) { return __builtin_bit_cast(u32, __builtin_convertvector((f32x2){a, b} * sc, f16x2)); }
__device__ __forceinline__ char* peer_ws(const Params& P, int p) {
    u64 wp = (u64)P.ws[0];
#pragma unroll
    for (int i = 1; i < NR; ++i) { u64 q = (u64)P.ws[i]; asm volatile("" : "+s"(q)); wp = p == i ? q : wp; }
    return (char*)wp;
}


__device__ __forceinline__ bool give_up(u32& spins, u64 t0, u32* tmo, u32 call) {
    if ((++spins & 63u) != 0u) return false;
    if (uni(ld4_sys(tmo)) == call) return true;
    if (now() - t0 > DEADLINE) { st4_sys(tmo, call); return true; }
    return false;
}

__device__ __forceinline__ float block_scale(const f32x4 (&v)[16], u32& code, u32& hm) {
    float m0 = 0.f, m1 = 0.f, m2 = 0.f, m3 = 0.f;
#pragma unroll
    for (int i = 0; i < 16; i += 2)
        asm("v_max3_f32 %0, %0, |%4|, |%5|\n\tv_max3_f32 %1, %1, |%6|, |%7|\n\tv_max3_f32 %2, %2, |%8|, |%9|\n\tv_max3_f32 %3, %3, |%10|, |%11|"
            : "+v"(m0), "+v"(m1), "+v"(m2), "+v"(m3)
            : "v"(v[i][0]), "v"(v[i][1]), "v"(v[i][2]), "v"(v[i][3]), "v"(v[i + 1][0]), "v"(v[i + 1][1]), "v"(v[i + 1][2]), "v"(v[i + 1][3]));
    const u32 mb = wave_umax(max(max(__float_as_uint(m0), __float_as_uint(m1)), max(__float_as_uint(m2), __float_as_uint(m3))));
    code = ((mb >> 23) + 2u) >> 3; code = code < 2u ? 2u : code;
    const float sc = __uint_as_float((263u - 8u * code) << 23);
    hm = uni(__builtin_bit_cast(unsigned short, (_Float16)(__uint_as_float(mb) * sc)));
    if (mb == 0u) code = 63u;
    return sc;
}
__device__ __forceinline__ void load_x(const Params& P, int idx, int j, int lane, f32x4 (&v)[16]) {
    const GAS float* p = (const GAS float*)P.x + (size_t)(idx * 64 + (lane & 15)) * KL + j * 64 + (lane >> 4) * 4;
#pragma unroll
    for (int i = 0; i < 16; ++i) v[i] = __builtin_nontemporal_load((const GAS f32x4*)(p + (i >> 2) * 16 * KL + (i & 3) * 16));
}
__device__ __forceinline__ void push_task(const Params& P, int idx, int lane, u32 craw, const f32x4 (&v)[16]) {
    asm volatile("" :: "s"(P.ws[0]), "s"(P.ws[1]), "s"(P.ws[2]), "s"(P.ws[3]), "s"(P.ws[4]), "s"(P.ws[5]), "s"(P.ws[6]), "s"(P.ws[7]));
    __builtin_amdgcn_sched_barrier(0);
    u32 code, hm;
    const float sc = block_scale(v, code, hm);
    const u32 call = uni(craw) + 1u;
    const u32 dest = (call & 1u) * U_PAR + ((idx >> 1) * 64 + P.rank) * 2 + (idx & 1);
    const u32 voff = (u32)OFF_A + dest * 8192u + lane * 16u;
#pragma unroll
    for (int f = 0; f < 8; ++f) {
        const u32x4 hv = {cvt2(v[2 * f][0], v[2 * f][1], sc), cvt2(v[2 * f][2], v[2 * f][3], sc), cvt2(v[2 * f + 1][0], v[2 * f + 1][1], sc), cvt2(v[2 * f + 1][2], v[2 * f + 1][3], sc)};
        VMCNT(8);
        asm volatile("global_store_dwordx4 %0, %1, %2 sc0 sc1\n\tglobal_store_dwordx4 %0, %1, %3 sc0 sc1\n\tglobal_store_dwordx4 %0, %1, %4 sc0 sc1\n\t"
                     "global_store_dwordx4 %0, %1, %5 sc0 sc1\n\tglobal_store_dwordx4 %0, %1, %6 sc0 sc1\n\tglobal_store_dwordx4 %0, %1, %7 sc0 sc1\n\t"
                     "global_store_dwordx4 %0, %1, %8 sc0 sc1\n\tglobal_store_dwordx4 %0, %1, %9 sc0 sc1\n\ts_nop 1"
                     :: "v"(voff + f * 1024), "v"(hv), "s"((u64)P.ws[0]), "s"((u64)P.ws[1]), "s"((u64)P.ws[2]), "s"((u64)P.ws[3]),
                        "s"((u64)P.ws[4]), "s"((u64)P.ws[5]), "s"((u64)P.ws[6]), "s"((u64)P.ws[7]) : "memory");
        __builtin_amdgcn_sched_barrier(0);
    }
    VMCNT(0);
    if (lane < NR) st8_sys(peer_ws(P, lane) + OFF_FA + (size_t)dest * 8, (u32x2){call, code});
}
__device__ __forceinline__ void load_w(const Params& P, int idx, int j, int lane, f32x4 (&v)[16]) {
    const GAS float* p = (const GAS float*)P.w + (size_t)(j * 64 + (lane >> 4) * 4) * GN + idx * 64 + (lane & 15) * 4;
#pragma unroll
    for (int i = 0; i < 16; ++i) v[i] = __builtin_nontemporal_load((const GAS f32x4*)(p + (size_t)((i >> 2) * 16 + (i & 3)) * GN));
}
__device__ __forceinline__ void task_of(int q, int half, int& j, int& idx, bool& isA) {
    if (q < 896) { j = q / 112; const int u = q % 112; isA = u < 32; idx = isA ? u : (u < 96 ? u + 32 : u - 96); }
    else { const int u = q - 896, v = u % 192; j = v / 24; isA = false; idx = 16 + (u / 192) * 24 + v % 24; }
    idx = idx * 2 + half;
}
__device__ __forceinline__ void stage_task(const Params& P, LAS char* scr, u32 lquad, u32 lpair, int q, int half, int lane, GAS u32* cnt, const f32x4 (&v)[16]) {
    int j, idx; bool isA;
    task_of(q, half, j, idx, isA);
    const u32 craw = __hip_atomic_load(cnt, __ATOMIC_RELAXED, __HIP_MEMORY_SCOPE_AGENT);
    *(LAS u32x4*)(scr + lane * 16) = (u32x4){0u, 0u, 0u, 0u}; *(LAS u32x4*)(scr + 1024 + lane * 16) = (u32x4){0u, 0u, 0u, 0u};
    __builtin_amdgcn_sched_barrier(0);
    u32 code, hm;
    const float sc = block_scale(v, code, hm);
    const u32 bb = hm >= 4095u ? hm - 4095u : 0u, b2 = bb * 0x10001u;
    u32x4 p[8];
    if (isA) {
#pragma unroll
        for (int f = 0; f < 8; ++f)
#pragma unroll
            for (int k = 0; k < 4; ++k) p[f][k] = cvt2(v[f * 2 + (k >> 1)][(k & 1) * 2], v[f * 2 + (k >> 1)][(k & 1) * 2 + 1], sc);
        CFENCE();
    } else {
#pragma unroll
        for (int f = 0; f < 8; ++f)
#pragma unroll
            for (int k = 0; k < 4; ++k) p[f][k] = cvt2(v[(f & 1) * 8 + k * 2][f >> 1], v[(f & 1) * 8 + k * 2 + 1][f >> 1], sc);
        asm volatile("s_nop 0" ::: "memory");
    }
    char* ws = P.me;
    const u32 call = uni(craw) + 1u;
    const bool all = isA || idx < 128;
    const int owner = (idx - 128) >> 4, lp = idx < 128 ? 4 + (idx >> 2) : (idx >> 2) & 3;
    const bool local = all || owner == P.rank, exprt = all || owner != P.rank;
    const u32 dest = (call & 1u) * U_PAR + (isA ? (((idx >> 1) * 8 + j) * 8 + P.rank) * 2 + half : U_B + (((lp * 8 + j) * 8 + P.rank) * 4 + (idx & 3)));
    u32 size16 = 0;
    if (exprt) {
        u32 w[32], T2[8], T3[8], len[8], e = 0, tot = 0;
#pragma unroll
        for (int f = 0; f < 8; ++f) {
            u32 x[4], c[4], g[4];
            const u32 h[4] = {p[f][0], p[f][1], p[f][2], p[f][3]};
#pragma unroll
            for (int k = 0; k < 4; ++k) x[k] = PW((u16x2)(PU(h[k] & 0x7fff7fffu) - PU(b2)));
            pk_classify<10>(x, c, g, T2[f], T3[f], len[f]);
#pragma unroll
            for (int k = 0; k < 4; ++k) w[f * 4 + k] = (h[k] & 0x80008000u) | (c[k] << 10) | (x[k] & 0x03ff03ffu);
            tot += len[f];
            asm volatile("" : "+v"(w[f * 4]), "+v"(w[f * 4 + 1]), "+v"(w[f * 4 + 2]), "+v"(w[f * 4 + 3]), "+v"(T2[f]), "+v"(T3[f]));
            emit_escapes(g, h, f * 512 + lane * 8u, scr + 4, 384u, e);
        }
        const u32 inc = wave_scan(tot), all2 = __builtin_amdgcn_readlane(inc, 63);
        size16 = (6656u + 4u + e * 4u + ((pk_sum(all2) + 7u) >> 3) + 15u) >> 4;
        if (size16 >= 512u) size16 = 512u;
        u32 off16 = 0;
        if (half == 0) { if (lane == 0) lds_st(lpair, size16); }
        else {
            const u64 t0 = now(); u32 sp = 0;
            while ((off16 = lds_ldu(lpair)) == 0u) {
                if ((++sp & 1023u) == 0u && now() - t0 > 4 * DEADLINE) { st4_sys(ws + OFF_TMO, call); break; }
                __builtin_amdgcn_s_sleep(1);
            }
        }
        char* d = ws + (call & 1u) * X_PAR + off16 * 16u + lane * 16 +
                  (isA ? OFF_AX + (size_t)((idx >> 1) * 8 + j) * 16384 : OFF_BX + (size_t)(((idx >> 2) * 8 + j) * 2 + ((idx >> 1) & 1)) * 16384);
        if (size16 == 512u) {
#pragma unroll
            for (int f = 0; f < 8; ++f) st16_sys(d + f * 1024, p[f]);
        } else {
            u32 bp = (e + 1u) * 32u * 0x10001u + (all2 << 16) + inc - tot;
#pragma unroll
            for (int f = 0; f < 8; f += 2) {
                or_bits(scr, bp & 0xffffu, T2[f] | (T2[f + 1] << (len[f] & 31u)));
                or_bits(scr, bp >> 16, T3[f]);
                bp += len[f];
                or_bits(scr, bp >> 16, T3[f + 1]);
                bp += len[f + 1];
            }
            if (lane == 0) *(LAS u32*)scr = e;
#pragma unroll
            for (int i = 0; i < 24; ++i) w[i] |= (w[26 + (i >> 2)] << (12 - 3 * (i & 3))) & 0x70007000u;
#pragma unroll
            for (int i = 0; i < 6; ++i) w[24 + i / 3] |= (w[26 + i] >> (1 + i % 3)) & (0x40004000u >> (i % 3));
#pragma unroll
            for (int f = 0; f < 6; ++f) st16_sys(d + f * 1024, (u32x4){w[f * 4], w[f * 4 + 1], w[f * 4 + 2], w[f * 4 + 3]});
            st8_sys(d + 6144 - lane * 8, (u32x2){w[24], w[25]});
            CFENCE();
            for (u32 g = lane; g + 416u < size16; g += 64u) st16_sys(d + 6656 + (g - lane) * 16u, *(LAS u32x4*)(scr + g * 16u));
        }
    }
    if (local) {
        char* d = ws + OFF_A + (size_t)dest * 8192 + lane * 16;
#pragma unroll
        for (int f = 0; f < 8; ++f) st16_sys(d + f * 1024, p[f]);
    }
    VMCNT(0);
    if (local && lane == 0) st8_sys(ws + OFF_FA + (size_t)dest * 8, (u32x2){call, code});
    if (exprt) {
        const int hb = (lp % 12) * 2 + ((idx >> 1) & 1);
        const int item = isA ? j * 56 + (idx >> 1) : (lp < 12 ? j * 56 + 32 + hb : 256 + (lp / 12) * 192 + j * 24 + hb);
        const u32 info = size16 | (bb << 10) | (code << 25);
        const size_t ro = OFF_REC + (call & 1u) * REC_PAR + P.rank * REC_SRC;
        if (q < 112) {
            if (lane < NR && lane != P.rank && (all || lane == owner)) st8_sys(peer_ws(P, lane) + ro + item * 16 + half * 8, (u32x2){call, info});
        } else {
            u32 old = 0;
            if (lane == 0) asm volatile("ds_write_b32 %1, %2\n\tds_add_rtn_u32 %0, %3, %4\n\ts_waitcnt lgkmcnt(0)"
                                        : "=&v"(old) : "v"(lquad + ((item & 3) * 2 + half) * 4), "v"(info), "v"(lquad + 32), "v"(1u) : "memory");
            if (uni(old) == 7u) {
                u32x2 in2; asm volatile("ds_read_b64 %0, %1\n\ts_waitcnt lgkmcnt(0)" : "=&v"(in2) : "v"(lquad + (lane & 3) * 8) : "memory");
                const int r = lane >> 2;
                if (lane < 32 && r != P.rank && (all || r == owner)) st16_sys(peer_ws(P, r) + ro + ((item & ~3) + (lane & 3)) * 16, (u32x4){call, in2.x, call, in2.y});
            }
        }
    }
    CFENCE();
}

__device__ __forceinline__ void scout(const Params& P, int src, int li, int lane, u32 call) {
    char* ws = P.me;
    u32* tmo = (u32*)(ws + OFF_TMO);
    const char* recs = ws + OFF_REC + (call & 1u) * REC_PAR;
    const u32 xpar = (call & 1u) * (u32)X_PAR;
    const u64 base = (u64)P.ws[src];
    const u64 tbase = (u64)call << 12;
    u32 seq = 32 + li,
        isrc = src, cand = src, pieces = 0, nq = 0, k = 0;
    __builtin_amdgcn_s_setprio(2);
    if (lane == 0) asm volatile("global_atomic_umax_x2 %0, %1, off\n\ts_waitcnt vmcnt(0)" :: "v"(ws + OFF_TICK + src * 256), "v"(tbase + (ITEM0 + LPP)) : "memory");
    const u32 s8 = lane & 7, ahead = ((((src - s8 - 1) & 7) - ((P.rank - s8 - 1) & 7) + 7) % 7);
    const u32 hfrom = (s8 != (u32)P.rank && s8 != (u32)src && ahead >= 2u) ? R2_ITEM + ahead * ROT + 5u : ~0u;
    while (seq < (u32)N_ITEMS || isrc != (u32)src) {
        const bool isin = seq < (u32)N_IN_ITEMS;
        u32 item = seq;
        if (isin && seq >= R2_ITEM) { item += ((P.rank - isrc - 1) & 7) * ROT; if (item >= (u32)N_IN_ITEMS) item -= N_IN_ITEMS - R2_ITEM; }
        const char* rp = recs + isrc * REC_SRC + (isin ? item * 16u + (lane & 1) * 8u : REC_OUT + ((item - N_IN_ITEMS) & 511u) * 64u + s8 * 8u);
        u32x2 r; bool dead = false, relay = false;
        u64 tk; u32 prog = 0;
        asm volatile("global_load_dwordx2 %0, %1, off sc0 sc1" : "=v"(r) : "v"(rp) : "memory");
        if (lane == 0) asm volatile("global_atomic_add_x2 %0, %1, %2, off sc0" : "=v"(tk) : "v"(ws + OFF_TICK + cand * 256), "v"(1ull) : "memory");
        const bool look = ((li ^ src) & 3) == 0 && seq - (R2_ITEM + 2 * ROT) < (u32)N_IN_ITEMS + 80u - (R2_ITEM + 2 * ROT);
        if (look && lane < 8) asm volatile("global_atomic_add %0, %1, %2, off sc0" : "=v"(prog) : "v"(ws + OFF_TICK + s8 * 256), "v"(0u) : "memory");
        u32 j = 0, u = 0;
        if (item < 448u) { j = item / 56u; u = item % 56u; }
        else if (isin) { const u32 v = item - 448u, w = v % 192u; j = w / 24u; u = 56u + (v / 192u) * 24u + w % 24u; }
        const u32 dest = (call & 1u) * U_PAR + (u >= 32u ? U_B + ((((u - 32u) >> 1) * 8 + j) * 8 + isrc) * 4 + (u & 1u) * 2 : ((u * 8 + j) * 8 + isrc) * 2);
        asm volatile("s_waitcnt vmcnt(0)" : "+v"(r), "+v"(tk), "+v"(prog) :: "memory");
        if (isrc != (u32)src && isin && seq >= R2_ITEM) {
            const u32x2 f = ld8_sys((const char*)base + OFF_FA + (size_t)(dest + (lane & 1)) * 8);
            relay = __builtin_amdgcn_ballot_w64(f.x == call) == ~0ull;
            if (relay) r = (u32x2){call, 512u | (f.y << 25)};
        }
        if (seq < (u32)N_ITEMS && __builtin_amdgcn_ballot_w64(r.x >= call) != ~0ull) {
            const u64 t0 = now(); u32 sp = 0;
            do {
                if (give_up(sp, t0, tmo, call)) { dead = true; break; }
                __builtin_amdgcn_s_sleep(2);
                r = ld8_sys(rp);
            } while (__builtin_amdgcn_ballot_w64(r.x >= call) != ~0ull);
        }
        u32 next = uni((u32)tk) - (u32)tbase;
        if (next < 32u + LPP) next -= LPP;
        const u32 nsrc = cand;
        cand = src;
        if (look) {
            const u32 ps = prog - (u32)tbase;
            const bool ok = lane < 8 && ps >= hfrom && ps + 12u <= (u32)N_IN_ITEMS && ps + (li == src ? HELP_LAG : HELP_LAG2) <= (u32)__builtin_amdgcn_readlane(ps, src);
            if (__builtin_amdgcn_ballot_w64(ok)) cand = wave_umax(ok ? ((4096u - ps) << 3) | s8 : 0u) & 7u;
        }
        if (seq >= (u32)N_ITEMS) { seq = next; isrc = nsrc; continue; }
        u32 soff, bytes, nsub; u32x4 d;
        if (isin) {
            if (relay) soff = (u32)OFF_A + dest * 8192u;
            else if (u >= 32u) { const u32 lp = (u - 32u) >> 1, gp = lp < 4u ? 32u + P.rank * 4u + lp : lp - 4u; soff = (u32)OFF_BX + xpar + (gp * 8 + j) * 32768u + (u & 1u) * 16384u; }
            else soff = (u32)OFF_AX + xpar + (u * 8 + j) * 16384u;
            const u32 s = lane & 1, sz = min(r.y & 1023u, 512u), sb = dead ? 0u : sz * 16u;
            const u32 b0 = __builtin_amdgcn_readlane(sb, 0);
            bytes = b0 + __builtin_amdgcn_readlane(sb, 1); nsub = 2;
            d.x = pieces * 1024u + (s ? b0 : 0u); d.y = pieces + (((s ? bytes : b0) + 1023u) >> 10);
            d.z = dead ? (62u << 12) | (2u << 24) : sz | ((r.y >> 25) << 12); d.w = ((r.y >> 10) & 0x7fffu) | ((dest + s) << 16);
        } else {
            const u32 o = item - N_IN_ITEMS;
            soff = (u32)OFF_O + o * (u32)QSLOT;
            const u32 a = __builtin_amdgcn_ds_bpermute((lane & 3) * 8, r.y), b = __builtin_amdgcn_ds_bpermute((lane & 3) * 8 + 4, r.y);
            const u32 sb = dead ? 0u : min(a & 1023u, 512u) * 16u, n = min((a >> 10) & 0xfffu, 2048u), e = b & 0xfffu, b16 = (b >> 12) & 0x7fffu;
            const u32 s0 = __builtin_amdgcn_readlane(sb, 0), s1 = s0 + __builtin_amdgcn_readlane(sb, 1), s2 = s1 + __builtin_amdgcn_readlane(sb, 2);
            const u32 off = (lane & 3) == 0 ? 0u : (lane & 3) == 1 ? s0 : (lane & 3) == 2 ? s1 : s2;
            bytes = s2 + __builtin_amdgcn_readlane(sb, 3); nsub = 4;
            d.x = pieces * 1024u + off; d.y = pieces + ((off + sb + 1023u) >> 10); d.z = n | (e << 12) | ((dead ? 3u : sb == 8192u ? 4u : 1u) << 24);
            d.w = b16 | ((o * 4u + (lane & 3)) << 16) | (isrc << 29);
        }
        const u32 np = (bytes + 1023u) >> 10;
        u64 ibase = base;
        if (__builtin_expect(isrc != (u32)src && !relay, 0)) ibase = (u64)P.ws[isrc & 7];
        {
            u64 t0 = 0; u32 sp = 0;
            for (;;) {
                u32 nx = lds_ld(lane < 6 ? L_FCTL + 32 + lane * 4 : L_START);
                u32 qm = __builtin_amdgcn_readlane(nx, 0);
#pragma unroll
                for (int i = 1; i < 6; ++i) qm = min(qm, (u32)__builtin_amdgcn_readlane(nx, i));
                const u32 first = qm < nq ? lds_ldu(L_DESC + (qm % NQ) * 16) >> 10 : pieces;
                if ((u32)__builtin_amdgcn_readlane(nx, 6) + 1u >= k && pieces + np - first <= RING / 1024 && (qm >= nq || nq + nsub - qm <= NQ)) break;
                if ((++sp & 1023u) == 0u) { if (!t0) t0 = now(); else if (now() - t0 > 4 * DEADLINE) { st4_sys(tmo, call); break; } }
                __builtin_amdgcn_s_sleep(4);
            }
        }
        if (lane < (int)nsub) asm volatile("ds_write_b128 %0, %1" :: "v"(L_DESC + ((nq + lane) % NQ) * 16), "v"(d) : "memory");
        nq += nsub;
        if (lane == 0) {
            const u64 ad = ibase + soff;
            const u32x4 it = {(u32)ad, bytes, (u32)(ad >> 32), k + 1};
            asm volatile("ds_write_b128 %0, %1" :: "v"(L_ITEM + (k & 3u) * 16), "v"(it) : "memory");
            lds_st(L_FCTL + 4, nq);
        }
        pieces += np; ++k;
        seq = next; isrc = nsrc;
    }
    if (lane == 0) {
        const u32x4 it = {0u, 0u, 0u, (k + 1) | 0x80000000u};
        asm volatile("ds_write_b128 %0, %1" :: "v"(L_ITEM + (k & 3u) * 16), "v"(it) : "memory");
        lds_st(L_FCTL + 4, nq | 0x80000000u);
    }
}

__device__ __forceinline__ void loader(const Params& P, int lane, u32 call) {
    const u32 voff = lane * 16, vland = L_LAND + lane * 4, vstart = L_START + lane * 4;
    u32* tmo = (u32*)(P.me + OFF_TMO);
    int issued = 0, landed = 0;
    u32 k = 0;
    asm volatile("fk_loader:\n\ts_setprio 3");
#define PIECE(sp) do { asm volatile("s_mov_b32 m0, %2\n\ts_waitcnt vmcnt(%3)\n\tglobal_load_lds_dwordx4 %0, %1 sc0 sc1" \
                                    :: "v"(voff), "s"(sp), "s"(((u32)issued * 1024u) & (RING - 1)), "n"(WIN - 1) : "memory"); } while (0)
    u32x4 it = lds_ld16(L_ITEM);
    for (;;) {
        ++k;
        if (__builtin_expect(uni(it.w) != k, 0)) {
            const u32 ia = L_ITEM + ((k - 1) & 3u) * 16;
#define STEP(n) if (uni(it.w) != k) { asm volatile("s_waitcnt vmcnt(%0)" :: "n"(n) : "memory"); landed = max(landed, issued - (n)); lds_st(vland, (u32)landed); it = lds_ld16(ia); }
            STEP(WIN - 1) STEP(WIN - 2) STEP(WIN - 3) STEP(WIN - 4) STEP(WIN - 5) STEP(WIN - 6) STEP(0)
#undef STEP
            const u64 t0 = now(); u32 sp = 0;
            while ((uni(it.w) & 0x7fffffffu) != k) {
                if ((++sp & 1023u) == 0u && now() - t0 > 4 * DEADLINE) { st4_sys(tmo, call); return; }
                __builtin_amdgcn_s_sleep(1);
                it = lds_ld16(ia);
            }
            if (uni(it.w) != k) return;
        }
        u64 sp = ((u64)uni(it.z) << 32) | uni(it.x);
        u32 left = uni(it.y);
        asm volatile("s_nop 4" : "+s"(sp));
        lds_st(vstart, k);
#pragma unroll 1
        for (; left > 1024u; left -= 1024u, sp += 1024) {
            PIECE(sp);
            landed = max(landed, issued - (WIN - 1));
            lds_st(vland, (u32)landed);
            ++issued;
        }
        asm volatile("ds_read_b128 %0, %1" : "=v"(it) : "v"(L_ITEM + (k & 3u) * 16) : "memory");
        if (left) {
            if (voff < left) PIECE(sp);
            landed = max(landed, issued - (WIN - 1));
            lds_st(vland, (u32)landed);
            ++issued;
        }
        asm volatile("s_waitcnt lgkmcnt(0)" : "+v"(it) :: "memory");
    }
#undef PIECE
}

template <int G> __device__ __forceinline__ void expand_dense(LAS char* L, LAS char* scr, u32 lane, u32 n8, u32 o_lo, u32 o_mid, u32 o_top, u32 o_sec, u32 b2) {
    u32x4 lo[G]; u32x2 mid[G]; u32 top[G], u[G][4], fl[G][4], ac[G][4], lw[G], t0[G], t1[G], bp[G], tot = 0;
#pragma unroll
    for (int i = 0; i < G; ++i) {
        const u32 g = i * 64 + lane;
        mid[i] = *(LAS u32x2*)(L + ((o_mid + g * 8u) & (RING - 1)));
        top[i] = *(LAS unsigned char*)(L + ((o_top + g) & (RING - 1)));
    }
#pragma unroll
    for (int i = 0; i < G; ++i) lo[i] = *(LAS u32x4*)(L + ((o_lo + (i * 64 + lane) * 16u) & (RING - 1)));
#pragma unroll
    for (int i = 0; i < G; ++i) {
        if (i == G - 1) top[i] = i * 64 + lane < n8 ? top[i] : 0u;
        const u32 tt = (top[i] | (top[i] << 12)) << 8;
#pragma unroll
        for (int k = 0; k < 4; ++k) {
            u[i][k] = __builtin_amdgcn_perm(0u, mid[i][k >> 1], (k & 1) ? 0x0c030c02u : 0x0c010c00u) | ((tt >> k) & 0x01000100u);
            fl[i][k] = PW((i16x2)(PI(u[i][k] + 0x7e807e80u) >> (i16x2){15, 15}));
        }
        lw[i] = pk_offsets<2>(fl[i], ac[i]);
        bp[i] = tot; tot += pk_sum(lw[i]);
    }
    const u32 inc = wave_scan(tot);
#pragma unroll
    for (int i = 0; i < G; ++i) {
        bp[i] += inc - tot;
        const u32 a = o_sec + (bp[i] >> 5) * 4u;
        t0[i] = *(LAS u32*)(L + (a & (RING - 1))); t1[i] = *(LAS u32*)(L + ((a + 4u) & (RING - 1)));
    }
    u32 s2[G][4];
    tot = 0;
#pragma unroll
    for (int i = 0; i < G; ++i) {
        const u32 TT = pk_split(__builtin_amdgcn_alignbit(t1[i], t0[i], bp[i]), lw[i]);
#pragma unroll
        for (int k = 0; k < 4; ++k) { s2[i][k] = pk_field(TT, ac[i][k], fl[i][k], 0x00030003u); fl[i][k] = pk_is3(s2[i][k]); }
        lw[i] = pk_offsets<3>(fl[i], ac[i]);
        bp[i] = tot; tot += pk_sum(lw[i]);
    }
    const u32 base = wave_scan(tot) - tot + __builtin_amdgcn_readlane(inc, 63);
#pragma unroll
    for (int i = 0; i < G; ++i) {
        bp[i] += base;
        const u32 a = o_sec + (bp[i] >> 5) * 4u;
        t0[i] = *(LAS u32*)(L + (a & (RING - 1))); t1[i] = *(LAS u32*)(L + ((a + 4u) & (RING - 1)));
    }
#pragma unroll
    for (int i = 0; i < G; ++i) {
        const u32 TT = pk_split(__builtin_amdgcn_alignbit(t1[i], t0[i], bp[i]), lw[i]);
        u32 r[8];
#pragma unroll
        for (int k = 0; k < 4; ++k) {
            const u32 h = pk_expand(s2[i][k] + pk_field(TT, ac[i][k], fl[i][k], 0x00070007u), 0xff80ff80u, u[i][k] + b2);
            r[2 * k] = __builtin_amdgcn_perm(h, lo[i][k], 0x05040100u); r[2 * k + 1] = __builtin_amdgcn_perm(h, lo[i][k], 0x07060302u);
        }
        *(LAS u32x4*)(scr + (i * 64 + lane) * 32u) = (u32x4){r[0], r[1], r[2], r[3]};
        *(LAS u32x4*)(scr + (i * 64 + lane) * 32u + 16u) = (u32x4){r[4], r[5], r[6], r[7]};
    }
}

__device__ __forceinline__ void decoder(const Params& P, LAS char* L, int dw, int lane, u32 call) {
    char* ws = P.me;
    u32* tmo = (u32*)(ws + OFF_TMO);
    LAS char* scr = L + L_SCR + dw * SCR_B;
    u32 qn = dw, pend = 0;
    auto flush = [&]() { if (pend) { VMCNT(0); if (lane == 0) st8_sys(ws + OFF_FA + (size_t)(pend & 0xffffu) * 8, (u32x2){call, (pend >> 16) & 63u}); pend = 0; } };
    auto release = [&]() { if (lane == 0) asm volatile("ds_add_rtn_u32 %0, %1, %2\n\ts_waitcnt lgkmcnt(0)\n\tds_write_b32 %3, %0"
                                                       : "=&v"(qn) : "v"(L_FCTL + 8), "v"(1u), "v"(L_FCTL + 32 + dw * 4) : "memory"); };
    for (;;) {
        const u32 q = uni(qn);
        u32x4 dv;
        {
            u64 t0 = 0; u32 sp = 0;
            for (;;) {
                u32 nq, ld;
                asm volatile("ds_read_b32 %0, %3\n\tds_read_b128 %1, %4\n\tds_read_b32 %2, %5\n\ts_waitcnt lgkmcnt(0)"
                             : "=&v"(nq), "=&v"(dv), "=&v"(ld) : "v"(L_FCTL + 4), "v"(L_DESC + (q % NQ) * 16), "v"(L_LAND) : "memory");
                nq = uni(nq);
                if ((nq & 0x7fffffffu) > q && uni(ld) >= uni(dv.y)) break;
                flush();
                if ((nq & 0x7fffffffu) <= q && (nq >> 31)) return;
                if ((++sp & 1023u) == 0u) { if (!t0) t0 = now(); else if (now() - t0 > 4 * DEADLINE) { st4_sys(tmo, call); return; } }
                __builtin_amdgcn_s_sleep(1);
            }
        }
        const u32 pos = uni(dv.x), z = uni(dv.z), wv = uni(dv.w);
        const u32 type = z >> 24, dest = wv >> 16;
        if (type == 0u || type == 2u) {
            if (type == 0u) {
                const u32 sz = z & 0xfffu, b2 = (wv & 0xffffu) * 0x10001u, pl = pos + lane * 16u;
                char* d = ws + OFF_A + (size_t)dest * 8192;
                u32x4 c[8];
#pragma unroll
                for (int i = 0; i < 6; ++i) c[i] = *(LAS u32x4*)(L + ((pl + i * 1024u) & (RING - 1)));
                if (sz == 512u) {
                    c[6] = *(LAS u32x4*)(L + ((pl + 6144u) & (RING - 1))); c[7] = *(LAS u32x4*)(L + ((pl + 7168u) & (RING - 1)));
                    LGKM0();
#pragma unroll
                    for (int i = 0; i < 8; ++i) asm volatile("" : "+v"(c[i]));
                    release();
                    flush();
#pragma unroll
                    for (int f = 0; f < 8; ++f) st16_sys(d + f * 1024 + lane * 16, c[f]);
                } else {
                    u32x2 c6 = *(LAS u32x2*)(L + ((pos + 6144u + lane * 8u) & (RING - 1)));
                    for (u32 g = lane; g + 416u < sz; g += 64u) *(LAS u32x4*)(scr + g * 16u) = *(LAS u32x4*)(L + ((pos + 6656u + g * 16u) & (RING - 1)));
                    LGKM0();
#pragma unroll
                    for (int i = 0; i < 6; ++i) asm volatile("" : "+v"(c[i]));
                    asm volatile("" : "+v"(c6));
                    release();
                    const u32 e = min(uni(*(LAS u32*)scr), 384u);
                    u32 w[32], fl[32], ac[32], len[8], lo[8], tot = 0;
#pragma unroll
                    for (int i = 0; i < 24; ++i) w[i] = c[i >> 2][i & 3];
                    w[24] = c6.x; w[25] = c6.y;
#pragma unroll
                    for (int i = 0; i < 6; ++i) {
                        u32 t = (w[24 + i / 3] << (1 + i % 3)) & 0x80008000u;
#pragma unroll
                        for (int k = 0; k < 4; ++k) t |= (w[i * 4 + k] >> (12 - 3 * k)) & (0x00070007u << (3 * k));
                        w[26 + i] = t;
                    }
#pragma unroll
                    for (int f = 0; f < 8; ++f) {
#pragma unroll
                        for (int k = 0; k < 4; ++k) fl[f * 4 + k] = PW((i16x2)(PI((w[f * 4 + k] & 0x0c000c00u) + 0x74007400u) >> (i16x2){15, 15}));
                        lo[f] = pk_offsets<2>(fl + f * 4, ac + f * 4);
                        len[f] = pk_sum(lo[f]); tot += len[f];
                    }
                    const u32 inc = wave_scan(tot);
                    u32 bp = (e + 1u) * 32u + inc - tot;
                    u32 T[8], s2[32];
#pragma unroll
                    for (int f = 0; f < 8; f += 2) {
                        const LAS u32* a = (const LAS u32*)(scr + (bp >> 5) * 4u);
                        T[f] = __builtin_amdgcn_alignbit(a[1], a[0], bp);
                        T[f + 1] = T[f] >> len[f];
                        bp += len[f] + len[f + 1];
                    }
                    tot = 0;
#pragma unroll
                    for (int f = 0; f < 8; ++f) {
                        const u32 TT = pk_split(T[f], lo[f]);
#pragma unroll
                        for (int k = 0; k < 4; ++k) { const int i = f * 4 + k; s2[i] = pk_field(TT, ac[i], fl[i], 0x00030003u); fl[i] = pk_is3(s2[i]); }
                        lo[f] = pk_offsets<3>(fl + f * 4, ac + f * 4);
                        len[f] = pk_sum(lo[f]); tot += len[f];
                    }
                    flush();
                    bp = (e + 1u) * 32u + __builtin_amdgcn_readlane(inc, 63) + wave_scan(tot) - tot;
#pragma unroll
                    for (int f = 0; f < 8; ++f) {
                        const LAS u32* a = (const LAS u32*)(scr + (bp >> 5) * 4u);
                        T[f] = __builtin_amdgcn_alignbit(a[1], a[0], bp);
                        bp += len[f];
                    }
#pragma unroll
                    for (int f = 0; f < 8; ++f) {
                        const u32 TT = pk_split(T[f], lo[f]);
                        u32x4 o;
#pragma unroll
                        for (int k = 0; k < 4; ++k) {
                            const int i = f * 4 + k;
                            o[k] = pk_expand(s2[i] + pk_field(TT, ac[i], fl[i], 0x00070007u), 0xfc00fc00u, PW((u16x2)(PU(w[i] & 0x8fff8fffu) + PU(b2))));
                        }
                        st16_sys(d + f * 1024 + lane * 16, o);
                    }
                    for (u32 g = lane; g < e; g += 64u) {
                        const u32 rr = *(LAS u32*)(scr + 4u + g * 4u);
                        asm volatile("global_store_short_d16_hi %0, %1, off sc0 sc1" :: "v"(d + (rr & 4095u) * 2u), "v"(rr) : "memory");
                    }
                }
            } else { release(); flush(); }
            pend = dest | (((z >> 12) & 63u) << 16) | 0x80000000u;
            continue;
        }
        flush();
        const u32 tile = (dest >> 5) & 63u, wq = (dest >> 2) & 7u, su = dest & 3u;
        const u32 row0 = (tile >> 2) * 256u + (su >> 1) * 128u + (wq >> 2) * 64u + (su & 1u) * 32u + (lane >> 4) * 4u;
        const u32 col0 = 8192u + (dest >> 13) * 1024u + (tile & 3u) * 256u + (wq & 3u) * 64u + (lane & 15) * 4u;
        GAS float* o = (GAS float*)P.out + (size_t)row0 * GN + col0;
        if (type == 3u) {
            release();
            const float qn = __builtin_nanf("");
#pragma unroll
            for (int g = 0; g < 8; ++g) *(GAS f32x4*)(o + (size_t)((g >> 2) * 16 + (g & 3)) * GN) = (f32x4){qn, qn, qn, qn};
            continue;
        }
        if (type == 4u) {
            u32x4 c[8];
#pragma unroll
            for (int g = 0; g < 8; ++g) c[g] = *(LAS u32x4*)(L + ((pos + g * 1024u + lane * 16u) & (RING - 1)));
            LGKM0();
#pragma unroll
            for (int g = 0; g < 8; ++g) asm volatile("" : "+v"(c[g]));
            release();
#pragma unroll
            for (int g = 0; g < 8; ++g) st_out(o + (size_t)((g >> 2) * 16 + (g & 3)) * GN, __builtin_bit_cast(f32x4, c[g]));
            continue;
        }
        const u32 n = z & 0xfffu, e = (z >> 12) & 0xfffu, b2 = (wv & 0xffffu) * 0x10001u;
        const u32 nr = (n & 511u) <= 64u ? n & 511u : 0u, n8 = (n - nr + 7u) >> 3, G = (n8 + 63u) >> 6;
        u32 mk = *(LAS u32*)(L + ((pos + lane * 4u) & (RING - 1)));
        const u32 o_lo = pos + 256u, o_mid = o_lo + n8 * 16u, o_top = o_mid + n8 * 8u, o_raw = o_top + ((n8 + 3u) & ~3u);
        const u32 o_esc = o_raw + nr * 4u, o_sec = o_esc + e * 4u;
        if (nr) *(LAS u32*)(scr + n8 * 32u + lane * 4u) = *(LAS u32*)(L + ((o_raw + lane * 4u) & (RING - 1)));
        if (G == 3u) expand_dense<3>(L, scr, lane, n8, o_lo, o_mid, o_top, o_sec, b2);
        else if (G == 2u) expand_dense<2>(L, scr, lane, n8, o_lo, o_mid, o_top, o_sec, b2);
        else if (G == 1u) expand_dense<1>(L, scr, lane, n8, o_lo, o_mid, o_top, o_sec, b2);
        else if (G == 4u) expand_dense<4>(L, scr, lane, n8, o_lo, o_mid, o_top, o_sec, b2);
        CFENCE();
#pragma unroll 1
        for (u32 g = lane; g < e; g += 64u) {
            const u32 rr = *(LAS u32*)(L + ((o_esc + g * 4u) & (RING - 1)));
            *(LAS unsigned short*)(scr + (rr & 2047u) * 4u + 2u) = (unsigned short)(rr >> 16);
        }
        LGKM0();
        asm volatile("" : "+v"(mk));
        release();
        {
            u32 ad = L_SCR + dw * SCR_B + wave_scan(__builtin_popcount(mk)) * 4u;
            u32 r[32], m[32];
#pragma unroll
            for (int i = 31; i >= 0; --i) {
                m[i] = (u32)__builtin_amdgcn_sbfe((int)mk, i, 1);
                asm("v_lshl_add_u32 %0, %1, 2, %2" : "=v"(ad) : "v"(m[i]), "v"(ad));
                r[i] = *(LAS u32*)(size_t)ad;
            }
            __builtin_amdgcn_sched_barrier(0);
#pragma unroll
            for (int g = 7; g >= 0; --g)
                st_out(o + (size_t)((g >> 2) * 16 + (g & 3)) * GN,
                       __builtin_bit_cast(f32x4, (u32x4){r[g * 4] & m[g * 4], r[g * 4 + 1] & m[g * 4 + 1], r[g * 4 + 2] & m[g * 4 + 2], r[g * 4 + 3] & m[g * 4 + 3]}));
        }
        CFENCE();
    }
}

__device__ __forceinline__ void gemm_wg(const Params& P, LAS char* L, int pm, int lp0, int wave, int lane, u32 call) {
    char* ws = P.me;
    const int wr = wave >> 2, wc = wave & 3;
    const u32 voffA = wave * 2048 + lane * 16, voffB = (wave >> 1) * 8192 + (wave & 1) * 2048 + lane * 16;
    const u32 boff = wc * 4096 + lane * 16, ldw = wave * 2048;
    auto lane_id = []() -> u32 { u32 l; asm volatile("v_mbcnt_lo_u32_b32 %0, -1, 0\n\tv_mbcnt_hi_u32_b32 %0, -1, %0" : "=v"(l)); return l; };
    constexpr size_t KA = 16384, KB = 32768, HA = 64 * KA, HB = 4096;
#define SA(b, h) (((b) * 2 + (h)) * 16384)
#define SB(b, h) ((4 + (b) * 2 + (h)) * 16384)
#define STAGE(bufoff, gbase, voff) asm volatile("s_mov_b32 m0, %2\n\ts_nop 0\n\tglobal_load_lds_dwordx4 %0, %1 sc0 sc1\n\tglobal_load_lds_dwordx4 %0, %1 offset:1024 sc0 sc1" \
        :: "v"(voff), "s"((u64)(gbase)), "s"((u32)(bufoff) + ldw) : "memory")
#define LDA(dst, b, h) do { _Pragma("unroll") for (int m = 0; m < 4; ++m) _Pragma("unroll") for (int k = 0; k < 2; ++k) \
        dst[m][k] = *(const LAS f16x8*)(L + SA(b, h) + ((h) ? aoff1 : aoff0) + m * 2048 + k * 1024); } while (0)
#define LDB(dst, b, h) do { _Pragma("unroll") for (int n = 0; n < 2; ++n) _Pragma("unroll") for (int k = 0; k < 2; ++k) \
        dst[n][k] = *(const LAS f16x8*)(L + SB(b, h) + boff + n * 2048 + k * 1024); } while (0)
#define MMA(ai, bj, At, Bt) do { __builtin_amdgcn_sched_barrier(0);         \
        _Pragma("unroll") for (int k = 0; k < 2; ++k) _Pragma("unroll") for (int mm = 0; mm < 4; ++mm) _Pragma("unroll") for (int nn = 0; nn < 2; ++nn) { \
            const int m = k ? 3 - mm : mm, n = (mm & 1) ? 1 - nn : nn; \
            acc[ai][bj][m][n] = __builtin_amdgcn_mfma_f32_16x16x32_f16(At[m][k], Bt[n][k], acc[ai][bj][m][n], 0, 0, 0); \
            __builtin_amdgcn_sched_barrier(0); } } while (0)
#define WAIT_L(n) asm volatile("s_waitcnt lgkmcnt(" #n ")" ::: "memory")
#define WAIT_L0 __builtin_amdgcn_s_waitcnt(0xC07F)
#define RBAR __builtin_amdgcn_s_barrier()
#define SCHED __builtin_amdgcn_sched_barrier(0)
    auto flag_ptr = [&](int lp, int j, int ent) -> const char* {
        return ws + OFF_FA + (size_t)((call & 1u) * U_PAR + (ent < 32 ? ((pm * 2 + (ent >> 4)) * 8 + j) * 16 + (ent & 15) : U_B + (lp * 8 + j) * 32 + (ent - 32))) * 8; };
    auto poll = [&](int lp, int j) {
        const u32 ln = lane_id();
        const char* f = flag_ptr(lp, j, ln);
        const u64 t0 = now(); u32 sp = 0; u32x2 v;
        for (;;) {
            v = ld8_sys(f);
            if (__builtin_amdgcn_ballot_w64(v.x >= call) == ~0ull) break;
            if (give_up(sp, t0, (u32*)(ws + OFF_TMO), call)) { v.y = 62u; break; }
            __builtin_amdgcn_s_sleep(4);
        }
        asm volatile("ds_write_b64 %0, %1\n\ts_waitcnt lgkmcnt(0)" :: "v"(L_T2 + ln * 8), "v"(v) : "memory");
    };
    auto codes = [&](u32 tb) -> u32 {
        const u32 r = lane_id() & 7;
        const u32 a0 = *(const LAS u32*)(L + tb + wr * 8 + 4 + r * 16), a1 = *(const LAS u32*)(L + tb + 128 + wr * 8 + 4 + r * 16);
        const u32 b = *(const LAS u32*)(L + tb + 256 + wc * 8 + 4 + r * 32);
        return a0 | (a1 << 8) | (b << 16);
    };
    f32x4 acc[2][2][4][2];
    f16x8 At[4][2], B0[2][2], B1[2][2];
    u32 cur = 0, cmax = 0, bad = 0;
    u32 aoff0 = wr * 8192 + lane * 16, aoff1 = aoff0;
    u32 nm, nmn = 0;
    auto changes = [&](u32 nw, u32 old) -> u32 {
        const u32 prev = __builtin_amdgcn_update_dpp((int)__builtin_amdgcn_readlane(old, 7), (int)nw, 0x111, 0xf, 0xf, false);
        return (u32)__builtin_amdgcn_ballot_w64(nw != prev) & 255u; };
#define ADJUST(r, buf) do { if (__builtin_expect((nm >> (r)) & 1u, 0)) { const u32 cn_ = __builtin_amdgcn_readlane(pk, (r)), cb_ = (cn_ >> 16) & 63u; \
            const u32 l16_ = lane_id() * 16; \
            _Pragma("unroll") for (int ai = 0; ai < 2; ++ai) { \
                const u32 ca_ = (cn_ >> (8 * ai)) & 63u, c_ = ca_ + cb_, o_ = (cur >> (8 * ai)) & 255u, x_ = (cmax >> (8 * ai)) & 255u; \
                u32 ab_ = wr * 8192; int sh_ = 0; \
                if (ca_ == 62u || cb_ == 62u) bad = 1; \
                else if (ca_ != 63u && cb_ != 63u) { \
                    if (c_ + 10u < x_) { ab_ = L_ZERO - SA(buf, ai); nm = 255u; nmn |= 1u; }      \
                    else { sh_ = 8 * ((int)o_ - (int)c_); \
                        cur = (cur & ~(255u << (8 * ai))) | (c_ << (8 * ai)); \
                        if (c_ > x_) cmax = (cmax & ~(255u << (8 * ai))) | (c_ << (8 * ai)); } } \
                asm volatile("v_add_u32 %0, %1, %2" : "+v"(ai ? aoff1 : aoff0) : "s"(ab_), "v"(l16_)); \
                if (sh_ != 0) { \
                    asm volatile("s_nop 15" ::: "memory");        \
                    _Pragma("unroll") for (int bj = 0; bj < 2; ++bj) _Pragma("unroll") for (int m = 0; m < 4; ++m) _Pragma("unroll") for (int n = 0; n < 2; ++n) \
                        _Pragma("unroll") for (int e_ = 0; e_ < 4; ++e_) asm volatile("v_ldexp_f32 %0, %0, %1" : "+v"(acc[ai][bj][m][n][e_]) : "s"(sh_)); } } } } while (0)
#pragma unroll
    for (int a = 0; a < 2; ++a)
#pragma unroll
        for (int b = 0; b < 2; ++b)
#pragma unroll
            for (int m = 0; m < 4; ++m)
#pragma unroll
                for (int n = 0; n < 2; ++n) acc[a][b][m][n] = (f32x4){0.f, 0.f, 0.f, 0.f};
    const char* const cA = ws + OFF_A + (size_t)(call & 1u) * U_PAR * 8192 + (size_t)pm * 2 * HA;
    const char* cB = ws + OFF_A + (size_t)((call & 1u) * U_PAR + U_B) * 8192 + (size_t)lp0 * 64 * KB;
    *(LAS u32x4*)(L + L_ZERO + wave * 1024 + lane * 16) = (u32x4){0u, 0u, 0u, 0u};
    if (wave == 0) poll(lp0, 0);
    LGKM0();
    BAR();
    u32 pk = codes(L_T2), pkn = pk;
    nm = changes(pk, pk) | 1u;
    STAGE(SB(0, 0), cB, voffB); STAGE(SA(0, 0), cA, voffA); STAGE(SB(0, 1), cB + HB, voffB); STAGE(SA(0, 1), cA + HA, voffA);
    if (wr == 1) RBAR;
    VMCNT(4); RBAR;
    STAGE(SB(1, 0), cB + KB, voffB); STAGE(SA(1, 0), cA + KA, voffA); STAGE(SB(1, 1), cB + HB + KB, voffB);
    VMCNT(6); RBAR;
#pragma unroll 1
    for (int rd = 0; rd < 3; ++rd) {
        const int lp = lp0 + rd * 12;
        const char* nB = rd < 2 ? cB + 12 * 64 * KB : cB;
#define TRIP(r, a1, a2, b2, HOOK) do { \
            ADJUST(r, 0); \
            LDB(B0, 0, 0); SCHED; LDA(At, 0, 0); STAGE(SA(1, 1), (a1) + HA, voffA); \
            WAIT_L(8); RBAR; WAIT_L0; MMA(0, 0, At, B0); RBAR; SCHED; \
            LDB(B1, 0, 1); STAGE(SB(0, 0), (b2), voffB); \
            RBAR; WAIT_L0; MMA(0, 1, At, B1); RBAR; \
            LDA(At, 0, 1); STAGE(SA(0, 0), (a2), voffA); \
            RBAR; WAIT_L0; MMA(1, 0, At, B0); RBAR; SCHED; \
            STAGE(SB(0, 1), (b2) + HB, voffB); \
            VMCNT(6); RBAR; MMA(1, 1, At, B1); RBAR; \
            ADJUST((r) + 1, 1); \
            LDB(B0, 1, 0); SCHED; LDA(At, 1, 0); STAGE(SA(0, 1), (a2) + HA, voffA); \
            WAIT_L(8); RBAR; WAIT_L0; MMA(0, 0, At, B0); RBAR; SCHED; \
            LDB(B1, 1, 1); STAGE(SB(1, 0), (b2) + KB, voffB); \
            RBAR; WAIT_L0; MMA(0, 1, At, B1); RBAR; \
            LDA(At, 1, 1); STAGE(SA(1, 0), (a2) + KA, voffA); \
            RBAR; WAIT_L0; MMA(1, 0, At, B0); RBAR; SCHED; \
            STAGE(SB(1, 1), (b2) + KB + HB, voffB); \
            HOOK; \
            VMCNT(6); RBAR; MMA(1, 1, At, B1); RBAR; } while (0)
#pragma unroll 1
        for (int j = 0; j < 8; ++j) {
            const char* ga = cA + (size_t)j * 8 * KA; const char* gb = cB + (size_t)j * 8 * KB;
            const char* na = j == 7 ? cA : ga + 8 * KA; const char* nb = j == 7 ? nB : gb + 8 * KB;
            const bool more = j < 7 || rd < 2;
            const int nlp = j < 7 ? lp : lp + 12, nj = (j + 1) & 7;
            TRIP(0, ga + KA, ga + 2 * KA, gb + 2 * KB, do { } while (0));
            TRIP(2, ga + 3 * KA, ga + 4 * KA, gb + 4 * KB,
                 if (__builtin_expect(wave == 0 && more, 0)) { const u32 ln = lane_id(); const char* fp = flag_ptr(nlp, nj, ln * 2);
                     if (ln < 32) asm volatile("s_mov_b32 m0, %1\n\ts_nop 0\n\tglobal_load_lds_dwordx4 %0, off sc0 sc1" :: "v"(fp), "s"(L_T1) : "memory"); });
            TRIP(4, ga + 5 * KA, ga + 6 * KA, gb + 6 * KB, {
                 const u32 tag = *(const LAS u32*)(L + L_T1 + lane_id() * 8);
                 pkn = codes(L_T1);
                 if (__builtin_expect((__builtin_amdgcn_ballot_w64(tag >= call) != ~0ull) & more, 0)) {
                     if (wave == 0) poll(nlp, nj);
                     BAR();
                     pkn = codes(L_T2);
                 }
                 nmn |= changes(pkn, pk); });
            TRIP(6, ga + 7 * KA, na, nb, { pk = pkn; nm = nmn; nmn = 0; });
        }
        {
            const int ln = lane_id();
            const int gp = lp < 4 ? 32 + P.rank * 4 + lp : lp - 4;
            GAS float* o = (GAS float*)P.out + (size_t)(pm * 256 + wr * 64 + (ln >> 4) * 4) * GN + gp * 256 + wc * 64 + (ln & 15) * 4;
#pragma unroll
            for (int ai = 0; ai < 2; ++ai) {
                const int sh = 8 * (int)((cur >> (8 * ai)) & 255u) - 272;
#pragma unroll
                for (int m = 0; m < 4; ++m)
#pragma unroll
                    for (int r = 0; r < 4; ++r) {
                        f32x4 v;
#pragma unroll
                        for (int i = 0; i < 4; ++i) {
                            const float y = __builtin_elementwise_maximum(__builtin_ldexpf(acc[ai][i >> 1][m][i & 1][r], sh), 0.0f);
                            v[i] = __uint_as_float(bad ? 0x7fc00000u : __float_as_uint(y) & 0x7fffffffu);
                            acc[ai][i >> 1][m][i & 1][r] = 0.f;
                        }
                        st_out(o + (size_t)(ai * 128 + m * 16 + r) * GN, v);
                    }
            }
            if (lp < 4) {
                VMCNT(0);
                if (ln == 0) st4_sys(ws + OFF_OF + (size_t)((pm * 4 + lp) * 8 + wave) * 4, call);
            }
            cur = 0; cmax = 0; bad = 0; nm |= 1u;
        }
        cB = nB;
    }
    asm volatile("fk_gemm:\n\ts_waitcnt vmcnt(0)" ::: "memory");
    if (wr == 0) RBAR;
#undef SA
#undef SB
#undef STAGE
#undef LDA
#undef LDB
#undef MMA
#undef ADJUST
#undef TRIP
}

template <int G> __device__ __forceinline__ void pack_dense(LAS char* dense, LAS char* img, u32 lane, u32 n8, u32 o_mid, u32 o_top, u32 o_esc, u32 b2,
                                                            u32* T, u32* T3, u32* bp, u32& tot, u32& e) {
    u32x4 a[G], b[G], lo[G]; u32x2 mid[G]; u32 hi[G][4], ge[G][4], top[G];
#pragma unroll
    for (int i = 0; i < G; ++i) { a[i] = *(LAS u32x4*)(dense + (i * 64 + lane) * 32u); b[i] = *(LAS u32x4*)(dense + (i * 64 + lane) * 32u + 16u); }
#pragma unroll
    for (int i = 0; i < G; ++i) {
        const u32 dd[8] = {a[i].x, a[i].y, a[i].z, a[i].w, b[i].x, b[i].y, b[i].z, b[i].w};
        u32 x[4], c[4], f9[4], len, tp = 0;
#pragma unroll
        for (int k = 0; k < 4; ++k) {
            lo[i][k] = __builtin_amdgcn_perm(dd[2 * k + 1], dd[2 * k], 0x05040100u);
            hi[i][k] = __builtin_amdgcn_perm(dd[2 * k + 1], dd[2 * k], 0x07060302u);
            x[k] = PW((u16x2)(PU(hi[i][k]) - PU(b2)));
            if (i == G - 1) x[k] = i * 64 + lane < n8 ? x[k] : 0u;
        }
        pk_classify<7>(x, c, ge[i], T[i], T3[i], len);
#pragma unroll
        for (int k = 0; k < 4; ++k) { f9[k] = (c[k] << 7) | (x[k] & 0x007f007fu); tp |= ((c[k] >> 1) & 0x00010001u) << k; }
        mid[i] = (u32x2){__builtin_amdgcn_perm(f9[1], f9[0], 0x06040200u), __builtin_amdgcn_perm(f9[3], f9[2], 0x06040200u)};
        top[i] = tp | (tp >> 12);
        bp[i] = tot; tot += len;
    }
#pragma unroll
    for (int i = 0; i < G; ++i) {
        const u32 g = i * 64 + lane;
        if (i < G - 1 || g < n8) {
            *(LAS u32x4*)(img + 256u + g * 16u) = lo[i];
            *(LAS u32x2*)(img + o_mid + g * 8u) = mid[i];
            *(LAS unsigned char*)(img + o_top + g) = (unsigned char)top[i];
        }
    }
#pragma unroll
    for (int i = 0; i < G; ++i) emit_escapes(ge[i], hi[i], (i * 64 + lane) * 8u, img + o_esc, (8192u - o_esc) >> 2, e);
}
__device__ __forceinline__ void packer(const Params& P, LAS char* L, int pw, int wave, int lane, u32 call) {
    char* ws = P.me;
    LAS char* dense = L + wave * PK_B;
    LAS char* img = dense + PK_IMG;
    const auto rs = __builtin_amdgcn_make_buffer_rsrc((void*)P.out, 0, (u32)GM * GN * 4u, 0x00020000);
    auto wait_item = [&](int it) -> u32 {
        const u64 t0 = now(); u32 sp = 0;
        while (uni(ld4_sys(ws + OFF_OF + it * 4)) < call) {
            if (give_up(sp, t0, (u32*)(ws + OFF_TMO), call)) return 0x7fc00000u;
            __builtin_amdgcn_s_sleep(8);
        }
        return 0u;
    };
    u32x4 nx[8];
    auto fetch_block = [&](int it, int su) {
        const int tile = it >> 3, wq = it & 7;
        const u32 so = ((u32)((tile >> 2) * 256 + (su >> 1) * 128 + (wq >> 2) * 64 + (su & 1) * 32) * GN + 8192 + P.rank * 1024 + (tile & 3) * 256 + (wq & 3) * 64) * 4u;
#pragma unroll
        for (int g = 0; g < 8; ++g) nx[g] = __builtin_amdgcn_raw_buffer_load_b128(rs, ((lane >> 4) * 4 * GN + (lane & 15) * 4) * 4, so + ((g >> 2) * 16 + (g & 3)) * GN * 4, 17);
    };
    auto publish = [&](int it, u32 ea, u32 eb) {
        if (lane < 32 && (lane >> 2) != P.rank)
            st16_sys(peer_ws(P, lane >> 2) + OFF_REC + (call & 1u) * REC_PAR + P.rank * REC_SRC + REC_OUT + it * 64 + (lane & 3) * 16, (u32x4){call, ea, call, eb});
    };
    {
        const char* bx = ws + OFF_BX + (call & 1u) * X_PAR + (size_t)8 * 8 * 32768 + lane * 16;
        u32 acc = 0;
#pragma unroll 1
        for (int i = 0; i < 6; ++i) {
            const int sl = pw * 3 + (i >> 1);
            u32x4 t[7];
#pragma unroll
            for (int g = 0; g < 7; ++g) t[g] = *(const GAS u32x4*)(bx + (size_t)sl * 16384 + ((i & 1) * 7 + g) * 1024);
#pragma unroll
            for (int g = 0; g < 7; ++g) acc ^= t[g].x;
        }
        asm volatile("" :: "v"(acc));
    }
    u32 nanm = wait_item(pw), nann = 0, off16 = 0, ea = 0, eb = 0;
    fetch_block(pw, 0);
#pragma unroll
    for (int g = 0; g < 8; ++g) asm volatile("" : "+v"(nx[g]));
#pragma unroll 1
    for (int bk = 0; bk < N_OUT_ITEMS / 16; ++bk) {
        {
            const int it = pw + (bk >> 2) * 64, su = bk & 3;
            char* og = ws + OFF_O + (size_t)it * QSLOT;
            u32 v[32]; u32 mx = 0;
#pragma unroll
            for (int r = 0; r < 32; ++r) { v[r] = nx[r >> 2][r & 3] | nanm; mx = max(mx, v[r]); }
            { const int bn = bk + 1 < N_OUT_ITEMS / 16 ? bk + 1 : bk; fetch_block(pw + (bn >> 2) * 64, bn & 3); }
            mx = wave_umax(mx) >> 16;
            const u32 b16 = mx >= 511u ? mx - 511u : 0u, b2 = b16 * 0x10001u;
            u32 f[32], mk0 = 0, mk1 = 0;
#pragma unroll
            for (int r = 0; r < 32; ++r) asm("v_min_u32 %0, 1, %1" : "=v"(f[r]) : "v"(v[r]));
#pragma unroll
            for (int r = 0; r < 16; ++r) asm("v_lshl_or_b32 %0, %2, %4, %0\n\tv_lshl_or_b32 %1, %3, %5, %1" : "+v"(mk0), "+v"(mk1) : "v"(f[r]), "v"(f[16 + r]), "n"(r), "n"(16 + r));
            const u32 mk = mk0 | mk1, pc = __builtin_popcount(mk), inc = wave_scan(pc), n = __builtin_amdgcn_readlane(inc, 63);
            {
                u32 ad = wave * PK_B + (inc - pc) * 4u;
#pragma unroll
                for (int r = 0; r < 32; r += 8) {
                    u32 a[8]; u64 m[8];
#pragma unroll
                    for (int i = 0; i < 8; ++i) { a[i] = ad; asm("v_lshl_add_u32 %0, %1, 2, %2" : "=v"(ad) : "v"(f[r + i]), "v"(a[i])); m[i] = __builtin_amdgcn_ballot_w64(v[r + i] != 0u); }
                    asm volatile("s_mov_b64 exec, %16\n\tds_write_b32 %0, %8\n\ts_mov_b64 exec, %17\n\tds_write_b32 %1, %9\n\t"
                                 "s_mov_b64 exec, %18\n\tds_write_b32 %2, %10\n\ts_mov_b64 exec, %19\n\tds_write_b32 %3, %11\n\t"
                                 "s_mov_b64 exec, %20\n\tds_write_b32 %4, %12\n\ts_mov_b64 exec, %21\n\tds_write_b32 %5, %13\n\t"
                                 "s_mov_b64 exec, %22\n\tds_write_b32 %6, %14\n\ts_mov_b64 exec, %23\n\tds_write_b32 %7, %15\n\ts_mov_b64 exec, -1"
                                 :: "v"(a[0]), "v"(a[1]), "v"(a[2]), "v"(a[3]), "v"(a[4]), "v"(a[5]), "v"(a[6]), "v"(a[7]),
                                    "v"(v[r]), "v"(v[r + 1]), "v"(v[r + 2]), "v"(v[r + 3]), "v"(v[r + 4]), "v"(v[r + 5]), "v"(v[r + 6]), "v"(v[r + 7]),
                                    "s"(m[0]), "s"(m[1]), "s"(m[2]), "s"(m[3]), "s"(m[4]), "s"(m[5]), "s"(m[6]), "s"(m[7]) : "memory");
                }
            }
            if (lane < 8) *(LAS u32*)(dense + (n + lane) * 4u) = b16 << 16;
            *(LAS u32*)(img + lane * 4) = mk;
            const u32 nr = (n & 511u) <= 64u ? n & 511u : 0u, n8 = (n - nr + 7u) >> 3, G = (n8 + 63u) >> 6;
            const u32 o_mid = 256u + n8 * 16u, o_top = o_mid + n8 * 8u, o_raw = o_top + ((n8 + 3u) & ~3u), o_esc = o_raw + nr * 4u;
            CFENCE();
            if (nr) *(LAS u32*)(img + o_raw + lane * 4u) = *(LAS u32*)(dense + n8 * 32u + lane * 4u);
            CFENCE();
            u32 e = 0, tot = 0, T[4] = {0u, 0u, 0u, 0u}, T3[4] = {0u, 0u, 0u, 0u}, bp[4] = {0u, 0u, 0u, 0u};
            if (G == 3u) pack_dense<3>(dense, img, lane, n8, o_mid, o_top, o_esc, b2, T, T3, bp, tot, e);
            else if (G == 2u) pack_dense<2>(dense, img, lane, n8, o_mid, o_top, o_esc, b2, T, T3, bp, tot, e);
            else if (G == 1u) pack_dense<1>(dense, img, lane, n8, o_mid, o_top, o_esc, b2, T, T3, bp, tot, e);
            else if (G == 4u) pack_dense<4>(dense, img, lane, n8, o_mid, o_top, o_esc, b2, T, T3, bp, tot, e);
#pragma unroll
            for (int g = 0; g < 8; ++g) asm volatile("" : "+v"(nx[g]));
            VMCNT(0);
            if (su == 0) { if (bk) publish(it - 64, ea, eb); off16 = 0; }
            if (su == 2 && it + 64 < N_OUT_ITEMS) nann = wait_item(it + 64);
            const u32 binc = wave_scan(tot), all2 = __builtin_amdgcn_readlane(binc, 63), o_sec = o_esc + e * 4u;
            u32 size16 = (o_sec + ((pk_sum(all2) + 7u) >> 3) + 15u) >> 4;
            if (size16 >= 512u) {
                size16 = 512u;
#pragma unroll
                for (int g = 0; g < 8; ++g) st16_sys(og + off16 * 16u + g * 1024 + lane * 16, (u32x4){v[g * 4], v[g * 4 + 1], v[g * 4 + 2], v[g * 4 + 3]});
            } else {
#pragma unroll
                for (int i = 0; i < 6; ++i) *(LAS u32*)(img + o_sec + i * 256 + lane * 4) = 0u;
                CFENCE();
#pragma unroll
                for (int i = 0; i < 4; ++i) if (i < (int)G) {
                    const u32 bq = bp[i] + binc - tot + (all2 << 16);
                    or_bits(img + o_sec, bq & 0xffffu, T[i]);
                    or_bits(img + o_sec, bq >> 16, T3[i]);
                }
                CFENCE();
#pragma unroll 1
                for (u32 g = lane; g < size16 + lane; g += 256u) {
                    u32x4 t[4];
#pragma unroll
                    for (int i = 0; i < 4; ++i) t[i] = *(LAS u32x4*)(img + (g + i * 64) * 16u);
#pragma unroll
                    for (int i = 0; i < 4; ++i) if (g + i * 64 < size16) st16_sys(og + (off16 + g + i * 64) * 16u, t[i]);
                }
            }
            if ((lane & 3) == su) { ea = size16 | (n << 10); eb = (e & 0xfffu) | (b16 << 12); }
            off16 += size16;
            if (su == 3) nanm = nann;
            LGKM0();
            CFENCE();
        }
    }
    VMCNT(0);
    publish(pw + N_OUT_ITEMS - 64, ea, eb);
}

__global__ __launch_bounds__(NTHR, 2) __attribute__((amdgpu_waves_per_eu(2, 2))) void fused_kernel(Params P) {
    extern __shared__ __attribute__((aligned(16))) char smem[];
    LAS char* L = (LAS char*)smem;
    {
        u64 pc0; int e0, e1;
        asm volatile("fk_begin:\n\ts_getpc_b64 %0\n\ts_mov_b32 %1, fk_loader-fk_begin\n\ts_mov_b32 %2, fk_gemm-fk_begin" : "=s"(pc0), "=s"(e0), "=s"(e1));
        const int o = (((threadIdx.x >> 4) & 31) * FS + (blockIdx.x >> 3)) * 512 + (threadIdx.x & 3) * 128;
        if (blockIdx.x < FS * 8 && (threadIdx.x & 12) == 0 && o < max(e0, e1))
            asm volatile("s_mov_b32 m0, %1\n\ts_nop 0\n\tglobal_load_lds_dword %0, off" :: "v"(pc0 + o), "s"(L_ZERO) : "memory");
    }
    asm volatile("" :: "s"(P.x), "s"(P.w), "s"(P.me), "s"(P.rank));
    const int tid = threadIdx.x, lane = tid & 63, wave = __builtin_amdgcn_readfirstlane(tid >> 6);
    const int b = blockIdx.x, xg = b & 7, s = b >> 3;
    const bool fetch = s < FS;
    constexpr int NF = FS * 8;
    GAS u32* const cnt = (GAS u32*)(P.me + OFF_CNT + (b * 8 + wave) * 8);
    const int pidx = b - (NF + 120);
    constexpr int NU = 112 - ITEM0;
    const int q0 = ITEM0 + (b - NF);
    const bool loaded = !fetch && q0 < 112 && wave < 2;
    f32x4 v[16];
    if (wave == 7 && (unsigned)pidx < (unsigned)NPUSH) {
        load_x(P, pidx, 0, lane, v);
        const u32 craw = __hip_atomic_load(cnt, __ATOMIC_RELAXED, __HIP_MEMORY_SCOPE_AGENT);
        __builtin_amdgcn_sched_barrier(0);
        BAR();
        __builtin_amdgcn_sched_barrier(0);
        push_task(P, pidx, lane, craw, v);
    } else {
        if (loaded) { if (q0 < 32) load_x(P, q0 * 2 + (wave & 1), 0, lane, v); else load_w(P, (q0 < 96 ? q0 + 32 : q0 - 96) * 2 + (wave & 1), 0, lane, v); }
        __builtin_amdgcn_sched_barrier(0);
        if (tid < 256) lds_st(L_ITEM + tid * 4, !fetch ? 0u : tid == 18 ? 6u : tid >= 24 && tid < 32 ? (u32)(tid - 24) : 0u);
        LGKM0();
        BAR();
    }
    if (!fetch) {
        const u64 tstart = now();
        const bool pack = s >= FS + GS;
        auto fetch_task = [&](int q) {
            const u64 go = pack ? (u64)PACE4 : (u64)(PACE2 + (q - 112) * PACE3 / 112);
            while (now() - tstart < go) __builtin_amdgcn_s_sleep(4);
            int j, idx; bool isA;
            task_of(q, wave & 1, j, idx, isA);
            if (isA) load_x(P, idx, j, lane, v); else load_w(P, idx, j, lane, v);
        };
        int qn = 112 + (pack ? GS * 8 + xg : (b - NF + GS * 8 - NU) % (GS * 8)) * 4 + (wave >> 1), q = q0;
        const int qs = pack ? 32 : N_PAIRS;
        u32 lq = L_ITEM;
        if (!loaded) { q = qn; qn += qs; lq += 64; fetch_task(q); }
#pragma unroll 1
        for (;;) {
            stage_task(P, L + wave * 4096, lq, lq + 48 + (wave >> 1) * 4, q, wave & 1, lane, cnt, v);
            if (qn >= N_PAIRS) break;
            q = qn; qn += qs; lq += 64;
            fetch_task(q);
        }
    }
    const u32 call = uni(__hip_atomic_load(cnt, __ATOMIC_RELAXED, __HIP_MEMORY_SCOPE_AGENT)) + 1u;
    if (lane == 0) st4_sys((void*)cnt, call);
    if (fetch) {
        const int src = (P.rank + 1 + s) & 7;
        if (wave == 0) { loader(P, lane, call); }
        else if (wave == 1) scout(P, src, xg, lane, call);
        else { decoder(P, L, wave - 2, lane, call); }
    } else if (s < FS + GS) {
        const int g = s - FS;
        gemm_wg(P, L, (xg >> 1) * 4 + g / 6, (xg & 1) * 6 + g % 6, wave, lane, call);
    } else {
        BAR();
        packer(P, L, wave * 8 + xg, wave, lane, call);
    }
}

}

extern "C" void kernel_launch(int rank, int nranks, ncclComm_t comm, hipStream_t stream,
                              void* const* d_in, const int* in_sizes, int n_in,
                              void* d_out, int out_size,
                              const int* in_dtypes, int out_dtype,
                              void* const* d_out_all,
                              void* const* d_ws_all, size_t ws_bytes) {
    (void)comm; (void)in_sizes; (void)n_in; (void)out_size; (void)in_dtypes; (void)out_dtype; (void)d_out_all;
    if (nranks != NR || (unsigned)rank >= (unsigned)NR || ws_bytes < WS_NEED) { fprintf(stderr, "unsupported configuration\n"); return; }
    static bool ready[NR];
    if (!ready[rank]) {
        HIP_CHECK(hipFuncSetAttribute((const void*)fused_kernel, hipFuncAttributeMaxDynamicSharedMemorySize, (int)LDS_BYTES));
        ready[rank] = true;
    }
    Params P;
    P.x = (const float*)d_in[0]; P.w = (const float*)d_in[1]; P.out = (float*)d_out;
    for (int i = 0; i < NR; ++i) P.ws[i] = (char*)d_ws_all[i];
    P.me = P.ws[rank]; P.rank = rank; P.pad = 0;
    hipLaunchKernelGGL(fused_kernel, dim3(NWG), dim3(NTHR), LDS_BYTES, stream, P);
    HIP_CHECK(hipGetLastError());
}
```
